```python
import numpy as np

kernel_name = "gpu_ag_gemm_m8192_k8192_n8192_bf16_silu"


ABSMAX_RTOL = 0.008

COMPUTE_DTYPE = "bfloat16"
EPILOGUE = "silu"
IN_QUANT = None
OUT_QUANT = None
IN_DTYPES = {"x": "bfloat16", "w_mat": "bfloat16"}
WORLD_SIZE = 8

def _to_bf16(x):
    u = np.ascontiguousarray(x, dtype=np.float32).view(np.uint32)
    return ((u + 0x7FFF + ((u >> 16) & 1)) & 0xFFFF0000).view(np.float32)

def setup_inputs():

    x = _to_bf16(np.random.randn(8192, 8192).astype(np.float32))
    w_mat = _to_bf16((np.random.randn(8192, 8192) / np.sqrt(8192)).astype(np.float32))
    return {"x": x, "w_mat": w_mat}

def _epilogue(y):
    return (y / (1.0 + np.exp(-np.clip(y, -60.0, 60.0)))).astype(np.float32)

def reference(x, w_mat):
    y = (x @ w_mat).astype(np.float32)
    return _epilogue(y)
```

```cpp
#include <hip/hip_runtime.h>
#include <rccl/rccl.h>
#include <cstdio>

#define CODEC 1
#define INFL 6
#define A_BITS " sc0 sc1"

namespace agg {
#define LAS __attribute__((address_space(3)))
typedef unsigned u32;
typedef unsigned long long u64;
typedef unsigned u32x4 __attribute__((ext_vector_type(4)));
typedef unsigned u32x2 __attribute__((ext_vector_type(2)));
typedef short s16x4 __attribute__((ext_vector_type(4)));
typedef short s16x8 __attribute__((ext_vector_type(8)));
typedef unsigned short u16x2 __attribute__((ext_vector_type(2)));
typedef float f32x4 __attribute__((ext_vector_type(4)));

constexpr int NR = 8, KD = 8192, NS = 1024;
constexpr int NCH = 64;
constexpr int NRF = 64;
constexpr int NB = 4;
constexpr int CPP = NRF / NB;
constexpr int ND = 7;
constexpr int SEG_STRIDE = NB * 4096 + 1024;
constexpr int SEG_P = NB * 2048, SEG_H = NB * 2560, SEG_V = SEG_H + NB * 4;
constexpr int NSLOT = 7;
constexpr int GRID = 256, NGEMM = 128;
constexpr int LDS_BYTES = 131072 + 1024;
static_assert(NSLOT * SEG_STRIDE <= 131072, "ring");
static_assert(7 * CPP <= GRID - NGEMM, "pullers");
constexpr u32 XCAP = 64;

constexpr size_t SHARD_B = (size_t)16 << 20;
constexpr size_t DEC_OFF = 0;
constexpr size_t ENC_OFF = NR * SHARD_B;
constexpr size_t ENC_SZ = (size_t)NCH * CPP * SEG_STRIDE;
constexpr size_t F1_OFF = ENC_OFF + 2 * ENC_SZ;
constexpr size_t F1_SZ = (size_t)NR * CPP * NCH * 8;
constexpr size_t F2_OFF = F1_OFF + 2 * F1_SZ;
constexpr size_t F2_SZ = (size_t)NR * NRF * NCH * 4;
constexpr size_t CNT_OFF = F2_OFF + F2_SZ;
constexpr size_t TMO_OFF = CNT_OFF + GRID * 128;
constexpr size_t DBG_OFF = TMO_OFF + 128;
constexpr size_t WS_NEED = DBG_OFF + (1 << 20);
constexpr u64 DEADLINE = 50000000ull;

struct Params { char* ws[NR]; char* myws; const char* x; const char* w; char* out; int rank; int pad; };

#define STAMP(i) do {} while (0)

#define WAIT_V(n) asm volatile("s_waitcnt vmcnt(" #n ")" ::: "memory")
#define WAIT_L(n) asm volatile("s_waitcnt lgkmcnt(" #n ")" ::: "memory")
#define BAR __builtin_amdgcn_s_barrier()
#define SCHED __builtin_amdgcn_sched_barrier(0)
#define LBAR do { WAIT_L(0); BAR; asm volatile("" ::: "memory"); } while (0)
template <int N> __device__ __forceinline__ void wait_v() { asm volatile("s_waitcnt vmcnt(%0)" :: "i"(N) : "memory"); }

__device__ __forceinline__ u32 rfl(u32 v) { return __builtin_amdgcn_readfirstlane(v); }
template <class T> __device__ __forceinline__ T* uni(T* q) {
  u64 v = (u64)q; v = ((u64)rfl((u32)(v >> 32)) << 32) | rfl((u32)v);
  asm volatile("s_nop 4" : "+s"(v));
  return (T*)v;
}
__device__ __forceinline__ u32 bfi(u32 m, u32 a, u32 b) { return (a & m) | (b & ~m); }
__device__ __forceinline__ void st4x4(char* base, u32 off, u32x4 a, u32x4 b, u32x4 c, u32x4 d) {
  asm volatile("global_store_dwordx4 %0, %1, %5 sc0 sc1\n\tglobal_store_dwordx4 %0, %2, %5 offset:1024 sc0 sc1\n\t"
               "global_store_dwordx4 %0, %3, %5 offset:2048 sc0 sc1\n\tglobal_store_dwordx4 %0, %4, %5 offset:3072 sc0 sc1\n\ts_nop 1"
               :: "v"(off), "v"(a), "v"(b), "v"(c), "v"(d), "s"(base) : "memory");
}
__device__ __forceinline__ void st4x2(char* base, u32 off, u32x4 a, u32x4 b) {
  asm volatile("global_store_dwordx4 %0, %1, %3 sc0 sc1\n\tglobal_store_dwordx4 %0, %2, %3 offset:1024 sc0 sc1\n\ts_nop 1"
               :: "v"(off), "v"(a), "v"(b), "s"(base) : "memory");
}
__device__ __forceinline__ void st4x1(char* base, u32 off, u32x4 a) {
  asm volatile("global_store_dwordx4 %0, %1, %2 sc0 sc1\n\ts_nop 1" :: "v"(off), "v"(a), "s"(base) : "memory");
}
__device__ __forceinline__ bool give_up(const Params& p, u32 call, u64 t0) {
  u32* tmo = (u32*)(p.myws + TMO_OFF);
  if (rfl(__hip_atomic_load(tmo, __ATOMIC_RELAXED, __HIP_MEMORY_SCOPE_AGENT)) == call) return true;
  if (__builtin_amdgcn_s_memrealtime() - t0 > DEADLINE) {
    __hip_atomic_store(tmo, call, __ATOMIC_RELAXED, __HIP_MEMORY_SCOPE_AGENT);
    WAIT_V(0);
    return true;
  }
  return false;
}
__device__ __forceinline__ u32 wave_scan(u32 v) {
  v += __builtin_amdgcn_update_dpp(0u, v, 0x111, 0xf, 0xf, false);
  v += __builtin_amdgcn_update_dpp(0u, v, 0x112, 0xf, 0xf, false);
  v += __builtin_amdgcn_update_dpp(0u, v, 0x114, 0xf, 0xf, false);
  v += __builtin_amdgcn_update_dpp(0u, v, 0x118, 0xf, 0xf, false);
  v += __builtin_amdgcn_update_dpp(0u, v, 0x142, 0xa, 0xf, false);
  v += __builtin_amdgcn_update_dpp(0u, v, 0x143, 0xc, 0xf, false);
  return v;
}
struct Mv { u32 a[5]; };
__device__ __forceinline__ Mv move_masks(u32 m) {
  Mv r; u32 mk = ~m << 1;
#pragma unroll
  for (int i = 0; i < 5; ++i) {
    u32 mp = mk ^ (mk << 1); mp ^= mp << 2; mp ^= mp << 4; mp ^= mp << 8; mp ^= mp << 16;
    const u32 mv = mp & m; r.a[i] = mv; m = (m ^ mv) | (mv >> (1 << i)); mk &= ~mp;
  }
  return r;
}
__device__ __forceinline__ u32 pext(u32 x, u32 m, const Mv& v) {
  x &= m;
#pragma unroll
  for (int i = 0; i < 5; ++i) { const u32 t = x & v.a[i]; x = (x ^ t) | (t >> (1 << i)); }
  return x;
}
__device__ __forceinline__ u32 pdep(u32 x, u32 m, const Mv& v) {
#pragma unroll
  for (int i = 4; i >= 0; --i) x = bfi(v.a[i], x << (1 << i), x);
  return x & m;
}
__device__ __forceinline__ void transpose4(u32 (&n)[4]) {
#pragma unroll
  for (int r = 0; r < 2; ++r) { const u32 a = n[r], c = n[r + 2]; n[r] = bfi(0x33333333u, a, c << 2); n[r + 2] = bfi(0x33333333u, a >> 2, c); }
#pragma unroll
  for (int r = 0; r < 4; r += 2) { const u32 a = n[r], c = n[r + 1]; n[r] = bfi(0x55555555u, a, c << 1); n[r + 1] = bfi(0x55555555u, a >> 1, c); }
}
__device__ __forceinline__ void or_bits(LAS u32* area, u32 bit, u32 val) {
  const u64 t = (u64)val << (bit & 31);
  LAS u32* q = area + (bit >> 5);
  __hip_atomic_fetch_or(q, (u32)t, __ATOMIC_RELAXED, __HIP_MEMORY_SCOPE_WORKGROUP);
  __hip_atomic_fetch_or(q + 1, (u32)(t >> 32), __ATOMIC_RELAXED, __HIP_MEMORY_SCOPE_WORKGROUP);
}
__device__ __forceinline__ u32 get_bits(const LAS u32* area, u32 bit) {
  const LAS u32* q = area + (bit >> 5);
  return __builtin_amdgcn_alignbit(q[1], q[0], bit & 31);
}

constexpr int ETAIL = SEG_STRIDE - SEG_P;
__device__ __forceinline__ void encode_phase(const Params& p, LAS char* lds, int e, u32 call, int w, int lane, const u32x4 (&X)[4][4]) {
  char* enc = uni(p.myws + ENC_OFF + (call & 1) * ENC_SZ);
  char* dec = uni(p.myws + DEC_OFF + (size_t)p.rank * SHARD_B);
  LAS u32* szw = (LAS u32*)(lds + 131072 + 64);
  const int sg = w / NB, rb = w % NB;
  LAS char* img = lds + sg * ETAIL;
#pragma unroll
  for (int r = 0; r < 4; ++r) {
    const int B = (r * NGEMM + e) * 8 + w, c = B >> 6, rf = B & 63, g = rf / NB;
    const u32 segoff = (u32)(c * CPP + g) * SEG_STRIDE;
    st4x4(dec, (u32)B * 4096u + lane * 16, X[r][0], X[r][1], X[r][2], X[r][3]);
    u32 len;
    {
      LAS u32x4* z = (LAS u32x4*)(img + (SEG_V - SEG_P) + rb * 1536);
      const u32x4 zero = {0u, 0u, 0u, 0u};
      z[lane] = zero; if (lane < 36) z[64 + lane] = zero;
    }
    u32 R[16], sum = 0;
#pragma unroll
    for (int i = 0; i < 16; ++i) { const u32 t = X[r][i >> 2][i & 3]; R[i] = __builtin_amdgcn_alignbit(t, t, 31); sum = __builtin_amdgcn_udot4(R[i], 0x01000100u, sum, false); }
    sum = __builtin_amdgcn_readlane(wave_scan(sum), 63);
    int basei = (int)((sum + 1741u) >> 11) - 10;
    basei = basei < 0 ? 0 : (basei > 240 ? 240 : basei);
    const u32 base = (u32)basei;
    const u16x2 bsub = {(unsigned short)(base << 8), (unsigned short)(base << 8)};
    u32 S[8], V[8];
#pragma unroll
    for (int q = 0; q < 8; ++q) {
      const u32 a = __builtin_bit_cast(u32, (u16x2)(__builtin_bit_cast(u16x2, R[2 * q]) - bsub));
      const u32 b = __builtin_bit_cast(u32, (u16x2)(__builtin_bit_cast(u16x2, R[2 * q + 1]) - bsub));
      S[q] = __builtin_amdgcn_perm(b, a, 0x06040200u); V[q] = __builtin_amdgcn_perm(b, a, 0x07050301u);
    }
    u32 d[4], h[4];
#pragma unroll
    for (int q = 0; q < 4; ++q) { d[q] = bfi(0x0f0f0f0fu, V[2 * q], V[2 * q + 1] << 4); h[q] = bfi(0x0f0f0f0fu, V[2 * q] >> 4, V[2 * q + 1]); }
    transpose4(d); transpose4(h);
    u32 Xm = (h[0] | h[1] | h[2] | h[3]) | (d[3] & d[2] & d[1]);
    const u32 m1 = d[3] & ~d[2];
    const u32 P1 = d[1] & m1 & ~Xm, P0 = (d[0] & m1) | Xm;
    const u32 E1 = ~(P1 | P0);
    const u32 Q1 = d[3] & ~d[0], Q0 = d[2] & bfi(d[3], ~d[0], d[1] & d[0]);
    const u32 E2 = E1 & ~(Q1 | Q0);
    const u32 nx = __builtin_popcount(Xm);
    const u32 cnt = (u32)__builtin_popcount(E1) | ((u32)__builtin_popcount(E2) << 12) | ((nx > 3u ? 3u : nx) << 24);
    const u32 inc = wave_scan(cnt), tot = __builtin_amdgcn_readlane(inc, 63), exc = inc - cnt;
    const u32 N1 = tot & 0xfffu, N2 = (tot >> 12) & 0xfffu, NX = tot >> 24;
    const bool raw = NX > XCAP || __any(nx > 3u);
    const u32 bitw = (2 * N1 + 3 * N2 + 31) >> 5;
    const u32 vd = raw ? 384u : bitw + NX;
    if (lane == 0) szw[w] = vd;
    LBAR;
    u32 voff = 0, vtot = 0;
#pragma unroll
    for (int i = 0; i < NB; ++i) { const u32 s = rfl(szw[sg * NB + i]); vtot += s; if (i < rb) voff += s; }
    LAS u32* var = (LAS u32*)(img + (SEG_V - SEG_P)) + voff;
    if (lane == 0) ((LAS u32*)(img + (SEG_H - SEG_P)))[rb] = base | ((raw ? 255u : NX) << 8) | (voff << 16);
    if (!raw) {
      *(LAS u32x2*)(img + rb * 512 + lane * 8) = (u32x2){P1, P0};
      const u32 o1 = exc & 0xfffu, o2 = 2 * N1 + ((exc >> 12) & 0xfffu);
      { const Mv mv = move_masks(E1); or_bits(var, o1, pext(Q1, E1, mv)); or_bits(var, N1 + o1, pext(Q0, E1, mv)); }
      { const Mv mv = move_masks(E2); or_bits(var, o2, pext(d[2], E2, mv)); or_bits(var, o2 + N2, pext(d[1] | d[3], E2, mv)); or_bits(var, o2 + 2 * N2, pext(d[0], E2, mv)); }
      u32 xo = bitw + (exc >> 24);
      while (__any(Xm != 0u)) {
        if (Xm != 0u) {
          const u32 j = __builtin_ctz(Xm); Xm &= Xm - 1;
          const u32 vi = 2 * (j & 3) + ((j >> 2) & 1), by = j >> 3;
          u32 vs = V[0];
#pragma unroll
          for (int i = 1; i < 8; ++i) vs = vi == (u32)i ? V[i] : vs;
          const u32 ev = (base + (vs >> (8 * by))) & 255u;
          __hip_atomic_fetch_or(var + xo, (u32)lane | (vi << 6) | (by << 9) | ((ev ^ (base + 9u)) << 16), __ATOMIC_RELAXED, __HIP_MEMORY_SCOPE_WORKGROUP);
          ++xo;
        }
      }
    } else {
      u32 E[8];
#pragma unroll
      for (int q = 0; q < 8; ++q) E[q] = __builtin_amdgcn_perm(R[2 * q + 1], R[2 * q], 0x07050301u);
      *(LAS u32x2*)(img + rb * 512 + lane * 8) = (u32x2){E[0], E[1]};
#pragma unroll
      for (int q = 2; q < 8; ++q) __hip_atomic_fetch_or(var + (q - 2) * 64 + lane, E[q], __ATOMIC_RELAXED, __HIP_MEMORY_SCOPE_WORKGROUP);
    }
    st4x2(enc, segoff + rb * 2048 + lane * 16, (u32x4){S[0], S[1], S[2], S[3]}, (u32x4){S[4], S[5], S[6], S[7]});
    LBAR;
    len = (SEG_V + vtot * 4 + 15) & ~15u;
    for (u32 o = rb * 1024 + lane * 16; o < len - SEG_P; o += NB * 1024) st4x1(enc, segoff + SEG_P + o, *(LAS u32x4*)(img + o));
    WAIT_V(0);
    if (lane == 0) __hip_atomic_store((u32*)(p.myws + F2_OFF) + ((p.rank * NRF + rf) * NCH + c), call, __ATOMIC_RELAXED, __HIP_MEMORY_SCOPE_SYSTEM);
    LBAR;
    if (rb == 0 && lane < NR && lane != p.rank) {
      u64* f = (u64*)(p.ws[lane] + F1_OFF + (call & 1) * F1_SZ) + ((p.rank * CPP + g) * NCH + c);
      __hip_atomic_store(f, ((u64)call << 32) | len, __ATOMIC_RELAXED, __HIP_MEMORY_SCOPE_SYSTEM);
    }
    if (r == 0) STAMP(1);
  }
}

__device__ __forceinline__ u32 lds_ld(LAS u32* q) { return rfl(__hip_atomic_load(q, __ATOMIC_ACQUIRE, __HIP_MEMORY_SCOPE_WORKGROUP)); }
__device__ __forceinline__ void lds_post(LAS u32* q, u32 v) { asm volatile("ds_write_b32 %0, %1" :: "v"((u32)(size_t)q), "v"(v) : "memory"); }
__device__ __forceinline__ u32 lds_peek(LAS u32* q) { u32 v; asm volatile("ds_read_b32 %0, %1\n\ts_waitcnt lgkmcnt(0)" : "=&v"(v) : "v"((u32)(size_t)q) : "memory"); return rfl(v); }
constexpr u32 ABORT = 0x40000000u;

__device__ __forceinline__ void pull_role(const Params& p, LAS char* lds, int src, int g, u32 call, int w, int lane) {
  LAS u32* landed = (LAS u32*)(lds + 131072 + 128);
  LAS u32* done = (LAS u32*)(lds + 131072 + 192);
  if (w == 0) {
    const u64* f1 = (const u64*)(p.myws + F1_OFF + (call & 1) * F1_SZ) + ((src * CPP + g) * NCH + lane);
    const char* rsrc = uni(p.ws[src] + ENC_OFF + (call & 1) * ENC_SZ + (size_t)g * SEG_STRIDE);
    u64 ready = 0; u32 lens = 0; int slot = 0; u32 use = 0;
    for (int c = 0; c < NCH; ++c) {
      if (!((ready >> c) & 1)) {
        WAIT_V(0); lds_post(landed, c);
        const u64 t0 = __builtin_amdgcn_s_memrealtime();
        for (;;) {
          const u64 v = __hip_atomic_load(f1, __ATOMIC_RELAXED, __HIP_MEMORY_SCOPE_SYSTEM);
          lens = (u32)v; ready = __ballot((u32)(v >> 32) >= call);
          if ((ready >> c) & 1) break;
          if (give_up(p, call, t0)) { lds_post(landed, ABORT); return; }
          __builtin_amdgcn_s_sleep(2);
        }
        if (c == 0) STAMP(1);
      }
      if (lds_peek(done + slot) < use) {
        WAIT_V(0); lds_post(landed, c);
        while (lds_peek(done + slot) < use) __builtin_amdgcn_s_sleep(1);
      }
      u32 len = __builtin_amdgcn_readlane(lens, c); len = len > (u32)SEG_STRIDE ? (u32)SEG_STRIDE : len;
      const char* gs = rsrc + (size_t)c * (CPP * SEG_STRIDE);
      const u32 ls = (u32)(size_t)lds + slot * SEG_STRIDE;
      for (u32 o = 0; o < len; o += 1024) {
        wait_v<INFL - 1>();
        if (o == (INFL - 1) * 1024) lds_post(landed, c);
        if (o + lane * 16 < len) asm volatile("s_mov_b32 m0, %2\n\ts_nop 0\n\tglobal_load_lds_dwordx4 %0, %1 sc0 sc1" :: "v"(o + lane * 16), "s"(gs), "s"(ls + o) : "memory");
      }
      if (++slot == NSLOT) { slot = 0; use += NB; }
    }
    WAIT_V(0); lds_post(landed, NCH);
    STAMP(2);
    return;
  }
  if (w > ND) return;
  char* dec = uni(p.myws + DEC_OFF + (size_t)src * SHARD_B);
  u32* f2 = (u32*)(p.myws + F2_OFF) + (src * NRF + g * NB) * NCH;
  u32 have = 0;
  for (int q = w - 1; q < NCH * NB; q += ND) {
    const int c = q / NB, rb = q % NB, slot = c % NSLOT;
    while (have <= (u32)c) { have = lds_peek(landed); if (have <= (u32)c) __builtin_amdgcn_s_sleep(1); }
    if (have == ABORT) return;
    const LAS char* ls = lds + slot * SEG_STRIDE;
    u32x4 O[4];
    const u32x4 Sa = *(const LAS u32x4*)(ls + rb * 2048 + lane * 16), Sb = *(const LAS u32x4*)(ls + rb * 2048 + 1024 + lane * 16);
    const u32x2 Pp = *(const LAS u32x2*)(ls + SEG_P + rb * 512 + lane * 8);
    const u32 hdr = rfl(*(const LAS u32*)(ls + SEG_H + rb * 4));
    const u32 base = hdr & 255u, NX = (hdr >> 8) & 255u;
    const LAS u32* var = (const LAS u32*)(ls + SEG_V) + (hdr >> 16);
    u32 E[8];
    if (NX != 255u) {
      const u32 P1 = Pp[0], P0 = Pp[1], E1 = ~(P1 | P0);
      const u32 n1 = __builtin_popcount(E1), i1 = wave_scan(n1), N1 = __builtin_amdgcn_readlane(i1, 63), o1 = i1 - n1;
      const u32 q1 = get_bits(var, o1), q0 = get_bits(var, N1 + o1);
      const Mv ma = move_masks(E1);
      const u32 Q1 = pdep(q1, E1, ma), Q0 = pdep(q0, E1, ma), E2 = E1 & ~(Q1 | Q0);
      const u32 n2 = __builtin_popcount(E2), i2 = wave_scan(n2), N2 = __builtin_amdgcn_readlane(i2, 63), o2 = 2 * N1 + i2 - n2;
      const u32 r2 = get_bits(var, o2), r1 = get_bits(var, o2 + N2), r0 = get_bits(var, o2 + 2 * N2);
      const u32 xe = var[((2 * N1 + 3 * N2 + 31) >> 5) + lane];
      const Mv mb = move_masks(E2);
      const u32 R2 = pdep(r2, E2, mb), R1 = pdep(r1, E2, mb), R0 = pdep(r0, E2, mb);
      const u32 t7 = R2 & R1 & R0, q7 = ~Q1 & Q0;
      u32 d[4] = {P0 | q7 | R0, P1 | q7 | (R1 & ~t7), Q0 | R2, ~E1 | Q1 | t7};
      transpose4(d);
      const u32 b4 = base * 0x01010101u;
#pragma unroll
      for (int i = 0; i < 4; ++i) { E[2 * i] = (d[i] & 0x0f0f0f0fu) + b4; E[2 * i + 1] = ((d[i] >> 4) & 0x0f0f0f0fu) + b4; }
      for (u32 i = 0; i < NX && i < 64u; ++i) {
        const u32 en = __builtin_amdgcn_readlane(xe, i);
        const u32 dv = (u32)lane == (en & 63u) ? ((en >> 16) & 255u) << (8 * ((en >> 9) & 3u)) : 0u, vi = (en >> 6) & 7u;
#pragma unroll
        for (int k = 0; k < 8; ++k) E[k] ^= vi == (u32)k ? dv : 0u;
      }
    } else {
      E[0] = Pp[0]; E[1] = Pp[1];
#pragma unroll
      for (int k = 2; k < 8; ++k) E[k] = var[(k - 2) * 64 + lane];
    }
#pragma unroll
    for (int k = 0; k < 8; ++k) {
      const u32 s = k < 4 ? Sa[k & 3] : Sb[k & 3];
      const u32 a = __builtin_amdgcn_perm(E[k], s, 0x05010400u), b = __builtin_amdgcn_perm(E[k], s, 0x07030602u);
      O[k >> 1][(k & 1) * 2] = __builtin_amdgcn_alignbit(a, a, 1); O[k >> 1][(k & 1) * 2 + 1] = __builtin_amdgcn_alignbit(b, b, 1);
    }
    st4x4(dec, (u32)(c * NRF + g * NB + rb) * 4096u + lane * 16, O[0], O[1], O[2], O[3]);
    if (lane == 0) __hip_atomic_fetch_add(done + slot, 1u, __ATOMIC_RELAXED, __HIP_MEMORY_SCOPE_WORKGROUP);
    WAIT_V(0);
    if (lane == 0) __hip_atomic_store(f2 + rb * NCH + c, call, __ATOMIC_RELAXED, __HIP_MEMORY_SCOPE_SYSTEM);
    if (q < ND) STAMP(1);
  }
  STAMP(2);
}

__device__ __forceinline__ int wait_chunks(const Params& p, const u32* f, int need, u32 call) {
  const u64 t0 = __builtin_amdgcn_s_memrealtime();
  for (;;) {
    const u32 a = __hip_atomic_load(f, __ATOMIC_RELAXED, __HIP_MEMORY_SCOPE_SYSTEM);
    const u32 b = __hip_atomic_load(f + 8 * NCH, __ATOMIC_RELAXED, __HIP_MEMORY_SCOPE_SYSTEM);
    const u64 miss = __ballot(a < call || b < call);
    const int n = miss ? __builtin_ctzll(miss) : NCH;
    if (n >= need) return n;
    if (give_up(p, call, t0)) return NCH;
    __builtin_amdgcn_s_sleep(4);
  }
}
__device__ __forceinline__ u32 cvt_pk(float lo, float hi) { u32 r; asm("v_cvt_pk_bf16_f32 %0, %1, %2" : "=v"(r) : "v"(lo), "v"(hi)); return r; }
__device__ __forceinline__ float silu(float y) {
  const float c = __builtin_amdgcn_fmed3f(y, -60.0f, 60.0f);
  return y * __builtin_amdgcn_rcpf(1.0f + __builtin_amdgcn_exp2f(c * -1.44269504088896341f));
}

__device__ __forceinline__ void gemm_role(const Params& p, LAS char* lds, int src, int tm, int tn, u32 call, int w, int lane) {
  constexpr int HT = 16384;
  const int wr = w >> 2, wc = w & 3;
  const char* Ab = uni(p.myws + DEC_OFF + (size_t)src * SHARD_B + (size_t)tm * 16 * 4096);
  const char* Bb = uni(p.w + tn * 512);
  const u32 voffA = w * 4096 + lane * 16;
  const u32 voffB = (4 * w + (lane >> 4)) * 2048 + (((lane & 15) ^ (((lane >> 4) << 2) | (w & 3))) << 4);
  const u32* flg = (const u32*)(p.myws + F2_OFF) + (src * NRF + tm * 16 + w) * NCH + lane;
  const u32 lds0 = (u32)(size_t)lds, ldsA = lds0 + w * 2048, ldsB = lds0 + w * 1024;
  const int aoff = wr * 8192 + lane * 16;
  u32 boff[2][2];
  { const int g = lane >> 4, q = (lane & 15) >> 2, pp = lane & 3;
#pragma unroll
    for (int n = 0; n < 2; ++n)
#pragma unroll
      for (int t = 0; t < 2; ++t) { boff[n][t] = lds0 + 4 * HT + 256 * (8 * g + 4 * t + q) + 16 * ((4 * wc + 2 * n + (pp >> 1)) ^ ((q << 2) | ((2 * g + t) & 3))) + 8 * (pp & 1);
        asm volatile("" : "+v"(boff[n][t])); } }
#define SA(b, h) (((b) * 2 + (h)) * HT)
#define SB(b, h) ((4 + (b) * 2 + (h)) * HT)
#define STA(buf, gp) asm volatile("s_mov_b32 m0, %2\n\ts_nop 0\n\tglobal_load_lds_dwordx4 %0, %1" A_BITS "\n\tglobal_load_lds_dwordx4 %0, %1 offset:1024" A_BITS \
    :: "v"(voffA), "s"(gp), "s"(ldsA + (buf)) : "memory")
#define STB(buf, gp) asm volatile("s_mov_b32 m0, %3\n\ts_nop 0\n\tglobal_load_lds_dwordx4 %0, %1\n\ts_add_u32 m0, m0, 0x2000\n\ts_nop 0\n\tglobal_load_lds_dwordx4 %0, %2" \
    :: "v"(voffB), "s"(gp), "s"((gp) + 65536), "s"(ldsB + (buf)) : "memory", "scc")
#define LDA(dst, b, h) do { _Pragma("unroll") for (int m = 0; m < 4; ++m) _Pragma("unroll") for (int k = 0; k < 2; ++k) \
    dst[m][k] = *(const LAS s16x8*)(lds + SA(b, h) + aoff + m * 2048 + k * 1024); } while (0)
#define LDB(dst, b, h) do { _Pragma("unroll") for (int n = 0; n < 2; ++n) _Pragma("unroll") for (int k = 0; k < 2; ++k) { \
    const s16x4 lo_ = __builtin_amdgcn_ds_read_tr16_b64_v4i16((LAS s16x4*)(size_t)(boff[n][0] + SA(b, h) + k * 8192)); \
    const s16x4 hi_ = __builtin_amdgcn_ds_read_tr16_b64_v4i16((LAS s16x4*)(size_t)(boff[n][1] + SA(b, h) + k * 8192)); \
    dst[n][k] = __builtin_shufflevector(lo_, hi_, 0, 1, 2, 3, 4, 5, 6, 7); } } while (0)
#define MMA(ai, bj, At, Bt) do { __builtin_amdgcn_s_setprio(1); _Pragma("unroll") for (int m = 0; m < 4; ++m) _Pragma("unroll") for (int n = 0; n < 2; ++n) \
    _Pragma("unroll") for (int k = 0; k < 2; ++k) acc[ai][bj][m][n] = __builtin_amdgcn_mfma_f32_16x16x32_bf16(__builtin_bit_cast(bf8, Bt[n][k]), __builtin_bit_cast(bf8, At[m][k]), acc[ai][bj][m][n], 0, 0, 0); \
    __builtin_amdgcn_s_setprio(0); } while (0)
  typedef __bf16 bf8 __attribute__((ext_vector_type(8)));
  f32x4 acc[2][2][4][2];
#pragma unroll
  for (int a = 0; a < 2; ++a)
#pragma unroll
    for (int b = 0; b < 2; ++b)
#pragma unroll
      for (int m = 0; m < 4; ++m)
#pragma unroll
        for (int n = 0; n < 2; ++n) acc[a][b][m][n] = (f32x4){0.f, 0.f, 0.f, 0.f};
  s16x8 At[4][2], B0[2][2], B1[2][2];
  int avail = wait_chunks(p, flg, 1, call);
  STAMP(3);
  STB(SB(0, 0), Bb); STA(SA(0, 0), Ab); STB(SB(0, 1), Bb + 256); STA(SA(0, 1), Ab + 32768);
  if (wr == 1) BAR;
  WAIT_V(4); BAR;
  STB(SB(1, 0), Bb + 131072); STA(SA(1, 0), Ab + 2048); STB(SB(1, 1), Bb + 131072 + 256);
  WAIT_V(6); BAR;
  for (int c = 0; c < NCH; ++c) {
    const int cn = c + 1 < NCH ? c + 1 : c;
    const char* a1 = Ab + (size_t)c * 262144 + 2048;
    const char* a2 = Ab + (size_t)cn * 262144;
    const char* b2 = Bb + (size_t)cn * 262144;
    LDB(B0, 0, 0); SCHED; LDA(At, 0, 0); STA(SA(1, 1), a1 + 32768);
    WAIT_L(8); BAR; WAIT_L(0); MMA(0, 0, At, B0); BAR; SCHED;
    LDB(B1, 0, 1); STB(SB(0, 0), b2);
    BAR; WAIT_L(0); MMA(0, 1, At, B1); BAR;
    if (avail <= cn) avail = wait_chunks(p, flg, cn + 1, call);
    LDA(At, 0, 1); STA(SA(0, 0), a2);
    BAR; WAIT_L(0); MMA(1, 0, At, B0); BAR; SCHED;
    STB(SB(0, 1), b2 + 256);
    WAIT_V(6); BAR; MMA(1, 1, At, B1); BAR;
    LDB(B0, 1, 0); SCHED; LDA(At, 1, 0); STA(SA(0, 1), a2 + 32768);
    WAIT_L(8); BAR; WAIT_L(0); MMA(0, 0, At, B0); BAR; SCHED;
    LDB(B1, 1, 1); STB(SB(1, 0), b2 + 131072);
    BAR; WAIT_L(0); MMA(0, 1, At, B1); BAR;
    LDA(At, 1, 1); STA(SA(1, 0), a2 + 2048);
    BAR; WAIT_L(0); MMA(1, 0, At, B0); BAR; SCHED;
    STB(SB(1, 1), b2 + 131072 + 256);
    WAIT_V(6); BAR; MMA(1, 1, At, B1); BAR;
  }
  if (wr == 0) BAR;
  STAMP(4);
  const bool bad = rfl(__hip_atomic_load((u32*)(p.myws + TMO_OFF), __ATOMIC_RELAXED, __HIP_MEMORY_SCOPE_AGENT)) == call;
  int tz = threadIdx.x; asm volatile("" : "+v"(tz));
  const int fr = tz & 15, fq = (tz >> 4) & 3;
  char* ob = p.out + ((size_t)(src * 1024 + tm * 256 + wr * 64 + fr) * NS + tn * 256 + wc * 32 + ((fq & 1) << 4) + ((fq >> 1) << 3)) * 2;
#pragma unroll
  for (int ai = 0; ai < 2; ++ai)
#pragma unroll
    for (int m = 0; m < 4; ++m)
#pragma unroll
      for (int bj = 0; bj < 2; ++bj) {
        const f32x4 u = acc[ai][bj][m][0], v = acc[ai][bj][m][1];
        const u32 a0 = cvt_pk(silu(u[0]), silu(u[1])), a1 = cvt_pk(silu(u[2]), silu(u[3]));
        const u32 b0 = cvt_pk(silu(v[0]), silu(v[1])), b1 = cvt_pk(silu(v[2]), silu(v[3]));
        const auto s0 = __builtin_amdgcn_permlane16_swap(a0, b0, false, false);
        const auto s1 = __builtin_amdgcn_permlane16_swap(a1, b1, false, false);
        u32x4 o = {s0[0], s1[0], s0[1], s1[1]};
        if (bad) o = (u32x4){0x7fc07fc0u, 0x7fc07fc0u, 0x7fc07fc0u, 0x7fc07fc0u};
        *(u32x4*)(ob + (size_t)(ai * 128 + m * 16) * (NS * 2) + bj * 256) = o;
      }
  STAMP(5);
}

__global__ __launch_bounds__(512, 2) void ag_gemm(Params p) {
  extern __shared__ __attribute__((aligned(16))) char smem[];
  LAS char* lds = (LAS char*)smem;
  const int b = blockIdx.x, lane = threadIdx.x & 63, w = rfl(threadIdx.x >> 6);
  STAMP(0);
  const bool is_gemm = b < NGEMM;
  if (!is_gemm && b - NGEMM >= 7 * CPP) return;
  u32x4 X[4][4];
  if (is_gemm) {
#pragma unroll
    for (int r = 0; r < 4; ++r) {
      const int B = (r * NGEMM + b) * 8 + w, c = B >> 6, rf = B & 63;
      const char* xp = p.x + ((size_t)(rf * 16 + (lane & 15)) * KD + c * 128 + (lane >> 4) * 8) * 2;
#pragma unroll
      for (int k = 0; k < 4; ++k) X[r][k] = *(const u32x4*)(xp + k * 64);
    }
  }
  LAS u32* sw = (LAS u32*)(lds + 131072);
  if (threadIdx.x < 64) {
    if (lane == 0) { u32* cp = (u32*)(p.myws + CNT_OFF + b * 128); const u32 v = *cp + 1; *cp = v; sw[0] = v; }
    else if (lane >= 32) sw[lane] = 0;
  }
  LBAR;
  const u32 call = rfl(sw[0]);
  if (is_gemm) {
    encode_phase(p, lds, b, call, w, lane, X);
    STAMP(2);
    gemm_role(p, lds, (p.rank + (b & 7)) & 7, b >> 5, (b >> 3) & 3, call, w, lane);
  } else {
    const int i = b - NGEMM;
    pull_role(p, lds, (p.rank + 1 + i % 7) & 7, i / 7, call, w, lane);
  }
}
}

extern "C" void kernel_launch(int rank, int nranks, ncclComm_t comm, hipStream_t stream,
                              void* const* d_in, const int* in_sizes, int n_in,
                              void* d_out, int out_size, const int* in_dtypes, int out_dtype,
                              void* const* d_out_all, void* const* d_ws_all, size_t ws_bytes) {
  using namespace agg;
  if (nranks != NR || (unsigned)rank >= (unsigned)NR || ws_bytes < WS_NEED || n_in < 2) { fprintf(stderr, "ag_gemm: unexpected configuration\n"); return; }
  Params p;
  for (int i = 0; i < NR; ++i) p.ws[i] = (char*)d_ws_all[i];
  p.myws = p.ws[rank]; p.x = (const char*)d_in[0]; p.w = (const char*)d_in[1]; p.out = (char*)d_out; p.rank = rank; p.pad = 0;
  hipError_t e = hipFuncSetAttribute((const void*)ag_gemm, hipFuncAttributeMaxDynamicSharedMemorySize, LDS_BYTES);
  if (e != hipSuccess) { fprintf(stderr, "HIP error %s\n", hipGetErrorString(e)); return; }
  hipLaunchKernelGGL(ag_gemm, dim3(GRID), dim3(512), LDS_BYTES, stream, p);
  e = hipGetLastError();
  if (e != hipSuccess) fprintf(stderr, "HIP error %s\n", hipGetErrorString(e));
}
```

```cpp
#include <hip/hip_runtime.h>
#include <rccl/rccl.h>
#include <cstdio>

#define CODEC 1
#define INFL 6
#define A_BITS " sc0 sc1"

namespace agg {
#define LAS __attribute__((address_space(3)))
typedef unsigned u32;
typedef unsigned long long u64;
typedef unsigned u32x4 __attribute__((ext_vector_type(4)));
typedef unsigned u32x2 __attribute__((ext_vector_type(2)));
typedef short s16x4 __attribute__((ext_vector_type(4)));
typedef short s16x8 __attribute__((ext_vector_type(8)));
typedef unsigned short u16x2 __attribute__((ext_vector_type(2)));
typedef float f32x4 __attribute__((ext_vector_type(4)));

constexpr int NR = 8, KD = 8192, NS = 1024;
constexpr int NCH = 64;
constexpr int NRF = 64;
constexpr int NB = 4;
constexpr int SPC = NRF / NB;
constexpr int NSEG = NCH * SPC;
#define CPP 8
constexpr int ND = 6;
#define NENC 32
constexpr int SEG_STRIDE = NB * 4096 + 1024;
constexpr int SEG_P = NB * 2048, SEG_H = NB * 2560, SEG_V = SEG_H + NB * 4;
constexpr int NSLOT = 7;
constexpr int GRID = 256, NGEMM = 128;
constexpr int LDS_BYTES = 131072 + 1024;
static_assert(NSLOT * SEG_STRIDE <= 131072, "ring");
static_assert(7 * CPP + NENC <= GRID - NGEMM, "roles");
constexpr u32 XCAP = 64;

constexpr size_t SHARD_B = (size_t)16 << 20;
constexpr size_t DEC_OFF = 0;
constexpr size_t ENC_OFF = NR * SHARD_B;
constexpr size_t ENC_SZ = (size_t)NSEG * SEG_STRIDE;
constexpr size_t F1_OFF = ENC_OFF + 2 * ENC_SZ;
constexpr size_t F1_SZ = (size_t)NR * NSEG * 8;
constexpr size_t F2_OFF = F1_OFF + 2 * F1_SZ;
constexpr size_t F2_SZ = (size_t)NR * NRF * NCH * 4;
constexpr size_t CNT_OFF = F2_OFF + F2_SZ;
constexpr size_t TKT_OFF = CNT_OFF + GRID * 128;
constexpr size_t TMO_OFF = TKT_OFF + NR * 128;
constexpr size_t DBG_OFF = TMO_OFF + 128;
constexpr size_t WS_NEED = DBG_OFF + (1 << 20);
constexpr u64 DEADLINE = 50000000ull;

struct Params { char* ws[NR]; char* myws; const char* x; const char* w; char* out; int rank; int pad; };

#define GMASK 0
__device__ __forceinline__ int role_id() {
  const int x = blockIdx.x & 7, j = blockIdx.x >> 3;
  if (GMASK == 0) return blockIdx.x;
  if ((GMASK >> x) & 1) { const int gi = __builtin_popcount(GMASK & ((1 << x) - 1)); return (j & 15) * 8 + gi * 2 + (j >> 4); }
  return NGEMM + __builtin_popcount(~GMASK & ((1 << x) - 1)) + 4 * j;
}
#define STAMP(i) do {} while (0)
#define STAMPC(i) do {} while (0)

#define WAIT_V(n) asm volatile("s_waitcnt vmcnt(" #n ")" ::: "memory")
#define WAIT_L(n) asm volatile("s_waitcnt lgkmcnt(" #n ")" ::: "memory")
#define BAR __builtin_amdgcn_s_barrier()
#define SCHED __builtin_amdgcn_sched_barrier(0)
#define LBAR do { WAIT_L(0); BAR; asm volatile("" ::: "memory"); } while (0)
template <int N> __device__ __forceinline__ void wait_v() { asm volatile("s_waitcnt vmcnt(%0)" :: "i"(N) : "memory"); }

__device__ __forceinline__ u32 rfl(u32 v) { return __builtin_amdgcn_readfirstlane(v); }
template <class T> __device__ __forceinline__ T* uni(T* q) {
  u64 v = (u64)q; v = ((u64)rfl((u32)(v >> 32)) << 32) | rfl((u32)v);
  asm volatile("s_nop 4" : "+s"(v));
  return (T*)v;
}
__device__ __forceinline__ void st4x4(char* base, u32 off, u32x4 a, u32x4 b, u32x4 c, u32x4 d) {
  asm volatile("global_store_dwordx4 %0, %1, %5 sc0 sc1\n\tglobal_store_dwordx4 %0, %2, %5 offset:1024 sc0 sc1\n\t"
               "global_store_dwordx4 %0, %3, %5 offset:2048 sc0 sc1\n\tglobal_store_dwordx4 %0, %4, %5 offset:3072 sc0 sc1\n\ts_nop 1"
               :: "v"(off), "v"(a), "v"(b), "v"(c), "v"(d), "s"(base) : "memory");
}
#define D_BITS " sc0 sc1"
__device__ __forceinline__ void st4x4d(char* base, u32 off, u32x4 a, u32x4 b, u32x4 c, u32x4 d) {
  asm volatile("global_store_dwordx4 %0, %1, %5" D_BITS "\n\tglobal_store_dwordx4 %0, %2, %5 offset:1024" D_BITS "\n\t"
               "global_store_dwordx4 %0, %3, %5 offset:2048" D_BITS "\n\tglobal_store_dwordx4 %0, %4, %5 offset:3072" D_BITS "\n\ts_nop 1"
               :: "v"(off), "v"(a), "v"(b), "v"(c), "v"(d), "s"(base) : "memory");
}
__device__ __forceinline__ void st4x2(char* base, u32 off, u32x4 a, u32x4 b) {
  asm volatile("global_store_dwordx4 %0, %1, %3 sc0 sc1\n\tglobal_store_dwordx4 %0, %2, %3 offset:1024 sc0 sc1\n\ts_nop 1"
               :: "v"(off), "v"(a), "v"(b), "s"(base) : "memory");
}
__device__ __forceinline__ void st4x1(char* base, u32 off, u32x4 a) {
  asm volatile("global_store_dwordx4 %0, %1, %2 sc0 sc1\n\ts_nop 1" :: "v"(off), "v"(a), "s"(base) : "memory");
}
__device__ __forceinline__ bool give_up(const Params& p, u32 call, u64 t0) {
  u32* tmo = (u32*)(p.myws + TMO_OFF);
  if (rfl(__hip_atomic_load(tmo, __ATOMIC_RELAXED, __HIP_MEMORY_SCOPE_AGENT)) == call) return true;
  if (__builtin_amdgcn_s_memrealtime() - t0 > DEADLINE) {
    __hip_atomic_store(tmo, call, __ATOMIC_RELAXED, __HIP_MEMORY_SCOPE_AGENT);
    WAIT_V(0);
    return true;
  }
  return false;
}
__device__ __forceinline__ u32 wave_scan(u32 v) {
  v += __builtin_amdgcn_update_dpp(0u, v, 0x111, 0xf, 0xf, false);
  v += __builtin_amdgcn_update_dpp(0u, v, 0x112, 0xf, 0xf, false);
  v += __builtin_amdgcn_update_dpp(0u, v, 0x114, 0xf, 0xf, false);
  v += __builtin_amdgcn_update_dpp(0u, v, 0x118, 0xf, 0xf, false);
  v += __builtin_amdgcn_update_dpp(0u, v, 0x142, 0xa, 0xf, false);
  v += __builtin_amdgcn_update_dpp(0u, v, 0x143, 0xc, 0xf, false);
  return v;
}
#define BOP(a, b, c, expr) (u32)__builtin_amdgcn_bitop3_b32((a), (b), (c), [] { constexpr u32 A = 0xf0, B = 0xcc, C = 0xaa; (void)A; (void)B; (void)C; return (expr) & 0xffu; }())
#define BFI(m, a, b) BOP(m, a, b, (A & B) | (~A & C))
struct Mv { u32 a0, a1, a2, c8, c16; };
__device__ __forceinline__ u32 byte_prefix_xor(u32 v) {
  v = BOP(v, v << 1, 0xfefefefeu, A ^ (B & C)); v = BOP(v, v << 2, 0xfcfcfcfcu, A ^ (B & C)); return BOP(v, v << 4, 0xf0f0f0f0u, A ^ (B & C));
}
__device__ __forceinline__ Mv move_masks(u32 m) {
  Mv r;
  r.c16 = __builtin_popcount(m & 0xffffu);
  r.c8 = (u32)__builtin_popcount(m & 0xffu) | ((u32)__builtin_popcount(m & 0xff0000u) << 16);
  u32 mk = BOP(m << 1, 0xfefefefeu, 0u, ~A & B);
  u32 mp = byte_prefix_xor(mk);
  r.a0 = mp & m; m = BOP(m, r.a0, r.a0 >> 1, (A ^ B) | C); mk = BOP(mk, mp, 0u, A & ~B);
  mp = byte_prefix_xor(mk);
  r.a1 = mp & m; m = BOP(m, r.a1, r.a1 >> 2, (A ^ B) | C); mk = BOP(mk, mp, 0u, A & ~B);
  r.a2 = BOP(0x80808080u - mk, 0x80808080u, m, (A ^ B) & C);
  return r;
}
__device__ __forceinline__ u32 pext(u32 x, u32 m, const Mv& v) {
  x &= m;
  u32 t = x & v.a0; x = BOP(x, t, t >> 1, (A ^ B) | C);
  t = x & v.a1; x = BOP(x, t, t >> 2, (A ^ B) | C);
  t = x & v.a2; x = BOP(x, t, t >> 4, (A ^ B) | C);
  t = __builtin_amdgcn_perm(0u, x, 0x0c030c01u);
  t = __builtin_bit_cast(u32, (u16x2)(__builtin_bit_cast(u16x2, t) << __builtin_bit_cast(u16x2, v.c8)));
  x = BOP(x, 0x00ff00ffu, t, (A & B) | C);
  return BOP(x, 0xffffu, (x >> 16) << v.c16, (A & B) | C);
}
__device__ __forceinline__ u32 pdep(u32 x, u32 m, const Mv& v) {
  x = __builtin_amdgcn_perm(x >> v.c16, x, 0x05040100u);
  const u32 t = __builtin_bit_cast(u32, (u16x2)(__builtin_bit_cast(u16x2, x) >> __builtin_bit_cast(u16x2, v.c8)));
  x = __builtin_amdgcn_perm(t, x, 0x06020400u);
  x = BFI(v.a2, x << 4, x); x = BFI(v.a1, x << 2, x); x = BFI(v.a0, x << 1, x);
  return x & m;
}
__device__ __forceinline__ void transpose4(u32 (&n)[4]) {
#pragma unroll
  for (int r = 0; r < 2; ++r) { const u32 a = n[r], c = n[r + 2]; n[r] = BFI(0x33333333u, a, c << 2); n[r + 2] = BFI(0x33333333u, a >> 2, c); }
#pragma unroll
  for (int r = 0; r < 4; r += 2) { const u32 a = n[r], c = n[r + 1]; n[r] = BFI(0x55555555u, a, c << 1); n[r + 1] = BFI(0x55555555u, a >> 1, c); }
}
__device__ __forceinline__ void or_bits(LAS u32* area, u32 bit, u32 val) {
  const u64 t = (u64)val << (bit & 31);
  LAS u32* q = area + (bit >> 5);
  __hip_atomic_fetch_or(q, (u32)t, __ATOMIC_RELAXED, __HIP_MEMORY_SCOPE_WORKGROUP);
  __hip_atomic_fetch_or(q + 1, (u32)(t >> 32), __ATOMIC_RELAXED, __HIP_MEMORY_SCOPE_WORKGROUP);
}
__device__ __forceinline__ u32 get_bits(const LAS u32* area, u32 bit) {
  const LAS u32* q = area + (bit >> 5);
  return __builtin_amdgcn_alignbit(q[1], q[0], bit & 31);
}

constexpr int ETAIL = SEG_STRIDE - SEG_P;
__device__ __forceinline__ void load_block(const Params& p, int seg, int rb, int lane, u32x4 (&X)[4]) {
  const int c = seg / SPC, rf = (seg % SPC) * NB + rb;
  const char* xp = p.x + ((size_t)(rf * 16 + (lane & 15)) * KD + c * 128 + (lane >> 4) * 8) * 2;
#pragma unroll
  for (int k = 0; k < 4; ++k) X[k] = *(const u32x4*)(xp + k * 64);
}
__device__ __forceinline__ int enc_seg(int r, int e, int sg) { return r == 0 ? (sg == 0 ? e : NSEG) : NENC + ((r - 1) * NENC + e) * 2 + sg; }
__device__ __forceinline__ void encode_role(const Params& p, LAS char* lds, int e, u32 call, int w, int lane, u32x4 (&Xn)[4]) {
  char* enc = uni(p.myws + ENC_OFF + (call & 1) * ENC_SZ);
  LAS u32* szw = (LAS u32*)(lds + 131072 + 64);
  const int sg = w / NB, rb = w % NB;
  LAS char* img = lds + sg * ETAIL;
  for (int r = 0; enc_seg(r, e, 0) < NSEG; ++r) {
    const int seg = enc_seg(r, e, sg), segn = enc_seg(r + 1, e, sg);
    if (seg >= NSEG) {
      if (segn < NSEG) load_block(p, segn, rb, lane, Xn);
      BAR; BAR;
      BAR;
      continue;
    }
    u32x4 X[4];
#pragma unroll
    for (int k = 0; k < 4; ++k) X[k] = Xn[k];
    asm volatile("" : "+v"(X[0]), "+v"(X[1]), "+v"(X[2]), "+v"(X[3]));
    if (segn < NSEG) load_block(p, segn, rb, lane, Xn);
    const u32 segoff = (u32)seg * SEG_STRIDE;
    u32 len;
    {
      LAS u32x4* z = (LAS u32x4*)(img + (SEG_V - SEG_P) + rb * 1536);
      const u32x4 zero = {0u, 0u, 0u, 0u};
      z[lane] = zero; if (lane < 36) z[64 + lane] = zero;
    }
    u32 R[16], sum = 0;
#pragma unroll
    for (int i = 0; i < 16; ++i) { const u32 t = X[i >> 2][i & 3]; R[i] = __builtin_amdgcn_alignbit(t, t, 31); sum = __builtin_amdgcn_udot4(R[i], 0x01000100u, sum, false); }
    sum = __builtin_amdgcn_readlane(wave_scan(sum), 63);
    int basei = (int)((sum + 1741u) >> 11) - 10;
    basei = basei < 0 ? 0 : (basei > 240 ? 240 : basei);
    const u32 base = (u32)basei;
    const u16x2 bsub = {(unsigned short)(base << 8), (unsigned short)(base << 8)};
    u32 S[8], V[8];
#pragma unroll
    for (int q = 0; q < 8; ++q) {
      const u32 a = __builtin_bit_cast(u32, (u16x2)(__builtin_bit_cast(u16x2, R[2 * q]) - bsub));
      const u32 b = __builtin_bit_cast(u32, (u16x2)(__builtin_bit_cast(u16x2, R[2 * q + 1]) - bsub));
      S[q] = __builtin_amdgcn_perm(b, a, 0x06040200u); V[q] = __builtin_amdgcn_perm(b, a, 0x07050301u);
    }
    u32 d[4], h[4];
#pragma unroll
    for (int q = 0; q < 4; ++q) { d[q] = BFI(0x0f0f0f0fu, V[2 * q], V[2 * q + 1] << 4); h[q] = BFI(0x0f0f0f0fu, V[2 * q] >> 4, V[2 * q + 1]); }
    transpose4(d); transpose4(h);
    u32 Xm = (h[0] | h[1] | h[2] | h[3]) | (d[3] & d[2] & d[1]);
    const u32 m1 = d[3] & ~d[2];
    const u32 P1 = d[1] & m1 & ~Xm, P0 = (d[0] & m1) | Xm;
    const u32 E1 = ~(P1 | P0);
    const u32 Q1 = d[3] & ~d[0], Q0 = d[2] & BOP(d[3], d[1], d[0], (A & ~C) | (~A & B & C));
    const u32 E2 = E1 & ~(Q1 | Q0);
    const u32 nx = __builtin_popcount(Xm);
    const u32 cnt = (u32)__builtin_popcount(E1) | ((u32)__builtin_popcount(E2) << 12) | ((nx > 3u ? 3u : nx) << 24);
    const u32 inc = wave_scan(cnt), tot = __builtin_amdgcn_readlane(inc, 63), exc = inc - cnt;
    const u32 N1 = tot & 0xfffu, N2 = (tot >> 12) & 0xfffu, NX = tot >> 24;
    const bool raw = NX > XCAP || __any(nx > 3u);
    const u32 bitw = (2 * N1 + 3 * N2 + 31) >> 5;
    const u32 vd = raw ? 384u : bitw + NX;
    if (lane == 0) szw[w] = vd;
    LBAR;
    u32 voff = 0, vtot = 0;
#pragma unroll
    for (int i = 0; i < NB; ++i) { const u32 s = rfl(szw[sg * NB + i]); vtot += s; if (i < rb) voff += s; }
    LAS u32* var = (LAS u32*)(img + (SEG_V - SEG_P)) + voff;
    if (lane == 0) ((LAS u32*)(img + (SEG_H - SEG_P)))[rb] = base | ((raw ? 255u : NX) << 8) | (voff << 16);
    if (!raw) {
      *(LAS u32x2*)(img + rb * 512 + lane * 8) = (u32x2){P1, P0};
      const u32 o1 = exc & 0xfffu, o2 = 2 * N1 + ((exc >> 12) & 0xfffu);
      { const Mv mv = move_masks(E1); or_bits(var, o1, pext(Q1, E1, mv)); or_bits(var, N1 + o1, pext(Q0, E1, mv)); }
      { const Mv mv = move_masks(E2); or_bits(var, o2, pext(d[2], E2, mv)); or_bits(var, o2 + N2, pext(d[1] | d[3], E2, mv)); or_bits(var, o2 + 2 * N2, pext(d[0], E2, mv)); }
      u32 xo = bitw + (exc >> 24);
      while (__any(Xm != 0u)) {
        if (Xm != 0u) {
          const u32 j = __builtin_ctz(Xm); Xm &= Xm - 1;
          const u32 vi = 2 * (j & 3) + ((j >> 2) & 1), by = j >> 3;
          u32 vs = V[0];
#pragma unroll
          for (int i = 1; i < 8; ++i) vs = vi == (u32)i ? V[i] : vs;
          const u32 ev = (base + (vs >> (8 * by))) & 255u;
          __hip_atomic_fetch_or(var + xo, (u32)lane | (vi << 6) | (by << 9) | ((ev ^ (base + 9u)) << 16), __ATOMIC_RELAXED, __HIP_MEMORY_SCOPE_WORKGROUP);
          ++xo;
        }
      }
    } else {
      u32 E[8];
#pragma unroll
      for (int q = 0; q < 8; ++q) E[q] = __builtin_amdgcn_perm(R[2 * q + 1], R[2 * q], 0x07050301u);
      *(LAS u32x2*)(img + rb * 512 + lane * 8) = (u32x2){E[0], E[1]};
#pragma unroll
      for (int q = 2; q < 8; ++q) __hip_atomic_fetch_or(var + (q - 2) * 64 + lane, E[q], __ATOMIC_RELAXED, __HIP_MEMORY_SCOPE_WORKGROUP);
    }
    st4x2(enc, segoff + rb * 2048 + lane * 16, (u32x4){S[0], S[1], S[2], S[3]}, (u32x4){S[4], S[5], S[6], S[7]});
    LBAR;
    len = (SEG_V + vtot * 4 + 15) & ~15u;
    for (u32 o = rb * 1024 + lane * 16; o < len - SEG_P; o += NB * 1024) st4x1(enc, segoff + SEG_P + o, *(LAS u32x4*)(img + o));
    WAIT_V(0);
    LBAR;
    if (rb == 0 && lane < NR && lane != p.rank) {
      u64* f = (u64*)(p.ws[lane] + F1_OFF + (call & 1) * F1_SZ) + (p.rank * NSEG + seg);
      __hip_atomic_store(f, ((u64)call << 32) | len, __ATOMIC_RELAXED, __HIP_MEMORY_SCOPE_SYSTEM);
    }
    if (r == 0) STAMP(1);
  }
  STAMP(2);
}

__device__ __forceinline__ u32 lds_ld(LAS u32* q) { return rfl(__hip_atomic_load(q, __ATOMIC_ACQUIRE, __HIP_MEMORY_SCOPE_WORKGROUP)); }
__device__ __forceinline__ void lds_post(LAS u32* q, u32 v) { asm volatile("ds_write_b32 %0, %1" :: "v"((u32)(size_t)q), "v"(v) : "memory"); }
__device__ __forceinline__ u32 lds_peek(LAS u32* q) { u32 v; asm volatile("ds_read_b32 %0, %1\n\ts_waitcnt lgkmcnt(0)" : "=&v"(v) : "v"((u32)(size_t)q) : "memory"); return rfl(v); }
constexpr u32 FIN = 0x40000000u;
#define LA 2u
#define GMAX 4u
#define GDIV 4u
constexpr int QD = 16;

__device__ __forceinline__ void pull_role(const Params& p, LAS char* lds, int src, int j, u32 call, int w, int lane) {
  LAS u32* landed = (LAS u32*)(lds + 131072 + 128);
  LAS u32* posted = landed + 1;
  LAS u32* started = landed + 2;
  LAS u32* done = (LAS u32*)(lds + 131072 + 192);
  LAS u32* queue = (LAS u32*)(lds + 131072 + 256);
  if (w == 1) {
    const u64* f1 = (const u64*)(p.myws + F1_OFF + (call & 1) * F1_SZ) + src * NSEG;
    u32* tk = (u32*)(p.myws + TKT_OFF + src * 128);
    const u32 tbase = call * 2048u;
    if (lane == 0) __hip_atomic_fetch_max(tk, tbase, __ATOMIC_RELAXED, __HIP_MEMORY_SCOPE_AGENT);
    WAIT_V(0);
    u32 seg = j, left = 1, win = 0; u64 v = 0;
    for (u32 k = 0;; ++k) {
      if (left == 0) {
        u32 st;
        while ((st = lds_peek(started)) + LA <= k) __builtin_amdgcn_s_sleep(1);
        left = ((u32)NSEG - seg) / (CPP * GDIV); left = seg >= (u32)NSEG || left < 1u ? 1u : (left > GMAX ? GMAX : left);
        if (left == 1u && st + LA - k >= 2u) left = 2u;
        if (k < 2) left = 1u;
        u32 t = 0;
        if (lane == 0) t = __hip_atomic_fetch_add(tk, left, __ATOMIC_RELAXED, __HIP_MEMORY_SCOPE_AGENT);
        win = seg;
        const u32 wi = win + lane;
        v = __hip_atomic_load(f1 + (wi < (u32)NSEG ? wi : (u32)NSEG - 1), __ATOMIC_RELAXED, __HIP_MEMORY_SCOPE_SYSTEM);
        seg = rfl(t) - tbase + CPP;
      }
      u32 ent = 0;
      if (seg < (u32)NSEG) {
        const u64 ok = __ballot((u32)(v >> 32) >= call);
        u32 len = 0;
        if (seg - win < 64u && ((ok >> (seg - win)) & 1)) len = __builtin_amdgcn_readlane((u32)v, seg - win);
        else {
          const u64 t0 = __builtin_amdgcn_s_memrealtime();
          for (u32 it = 1;; ++it) {
            const u64 u = __hip_atomic_load(f1 + seg, __ATOMIC_RELAXED, __HIP_MEMORY_SCOPE_SYSTEM);
            if (rfl((u32)(u >> 32)) >= call) { len = rfl((u32)u); break; }
            if ((it & 63u) == 0 && give_up(p, call, t0)) { seg = NSEG; break; }
            __builtin_amdgcn_s_sleep(1);
          }
        }
        if (seg < (u32)NSEG) {
          len = len > (u32)SEG_STRIDE ? (u32)SEG_STRIDE : (len < 16u ? 16u : len);
          ent = (seg << 16) | (len >> 4);
        }
      }
      lds_post(queue + (k % QD), ent); lds_post(posted, k + 1);
      if (k == 0) STAMP(1);
      if (ent == 0) return;
      ++seg; --left;
    }
  }
  if (w == 0) {
    const char* rsrc = uni(p.ws[src] + ENC_OFF + (call & 1) * ENC_SZ);
    __builtin_amdgcn_s_waitcnt(0x0f70);
    int slot = 0; u32 use = 0, k = 0;
    for (;; ++k) {
      u32 ent;
      for (u32 it = 0;; ++it) {
        u32 a, d;
        asm volatile("ds_read_b32 %0, %3\n\tds_read_b32 %1, %4\n\tds_read_b32 %2, %5\n\ts_waitcnt lgkmcnt(0)" : "=&v"(a), "=&v"(d), "=&v"(ent)
                     : "v"((u32)(size_t)posted), "v"((u32)(size_t)(done + slot)), "v"((u32)(size_t)(queue + (k % QD))) : "memory");
        if (rfl(a) > k && rfl(d) >= use) break;
        if (it == 8) { WAIT_V(0); lds_post(landed, k); }
        __builtin_amdgcn_s_sleep(1);
      }
      ent = rfl(ent);
      if (ent == 0) break;
      lds_post(started, k + 1);
      const u32 len = (ent & 0xffffu) << 4;
      const char* gs = rsrc + (size_t)(ent >> 16) * SEG_STRIDE;
      const u32 ls = (u32)(size_t)lds + slot * SEG_STRIDE;
      for (u32 o = 0; o < len; o += 1024) {
        wait_v<INFL - 1>();
        if (o == (INFL - 1) * 1024) lds_post(landed, k);
        if (o + lane * 16 < len) asm volatile("s_mov_b32 m0, %2\n\ts_nop 0\n\tglobal_load_lds_dwordx4 %0, %1 sc0 sc1" :: "v"(o + lane * 16), "s"(gs), "s"(ls + o) : "memory");
      }
      if (++slot == NSLOT) { slot = 0; use += NB; }
    }
    WAIT_V(0); lds_post(landed, k | FIN);
    STAMP(2);
    return;
  }
  char* dec = uni(p.myws + DEC_OFF + (size_t)src * SHARD_B);
  u32* f2 = (u32*)(p.myws + F2_OFF) + src * NRF * NCH;
  u32 have = 0, pend = ~0u;
#define PUBLISH() do { if (pend != ~0u) { WAIT_V(0); if (lane == 0) __hip_atomic_store(f2 + pend, call, __ATOMIC_RELAXED, __HIP_MEMORY_SCOPE_SYSTEM); pend = ~0u; } } while (0)
  for (u32 q = w - 2;; q += ND) {
    const u32 k = q / NB, rb = q % NB, slot = k % NSLOT;
    if (have <= k) have = lds_peek(landed);
    if ((have & ~FIN) <= k) {
      PUBLISH();
      while (have <= k) { __builtin_amdgcn_s_sleep(1); have = lds_peek(landed); }
      if ((have & ~FIN) <= k) break;
    }
    const u32 seg = (lds_peek(queue + (k % QD)) >> 16) & (NSEG - 1), c = seg / SPC, rf = (seg % SPC) * NB + rb;
    const LAS char* ls = lds + slot * SEG_STRIDE;
    u32x4 O[4];
    const u32x4 Sa = *(const LAS u32x4*)(ls + rb * 2048 + lane * 16), Sb = *(const LAS u32x4*)(ls + rb * 2048 + 1024 + lane * 16);
    const u32x2 Pp = *(const LAS u32x2*)(ls + SEG_P + rb * 512 + lane * 8);
    const u32 hdr = rfl(*(const LAS u32*)(ls + SEG_H + rb * 4));
    const u32 base = hdr & 255u, NX = (hdr >> 8) & 255u;
    const LAS u32* var = (const LAS u32*)(ls + SEG_V) + (hdr >> 16);
    u32 E[8];
    if (NX != 255u) {
      const u32 P1 = Pp[0], P0 = Pp[1], E1 = ~(P1 | P0);
      const u32 n1 = __builtin_popcount(E1), i1 = wave_scan(n1), N1 = __builtin_amdgcn_readlane(i1, 63), o1 = i1 - n1;
      const u32 q1 = get_bits(var, o1), q0 = get_bits(var, N1 + o1);
      const Mv ma = move_masks(E1);
      const u32 n2 = __builtin_popcount((u32)((u64)~(q1 | q0) << (32 - n1)));
      const u32 i2 = wave_scan(n2), N2 = __builtin_amdgcn_readlane(i2, 63), o2 = 2 * N1 + i2 - n2;
      const u32 r2 = get_bits(var, o2), r1 = get_bits(var, o2 + N2), r0 = get_bits(var, o2 + 2 * N2);
      const u32 xe = var[((2 * N1 + 3 * N2 + 31) >> 5) + lane];
      const u32 Q1 = pdep(q1, E1, ma), Q0 = pdep(q0, E1, ma), E2 = BOP(E1, Q1, Q0, A & ~(B | C));
      const Mv mb = move_masks(E2);
      const u32 R2 = pdep(r2, E2, mb), R1 = pdep(r1, E2, mb), R0 = pdep(r0, E2, mb);
      const u32 t7 = BOP(R2, R1, R0, A & B & C), q7 = BOP(Q1, Q0, 0u, ~A & B);
      u32 d[4] = {BOP(P0, q7, R0, A | B | C), BOP(P1 | q7, R1, t7, A | (B & ~C)), Q0 | R2, BOP(E1, Q1, t7, ~A | B | C)};
      transpose4(d);
      const u32 b4 = base * 0x01010101u;
#pragma unroll
      for (int i = 0; i < 4; ++i) { E[2 * i] = (d[i] & 0x0f0f0f0fu) + b4; E[2 * i + 1] = ((d[i] >> 4) & 0x0f0f0f0fu) + b4; }
      for (u32 i = 0; i < NX && i < 64u; ++i) {
        const u32 en = __builtin_amdgcn_readlane(xe, i);
        const u32 dv = (u32)lane == (en & 63u) ? ((en >> 16) & 255u) << (8 * ((en >> 9) & 3u)) : 0u;
        const int hot = (int)(0x80000000u >> ((en >> 6) & 7u));
#pragma unroll
        for (int k = 0; k < 8; ++k) E[k] = BOP(E[k], dv, (u32)((hot << k) >> 31), A ^ (B & C));
      }
    } else {
      E[0] = Pp[0]; E[1] = Pp[1];
#pragma unroll
      for (int k = 2; k < 8; ++k) E[k] = var[(k - 2) * 64 + lane];
    }
#pragma unroll
    for (int k = 0; k < 8; ++k) {
      const u32 s = k < 4 ? Sa[k & 3] : Sb[k & 3];
      const u32 a = __builtin_amdgcn_perm(E[k], s, 0x05010400u), b = __builtin_amdgcn_perm(E[k], s, 0x07030602u);
      O[k >> 1][(k & 1) * 2] = __builtin_amdgcn_alignbit(a, a, 1); O[k >> 1][(k & 1) * 2 + 1] = __builtin_amdgcn_alignbit(b, b, 1);
    }
    asm volatile("" : "+v"(O[0]), "+v"(O[1]), "+v"(O[2]), "+v"(O[3]));
    if (lane == 0) __hip_atomic_fetch_add(done + slot, 1u, __ATOMIC_RELAXED, __HIP_MEMORY_SCOPE_WORKGROUP);
    PUBLISH();
    st4x4d(dec, (c * NRF + rf) * 4096u + lane * 16, O[0], O[1], O[2], O[3]);
    pend = rf * NCH + c;
    if (q < NB) STAMP(1);
  }
  STAMP(2);
}

__device__ __forceinline__ int wait_chunks(const Params& p, const u32* f, int need, u32 call) {
  const u64 t0 = __builtin_amdgcn_s_memrealtime();
  for (u32 it = 1;; ++it) {
    const u32 a = __hip_atomic_load(f, __ATOMIC_RELAXED, __HIP_MEMORY_SCOPE_SYSTEM);
    const u32 b = __hip_atomic_load(f + 8 * NCH, __ATOMIC_RELAXED, __HIP_MEMORY_SCOPE_SYSTEM);
    const u64 miss = __ballot(a < call || b < call);
    const int n = miss ? __builtin_ctzll(miss) : NCH;
    if (n >= need) return n;
    if ((it & 63u) == 0 && give_up(p, call, t0)) return NCH;
    __builtin_amdgcn_s_sleep(2);
  }
}
__device__ __forceinline__ u32 cvt_pk(float lo, float hi) { u32 r; asm("v_cvt_pk_bf16_f32 %0, %1, %2" : "=v"(r) : "v"(lo), "v"(hi)); return r; }
__device__ __forceinline__ float silu(float y) {
  const float c = __builtin_amdgcn_fmed3f(y, -60.0f, 60.0f);
  return y * __builtin_amdgcn_rcpf(1.0f + __builtin_amdgcn_exp2f(c * -1.44269504088896341f));
}

__device__ __forceinline__ void gemm_role(const Params& p, LAS char* lds, int src, int tm, int tn, u32 call, int w, int lane) {
  constexpr int HT = 16384;
  const int wr = w >> 2, wc = w & 3;
  const bool own = src == p.rank;
  const char* Ab = uni(own ? p.x + (size_t)tm * 256 * KD * 2 : p.myws + DEC_OFF + (size_t)src * SHARD_B + (size_t)tm * 16 * 4096);
  const u32 sC = own ? 256u : 262144u, sH = own ? 128u * KD * 2 : 32768u, sK = own ? 128u : 2048u;
  const int sF = own ? 64 - 1024 : 0;
  const char* Bb = uni(p.w + tn * 512);
  const u32 voffA = own ? (u32)(w * 16 + (lane >> 2)) * (KD * 2) + (((lane & 3) ^ ((lane >> 4) & 2)) << 4) : (u32)(w * 4096 + lane * 16);
  const u32 voffB = (4 * w + (lane >> 4)) * 2048 + (((lane & 15) ^ (((lane >> 4) << 2) | (w & 3))) << 4);
  const u32* flg = (const u32*)(p.myws + F2_OFF) + (src * NRF + tm * 16 + w) * NCH + lane;
  const u32 lds0 = (u32)(size_t)lds, ldsA = lds0 + w * 2048, ldsB = lds0 + w * 1024;
  const int aoff = wr * 8192 + (own ? (lane & 15) * 64 + (((lane >> 4) ^ ((lane >> 2) & 2)) << 4) : lane * 16);
  u32 boff[2][2];
  { const int g = lane >> 4, q = (lane & 15) >> 2, pp = lane & 3;
#pragma unroll
    for (int n = 0; n < 2; ++n)
#pragma unroll
      for (int t = 0; t < 2; ++t) { boff[n][t] = lds0 + 4 * HT + 256 * (8 * g + 4 * t + q) + 16 * ((4 * wc + 2 * n + (pp >> 1)) ^ ((q << 2) | ((2 * g + t) & 3))) + 8 * (pp & 1);
        asm volatile("" : "+v"(boff[n][t])); } }
#define SA(b, h) (((b) * 2 + (h)) * HT)
#define SB(b, h) ((4 + (b) * 2 + (h)) * HT)
#define STA(buf, gp) asm volatile("s_mov_b32 m0, %3\n\ts_nop 0\n\tglobal_load_lds_dwordx4 %0, %1" A_BITS "\n\tglobal_load_lds_dwordx4 %0, %2 offset:1024" A_BITS \
    :: "v"(voffA), "s"(gp), "s"((gp) + sF), "s"(ldsA + (buf)) : "memory")
#define STB(buf, gp) asm volatile("s_mov_b32 m0, %3\n\ts_nop 0\n\tglobal_load_lds_dwordx4 %0, %1\n\ts_add_u32 m0, m0, 0x2000\n\ts_nop 0\n\tglobal_load_lds_dwordx4 %0, %2" \
    :: "v"(voffB), "s"(gp), "s"((gp) + 65536), "s"(ldsB + (buf)) : "memory", "scc")
#define LDA(dst, b, h) do { _Pragma("unroll") for (int m = 0; m < 4; ++m) _Pragma("unroll") for (int k = 0; k < 2; ++k) \
    dst[m][k] = *(const LAS s16x8*)(lds + SA(b, h) + aoff + m * 2048 + k * 1024); } while (0)
#define LDB(dst, b, h) do { _Pragma("unroll") for (int n = 0; n < 2; ++n) _Pragma("unroll") for (int k = 0; k < 2; ++k) { \
    const s16x4 lo_ = __builtin_amdgcn_ds_read_tr16_b64_v4i16((LAS s16x4*)(size_t)(boff[n][0] + SA(b, h) + k * 8192)); \
    const s16x4 hi_ = __builtin_amdgcn_ds_read_tr16_b64_v4i16((LAS s16x4*)(size_t)(boff[n][1] + SA(b, h) + k * 8192)); \
    dst[n][k] = __builtin_shufflevector(lo_, hi_, 0, 1, 2, 3, 4, 5, 6, 7); } } while (0)
#define MMA(ai, bj, At, Bt) do { __builtin_amdgcn_s_setprio(1); _Pragma("unroll") for (int m = 0; m < 4; ++m) _Pragma("unroll") for (int n = 0; n < 2; ++n) \
    _Pragma("unroll") for (int k = 0; k < 2; ++k) acc[ai][bj][m][n] = __builtin_amdgcn_mfma_f32_16x16x32_bf16(__builtin_bit_cast(bf8, Bt[n][k]), __builtin_bit_cast(bf8, At[m][k]), acc[ai][bj][m][n], 0, 0, 0); \
    __builtin_amdgcn_s_setprio(0); } while (0)
  typedef __bf16 bf8 __attribute__((ext_vector_type(8)));
  f32x4 acc[2][2][4][2];
#pragma unroll
  for (int a = 0; a < 2; ++a)
#pragma unroll
    for (int b = 0; b < 2; ++b)
#pragma unroll
      for (int m = 0; m < 4; ++m)
#pragma unroll
        for (int n = 0; n < 2; ++n) acc[a][b][m][n] = (f32x4){0.f, 0.f, 0.f, 0.f};
  s16x8 At[4][2], B0[2][2], B1[2][2];
  int avail = own ? NCH : wait_chunks(p, flg, 1, call);
  STAMP(3); STAMPC(8);
  STB(SB(0, 0), Bb); STA(SA(0, 0), Ab); STB(SB(0, 1), Bb + 256); STA(SA(0, 1), Ab + sH);
  if (wr == 1) BAR;
  WAIT_V(4); BAR;
  STB(SB(1, 0), Bb + 131072); STA(SA(1, 0), Ab + sK); STB(SB(1, 1), Bb + 131072 + 256);
  WAIT_V(6); BAR;
  for (int c = 0; c < NCH; ++c) {
    const int cn = c + 1 < NCH ? c + 1 : c;
    const char* a1 = Ab + (size_t)(c * sC + sK);
    const char* a2 = Ab + (size_t)(cn * sC);
    const char* b2 = Bb + (size_t)cn * 262144;
    LDB(B0, 0, 0); SCHED; LDA(At, 0, 0); STA(SA(1, 1), a1 + sH);
    WAIT_L(8); BAR; WAIT_L(0); MMA(0, 0, At, B0); BAR; SCHED;
    LDB(B1, 0, 1); STB(SB(0, 0), b2);
    BAR; WAIT_L(0); MMA(0, 1, At, B1); BAR;
    if (avail <= cn) avail = wait_chunks(p, flg, cn + 1, call);
    LDA(At, 0, 1); STA(SA(0, 0), a2);
    BAR; WAIT_L(0); MMA(1, 0, At, B0); BAR; SCHED;
    STB(SB(0, 1), b2 + 256);
    WAIT_V(6); BAR; MMA(1, 1, At, B1); BAR;
    LDB(B0, 1, 0); SCHED; LDA(At, 1, 0); STA(SA(0, 1), a2 + sH);
    WAIT_L(8); BAR; WAIT_L(0); MMA(0, 0, At, B0); BAR; SCHED;
    LDB(B1, 1, 1); STB(SB(1, 0), b2 + 131072);
    BAR; WAIT_L(0); MMA(0, 1, At, B1); BAR;
    LDA(At, 1, 1); STA(SA(1, 0), a2 + sK);
    BAR; WAIT_L(0); MMA(1, 0, At, B0); BAR; SCHED;
    STB(SB(1, 1), b2 + 131072 + 256);
    WAIT_V(6); BAR; MMA(1, 1, At, B1); BAR;
  }
  if (wr == 0) BAR;
  STAMP(4); STAMPC(9);
  const bool bad = rfl(__hip_atomic_load((u32*)(p.myws + TMO_OFF), __ATOMIC_RELAXED, __HIP_MEMORY_SCOPE_AGENT)) == call;
  int tz = threadIdx.x; asm volatile("" : "+v"(tz));
  const int fr = tz & 15, fq = (tz >> 4) & 3;
  char* ob = p.out + ((size_t)(src * 1024 + tm * 256 + wr * 64 + fr) * NS + tn * 256 + wc * 32 + ((fq & 1) << 4) + ((fq >> 1) << 3)) * 2;
#pragma unroll
  for (int ai = 0; ai < 2; ++ai)
#pragma unroll
    for (int m = 0; m < 4; ++m)
#pragma unroll
      for (int bj = 0; bj < 2; ++bj) {
        const f32x4 u = acc[ai][bj][m][0], v = acc[ai][bj][m][1];
        const u32 a0 = cvt_pk(silu(u[0]), silu(u[1])), a1 = cvt_pk(silu(u[2]), silu(u[3]));
        const u32 b0 = cvt_pk(silu(v[0]), silu(v[1])), b1 = cvt_pk(silu(v[2]), silu(v[3]));
        const auto s0 = __builtin_amdgcn_permlane16_swap(a0, b0, false, false);
        const auto s1 = __builtin_amdgcn_permlane16_swap(a1, b1, false, false);
        u32x4 o = {s0[0], s1[0], s0[1], s1[1]};
        if (bad) o = (u32x4){0x7fc07fc0u, 0x7fc07fc0u, 0x7fc07fc0u, 0x7fc07fc0u};
        *(u32x4*)(ob + (size_t)(ai * 128 + m * 16) * (NS * 2) + bj * 256) = o;
      }
  STAMP(5);
}

__global__ __launch_bounds__(512, 2) void ag_gemm(Params p) {
  extern __shared__ __attribute__((aligned(16))) char smem[];
  LAS char* lds = (LAS char*)smem;
  const int lane = threadIdx.x & 63, w = rfl(threadIdx.x >> 6);
  const int b = role_id();
  STAMP(0);
  const int i = b - NGEMM;
  if (i >= NENC + 7 * CPP) return;
  u32x4 X[4];
  if (i >= 0 && i < NENC && w < NB) load_block(p, i, w, lane, X);
  LAS u32* sw = (LAS u32*)(lds + 131072);
  if (threadIdx.x < 64) {
    if (lane == 0) { u32* cp = (u32*)(p.myws + CNT_OFF + b * 128); const u32 v = *cp + 1; *cp = v; sw[0] = v; }
    else if (lane >= 32) sw[lane] = 0;
  }
  LBAR;
  const u32 call = rfl(sw[0]);
  if (i < 0) gemm_role(p, lds, (p.rank + (b & 7)) & 7, b >> 5, (b >> 3) & 3, call, w, lane);
  else if (i < NENC) encode_role(p, lds, i, call, w, lane, X);
  else {
    pull_role(p, lds, (p.rank + 1 + (i - NENC) % 7) & 7, (i - NENC) / 7, call, w, lane);
  }
}
}

extern "C" void kernel_launch(int rank, int nranks, ncclComm_t comm, hipStream_t stream,
                              void* const* d_in, const int* in_sizes, int n_in,
                              void* d_out, int out_size, const int* in_dtypes, int out_dtype,
                              void* const* d_out_all, void* const* d_ws_all, size_t ws_bytes) {
  using namespace agg;
  if (nranks != NR || (unsigned)rank >= (unsigned)NR || ws_bytes < WS_NEED || n_in < 2) { fprintf(stderr, "ag_gemm: unexpected configuration\n"); return; }
  Params p;
  for (int i = 0; i < NR; ++i) p.ws[i] = (char*)d_ws_all[i];
  p.myws = p.ws[rank]; p.x = (const char*)d_in[0]; p.w = (const char*)d_in[1]; p.out = (char*)d_out; p.rank = rank; p.pad = 0;
  hipError_t e = hipFuncSetAttribute((const void*)ag_gemm, hipFuncAttributeMaxDynamicSharedMemorySize, LDS_BYTES);
  if (e != hipSuccess) { fprintf(stderr, "HIP error %s\n", hipGetErrorString(e)); return; }
  hipLaunchKernelGGL(ag_gemm, dim3(GRID), dim3(512), LDS_BYTES, stream, p);
  e = hipGetLastError();
  if (e != hipSuccess) fprintf(stderr, "HIP error %s\n", hipGetErrorString(e));
}
```

```cpp
#include <hip/hip_runtime.h>
#include <rccl/rccl.h>
#include <cstdio>

#define INFL 8
#define LEAD 10
#define UPR 1
#define LA 2u
#define GMAX 4u

namespace agg {
#define LAS __attribute__((address_space(3)))
typedef unsigned u32;
typedef unsigned long long u64;
typedef unsigned u32x4 __attribute__((ext_vector_type(4)));
typedef unsigned u32x2 __attribute__((ext_vector_type(2)));
typedef short s16x4 __attribute__((ext_vector_type(4)));
typedef short s16x8 __attribute__((ext_vector_type(8)));
typedef unsigned short u16x2 __attribute__((ext_vector_type(2)));
typedef float f32x4 __attribute__((ext_vector_type(4)));
typedef __bf16 bf8 __attribute__((ext_vector_type(8)));

constexpr int NR = 8, KD = 8192, NS = 1024;
constexpr int NCH = 64;
constexpr int NB = 4;
constexpr int ND = 6;
constexpr int SEG_STRIDE = NB * 4096 + 1024;
constexpr int SEG_P = NB * 2048, SEG_H = NB * 2560, SEG_V = SEG_H + NB * 4;
constexpr int NSLOT = 8, QD = 16;
constexpr int GRID = 256;
constexpr int CPL = 6, NPULL = 7 * CPL, NENC = 48 - NPULL, NUP = 160;
constexpr int LDS_MAIN = 147456, LDS_FLG = LDS_MAIN + 1024, LDS_BYTES = LDS_FLG + 2048;
static_assert(NSLOT * SEG_STRIDE <= LDS_MAIN, "ring");
constexpr u32 XCAP = 64;
constexpr int NSRC = 24 * 128, NJOB = 16 * 128, JOB0 = NUP + UPR * NUP * 2;
static_assert(JOB0 <= NJOB, "jobs");

constexpr size_t PANEL_B = (size_t)2 << 20;
constexpr size_t DEC_OFF = 0;
constexpr size_t ENC_OFF = 32 * PANEL_B;
constexpr size_t ENC_SZ = (size_t)NSRC * SEG_STRIDE;
constexpr size_t F1_OFF = ENC_OFF + 2 * ENC_SZ;
constexpr size_t F1_SZ = (size_t)NR * NSRC * 8;
constexpr size_t F2_OFF = F1_OFF + 2 * F1_SZ;
constexpr size_t F2_SZ = (size_t)32 * 8 * NCH * 4;
constexpr size_t CNT_OFF = F2_OFF + F2_SZ;
constexpr size_t TKT_OFF = CNT_OFF + GRID * 128;
constexpr size_t TMO_OFF = TKT_OFF + NR * 128;
constexpr size_t DBG_OFF = TMO_OFF + 128;
constexpr size_t WS_NEED = DBG_OFF + (1 << 20);
constexpr u64 DEADLINE = 50000000ull;

struct Params { char* ws[NR];   const char* x; const char* w; char* out; int rank; int pad; };
typedef const __attribute__((address_space(4))) Params CParams;

__host__ __device__ __forceinline__ int cls(int r) { return (r ^ (r >> 1)) & 1; }
__host__ __device__ __forceinline__ int slot_row(int r, int s) {
  const int h4 = 4 * cls(r), m = r & 1;
  int owner, piece;
  if (s < 8) { owner = r; piece = s; }
  else if (s < 12) { owner = r ^ 3; piece = 2 + (s & 1) + ((s & 2) << 1); }
  else if (s < 24) { const int ei = (s - 12) / 3, a = (s - 12) % 3; owner = r ^ (4 + ei); piece = h4 + (a == 0 ? m : a + 1); }
  else { const int y = r ^ (4 + ((s - 24) >> 1)); if (s & 1) { owner = r ^ 3; piece = 4 * cls(y) + (y & 1); } else { owner = y; piece = h4 + (m ^ 1); } }
  return owner * 1024 + piece * 128;
}
#define MIX 1
__host__ __device__ __forceinline__ int far_slot(int xi, int jj) { return !MIX ? xi * 8 + jj : (jj < 2 ? 2 * xi + jj : (jj == 2 ? 8 + xi : (jj < 6 ? 9 + 3 * xi + jj : 18 + 2 * xi + jj))); }
__host__ __device__ __forceinline__ u32 list_len(int d) { return d < 3 ? 768u : (d == 3 ? 512u : 640u); }
constexpr u32 D_FWD = 1u << 12, D_C = 1u << 13;
__host__ __device__ __forceinline__ u32 list_item(int d, int r, u32 pos) {
  u32 sp, dp, c, g = pos & 1, fl = 0;
  if (d == 3) { const u32 pi = (pos >> 1) & 3; c = pos >> 3; sp = 2 + (pi & 1) + ((pi & 2) << 1); dp = 8 + pi; }
  else if (d < 3) {
    if (pos >= 512u) {
      const u32 q = pos - 512u, ti = q >> 2, slot = ti < 12u ? ti : (ti < 24u ? ti - 12u : ti >> 1), part = ti < 12u ? 0u : (ti < 24u ? 1u : ti & 1u);
      const u32 code = (q & 3u) | (part << 2) | (slot << 3);
      return ((20u * 128u + (u32)(d - 1) * 256u + code) << 16) | D_C | code;
    }
    const u32 u = pos >> 2; u32 part;
    if (u < (u32)LEAD) { part = 0; c = u; }
    else if (u < 128u - LEAD) { const u32 v = u - LEAD; part = (v & 1u) ^ 1u; c = (v >> 1) + (part ? 0u : (u32)LEAD); }
    else { part = 1; c = u - 64u; }
    dp = 4u * (d - 1) + part * 2 + ((pos >> 1) & 1); sp = 8 + dp;
  } else {
    const u32 ei = d - 4, h4 = 4 * cls(r), m = r & 1; u32 rr; bool fast;
    if (pos < 24u) { c = pos / 6u; rr = pos % 6u; fast = true; }
    else if (pos < 564u) { const u32 k = (pos - 24u) / 36u, q = (pos - 24u) % 36u, j = q / 10u; rr = q % 10u; fast = rr < 6u; c = fast ? 4 * k + 4 + j : 3 * k + j; if (!fast) rr -= 6u; }
    else { c = 45u + ((pos - 564u) >> 2); rr = pos & 3u; fast = false; }
    const u32 a = rr >> 1;
    if (fast) { sp = h4 + (a == 0 ? m : a + 1); dp = 12 + 3 * ei + a; fl = a == 0 ? D_FWD : 0u; }
    else { sp = a ? 16u + ((d ^ 3) - 4) : h4 + (m ^ 1); dp = 24 + 2 * ei + a; }
  }
  return ((sp * 128u + c * 2 + g) << 16) | fl | (dp << 7) | (c << 1) | g;
}

#define STAMPV(i, v) do { (void)(v); } while (0)
#define STAMP(i) do {} while (0)
#define STAMPC(i) do {} while (0)

#define WAIT_V(n) asm volatile("s_waitcnt vmcnt(" #n ")" ::: "memory")
#define WAIT_L(n) asm volatile("s_waitcnt lgkmcnt(" #n ")" ::: "memory")
#define DRAIN do { WAIT_V(0); __builtin_amdgcn_s_waitcnt(0x0f70); } while (0)
#define BAR __builtin_amdgcn_s_barrier()
#define SCHED __builtin_amdgcn_sched_barrier(0)
#define LBAR do { WAIT_L(0); BAR; asm volatile("" ::: "memory"); } while (0)
template <int N> __device__ __forceinline__ void wait_v() { asm volatile("s_waitcnt vmcnt(%0)" :: "i"(N) : "memory"); }

__device__ __forceinline__ u32 rfl(u32 v) { return __builtin_amdgcn_readfirstlane(v); }
template <class T> __device__ __forceinline__ T* uni(T* q) {
  u64 v = (u64)q; v = ((u64)rfl((u32)(v >> 32)) << 32) | rfl((u32)v);
  asm volatile("s_nop 4" : "+s"(v));
  return (T*)v;
}
__device__ __forceinline__ void st4x4(char* base, u32 off, u32x4 a, u32x4 b, u32x4 c, u32x4 d) {
  asm volatile("global_store_dwordx4 %0, %1, %5 sc0 sc1\n\tglobal_store_dwordx4 %0, %2, %5 offset:1024 sc0 sc1\n\t"
               "global_store_dwordx4 %0, %3, %5 offset:2048 sc0 sc1\n\tglobal_store_dwordx4 %0, %4, %5 offset:3072 sc0 sc1\n\ts_nop 1"
               :: "v"(off), "v"(a), "v"(b), "v"(c), "v"(d), "s"(base) : "memory");
}
__device__ __forceinline__ void st4x2(char* base, u32 off, u32x4 a, u32x4 b) {
  asm volatile("global_store_dwordx4 %0, %1, %3 sc0 sc1\n\tglobal_store_dwordx4 %0, %2, %3 offset:1024 sc0 sc1\n\ts_nop 1"
               :: "v"(off), "v"(a), "v"(b), "s"(base) : "memory");
}
__device__ __forceinline__ void st4x1(char* base, u32 off, u32x4 a) {
  asm volatile("global_store_dwordx4 %0, %1, %2 sc0 sc1\n\ts_nop 1" :: "v"(off), "v"(a), "s"(base) : "memory");
}
__device__ __forceinline__ bool give_up(CParams& p, u32 call, u64 t0) {
  u32* tmo = (u32*)(p.ws[0] + TMO_OFF);
  if (rfl(__hip_atomic_load(tmo, __ATOMIC_RELAXED, __HIP_MEMORY_SCOPE_AGENT)) == call) return true;
  if (__builtin_amdgcn_s_memrealtime() - t0 > DEADLINE) {
    __hip_atomic_store(tmo, call, __ATOMIC_RELAXED, __HIP_MEMORY_SCOPE_AGENT);
    WAIT_V(0);
    return true;
  }
  return false;
}
__device__ __forceinline__ u64* flag_row(CParams& p, u32 call, int lane) {
  u32 lo = 0, hi = 0;
#pragma unroll
  for (int i = 0; i < NR; ++i) { u64 v = (u64)p.ws[i]; asm volatile("" : "+s"(v)); lo = lane == i ? (u32)v : lo; hi = lane == i ? (u32)(v >> 32) : hi; }
  return (u64*)(__attribute__((address_space(1))) u64*)((((u64)hi << 32) | lo) + F1_OFF + (call & 1) * F1_SZ) + (lane & 7) * NSRC;
}
__device__ __forceinline__ void post_seg(u64* row, u32 call, u32 seg, u32 len, u32 mask, int lane) {
  if (lane < NR && ((mask >> lane) & 1u)) __hip_atomic_store(row + seg, ((u64)call << 32) | len, __ATOMIC_RELAXED, __HIP_MEMORY_SCOPE_SYSTEM);
}
__device__ __forceinline__ u32 wave_scan(u32 v) {
  v += __builtin_amdgcn_update_dpp(0u, v, 0x111, 0xf, 0xf, false);
  v += __builtin_amdgcn_update_dpp(0u, v, 0x112, 0xf, 0xf, false);
  v += __builtin_amdgcn_update_dpp(0u, v, 0x114, 0xf, 0xf, false);
  v += __builtin_amdgcn_update_dpp(0u, v, 0x118, 0xf, 0xf, false);
  v += __builtin_amdgcn_update_dpp(0u, v, 0x142, 0xa, 0xf, false);
  v += __builtin_amdgcn_update_dpp(0u, v, 0x143, 0xc, 0xf, false);
  return v;
}
#define BOP(a, b, c, expr) (u32)__builtin_amdgcn_bitop3_b32((a), (b), (c), [] { constexpr u32 A = 0xf0, B = 0xcc, C = 0xaa; (void)A; (void)B; (void)C; return (expr) & 0xffu; }())
#define BFI(m, a, b) BOP(m, a, b, (A & B) | (~A & C))
struct Mv { u32 a0, a1, a2, c8, c16; };
__device__ __forceinline__ u32 byte_prefix_xor(u32 v) {
  v = BOP(v, v << 1, 0xfefefefeu, A ^ (B & C)); v = BOP(v, v << 2, 0xfcfcfcfcu, A ^ (B & C)); return BOP(v, v << 4, 0xf0f0f0f0u, A ^ (B & C));
}
__device__ __forceinline__ Mv move_masks(u32 m) {
  Mv r;
  r.c16 = __builtin_popcount(m & 0xffffu);
  r.c8 = (u32)__builtin_popcount(m & 0xffu) | ((u32)__builtin_popcount(m & 0xff0000u) << 16);
  u32 mk = BOP(m << 1, 0xfefefefeu, 0u, ~A & B);
  u32 mp = byte_prefix_xor(mk);
  r.a0 = mp & m; m = BOP(m, r.a0, r.a0 >> 1, (A ^ B) | C); mk = BOP(mk, mp, 0u, A & ~B);
  mp = byte_prefix_xor(mk);
  r.a1 = mp & m; m = BOP(m, r.a1, r.a1 >> 2, (A ^ B) | C); mk = BOP(mk, mp, 0u, A & ~B);
  r.a2 = BOP(0x80808080u - mk, 0x80808080u, m, (A ^ B) & C);
  return r;
}
__device__ __forceinline__ u32 pext(u32 x, u32 m, const Mv& v) {
  x &= m;
  u32 t = x & v.a0; x = BOP(x, t, t >> 1, (A ^ B) | C);
  t = x & v.a1; x = BOP(x, t, t >> 2, (A ^ B) | C);
  t = x & v.a2; x = BOP(x, t, t >> 4, (A ^ B) | C);
  t = __builtin_amdgcn_perm(0u, x, 0x0c030c01u);
  t = __builtin_bit_cast(u32, (u16x2)(__builtin_bit_cast(u16x2, t) << __builtin_bit_cast(u16x2, v.c8)));
  x = BOP(x, 0x00ff00ffu, t, (A & B) | C);
  return BOP(x, 0xffffu, (x >> 16) << v.c16, (A & B) | C);
}
__device__ __forceinline__ u32 pdep(u32 x, u32 m, const Mv& v) {
  x = __builtin_amdgcn_perm(x >> v.c16, x, 0x05040100u);
  const u32 t = __builtin_bit_cast(u32, (u16x2)(__builtin_bit_cast(u16x2, x) >> __builtin_bit_cast(u16x2, v.c8)));
  x = __builtin_amdgcn_perm(t, x, 0x06020400u);
  x = BFI(v.a2, x << 4, x); x = BFI(v.a1, x << 2, x); x = BFI(v.a0, x << 1, x);
  return x & m;
}
__device__ __forceinline__ void transpose4(u32 (&n)[4]) {
#pragma unroll
  for (int r = 0; r < 2; ++r) { const u32 a = n[r], c = n[r + 2]; n[r] = BFI(0x33333333u, a, c << 2); n[r + 2] = BFI(0x33333333u, a >> 2, c); }
#pragma unroll
  for (int r = 0; r < 4; r += 2) { const u32 a = n[r], c = n[r + 1]; n[r] = BFI(0x55555555u, a, c << 1); n[r + 1] = BFI(0x55555555u, a >> 1, c); }
}
__device__ __forceinline__ void or_bits(LAS u32* area, u32 bit, u32 val) {
  const u64 t = (u64)val << (bit & 31);
  LAS u32* q = area + (bit >> 5);
  if ((u32)t) __hip_atomic_fetch_or(q, (u32)t, __ATOMIC_RELAXED, __HIP_MEMORY_SCOPE_WORKGROUP);
  if ((u32)(t >> 32)) __hip_atomic_fetch_or(q + 1, (u32)(t >> 32), __ATOMIC_RELAXED, __HIP_MEMORY_SCOPE_WORKGROUP);
}
__device__ __forceinline__ u32 get_bits(const LAS u32* area, u32 bit) {
  const LAS u32* q = area + (bit >> 5);
  return __builtin_amdgcn_alignbit(q[1], q[0], bit & 31);
}

constexpr int ETAIL = SEG_STRIDE - SEG_P;
struct Blk {
  u32 S[8], V[8], P1, P0, Xm, q1, q0, r2, r1, r0, cnt, exc, base, N1, NX, bitw; bool raw;
  __device__ __forceinline__ u32 code(const u32x4 (&X)[4]) {
    u32 R[16], sum = 0;
#pragma unroll
    for (int i = 0; i < 16; ++i) { const u32 t = X[i >> 2][i & 3]; R[i] = __builtin_amdgcn_alignbit(t, t, 31); sum = __builtin_amdgcn_udot4(R[i], 0x01000100u, sum, false); }
    sum = __builtin_amdgcn_readlane(wave_scan(sum), 63);
    int basei = (int)((sum + 1741u) >> 11) - 10;
    basei = basei < 0 ? 0 : (basei > 240 ? 240 : basei);
    base = (u32)basei;
    const u16x2 bsub = {(unsigned short)(base << 8), (unsigned short)(base << 8)};
#pragma unroll
    for (int q = 0; q < 8; ++q) {
      const u32 a = __builtin_bit_cast(u32, (u16x2)(__builtin_bit_cast(u16x2, R[2 * q]) - bsub));
      const u32 b = __builtin_bit_cast(u32, (u16x2)(__builtin_bit_cast(u16x2, R[2 * q + 1]) - bsub));
      S[q] = __builtin_amdgcn_perm(b, a, 0x06040200u); V[q] = __builtin_amdgcn_perm(b, a, 0x07050301u);
    }
    u32 d[4], h[4];
#pragma unroll
    for (int q = 0; q < 4; ++q) { d[q] = BFI(0x0f0f0f0fu, V[2 * q], V[2 * q + 1] << 4); h[q] = BFI(0x0f0f0f0fu, V[2 * q] >> 4, V[2 * q + 1]); }
    transpose4(d); transpose4(h);
    Xm = (h[0] | h[1] | h[2] | h[3]) | (d[3] & d[2] & d[1]);
    const u32 m1 = d[3] & ~d[2];
    P1 = d[1] & m1 & ~Xm; P0 = (d[0] & m1) | Xm;
    const u32 E1 = ~(P1 | P0);
    const u32 Q1 = d[3] & ~d[0], Q0 = d[2] & BOP(d[3], d[1], d[0], (A & ~C) | (~A & B & C));
    const u32 E2 = E1 & ~(Q1 | Q0);
    const u32 nx = __builtin_popcount(Xm);
    cnt = (u32)__builtin_popcount(E1) | ((u32)__builtin_popcount(E2) << 12) | ((nx > 3u ? 3u : nx) << 24);
    const u32 inc = wave_scan(cnt), tot = __builtin_amdgcn_readlane(inc, 63);
    exc = inc - cnt; N1 = tot & 0xfffu; NX = tot >> 24;
    raw = (NX > XCAP) | (__ballot(nx > 3u) != 0);
    bitw = (2 * N1 + 3 * ((tot >> 12) & 0xfffu) + 31) >> 5;
    { const Mv mv = move_masks(E1); q1 = pext(Q1, E1, mv); q0 = pext(Q0, E1, mv); }
    { const Mv mv = move_masks(E2); r2 = pext(d[2], E2, mv); r1 = pext(d[1] | d[3], E2, mv); r0 = pext(d[0], E2, mv); }
    return raw ? 384u : bitw + NX;
  }
  __device__ __forceinline__ void pin() { asm volatile("" : "+v"(q1), "+v"(q0), "+v"(r2), "+v"(r1), "+v"(r0)); }
  __device__ __forceinline__ u32 pack(LAS char* img, const LAS u32* sz, int rb, int lane) {
    u32 voff = 0, vtot = 0;
#pragma unroll
    for (int i = 0; i < NB; ++i) { const u32 s = rfl(sz[i]); vtot += s; if (i < rb) voff += s; }
    LAS u32* var = (LAS u32*)(img + (SEG_V - SEG_P)) + voff;
    if (lane == 0) ((LAS u32*)(img + (SEG_H - SEG_P)))[rb] = base | ((raw ? 255u : NX) << 8) | (voff << 16);
    if (!raw) {
      *(LAS u32x2*)(img + rb * 512 + lane * 8) = (u32x2){P1, P0};
      const u32 o1 = 2 * (exc & 0xfffu), o2 = 2 * N1 + 3 * ((exc >> 12) & 0xfffu), n1 = cnt & 0xfffu, n2 = (cnt >> 12) & 0xfffu;
      if (__all(n1 <= 16u)) or_bits(var, o1, q1 | (q0 << n1));
      else { or_bits(var, o1, q1); or_bits(var, o1 + n1, q0); }
      if (__all(n2 <= 10u)) or_bits(var, o2, r2 | (r1 << n2) | (r0 << (2 * n2)));
      else { or_bits(var, o2, r2); or_bits(var, o2 + n2, r1); or_bits(var, o2 + 2 * n2, r0); }
      u32 xo = bitw + (exc >> 24);
      while (__any(Xm != 0u)) {
        if (Xm != 0u) {
          const u32 j = __builtin_ctz(Xm); Xm &= Xm - 1;
          const u32 vi = 2 * (j & 3) + ((j >> 2) & 1), by = j >> 3;
          const u32 k0 = 0u - (vi & 1u), k1 = 0u - ((vi >> 1) & 1u), k2 = 0u - (vi >> 2);
          const u32 vs = BFI(k2, BFI(k1, BFI(k0, V[7], V[6]), BFI(k0, V[5], V[4])), BFI(k1, BFI(k0, V[3], V[2]), BFI(k0, V[1], V[0])));
          const u32 ev = (base + (vs >> (8 * by))) & 255u;
          var[xo] = (u32)lane | (vi << 6) | (by << 9) | ((ev ^ (base + 9u)) << 16);
          ++xo;
        }
      }
    } else {
      u32 E[8]; const u32 b4 = base * 0x01010101u;
#pragma unroll
      for (int q = 0; q < 8; ++q) E[q] = ((V[q] & 0x7f7f7f7fu) + (b4 & 0x7f7f7f7fu)) ^ ((V[q] ^ b4) & 0x80808080u);
      *(LAS u32x2*)(img + rb * 512 + lane * 8) = (u32x2){E[0], E[1]};
#pragma unroll
      for (int q = 2; q < 8; ++q) var[(q - 2) * 64 + lane] = E[q];
    }
    return (SEG_V + vtot * 4 + 15) & ~15u;
  }
  __device__ __forceinline__ void store_bytes(char* enc, u32 seg, int rb, int lane) {
    st4x2(enc, seg * SEG_STRIDE + rb * 2048 + lane * 16, (u32x4){S[0], S[1], S[2], S[3]}, (u32x4){S[4], S[5], S[6], S[7]});
  }
};
__device__ __forceinline__ void img_zero(LAS char* img, int rb, int lane) {
  LAS u32x4* z = (LAS u32x4*)(img + (SEG_V - SEG_P) + rb * 1536);
  const u32x4 zero = {0u, 0u, 0u, 0u};
  z[lane] = zero; if (lane < 36) z[64 + lane] = zero;
}
__device__ __forceinline__ void img_store(char* enc, u32 seg, const LAS char* img, u32 len, int rb, int lane) {
  for (u32 o = rb * 1024 + lane * 16; o < len - SEG_P; o += NB * 1024) st4x1(enc, seg * SEG_STRIDE + SEG_P + o, *(const LAS u32x4*)(img + o));
}
struct Enc {
  char* enc; u64* row; LAS char* lds; u32 call; int w, lane; u32 par, pseg, plen, pmask;
  __device__ __forceinline__ void init(CParams& p, LAS char* l, u32 c, int w_, int lane_) {
    enc = uni(p.ws[0] + ENC_OFF + (c & 1) * ENC_SZ); row = flag_row(p, c, lane_); lds = l; call = c; w = w_; lane = lane_; par = 0; pseg = 0; plen = 0; pmask = 0;
  }
  __device__ __forceinline__ void post() { if ((w & (NB - 1)) == 0) post_seg(row, call, pseg, plen, pmask, lane); pmask = 0; }
  __device__ __forceinline__ void finish() { DRAIN; LBAR; post(); }
  template <class F> __device__ __forceinline__ void idle(F&& mid) { DRAIN; LBAR; post(); mid(); LBAR; par ^= 1; }
  template <class F> __device__ __forceinline__ void round(const u32x4 (&X)[4], u32 seg, u32 mask, F&& mid) {
    const int sg = w / NB, rb = w % NB;
    LAS char* img = lds + (par * 2 + sg) * ETAIL;
    LAS u32* szw = (LAS u32*)(lds + LDS_MAIN + 64) + par * 8;
    img_zero(img, rb, lane);
    Blk b; const u32 vd = b.code(X);
    if (lane == 0) szw[w] = vd;
    b.pin();
    DRAIN; LBAR;
    post();
    mid();
    b.store_bytes(enc, seg, rb, lane);
    const u32 len = b.pack(img, szw + sg * NB, rb, lane);
    LBAR;
    img_store(enc, seg, img, len, rb, lane);
    pseg = seg; plen = len; pmask = mask; par ^= 1;
  }
};
constexpr int XBUF = 65536;
__device__ __forceinline__ void load_job(CParams& p, LAS char* lds, int J, int buf, int w, int lane) {
  if (J >= NJOB) return;
  const int c = J >> 5, pn = (J >> 1) & 15, st = (J & 1) * NB + (w & (NB - 1));
  const char* q; u32 off, step;
  if (pn < 8) { q = p.x; off = ((u32)((pn * 8 + st) * 16 + (lane & 15)) * KD + c * 128 + (lane >> 4) * 8) * 2; step = 64; }
  else { q = p.w; off = (u32)(c * 128 + 4 * st + (lane >> 4)) * (NS * 2) + (pn - 8) * 256 + (((lane & 15) ^ (((lane >> 4) << 2) | (st & 3))) << 4); step = 65536; }
  q = uni(q);
  const u32 dst = (u32)(size_t)lds + XBUF + buf * 32768 + w * 4096;
#pragma unroll
  for (int k = 0; k < 4; ++k) asm volatile("s_mov_b32 m0, %2\n\ts_nop 0\n\tglobal_load_lds_dwordx4 %0, %1" :: "v"(off + k * step), "s"(q), "s"(dst + k * 1024) : "memory");
}
constexpr int NLATE = 64 + NENC;
struct Jobs {
  int nr, j0, j1, j2;
  __device__ __forceinline__ int job(int r) const { return r >= nr ? NJOB : (r == 0 ? j0 : (r <= UPR ? j1 + (r - 1) * 2 * NUP : j2 + (r - 1 - UPR) * 2 * NLATE)); }
};
__device__ __forceinline__ void encode_jobs(CParams& p, LAS char* lds, u32 call, int w, int lane, const Jobs& jb) {
  Enc e; e.init(p, lds, call, w, lane);
  const u32 near = cls(p.rank);
  for (int r = 0; r < jb.nr; ++r) {
    const int J = jb.job(r), J2 = jb.job(r + 2);
    auto mid = [&] { if (r != 0) load_job(p, lds, J2, r & 1, w, lane); };
    if (J < NJOB) {
      u32x4 X[4];
#pragma unroll
      for (int k = 0; k < 4; ++k) X[k] = *(const LAS u32x4*)(lds + XBUF + (r & 1) * 32768 + w * 4096 + k * 1024 + lane * 16);
      const u32 pn = (J >> 1) & 15;
      const u32 mask = pn < 8u ? (((pn >> 2) == near ? 0x90u : 0x60u) | ((pn & 2u) << 2)) : (pn < 12u ? 2u : 4u);
      e.round(X, pn * 128 + (J >> 5) * 2 + (J & 1), mask, mid);
    } else e.idle(mid);
    if (r == 0) { e.finish(); load_job(p, lds, J2, 0, w, lane); STAMP(1); }
  }
  e.finish();
  STAMP(2);
}

__device__ __forceinline__ void lds_post(LAS u32* q, u32 v) { asm volatile("ds_write_b32 %0, %1" :: "v"((u32)(size_t)q), "v"(v) : "memory"); }
__device__ __forceinline__ u32 lds_peek(LAS u32* q) { u32 v; asm volatile("ds_read_b32 %0, %1\n\ts_waitcnt lgkmcnt(0)" : "=&v"(v) : "v"((u32)(size_t)q) : "memory"); return rfl(v); }
constexpr u32 FIN = 0x40000000u;
__device__ __forceinline__ u32 c_offset(int rank, int d, u32 code) {
  return ((u32)(slot_row(rank ^ d, (code >> 3) & 31u) + (code & 1u) * 64) * NS + (d - 1) * 512 + ((code >> 2) & 1u) * 256 + ((code >> 1) & 1u) * 128) * 2;
}

__device__ __forceinline__ void pull_role(CParams& p, LAS char* lds, int d, int j, int ncu, u32 call, int w, int lane) {
  LAS u32* landed = (LAS u32*)(lds + LDS_MAIN + 128);
  LAS u32* posted = landed + 1;
  LAS u32* started = landed + 2;
  LAS u32* done = (LAS u32*)(lds + LDS_MAIN + 192);
  LAS u32* qitem = (LAS u32*)(lds + LDS_MAIN + 256);
  LAS u32* qlen = qitem + QD;
  const int rank = rfl(p.rank);
  if (w == 1) {
    const u64* f1 = (const u64*)(p.ws[0] + F1_OFF + (call & 1) * F1_SZ) + d * NSRC;
    u32* tk = (u32*)(p.ws[0] + TKT_OFF + d * 128);
    const u32 tbase = call * 8192u, n = list_len(d);
    if (lane == 0) __hip_atomic_fetch_max(tk, tbase, __ATOMIC_RELAXED, __HIP_MEMORY_SCOPE_AGENT);
    WAIT_V(0);
    u32 pos = j, left = 1, win = 0; u64 v = 0;
    for (u32 k = 0;; ++k) {
      if (left == 0) {
        u32 st;
        while ((st = lds_peek(started)) + LA <= k) __builtin_amdgcn_s_sleep(1);
        left = (n - pos) / (4u * ncu); left = pos >= n || left < 1u ? 1u : (left > GMAX ? GMAX : left);
        if (left == 1u && st + LA - k >= 2u) left = 2u;
        if (k < 2) left = 1u;
        u32 t = 0;
        if (lane == 0) t = __hip_atomic_fetch_add(tk, left, __ATOMIC_RELAXED, __HIP_MEMORY_SCOPE_AGENT);
        win = pos;
        const u32 wi = win + lane;
        v = __hip_atomic_load(f1 + (list_item(d, rank, wi < n ? wi : n - 1) >> 16), __ATOMIC_RELAXED, __HIP_MEMORY_SCOPE_SYSTEM);
        pos = rfl(t) - tbase + ncu;
      }
      u32 item = 0, len = 0;
      if (pos < n) {
        item = list_item(d, rank, pos);
        const u64 ok = __ballot((u32)(v >> 32) >= call);
        if (pos - win < 64u && ((ok >> (pos - win)) & 1)) len = __builtin_amdgcn_readlane((u32)v, pos - win);
        else {
          const u64 t0 = __builtin_amdgcn_s_memrealtime();
          for (u32 it = 1;; ++it) {
            const u64 u = __hip_atomic_load(f1 + (item >> 16), __ATOMIC_RELAXED, __HIP_MEMORY_SCOPE_SYSTEM);
            if (rfl((u32)(u >> 32)) >= call) { len = rfl((u32)u); break; }
            if ((it & 63u) == 0 && give_up(p, call, t0)) {
              if (d < 3) {
                if (pos + left <= 512u) { u32 t = 0; if (lane == 0) t = __hip_atomic_fetch_add(tk, 512u, __ATOMIC_RELAXED, __HIP_MEMORY_SCOPE_AGENT); pos = rfl(t) - tbase + ncu; left = 512u; }
                if (pos < 512u) pos = 512u;
                if (pos < n) *(u32*)(p.out + c_offset(rank, d, list_item(d, rank, pos))) = 0x7fc07fc0u;
              }
              pos = n; break;
            }
            __builtin_amdgcn_s_sleep(1);
          }
        }
        len = pos < n ? (len > (u32)SEG_STRIDE ? (u32)SEG_STRIDE : (len < 16u ? 16u : len & ~15u)) : 0u;
      }
      lds_post(qitem + (k % QD), item); lds_post(qlen + (k % QD), len); lds_post(posted, k + 1);
      if (k == 0) STAMP(1);
      if (len == 0) return;
      ++pos; --left;
    }
  }
  if (w == 0) {
    const char* rsrc = uni(p.ws[d] + ENC_OFF + (call & 1) * ENC_SZ);
    __builtin_amdgcn_s_waitcnt(0x0f70);
    int slot = 0; u32 use = 0, k = 0;
    for (;; ++k) {
      u32 item, len;
      for (u32 it = 0;; ++it) {
        u32 a, dn;
        asm volatile("ds_read_b32 %0, %4\n\tds_read_b32 %1, %5\n\tds_read_b32 %2, %6\n\tds_read_b32 %3, %6 offset:64\n\ts_waitcnt lgkmcnt(0)"
                     : "=&v"(a), "=&v"(dn), "=&v"(item), "=&v"(len)
                     : "v"((u32)(size_t)posted), "v"((u32)(size_t)(done + slot)), "v"((u32)(size_t)(qitem + (k % QD))) : "memory");
        if (rfl(a) > k && rfl(dn) >= use) break;
        if (it == 8) { WAIT_V(0); lds_post(landed, k); }
        __builtin_amdgcn_s_sleep(1);
      }
      len = rfl(len);
      if (len == 0) break;
      lds_post(started, k + 1);
      const char* gs = rsrc + (size_t)(rfl(item) >> 16) * SEG_STRIDE;
      const u32 ls = (u32)(size_t)lds + slot * SEG_STRIDE;
      for (u32 o = 0; o < len; o += 1024) {
        wait_v<INFL - 1>();
        if (o == (INFL - 1) * 1024) lds_post(landed, k);
        if (o + lane * 16 < len) asm volatile("s_mov_b32 m0, %2\n\ts_nop 0\n\tglobal_load_lds_dwordx4 %0, %1 sc0 sc1" :: "v"(o + lane * 16), "s"(gs), "s"(ls + o) : "memory");
      }
      if (++slot == NSLOT) { slot = 0; use += NB; }
    }
    WAIT_V(0); lds_post(landed, k | FIN);
    STAMP(2);
    return;
  }
  char* dec = uni(p.ws[0] + DEC_OFF);
  char* enc = uni(p.ws[0] + ENC_OFF + (call & 1) * ENC_SZ);
  u32* f2 = (u32*)(p.ws[0] + F2_OFF);
  u64* row = flag_row(p, call, lane);
  u32 have = 0, pend = ~0u, fseg = 0, flen = 0;
#define PUBLISH() do { if (pend != ~0u) { WAIT_V(0); if (lane == 0) __hip_atomic_store(f2 + pend, call, __ATOMIC_RELAXED, __HIP_MEMORY_SCOPE_SYSTEM); \
    if (flen) post_seg(row, call, fseg, flen, 1u << (d ^ 3), lane); pend = ~0u; flen = 0; } } while (0)
  for (u32 q = w - 2;; q += ND) {
    const u32 k = q / NB, rb = q % NB, slot = k % NSLOT;
    if (have <= k) have = lds_peek(landed);
    if ((have & ~FIN) <= k) {
      PUBLISH();
      while (have <= k) { __builtin_amdgcn_s_sleep(1); have = lds_peek(landed); }
      if ((have & ~FIN) <= k) break;
    }
    const u32 item = lds_peek(qitem + (k % QD));
    const LAS char* ls = lds + slot * SEG_STRIDE;
    u32x4 O[4];
    const u32x4 Sa = *(const LAS u32x4*)(ls + rb * 2048 + lane * 16), Sb = *(const LAS u32x4*)(ls + rb * 2048 + 1024 + lane * 16);
    const u32x2 Pp = *(const LAS u32x2*)(ls + SEG_P + rb * 512 + lane * 8);
    const u32 hdr = rfl(*(const LAS u32*)(ls + SEG_H + rb * 4));
    const u32 base = hdr & 255u, NX = (hdr >> 8) & 255u;
    const LAS u32* var = (const LAS u32*)(ls + SEG_V) + (hdr >> 16);
    u32 E[8];
    if (NX != 255u) {
      const u32 P1 = Pp[0], P0 = Pp[1], E1 = ~(P1 | P0);
      const u32 n1 = __builtin_popcount(E1), i1 = wave_scan(n1), N1 = __builtin_amdgcn_readlane(i1, 63), o1 = 2 * (i1 - n1);
      const u32 q1 = get_bits(var, o1), q0 = get_bits(var, o1 + n1);
      const Mv ma = move_masks(E1);
      const u32 n2 = __builtin_popcount((u32)((u64)~(q1 | q0) << (32 - n1)));
      const u32 i2 = wave_scan(n2), N2 = __builtin_amdgcn_readlane(i2, 63), o2 = 2 * N1 + 3 * (i2 - n2);
      const u32 r2 = get_bits(var, o2), r1 = get_bits(var, o2 + n2), r0 = get_bits(var, o2 + 2 * n2);
      const u32 xe = var[((2 * N1 + 3 * N2 + 31) >> 5) + lane];
      const u32 Q1 = pdep(q1, E1, ma), Q0 = pdep(q0, E1, ma), E2 = BOP(E1, Q1, Q0, A & ~(B | C));
      const Mv mb = move_masks(E2);
      const u32 R2 = pdep(r2, E2, mb), R1 = pdep(r1, E2, mb), R0 = pdep(r0, E2, mb);
      const u32 t7 = BOP(R2, R1, R0, A & B & C), q7 = BOP(Q1, Q0, 0u, ~A & B);
      u32 d[4] = {BOP(P0, q7, R0, A | B | C), BOP(P1 | q7, R1, t7, A | (B & ~C)), Q0 | R2, BOP(E1, Q1, t7, ~A | B | C)};
      transpose4(d);
      const u32 b4 = base * 0x01010101u;
#pragma unroll
      for (int i = 0; i < 4; ++i) { E[2 * i] = (d[i] & 0x0f0f0f0fu) + b4; E[2 * i + 1] = ((d[i] >> 4) & 0x0f0f0f0fu) + b4; }
      for (u32 i = 0; i < NX && i < 64u; ++i) {
        const u32 en = __builtin_amdgcn_readlane(xe, i);
        const u32 dv = (u32)lane == (en & 63u) ? ((en >> 16) & 255u) << (8 * ((en >> 9) & 3u)) : 0u;
        const int hot = (int)(0x80000000u >> ((en >> 6) & 7u));
#pragma unroll
        for (int k = 0; k < 8; ++k) E[k] = BOP(E[k], dv, (u32)((hot << k) >> 31), A ^ (B & C));
      }
    } else {
      E[0] = Pp[0]; E[1] = Pp[1];
#pragma unroll
      for (int k = 2; k < 8; ++k) E[k] = var[(k - 2) * 64 + lane];
    }
#pragma unroll
    for (int k = 0; k < 8; ++k) {
      const u32 s = k < 4 ? Sa[k & 3] : Sb[k & 3];
      const u32 a = __builtin_amdgcn_perm(E[k], s, 0x05010400u), b = __builtin_amdgcn_perm(E[k], s, 0x07030602u);
      O[k >> 1][(k & 1) * 2] = __builtin_amdgcn_alignbit(a, a, 1); O[k >> 1][(k & 1) * 2 + 1] = __builtin_amdgcn_alignbit(b, b, 1);
    }
    asm volatile("" : "+v"(O[0]), "+v"(O[1]), "+v"(O[2]), "+v"(O[3]));
    PUBLISH();
    if (item & D_C) {
      char* o = p.out + c_offset(rank, d, item) + (lane & 15) * (NS * 2) + rb * 64 + ((lane & 16) << 1) + ((lane & 32) >> 1);
#pragma unroll
      for (int m = 0; m < 4; ++m) *(u32x4*)(o + m * 16 * NS * 2) = O[m];
    } else {
      const u32 c = (item >> 1) & 63u, dp = (item >> 7) & 31u, st = (item & 1u) * NB + rb;
      st4x4(dec, dp * (u32)PANEL_B + (c * 8 + st) * 4096u + lane * 16, O[0], O[1], O[2], O[3]);
      pend = (dp * 8 + st) * NCH + c;
      if ((item & D_FWD) && rb == 0) {
        flen = lds_peek(qlen + (k % QD)); fseg = (12 + d) * 128 + (item & 127u);
        for (u32 o = lane * 16; o < flen; o += 1024) st4x1(enc, fseg * SEG_STRIDE + o, *(const LAS u32x4*)(ls + o));
      }
    }
    if (lane == 0) __hip_atomic_fetch_add(done + slot, 1u, __ATOMIC_RELAXED, __HIP_MEMORY_SCOPE_WORKGROUP);
    if (q < NB) STAMP(1);
  }
  STAMP(2);
}

__device__ __forceinline__ int wait_chunks(CParams& p, const u32* f, u32 o0, u32 o1, u32 o2, int need, u32 call) {
  const u64 t0 = __builtin_amdgcn_s_memrealtime();
  __builtin_amdgcn_s_waitcnt(0xc07f);
  for (u32 it = 1;; ++it) {
    const u32 a = __hip_atomic_load(f + o0, __ATOMIC_RELAXED, __HIP_MEMORY_SCOPE_SYSTEM);
    const u32 b = __hip_atomic_load(f + o1, __ATOMIC_RELAXED, __HIP_MEMORY_SCOPE_SYSTEM);
    const u32 c = __hip_atomic_load(f + o2, __ATOMIC_RELAXED, __HIP_MEMORY_SCOPE_SYSTEM);
    const u64 miss = __ballot(a < call || b < call || c < call);
    const int n = miss ? __builtin_ctzll(miss) : NCH;
    if (n >= need) return n;
    if ((it & 63u) == 0 && give_up(p, call, t0)) return NCH;
    __builtin_amdgcn_s_sleep(2);
  }
}
#define FL_ADDR(next) do { fbase = (next); const u32 c_ = fbase + (lane & 15); fvo = foff + 4 * (c_ < 63u ? c_ : 63u); } while (0)
#define FL_ISSUE() asm volatile("s_mov_b32 m0, %2\n\ts_nop 0\n\tglobal_load_lds_dword %0, %1 sc0 sc1" :: "v"(fvo), "s"(f2), "s"(lds0 + LDS_FLG + w * 256) : "memory")
#define FL_READ() fval = *(const volatile LAS u32*)(lds + LDS_FLG + w * 256 + lane * 4)
#define FL_USE() do { u64 m_ = __ballot(fval < call); m_ |= m_ >> 32; const u32 h_ = (u32)m_ | ((u32)m_ >> 16) | 0x10000u; \
    const int n_ = fbase + __builtin_ctz(h_); avail = n_ > avail ? n_ : avail; } while (0)
__device__ __forceinline__ u32 cvt_pk(float lo, float hi) { u32 r; asm("v_cvt_pk_bf16_f32 %0, %1, %2" : "=v"(r) : "v"(lo), "v"(hi)); return r; }
__device__ __forceinline__ float silu(float y) {
  const float c = __builtin_amdgcn_fmed3f(y, -60.0f, 60.0f);
  return y * __builtin_amdgcn_rcpf(1.0f + __builtin_amdgcn_exp2f(c * -1.44269504088896341f));
}
__device__ __forceinline__ u32x4 finish8(const f32x4& u, const f32x4& v, bool bad) {
  const u32 a0 = cvt_pk(silu(u[0]), silu(u[1])), a1 = cvt_pk(silu(u[2]), silu(u[3]));
  const u32 b0 = cvt_pk(silu(v[0]), silu(v[1])), b1 = cvt_pk(silu(v[2]), silu(v[3]));
  const auto s0 = __builtin_amdgcn_permlane16_swap(a0, b0, false, false);
  const auto s1 = __builtin_amdgcn_permlane16_swap(a1, b1, false, false);
  u32x4 o = {s0[0], s1[0], s0[1], s1[1]};
  if (bad) o = (u32x4){0x7fc07fc0u, 0x7fc07fc0u, 0x7fc07fc0u, 0x7fc07fc0u};
  return o;
}
__device__ __forceinline__ bool timed_out(CParams& p, u32 call) {
  return rfl(__hip_atomic_load((u32*)(p.ws[0] + TMO_OFF), __ATOMIC_RELAXED, __HIP_MEMORY_SCOPE_AGENT)) == call;
}
constexpr int HT = 16384;
#define A_SETUP(own) \
  const u32 sCa = (own) ? 256u : 32768u, sKa = (own) ? 128u : 2048u; \
  const u32 voffA0 = (own) ? (u32)(w * 16 + (lane >> 3)) * (KD * 2) + (((lane & 7) ^ (lane >> 4)) << 4) : (u32)(w * 4096 + lane * 16); \
  const u32 voffA1 = (own) ? (voffA0 ^ 64u) + 8 * KD * 2 - 1024 : voffA0;        \
  const u32 lds0 = (u32)(size_t)lds, ldsA = lds0 + w * 2048, ldsB = lds0 + w * 1024; \
  u32 aoff[2]; \
  aoff[0] = lds0 + wr * 8192 + ((own) ? (lane & 15) * 128 + (((lane >> 4) ^ ((lane >> 1) & 7)) << 4) : lane * 16); \
  aoff[1] = (own) ? aoff[0] ^ 64u : aoff[0] + 1024;
#define B_READ_OFF(n, t) (256 * (8 * (lane >> 4) + 4 * (t) + ((lane & 15) >> 2)) + 8 * (lane & 1) + \
    16 * ((4 * wc + 2 * (n) + ((lane & 3) >> 1)) ^ ((((lane & 15) >> 2) << 2) | ((2 * (lane >> 4) + (t)) & 3))))
#define VOFF_W ((u32)(4 * w + (lane >> 4)) * 2048u + (((lane & 15) ^ (((lane >> 4) << 2) | (w & 3))) << 4))
#define GB " sc0 sc1"
#define STA(buf, gp) asm volatile("s_mov_b32 m0, %3\n\ts_nop 0\n\tglobal_load_lds_dwordx4 %0, %2" GB "\n\tglobal_load_lds_dwordx4 %1, %2 offset:1024" GB \
    :: "v"(voffA0), "v"(voffA1), "s"(gp), "s"(ldsA + (buf)) : "memory")
#define STB(buf, gp) asm volatile("s_mov_b32 m0, %3\n\ts_nop 0\n\tglobal_load_lds_dwordx4 %0, %1" GB "\n\ts_add_u32 m0, m0, 0x2000\n\ts_nop 0\n\tglobal_load_lds_dwordx4 %0, %2" GB \
    :: "v"(voffB), "s"(gp), "s"((gp) + sIb), "s"(ldsB + (buf)) : "memory", "scc")
#define LDA(dst, ad, imm) do { _Pragma("unroll") for (int m = 0; m < 4; ++m) _Pragma("unroll") for (int k = 0; k < 2; ++k) \
    dst[m][k] = *(const LAS s16x8*)(size_t)(ad[k] + (imm) + m * 2048); } while (0)
#define LDB(dst, ad, imm) do { _Pragma("unroll") for (int n = 0; n < 2; ++n) _Pragma("unroll") for (int k = 0; k < 2; ++k) { \
    const s16x4 lo_ = __builtin_amdgcn_ds_read_tr16_b64_v4i16((LAS s16x4*)(size_t)(ad[n][0] + (imm) + k * 8192)); \
    const s16x4 hi_ = __builtin_amdgcn_ds_read_tr16_b64_v4i16((LAS s16x4*)(size_t)(ad[n][1] + (imm) + k * 8192)); \
    dst[n][k] = __builtin_shufflevector(lo_, hi_, 0, 1, 2, 3, 4, 5, 6, 7); } } while (0)
#define MMAX(ac, At, Bt, EXTRA) do { __builtin_amdgcn_s_setprio(1); EXTRA; _Pragma("unroll") for (int m = 0; m < 4; ++m) _Pragma("unroll") for (int n = 0; n < 2; ++n) \
    _Pragma("unroll") for (int k = 0; k < 2; ++k) ac[m][n] = __builtin_amdgcn_mfma_f32_16x16x32_bf16(__builtin_bit_cast(bf8, Bt[n][k]), __builtin_bit_cast(bf8, At[m][k]), ac[m][n], 0, 0, 0); \
    __builtin_amdgcn_s_setprio(0); } while (0)
#define MMA(ac, At, Bt) MMAX(ac, At, Bt, (void)0)
#define WEAVE do { _Pragma("unroll") for (int i_ = 0; i_ < 14; ++i_) { __builtin_amdgcn_sched_group_barrier(0x8, 1, 0); __builtin_amdgcn_sched_group_barrier(0x6, 1, 0); } } while (0)

__device__ __forceinline__ void gemm_big(CParams& p, LAS char* lds, int s0, int tn, u32 call, int w, int lane) {
  const int wr = w >> 2, wc = w & 3;
  const bool own = s0 < 8;
  const char* Ab = uni(own ? p.x + (size_t)s0 * PANEL_B : p.ws[0] + DEC_OFF + (size_t)s0 * PANEL_B);
  const char* Bb = uni(p.w + tn * 512);
  A_SETUP(own)
  const u32 voffB = VOFF_W; constexpr int sIb = 65536;
  const u32* flg = (const u32*)(p.ws[0] + F2_OFF) + (s0 * 8 + w) * NCH + lane;
  const char* f2 = uni(p.ws[0] + F2_OFF);
  const u32 foff = ((s0 + (lane >> 5)) * 8 + w) * NCH * 4;
  int fbase = 0, npoll = 0; u32 fval = 0, fvo = 0;
  u32 boff[2][2];
#pragma unroll
  for (int n = 0; n < 2; ++n)
#pragma unroll
    for (int t = 0; t < 2; ++t) { boff[n][t] = lds0 + 4 * HT + B_READ_OFF(n, t); asm volatile("" : "+v"(boff[n][t])); }
#define SA(b, h) (((b) * 2 + (h)) * HT)
#define SB(b, h) ((4 + (b) * 2 + (h)) * HT)
  f32x4 acc[2][2][4][2];
#pragma unroll
  for (int a = 0; a < 2; ++a)
#pragma unroll
    for (int b = 0; b < 2; ++b)
#pragma unroll
      for (int m = 0; m < 4; ++m)
#pragma unroll
        for (int n = 0; n < 2; ++n) acc[a][b][m][n] = (f32x4){0.f, 0.f, 0.f, 0.f};
  s16x8 At[4][2], B0[2][2], B1[2][2];
  int avail = own ? NCH : wait_chunks(p, flg, 0, 8 * NCH, 0, 1, call);
  STAMP(3); STAMPC(8);
  __builtin_amdgcn_s_waitcnt(0x0f70);
  FL_ADDR(1); FL_ISSUE();
  STB(SB(0, 0), Bb); STA(SA(0, 0), Ab); STB(SB(0, 1), Bb + 256); STA(SA(0, 1), Ab + PANEL_B);
  if (wr == 1) BAR;
  WAIT_V(4); BAR;
  STB(SB(1, 0), Bb + 131072); STA(SA(1, 0), Ab + sKa); STB(SB(1, 1), Bb + 131072 + 256);
  WAIT_V(6); BAR;
  for (int c = 0; c < NCH; ++c) {
    const int cn = c + 1 < NCH ? c + 1 : c;
    const char* a1 = Ab + (size_t)(c * sCa + sKa);
    const char* a2 = Ab + (size_t)(cn * sCa);
    const char* b2 = Bb + (size_t)cn * 262144;
    LDB(B0, boff, SA(0, 0)); SCHED; LDA(At, aoff, SA(0, 0)); STA(SA(1, 1), a1 + PANEL_B);
    WAIT_L(8); BAR; WAIT_L(0); MMA(acc[0][0], At, B0); BAR; SCHED;
    LDB(B1, boff, SA(0, 1)); FL_READ(); STB(SB(0, 0), b2);
    BAR; WAIT_L(0); SCHED; MMAX(acc[0][1], At, B1, FL_USE(); FL_ADDR(cn + 1); WEAVE); asm volatile("" : "+s"(avail), "+v"(fvo)); BAR;
    if (avail <= cn) { avail = wait_chunks(p, flg, 0, 8 * NCH, 0, cn + 1, call); ++npoll; }
    LDA(At, aoff, SA(0, 1)); STA(SA(0, 0), a2);
    BAR; WAIT_L(0); MMA(acc[1][0], At, B0); BAR; SCHED;
    STB(SB(0, 1), b2 + 256); FL_ISSUE();
    WAIT_V(7); BAR; MMA(acc[1][1], At, B1); BAR;
    LDB(B0, boff, SA(1, 0)); SCHED; LDA(At, aoff, SA(1, 0)); STA(SA(0, 1), a2 + PANEL_B);
    WAIT_L(8); BAR; WAIT_L(0); MMA(acc[0][0], At, B0); BAR; SCHED;
    LDB(B1, boff, SA(1, 1)); STB(SB(1, 0), b2 + 131072);
    BAR; WAIT_L(0); MMA(acc[0][1], At, B1); BAR;
    LDA(At, aoff, SA(1, 1)); STA(SA(1, 0), a2 + sKa);
    BAR; WAIT_L(0); MMA(acc[1][0], At, B0); BAR; SCHED;
    STB(SB(1, 1), b2 + 131072 + 256);
    WAIT_V(6); BAR; MMA(acc[1][1], At, B1); BAR;
  }
  if (wr == 0) BAR;
  STAMP(4); STAMPC(9); STAMPV(10, npoll);
  const bool bad = timed_out(p, call);
  int tz = threadIdx.x; asm volatile("" : "+v"(tz));
  char* ob = p.out + ((size_t)(wr * 64 + (tz & 15)) * NS + tn * 256 + wc * 32 + (tz & 16) + ((tz & 32) >> 2)) * 2;
#pragma unroll
  for (int ai = 0; ai < 2; ++ai) {
    char* oa = ob + (size_t)slot_row(p.rank, s0 + ai) * (NS * 2);
#pragma unroll
    for (int m = 0; m < 4; ++m)
#pragma unroll
      for (int bj = 0; bj < 2; ++bj) *(u32x4*)(oa + m * 16 * NS * 2 + bj * 256) = finish8(acc[ai][bj][m][0], acc[ai][bj][m][1], bad);
  }
  STAMP(5);
}

__device__ __forceinline__ void gemm_small(CParams& p, LAS char* lds, int s, int cp, u32 call, int w, int lane) {
  const int wr = w >> 2, wc = w & 3;
  const bool own = s < 8, far = cp < 4;
  const char* Ab = uni(own ? p.x + (size_t)s * PANEL_B : p.ws[0] + DEC_OFF + (size_t)s * PANEL_B);
  const char* Bb = uni(far ? p.ws[0] + DEC_OFF + (size_t)cp * 2 * PANEL_B : p.w + (cp - 4) * 512);
  A_SETUP(own)
  const u32 voffB = far ? (u32)(w * 4096 + lane * 16) : VOFF_W;
  const u32 sCb = far ? 32768u : 262144u, sKb = far ? 2048u : 131072u, sHb = far ? (u32)PANEL_B : 256u, sIb = far ? 1024u : 65536u;
  const u32* flg = (const u32*)(p.ws[0] + F2_OFF) + w * NCH + lane;
  const u32 fb0 = (far ? cp * 2 : s) * 8 * NCH, fb1 = (far ? cp * 2 + 1 : s) * 8 * NCH, fa = own ? fb0 : s * 8 * NCH;
  const char* f2 = uni(p.ws[0] + F2_OFF);
  const u32 foff = ((lane & 32 ? fa : (lane & 16 ? fb1 : fb0)) + w * NCH) * 4;
  int fbase = 0, npoll = 0; u32 fval = 0, fvo = 0;
  constexpr u32 TB = 3 * HT;
  u32 b0[2][2], bc[2][2], bn[2][2], ac[2] = {aoff[0], aoff[1]};
#pragma unroll
  for (int n = 0; n < 2; ++n)
#pragma unroll
    for (int t = 0; t < 2; ++t) { b0[n][t] = lds0 + B_READ_OFF(n, t); bc[n][t] = b0[n][t]; }
  f32x4 acc[2][4][2];
#pragma unroll
  for (int b = 0; b < 2; ++b)
#pragma unroll
    for (int m = 0; m < 4; ++m)
#pragma unroll
      for (int n = 0; n < 2; ++n) acc[b][m][n] = (f32x4){0.f, 0.f, 0.f, 0.f};
  s16x8 At[4][2], B0[2][2], B1[2][2];
  int avail = wait_chunks(p, flg, fa, fb0, fb1, 2, call);
  STAMP(3); STAMPC(8);
  __builtin_amdgcn_s_waitcnt(0x0f70);
  FL_ADDR(2);
  STB(HT, Bb); STA(0, Ab); STB(2 * HT, Bb + sHb); STB(TB + HT, Bb + sKb); STA(TB, Ab + sKa); FL_ISSUE(); STB(TB + 2 * HT, Bb + sKb + sHb); STB(2 * TB + HT, Bb + sCb);
  WAIT_V(11); BAR;
  LDB(B0, bc, HT);
  WAIT_L(0);
  if (wr == 1) BAR;
  u32 s0 = 0, s1 = TB, s2 = 2 * TB;
#define ST1(buf, gp) asm volatile("s_mov_b32 m0, %2\n\ts_nop 0\n\tglobal_load_lds_dwordx4 %0, %1" GB :: "v"(voffB), "s"(gp), "s"(ldsB + (buf)) : "memory")
#define KTILE(a2, b2, b3, V1, V2, ISS, CHECK, RD, X2) do { \
    STA(s2, a2); ST1(s2 + 2 * HT, (b2) + sHb); ISS; LDA(At, ac, 0); \
    _Pragma("unroll") for (int n = 0; n < 2; ++n) _Pragma("unroll") for (int t = 0; t < 2; ++t) { bn[n][t] = b0[n][t] + s1; asm volatile("" : "+v"(bn[n][t])); } \
    WAIT_V(V1); BAR; WAIT_L(0); MMA(acc[0], At, B0); BAR; SCHED; \
    CHECK; \
    ST1(s2 + 2 * HT + 8192, (b2) + sHb + sIb); STB(s0 + HT, b3); LDB(B1, bc, 2 * HT); SCHED; LDB(B0, bn, HT); \
    ac[0] = aoff[0] + s1; ac[1] = aoff[1] + s1; asm volatile("" : "+v"(ac[0]), "+v"(ac[1])); \
    WAIT_V(V2); WAIT_L(8); RD; BAR; WAIT_L(0); SCHED; MMAX(acc[1], At, B1, X2); asm volatile("" : "+s"(avail), "+v"(fvo)); BAR; SCHED; \
    _Pragma("unroll") for (int n = 0; n < 2; ++n) _Pragma("unroll") for (int t = 0; t < 2; ++t) bc[n][t] = bn[n][t]; \
    const u32 s_ = s0; s0 = s1; s1 = s2; s2 = s_; } while (0)
  for (int c = 0; c < NCH; ++c) {
    const int c1 = c + 1 < NCH ? c + 1 : NCH - 1, c2 = c + 2 < NCH ? c + 2 : NCH - 1;
    const char* a2 = Ab + (size_t)(c1 * sCa);
    const char* b2 = Bb + (size_t)(c1 * sCb);
    KTILE(a2, b2, b2 + sKb, 10, 9, (void)0, (void)0, FL_READ(), FL_USE(); FL_ADDR(c2 + 1); WEAVE);
    KTILE(a2 + sKa, b2 + sKb, Bb + (size_t)(c2 * sCb), 10, 10, FL_ISSUE(),
          if (avail <= c2) { avail = wait_chunks(p, flg, fa, fb0, fb1, c2 + 1, call); ++npoll; }, (void)0, (void)0);
  }
  if (wr == 0) BAR;
  STAMP(4); STAMPC(9); STAMPV(10, npoll);
  const u32 tmo = __hip_atomic_load((u32*)(p.ws[0] + TMO_OFF), __ATOMIC_RELAXED, __HIP_MEMORY_SCOPE_AGENT);
  int tz = threadIdx.x; asm volatile("" : "+v"(tz));
  u32x4 X[2][4];
#pragma unroll
  for (int bj = 0; bj < 2; ++bj)
#pragma unroll
    for (int m = 0; m < 4; ++m) { X[bj][m] = finish8(acc[bj][m][0], acc[bj][m][1], false); asm volatile("" : "+v"(X[bj][m])); }
  if (rfl(tmo) == call) {
#pragma unroll
    for (int bj = 0; bj < 2; ++bj)
#pragma unroll
      for (int m = 0; m < 4; ++m) X[bj][m] = (u32x4){0x7fc07fc0u, 0x7fc07fc0u, 0x7fc07fc0u, 0x7fc07fc0u};
  }
  if (far) {
    const int ln = tz & 63;
    char* enc = uni(p.ws[0] + ENC_OFF + (call & 1) * ENC_SZ);
    const u32 seg = 20 * 128 + (cp >> 1) * 256 + (s * 2 + (cp & 1)) * 4 + wr;
    LAS char* img = lds + wr * ETAIL;
    LAS u32* szw = (LAS u32*)(lds + LDS_MAIN + 64);
    STAMP(11);
    DRAIN; LBAR;
    STAMP(12);
    img_zero(img, wc, ln); img_zero(img + 2 * ETAIL, wc, ln);
    Blk b0_, b1_;
    const u32 v0 = b0_.code(X[0]), v1 = b1_.code(X[1]);
    b0_.store_bytes(enc, seg, wc, ln); b1_.store_bytes(enc, seg + 2, wc, ln);
    b0_.pin(); b1_.pin();
    if (ln == 0) { szw[w] = v0; szw[8 + w] = v1; }
    LBAR;
    STAMP(13);
    const u32 l0 = b0_.pack(img, szw + wr * NB, wc, ln), l1 = b1_.pack(img + 2 * ETAIL, szw + 8 + wr * NB, wc, ln);
    LBAR;
    STAMP(14);
    img_store(enc, seg, img, l0, wc, ln); img_store(enc, seg + 2, img + 2 * ETAIL, l1, wc, ln);
    u64* row = flag_row(p, call, ln);
    DRAIN; LBAR;
    if (wc < 2) post_seg(row, call, seg + 2 * wc, wc ? l1 : l0, 2u << (cp >> 1), ln);
  } else {
    char* ob = p.out + ((size_t)(slot_row(p.rank, s) + wr * 64 + (tz & 15)) * NS + (cp - 4) * 256 + wc * 32 + (tz & 16) + ((tz & 32) >> 2)) * 2;
#pragma unroll
    for (int m = 0; m < 4; ++m)
#pragma unroll
      for (int bj = 0; bj < 2; ++bj) *(u32x4*)(ob + m * 16 * NS * 2 + bj * 256) = X[bj][m];
  }
  STAMP(5);
}

__global__ __launch_bounds__(512, 2) void ag_gemm(Params) {
  CParams& p = *(CParams*)__builtin_amdgcn_kernarg_segment_ptr();
  extern __shared__ __attribute__((aligned(16))) char smem[];
  LAS char* lds = (LAS char*)smem;
  const int lane = threadIdx.x & 63, w = rfl(threadIdx.x >> 6), sg = w / NB;
  const int b = blockIdx.x, xi = (b >> 1) & 3, j = b >> 3;
  STAMP(0);
  const bool farx = b & 1, small = farx || (j >= 12 && j < 20), big = !farx && j < 12;
  const int o = (j - 20) * 4 + xi;
  Jobs jb = {0, NJOB, NJOB, NJOB};
  const bool enc = !small && !big && o >= NPULL;
  const int fs = far_slot(xi, j >> 2);
  const int v = enc ? 64 + o - NPULL : (farx ? (fs - 24) * 4 + (j & 3) : (small ? 32 + (j - 12) * 4 + xi : -1));
  if (small) { const int u = farx ? j * 4 + xi : 128 + (j - 12) * 4 + xi; jb.nr = UPR + 1; jb.j0 = sg ? NJOB : u; jb.j1 = NUP + 2 * u + sg; }
  if (v >= 0) { jb.nr = UPR + 1 + (NJOB - JOB0 - 2 * v + 2 * NLATE - 1) / (2 * NLATE); jb.j2 = JOB0 + 2 * v + sg; }
  const int nr = jb.nr, j0 = jb.job(0), j1 = jb.job(1);
  load_job(p, lds, j0, 0, w, lane);
  load_job(p, lds, j1, 1, w, lane);
  u32* cnt = (u32*)(p.ws[0] + CNT_OFF + b * 128);
  u32 call = *(const __attribute__((address_space(4))) u32*)cnt + 1;
  asm volatile("" : "+s"(call));
  if (threadIdx.x < 32) ((LAS u32*)(lds + LDS_MAIN + 128))[lane] = 0;
  LBAR;
  if (threadIdx.x == 511) *cnt = call;
  if (nr) {
    if (j1 < NJOB) WAIT_V(4); else WAIT_V(0);
    encode_jobs(p, lds, call, w, lane, jb);
  }
  if (big) gemm_big(p, lds, (xi * 3 + (j >> 2)) * 2, j & 3, call, w, lane);
  else if (small) gemm_small(p, lds, farx ? fs : 24 + xi * 2 + ((j - 12) >> 2), farx ? (j & 3) : 4 + (j & 3), call, w, lane);
  else if (o < NPULL) {
    pull_role(p, lds, 1 + o % 7, o / 7, CPL, call, w, lane);
  }
}
}

extern "C" void kernel_launch(int rank, int nranks, ncclComm_t comm, hipStream_t stream,
                              void* const* d_in, const int* in_sizes, int n_in,
                              void* d_out, int out_size, const int* in_dtypes, int out_dtype,
                              void* const* d_out_all, void* const* d_ws_all, size_t ws_bytes) {
  using namespace agg;
  if (nranks != NR || (unsigned)rank >= (unsigned)NR || ws_bytes < WS_NEED || n_in < 2) { fprintf(stderr, "ag_gemm: unexpected configuration\n"); return; }
  Params p;
  for (int i = 0; i < NR; ++i) p.ws[i] = (char*)d_ws_all[rank ^ i];
  p.x = (const char*)d_in[0]; p.w = (const char*)d_in[1]; p.out = (char*)d_out; p.rank = rank; p.pad = 0;
  hipError_t e = hipFuncSetAttribute((const void*)ag_gemm, hipFuncAttributeMaxDynamicSharedMemorySize, LDS_BYTES);
  if (e != hipSuccess) { fprintf(stderr, "HIP error %s\n", hipGetErrorString(e)); return; }
  hipLaunchKernelGGL(ag_gemm, dim3(GRID), dim3(512), LDS_BYTES, stream, p);
  e = hipGetLastError();
  if (e != hipSuccess) fprintf(stderr, "HIP error %s\n", hipGetErrorString(e));
}
```

```cpp
#include <hip/hip_runtime.h>
#include <rccl/rccl.h>
#include <cstdio>

#define INFL 11
#define OF1 5
#define OF2 10
#define OF3 10
#define OF4 9
#define FDIR 0xfffc000000000000ull
#define P3 19
#define O3 1024
#define LEAD 6
#define RS 0
#define G3 16
#define G4 16
#define HOPM 0
#define MIDR 2
#define UPR 1
#define LA 2u
#define DRAINIT 2
#define GMAXC 2u
#define GMAX 4u

namespace agg {
#define LAS __attribute__((address_space(3)))
typedef unsigned u32;
typedef unsigned long long u64;
typedef __attribute__((address_space(1))) unsigned long long gu64;
typedef unsigned u32x4 __attribute__((ext_vector_type(4)));
typedef unsigned u32x2 __attribute__((ext_vector_type(2)));
typedef short s16x4 __attribute__((ext_vector_type(4)));
typedef short s16x8 __attribute__((ext_vector_type(8)));
typedef unsigned short u16x2 __attribute__((ext_vector_type(2)));
typedef float f32x4 __attribute__((ext_vector_type(4)));
typedef __bf16 bf8 __attribute__((ext_vector_type(8)));

constexpr int NR = 8, KD = 8192, NS = 1024;
constexpr int NCH = 64;
constexpr int NB = 4;
constexpr int SEG_STRIDE = NB * 4096 + 1024;
constexpr int SEG_P = NB * 2048, SEG_H = NB * 2560, SEG_V = SEG_H + NB * 4;
#define NSLOT 8
constexpr int QD = 16;
constexpr int GRID = 256;
constexpr int LDS_MAIN = 147456, LDS_FLG = LDS_MAIN + 1024, LDS_BYTES = 1024 + LDS_FLG + 2048;
static_assert(NSLOT * SEG_STRIDE <= LDS_MAIN, "ring");
constexpr u32 XCAP = 64;
constexpr int NSRC = 28 * 128, NJOB = 16 * 128;

constexpr size_t PANEL_B = (size_t)2 << 20;
constexpr size_t DEC_OFF = 0;
constexpr size_t ENC_OFF = 32 * PANEL_B;
constexpr size_t ENC_SZ = (size_t)NSRC * SEG_STRIDE;
constexpr size_t F1_OFF = ENC_OFF + 2 * ENC_SZ;
constexpr size_t F1_SZ = (size_t)NR * NSRC * 8;
constexpr size_t F2_OFF = F1_OFF + 2 * F1_SZ;
constexpr size_t F2_SZ = (size_t)32 * 8 * NCH * 4;
constexpr size_t CNT_OFF = F2_OFF + F2_SZ;
constexpr size_t TKT_OFF = CNT_OFF + GRID * 128;
constexpr size_t MBX_OFF = TKT_OFF + NR * 128;
constexpr size_t TMO_OFF = MBX_OFF + NR * 128;
constexpr size_t DBG_OFF = TMO_OFF + 128;
constexpr size_t WS_NEED = DBG_OFF + (1 << 20);
constexpr u64 DEADLINE = 50000000ull;

struct Params { char* ws[NR];   const char* x; const char* w; char* out; int rank; int pad; };
typedef const __attribute__((address_space(4))) Params CParams;

__host__ __device__ __forceinline__ int cls(int r) { return (r ^ (r >> 1)) & 1; }
__host__ __device__ __forceinline__ int slot_row(int r, int s) {
  const int h4 = 4 * cls(r), m = r & 1;
  int owner, piece;
  if (s < 8) { owner = r; piece = s; }
  else if (s < 12) { owner = r ^ 3; piece = 2 + (s & 1) + ((s & 2) << 1); }
  else {
    const int y = r ^ (4 + (s & 3)), a = (s - 12) >> 2;
    if (a == 4) { owner = r ^ 3; piece = 4 * cls(y) + (y & 1); } else { owner = y; piece = h4 + (a == 0 ? m : (a == 3 ? m ^ 1 : a + 1)); }
  }
  return owner * 1024 + piece * 128;
}
struct Role { int kind, a, c, u; };
#define NPULL 40
#define LATE0 12
#define LATE1 16
constexpr int NENC = 40 - NPULL, NPA = NENC + 8 + (32 - LATE1) * 4, NPB = NENC + 88 + (32 - LATE0) * 4, NPC = NENC + 216;
__host__ __device__ __forceinline__ Role role_of(int b) {
  const int xi = (b & 7) >> 1, j = b >> 3, jj = j >> 2;
  if (b & 1) { const int s = jj < 2 ? 2 * xi + jj : 4 * jj + xi; return {0, s, j & 3, NENC + (s >= LATE1 ? 8 : 88) + (31 - s) * 4 + (j & 3)}; }
  if (j < 16) return {1, 64 + 12 * (4 * xi + jj), j & 3, NPA + 16 + j * 4 + xi};
  if (j < 20) return {1, 12 * xi, j & 3, NPA + (j - 16) * 4 + xi};
  if (j < 22) { const int i = 2 * xi + j - 20; return {2, 48 + 8 * (i >> 2), i & 3, NENC + i}; }
  const int o = (j - 22) * 4 + xi;
  return {o < NPULL ? 3 : 4, o, 0, o - NPULL};
}
__host__ __device__ __forceinline__ int link_of(int o) { return 1 + (o + 2) % 7; }
constexpr int LMAX = 896;
struct Lists { unsigned short e[3][LMAX]; unsigned short n[3]; unsigned short mark[3][4]; };
constexpr void merge(Lists& L, int t, int ns, const u64* has, const int* pace, const int* off) {
  int c[8] = {};
  for (;;) {
    int b = -1;
    for (int i = 0; i < ns; ++i) {
      while (c[i] < 64 && !((has[i] >> c[i]) & 1)) ++c[i];
      if (c[i] < 64 && (b < 0 || c[i] * pace[i] + off[i] < c[b] * pace[b] + off[b])) b = i;
    }
    if (b < 0) break;
    for (int g = 0; g < 2; ++g) L.e[t][L.n[t]++] = (unsigned short)(b << 7 | c[b] << 1 | g);
    if (++c[b] == 64 && b == 0) L.mark[t][0] = L.n[t] - 1;
  }
  L.mark[t][1] = L.n[t] - 1;
}
constexpr void pieces(Lists& L, int s0, int s1) {
  for (int bj = 0; bj < 2; ++bj) for (int s = s0; s < s1; ++s) for (int pt = 0; pt < 2; ++pt) for (int wr = 0; wr < 2; ++wr)
    if (wr || s < RS) L.e[0][L.n[0]++] = (unsigned short)(1 << 13 | wr | bj << 1 | pt << 2 | s << 3);
}
constexpr void hops(Lists& L, int k, int s0, int s1) {
  for (int bj = 0; bj < 2; ++bj) for (int s = s0 > RS ? s0 : RS; s < s1; s += 2) for (int d = 0; d < 2; ++d) L.e[2][L.n[2]++] = (unsigned short)(k << 13 | (s | bj) << 1 | d);
}
constexpr Lists make_lists() {
  Lists L = {};
  constexpr u64 ALL = ~0ull;
  { const u64 has[4] = {ALL, ALL, ALL, ALL}; const int pace[4] = {1, 1, 1, 1}, off[4] = {0, 0, LEAD, LEAD}; merge(L, 0, 4, has, pace, off); }
  pieces(L, 0, 12); L.mark[0][2] = L.n[0] - 1; pieces(L, 12, G3); pieces(L, G3, G4); L.mark[0][3] = L.n[0] - 1; pieces(L, G4, 32);
  { const u64 has[8] = {ALL, ALL, ALL, ALL, FDIR, FDIR, FDIR, FDIR}; const int pace[8] = {16, 16, 16, 16, P3, P3, P3, P3}, off[8] = {0, 0, 0, 0, O3, O3, O3, O3};
    merge(L, 1, 8, has, pace, off); }
  { const u64 has[5] = {ALL, ALL, ALL, ALL, ~FDIR}; const int pace[5] = {1, 1, 1, 1, 1}, off[5] = {0, OF1, OF2, OF3, OF4}; merge(L, 2, 5, has, pace, off); }
  hops(L, 2, 0, 12); hops(L, 3, 0, 12); L.mark[2][2] = L.n[2] - 1;
  if (HOPM == 0) { hops(L, 2, 12, G3); hops(L, 3, 12, G3); hops(L, 2, G3, G4); hops(L, 3, G3, G4); L.mark[2][3] = L.n[2] - 1; hops(L, 2, G4, 32); hops(L, 3, G4, 32); }
  else { hops(L, 2, 12, G3); hops(L, 2, G3, G4); hops(L, 3, 12, G3); L.mark[2][3] = L.n[2] - 1; hops(L, 2, G4, 32); hops(L, 3, G3, G4); hops(L, 3, G4, 32); }
  return L;
}
constexpr Lists LISTS = make_lists();
static_assert(OF4 > 0 && RS % 2 == 0 && G3 % 2 == 0 && G4 % 2 == 0, "lists");
__host__ __device__ __forceinline__ int list_type(int d) { return d < 3 ? 0 : (d == 3 ? 1 : 2); }
constexpr u32 D_FWD = 1u << 12, D_C = 1u << 13, D_REL = 1u << 14;
__host__ __device__ __forceinline__ u32 resolve(int d, int r, u32 en) {
  const u32 kind = en >> 13;
  if (kind) {
    const u32 o = kind == 1 ? d - 1 : en & 1u, e = (kind == 2 ? d : d ^ (o + 1)) & 3u;
    const u32 pc = (kind == 1 ? en & 255u : (en & 2u) | ((e & 2u) << 1) | (((en << 2) & 0xf8u) ^ ((e & 1u) << 3))) | (o << 8);
    return (((kind == 3 ? 24u : 20u) * 128u + pc) << 16) | (kind == 2 ? D_REL : D_C) | pc;
  }
  const u32 a = (en >> 7) & 7u, h4 = 4 * cls(r), m = r & 1; u32 sp, dp, fl = 0;
  if (d < 3) { dp = 4u * (d - 1) + a; sp = 8 + dp; }
  else if (d == 3) { if (a < 4) { sp = 2 + (a & 1) + ((a & 2) << 1); dp = 8 + a; } else { const int y = r ^ a; sp = 4 * cls(y) + (y & 1); dp = 24 + a; } }
  else if (a < 3) { sp = h4 + (a == 0 ? m : a + 1); dp = 8 + 4 * a + d; fl = a == 0 && !((FDIR >> ((en >> 1) & 63u)) & 1) ? D_FWD : 0u; }
  else { sp = a == 4 ? 12u + (d ^ 3) : h4 + (m ^ 1); dp = 8 + 4 * a + d; }
  return ((sp * 128u + (en & 127u)) << 16) | fl | (dp << 7) | (en & 127u);
}
__host__ __device__ __forceinline__ u32 input_mask(int r, u32 pn, u32 c) {
  return pn < 8u ? (((pn >> 2) == (u32)cls(r) ? 0x90u : 0x60u) | ((pn & 2u) || ((FDIR >> c) & 1) ? 8u : 0u)) : (pn < 12u ? 2u : 4u);
}
__host__ __device__ __forceinline__ u32 piece_mask(u32 o, u32 code) {
  return (code >> 3) >= (u32)RS && !(code & 1u) ? 16u << ((((code >> 1) ^ (code >> 3)) & 1u) | ((code >> 1) & 2u)) : 2u << o;
}

#define STAMPV(i, v) do { (void)(v); } while (0)
#define LASTPOLL do {} while (0)
#define CLKS(c) do {} while (0)
#define STAMP(i) do {} while (0)
#define STAMPC(i) do {} while (0)

#define WAIT_V(n) asm volatile("s_waitcnt vmcnt(" #n ")" ::: "memory")
#define WAIT_L(n) asm volatile("s_waitcnt lgkmcnt(" #n ")" ::: "memory")
#define DRAIN do { WAIT_V(0); __builtin_amdgcn_s_waitcnt(0x0f70); } while (0)
#define BAR __builtin_amdgcn_s_barrier()
#define SCHED __builtin_amdgcn_sched_barrier(0)
#define LBAR do { WAIT_L(0); BAR; asm volatile("" ::: "memory"); } while (0)
template <int N> __device__ __forceinline__ void wait_v() { asm volatile("s_waitcnt vmcnt(%0)" :: "i"(N) : "memory"); }

__device__ __forceinline__ u32 rfl(u32 v) { return __builtin_amdgcn_readfirstlane(v); }
template <class T> __device__ __forceinline__ T* uni(T* q) {
  u64 v = (u64)q; v = ((u64)rfl((u32)(v >> 32)) << 32) | rfl((u32)v);
  asm volatile("s_nop 4" : "+s"(v));
  return (T*)(__attribute__((address_space(1))) T*)v;
}
__device__ __forceinline__ void st4x4(char* base, u32 off, u32x4 a, u32x4 b, u32x4 c, u32x4 d) {
  asm volatile("global_store_dwordx4 %0, %1, %5 sc0 sc1\n\tglobal_store_dwordx4 %0, %2, %5 offset:1024 sc0 sc1\n\t"
               "global_store_dwordx4 %0, %3, %5 offset:2048 sc0 sc1\n\tglobal_store_dwordx4 %0, %4, %5 offset:3072 sc0 sc1\n\ts_nop 1"
               :: "v"(off), "v"(a), "v"(b), "v"(c), "v"(d), "s"(base) : "memory");
}
__device__ __forceinline__ void st4x2(char* base, u32 off, u32x4 a, u32x4 b) {
  asm volatile("global_store_dwordx4 %0, %1, %3 sc0 sc1\n\tglobal_store_dwordx4 %0, %2, %3 offset:1024 sc0 sc1\n\ts_nop 1"
               :: "v"(off), "v"(a), "v"(b), "s"(base) : "memory");
}
__device__ __forceinline__ void st4x1(char* base, u32 off, u32x4 a) {
  asm volatile("global_store_dwordx4 %0, %1, %2 sc0 sc1\n\ts_nop 1" :: "v"(off), "v"(a), "s"(base) : "memory");
}
__device__ __forceinline__ bool give_up(CParams& p, u32 call, u64 t0) {
  u32* tmo = (u32*)(p.ws[0] + TMO_OFF);
  if (rfl(__hip_atomic_load(tmo, __ATOMIC_RELAXED, __HIP_MEMORY_SCOPE_AGENT)) == call) return true;
  if (__builtin_amdgcn_s_memrealtime() - t0 > DEADLINE) {
    __hip_atomic_store(tmo, call, __ATOMIC_RELAXED, __HIP_MEMORY_SCOPE_AGENT);
    WAIT_V(0);
    return true;
  }
  return false;
}
__device__ __forceinline__ gu64* flag_row(CParams& p, u32 call, int lane) {
  u32 lo = 0, hi = 0;
#pragma unroll
  for (int i = 0; i < NR; ++i) { u64 v = (u64)p.ws[i]; asm volatile("" : "+s"(v)); lo = lane == i ? (u32)v : lo; hi = lane == i ? (u32)(v >> 32) : hi; }
  return (gu64*)((((u64)hi << 32) | lo) + F1_OFF + (call & 1) * F1_SZ) + (lane & 7) * NSRC;
}
__device__ __forceinline__ void post_seg(gu64* row, u32 call, u32 seg, u32 len, u32 mask, int lane) {
  if (lane < NR && ((mask >> lane) & 1u)) __hip_atomic_store(row + seg, ((u64)call << 32) | len, __ATOMIC_RELAXED, __HIP_MEMORY_SCOPE_SYSTEM);
}
constexpr u32 MAIL0 = 20 * 128;
__device__ __forceinline__ u32 mail_place(const char* mbx, u32 mask, int lane) {
  u32 t;
  asm volatile("s_mov_b64 exec, %4\n\tglobal_atomic_add %0, %1, %2, %3 sc0\n\ts_mov_b64 exec, -1" : "=&v"(t) : "v"(lane * 128), "v"(1u), "s"(mbx), "s"((u64)(mask & 0xffu)) : "memory");
  return t;
}
__device__ __forceinline__ void post_mail(gu64* row, u32 call, u32 place, u32 seg, u32 len, u32 mask, int lane) {
  asm volatile("" : "+v"(place));
  constexpr u32 n0 = LISTS.n[0] - LISTS.mark[0][1] - 1, n2 = LISTS.n[2] - LISTS.mark[2][1] - 1;
  post_seg(row, call, MAIL0 + (lane < 3 ? place % n0 : place % (n2 ? n2 : 1)), (seg << 16) | (len >> 4), mask, lane);
}
__device__ __forceinline__ u32 wave_scan(u32 v) {
  v += __builtin_amdgcn_update_dpp(0u, v, 0x111, 0xf, 0xf, false);
  v += __builtin_amdgcn_update_dpp(0u, v, 0x112, 0xf, 0xf, false);
  v += __builtin_amdgcn_update_dpp(0u, v, 0x114, 0xf, 0xf, false);
  v += __builtin_amdgcn_update_dpp(0u, v, 0x118, 0xf, 0xf, false);
  v += __builtin_amdgcn_update_dpp(0u, v, 0x142, 0xa, 0xf, false);
  v += __builtin_amdgcn_update_dpp(0u, v, 0x143, 0xc, 0xf, false);
  return v;
}
#define BOP(a, b, c, expr) (u32)__builtin_amdgcn_bitop3_b32((a), (b), (c), [] { constexpr u32 A = 0xf0, B = 0xcc, C = 0xaa; (void)A; (void)B; (void)C; return (expr) & 0xffu; }())
#define BFI(m, a, b) BOP(m, a, b, (A & B) | (~A & C))
struct Mv { u32 a0, a1, a2, c8, c16; };
__device__ __forceinline__ u32 byte_prefix_xor(u32 v) {
  v = BOP(v, v << 1, 0xfefefefeu, A ^ (B & C)); v = BOP(v, v << 2, 0xfcfcfcfcu, A ^ (B & C)); return BOP(v, v << 4, 0xf0f0f0f0u, A ^ (B & C));
}
__device__ __forceinline__ Mv calc_masks(u32 m) {
  Mv r;
  r.c16 = __builtin_popcount(m & 0xffffu);
  r.c8 = (u32)__builtin_popcount(m & 0xffu) | ((u32)__builtin_popcount(m & 0xff0000u) << 16);
  u32 mk = BOP(m << 1, 0xfefefefeu, 0u, ~A & B);
  u32 mp = byte_prefix_xor(mk);
  r.a0 = mp & m; m = BOP(m, r.a0, r.a0 >> 1, (A ^ B) | C); mk = BOP(mk, mp, 0u, A & ~B);
  mp = byte_prefix_xor(mk);
  r.a1 = mp & m; m = BOP(m, r.a1, r.a1 >> 2, (A ^ B) | C); mk = BOP(mk, mp, 0u, A & ~B);
  r.a2 = BOP(0x80808080u - mk, 0x80808080u, m, (A ^ B) & C);
  return r;
}
__device__ __forceinline__ u32 mask_entry(u32 b) { const Mv v = calc_masks(b); return v.c16 | (v.a0 << 8) | (v.a1 << 16) | (v.a2 << 24); }
__device__ __forceinline__ Mv move_masks(u32 m) {
  const u32 e0 = *(const LAS u32*)(size_t)((m & 255u) << 2), e1 = *(const LAS u32*)(size_t)(((m >> 8) & 255u) << 2);
  const u32 e2 = *(const LAS u32*)(size_t)(((m >> 16) & 255u) << 2), e3 = *(const LAS u32*)(size_t)((m >> 24) << 2);
  const u32 l02 = __builtin_amdgcn_perm(e2, e0, 0x05040100u), l13 = __builtin_amdgcn_perm(e3, e1, 0x05040100u);
  const u32 h02 = __builtin_amdgcn_perm(e2, e0, 0x07060302u), h13 = __builtin_amdgcn_perm(e3, e1, 0x07060302u);
  Mv r;
  r.a0 = __builtin_amdgcn_perm(l13, l02, 0x07030501u); r.a1 = __builtin_amdgcn_perm(h13, h02, 0x06020400u); r.a2 = __builtin_amdgcn_perm(h13, h02, 0x07030501u);
  r.c8 = l02; r.c16 = e0 + e1;
  return r;
}
__device__ __forceinline__ u32 pext(u32 x, u32 m, const Mv& v) {
  x &= m;
  u32 t = x & v.a0; x = BOP(x, t, t >> 1, (A ^ B) | C);
  t = x & v.a1; x = BOP(x, t, t >> 2, (A ^ B) | C);
  t = x & v.a2; x = BOP(x, t, t >> 4, (A ^ B) | C);
  t = __builtin_amdgcn_perm(0u, x, 0x0c030c01u);
  asm("v_pk_lshlrev_b16 %0, %1, %0" : "+v"(t) : "v"(v.c8));
  x = BOP(x, 0x00ff00ffu, t, (A & B) | C);
  return BOP(x, 0xffffu, (x >> 16) << (v.c16 & 31u), (A & B) | C);
}
__device__ __forceinline__ u32 pdep(u32 x, const Mv& v) {
  x = __builtin_amdgcn_perm(x >> (v.c16 & 31u), x, 0x05040100u);
  u32 t; asm("v_pk_lshrrev_b16 %0, %1, %2" : "=v"(t) : "v"(v.c8), "v"(x));
  x = __builtin_amdgcn_perm(t, x, 0x06020400u);
  x = BFI(v.a2, x << 4, x); x = BFI(v.a1, x << 2, x);
  return BFI(v.a0, x << 1, x);
}
__device__ __forceinline__ void transpose4(u32 (&n)[4]) {
#pragma unroll
  for (int r = 0; r < 2; ++r) { const u32 a = n[r], c = n[r + 2]; n[r] = BFI(0x33333333u, a, c << 2); n[r + 2] = BFI(0x33333333u, a >> 2, c); }
#pragma unroll
  for (int r = 0; r < 4; r += 2) { const u32 a = n[r], c = n[r + 1]; n[r] = BFI(0x55555555u, a, c << 1); n[r + 1] = BFI(0x55555555u, a >> 1, c); }
}
__device__ __forceinline__ void or_bits(LAS u32* area, u32 bit, u32 val) {
  const u64 t = (u64)val << (bit & 31);
  LAS u32* q = area + (bit >> 5);
  if ((u32)t) __hip_atomic_fetch_or(q, (u32)t, __ATOMIC_RELAXED, __HIP_MEMORY_SCOPE_WORKGROUP);
  if ((u32)(t >> 32)) __hip_atomic_fetch_or(q + 1, (u32)(t >> 32), __ATOMIC_RELAXED, __HIP_MEMORY_SCOPE_WORKGROUP);
}
__device__ __forceinline__ u32 mad3(u32 a, u32 c) { u32 r; asm("v_mad_u32_u24 %0, %1, 3, %2" : "=v"(r) : "v"(a), "s"(c)); return r; }
__device__ __forceinline__ u32 get_bits(u32 bit) {
  const LAS u32* q = (const LAS u32*)(size_t)((bit >> 3) & ~3u);
  return __builtin_amdgcn_alignbit(q[1], q[0], bit & 31);
}

constexpr int ETAIL = SEG_STRIDE - SEG_P;
struct Blk {
  u32 R[16], S[8], P1, P0, Xm, q1, q0, r2, r1, r0, cnt, exc, base, N1, NX, bitw; bool raw;
  __device__ __forceinline__ u32 code(const u32x4 (&X)[4]) {
    u32 sum = 0;
#pragma unroll
    for (int i = 0; i < 16; ++i) { const u32 t = X[i >> 2][i & 3]; R[i] = __builtin_amdgcn_alignbit(t, t, 31); if (i & 1) sum = __builtin_amdgcn_udot4(R[i], 0x01000100u, sum, false); }
#pragma unroll
    for (int q = 0; q < 8; ++q) S[q] = __builtin_amdgcn_perm(R[2 * q + 1], R[2 * q], 0x06040200u);
    sum = __builtin_amdgcn_readlane(wave_scan(sum), 63);
    int basei = (int)((sum + 870u) >> 10) - 10;
    basei = basei < 0 ? 0 : (basei > 240 ? 240 : basei);
    base = (u32)basei;
    const u16x2 bsub = {(unsigned short)(base << 8), (unsigned short)(base << 8)}, top = {0x0f00, 0x0f00};
    u32 V[8], d[4];
#pragma unroll
    for (int q = 0; q < 8; ++q) {
      const u16x2 a = __builtin_elementwise_min((u16x2)(__builtin_bit_cast(u16x2, R[2 * q]) - bsub), top);
      const u16x2 b = __builtin_elementwise_min((u16x2)(__builtin_bit_cast(u16x2, R[2 * q + 1]) - bsub), top);
      V[q] = __builtin_amdgcn_perm(__builtin_bit_cast(u32, b), __builtin_bit_cast(u32, a), 0x07050301u);
    }
#pragma unroll
    for (int q = 0; q < 4; ++q) d[q] = V[2 * q] | (V[2 * q + 1] << 4);
    transpose4(d);
    Xm = BOP(d[3], d[2], d[1], A & B & C);
    P1 = BOP(d[3], d[2], d[1], A & ~B & C); P0 = BOP(d[3], d[2], d[0], A & ~B & C) | Xm;
    const u32 E1 = ~(P1 | P0);
    const u32 Q1 = d[3] & ~d[0], Q0 = d[2] & BOP(d[3], d[1], d[0], (A & ~C) | (~A & B & C));
    const u32 E2 = E1 & ~(Q1 | Q0);
    const u32 nx = __builtin_popcount(Xm);
    cnt = (u32)__builtin_popcount(E1) | ((u32)__builtin_popcount(E2) << 12) | ((nx > 3u ? 3u : nx) << 24);
    const u32 inc = wave_scan(cnt), tot = __builtin_amdgcn_readlane(inc, 63);
    exc = inc - cnt; N1 = tot & 0xfffu; NX = tot >> 24;
    raw = (NX > XCAP) | (__ballot(nx > 3u) != 0);
    bitw = (2 * N1 + 3 * ((tot >> 12) & 0xfffu) + 31) >> 5;
    { const Mv mv = move_masks(E1); q1 = pext(Q1, E1, mv); q0 = pext(Q0, E1, mv); }
    { const Mv mv = move_masks(E2); r2 = pext(d[2], E2, mv); r1 = pext(d[1] | d[3], E2, mv); r0 = pext(d[0], E2, mv); }
    return raw ? 384u : bitw + NX;
  }
  __device__ __forceinline__ void pin() { asm volatile("" : "+v"(q1), "+v"(q0), "+v"(r2), "+v"(r1), "+v"(r0)); }
  __device__ __forceinline__ u32 pack(LAS char* img, const LAS u32* sz, int rb, int lane) {
    u32 voff = 0, vtot = 0;
#pragma unroll
    for (int i = 0; i < NB; ++i) { const u32 s = rfl(sz[i]); vtot += s; if (i < rb) voff += s; }
    LAS u32* var = (LAS u32*)(img + (SEG_V - SEG_P)) + voff;
    if (lane == 0) ((LAS u32*)(img + (SEG_H - SEG_P)))[rb] = base | ((raw ? 255u : NX) << 8) | (voff << 16);
    u32 E[8];
    if (NX) {
#pragma unroll
      for (int q = 0; q < 8; ++q) E[q] = __builtin_amdgcn_perm(R[2 * q + 1], R[2 * q], 0x07050301u);
    }
    if (!raw) {
      *(LAS u32x2*)(img + rb * 512 + lane * 8) = (u32x2){P1, P0};
      const u32 o1 = 2 * (exc & 0xfffu), o2 = 2 * N1 + 3 * ((exc >> 12) & 0xfffu), n1 = cnt & 0xfffu, n2 = (cnt >> 12) & 0xfffu;
      if (__all(n1 <= 16u)) or_bits(var, o1, q1 | (q0 << n1));
      else { or_bits(var, o1, q1); or_bits(var, o1 + n1, q0); }
      if (__all(n2 <= 10u)) or_bits(var, o2, r2 | (r1 << n2) | (r0 << (2 * n2)));
      else { or_bits(var, o2, r2); or_bits(var, o2 + n2, r1); or_bits(var, o2 + 2 * n2, r0); }
      u32 xo = bitw + (exc >> 24);
      if (NX) while (__any(Xm != 0u)) {
        if (Xm != 0u) {
          const u32 j = __builtin_ctz(Xm); Xm &= Xm - 1;
          const u32 vi = 2 * (j & 3) + ((j >> 2) & 1), by = j >> 3;
          const u32 k0 = 0u - (vi & 1u), k1 = 0u - ((vi >> 1) & 1u), k2 = 0u - (vi >> 2);
          const u32 vs = BFI(k2, BFI(k1, BFI(k0, E[7], E[6]), BFI(k0, E[5], E[4])), BFI(k1, BFI(k0, E[3], E[2]), BFI(k0, E[1], E[0])));
          const u32 ev = (vs >> (8 * by)) & 255u;
          var[xo] = (u32)lane | (vi << 6) | (by << 9) | ((ev ^ (base + 9u)) << 16);
          ++xo;
        }
      }
    } else {
      *(LAS u32x2*)(img + rb * 512 + lane * 8) = (u32x2){E[0], E[1]};
#pragma unroll
      for (int q = 2; q < 8; ++q) var[(q - 2) * 64 + lane] = E[q];
    }
    return (SEG_V + vtot * 4 + 15) & ~15u;
  }
  __device__ __forceinline__ void store_bytes(char* enc, u32 seg, int rb, int lane) {
    st4x2(enc, seg * SEG_STRIDE + rb * 2048 + lane * 16, (u32x4){S[0], S[1], S[2], S[3]}, (u32x4){S[4], S[5], S[6], S[7]});
  }
};
__device__ __forceinline__ void img_zero(LAS char* img, int rb, int lane) {
  LAS u32x4* z = (LAS u32x4*)(img + (SEG_V - SEG_P) + rb * 1536);
  const u32x4 zero = {0u, 0u, 0u, 0u};
  z[lane] = zero; if (lane < 36) z[64 + lane] = zero;
}
__device__ __forceinline__ void img_store(char* enc, u32 seg, const LAS char* img, u32 len, int rb, int lane) {
  for (u32 o = rb * 1024 + lane * 16; o < len - SEG_P; o += NB * 1024) st4x1(enc, seg * SEG_STRIDE + SEG_P + o, *(const LAS u32x4*)(img + o));
}
struct Enc {
  char* enc; gu64* row; LAS char* lds; u32 call; int w, lane; u32 par, pseg, plen, pmask, place;
  __device__ __forceinline__ void init(CParams& p, LAS char* l, u32 c, int w_, int lane_) {
    enc = uni(p.ws[0] + ENC_OFF + (c & 1) * ENC_SZ); row = flag_row(p, c, lane_); lds = l; call = c; w = w_; lane = lane_; par = 0; pseg = 0; plen = 0; pmask = 0;
  }
  __device__ __forceinline__ void post() { if ((w & (NB - 1)) == 0) post_seg(row, call, pseg, plen, pmask, lane); pmask = 0; }
  __device__ __forceinline__ void finish() { DRAIN; LBAR; post(); }
  __device__ __forceinline__ void draw(const char* mbx) { place = mail_place(mbx, (w & (NB - 1)) == 0 ? pmask : 0u, lane); }
  __device__ __forceinline__ void finish_mail() { DRAIN; LBAR; if ((w & (NB - 1)) == 0) post_mail(row, call, place, pseg, plen, pmask, lane); pmask = 0; }
  template <class F> __device__ __forceinline__ void idle(F&& mid) { DRAIN; LBAR; post(); mid(); LBAR; par ^= 1; }
  template <class F> __device__ __forceinline__ void round(const u32x4 (&X)[4], u32 seg, u32 mask, F&& mid) {
    const int sg = w / NB, rb = w % NB;
    LAS char* img = lds + (par * 2 + sg) * ETAIL;
    LAS u32* szw = (LAS u32*)(lds + LDS_MAIN + 64) + par * 8;
    img_zero(img, rb, lane);
    Blk b; const u32 vd = b.code(X);
    if (lane == 0) szw[w] = vd;
    b.pin();
    DRAIN; LBAR;
    post();
    mid();
    b.store_bytes(enc, seg, rb, lane);
    const u32 len = b.pack(img, szw + sg * NB, rb, lane);
    LBAR;
    img_store(enc, seg, img, len, rb, lane);
    pseg = seg; plen = len; pmask = mask; par ^= 1;
  }
};
constexpr int XBUF = 65536;
__device__ __forceinline__ void load_job(CParams& p, LAS char* lds, int J, int buf, int w, int lane) {
  if (J >= NJOB) return;
  const int c = J >> 5, pn = (J >> 1) & 15, st = (J & 1) * NB + (w & (NB - 1));
  const char* q; u32 off, step;
  if (pn < 8) { q = p.x; off = ((u32)((pn * 8 + st) * 16 + (lane & 15)) * KD + c * 128 + (lane >> 4) * 8) * 2; step = 64; }
  else { q = p.w; off = (u32)(c * 128 + (lane >> 1)) * (NS * 2) + (pn - 8) * 256 + st * 32 + (lane & 1) * 16; step = 65536; }
  q = uni(q);
  const u32 dst = (u32)(size_t)lds + XBUF + buf * 32768 + w * 4096;
#pragma unroll
  for (int k = 0; k < 4; ++k) asm volatile("s_mov_b32 m0, %2\n\ts_nop 0\n\tglobal_load_lds_dwordx4 %0, %1" :: "v"(off + k * step), "s"(q), "s"(dst + k * 1024) : "memory");
}
constexpr int JOB1 = NPC + UPR * 2 * NPC, JOB2 = JOB1 + MIDR * 2 * NPB;

struct Jobs {
  int nr, u, sg;
  __device__ __forceinline__ void init(int u_, int sg_) {
    u = u_; sg = sg_;
    nr = u < 0 ? 0 : (u >= NPB ? 1 + UPR : 1 + UPR + MIDR + (u >= NPA || JOB2 + 2 * u >= NJOB ? 0 : (NJOB - JOB2 - 2 * u + 2 * NPA - 1) / (2 * NPA)));
  }
  __device__ __forceinline__ int job(int r) const {
    if (r >= nr) return NJOB;
    if (r == 0) return sg ? NJOB : u;
    return 2 * u + sg + (r <= UPR ? NPC + (r - 1) * 2 * NPC : (r <= UPR + MIDR ? JOB1 + (r - 1 - UPR) * 2 * NPB : JOB2 + (r - 1 - UPR - MIDR) * 2 * NPA));
  }
};
__device__ __forceinline__ void encode_jobs(CParams& p, LAS char* lds, u32 call, int w, int lane, const Jobs& jb) {
  Enc e; e.init(p, lds, call, w, lane);
  for (int r = 0; r < jb.nr; ++r) {
    const int J = jb.job(r), J2 = jb.job(r + 2);
    auto mid = [&] { if (r != 0) load_job(p, lds, J2, r & 1, w, lane); };
    if (J < NJOB) {
      u32x4 X[4];
      const u32 pn = (J >> 1) & 15;
      LAS char* xb = lds + XBUF + (r & 1) * 32768 + w * 4096;
      if (pn < 8u) {
#pragma unroll
        for (int k = 0; k < 4; ++k) X[k] = *(const LAS u32x4*)(xb + k * 1024 + lane * 16);
      } else {
#pragma unroll
        for (int k = 0; k < 4; ++k) {
          LAS char* q = xb + k * 1024 + (lane >> 4) * 256 + (lane & 15) * 8;
          const s16x4 lo = __builtin_amdgcn_ds_read_tr16_b64_v4i16((LAS s16x4*)q), hi = __builtin_amdgcn_ds_read_tr16_b64_v4i16((LAS s16x4*)(q + 128));
          X[k] = __builtin_bit_cast(u32x4, __builtin_shufflevector(lo, hi, 0, 1, 2, 3, 4, 5, 6, 7));
        }
      }
      e.round(X, pn * 128 + (J >> 5) * 2 + (J & 1), input_mask(p.rank, pn, J >> 5), mid);
    } else e.idle(mid);
    if (r == 0) { e.finish(); load_job(p, lds, J2, 0, w, lane); STAMP(1); }
  }
  e.finish();
  STAMP(2);
}

__device__ __forceinline__ void lds_post(LAS u32* q, u32 v) { asm volatile("ds_write_b32 %0, %1" :: "v"((u32)(size_t)q), "v"(v) : "memory"); }
__device__ __forceinline__ u32 lds_peek(LAS u32* q) { u32 v; asm volatile("ds_read_b32 %0, %1\n\ts_waitcnt lgkmcnt(0)" : "=&v"(v) : "v"((u32)(size_t)q) : "memory"); return rfl(v); }
__device__ __forceinline__ void lds_inc(LAS u32* q) { asm volatile("s_mov_b64 exec, 1\n\tds_add_u32 %0, %1\n\ts_mov_b64 exec, -1" :: "v"((u32)(size_t)q), "v"(1u) : "memory"); }
template <int I> __device__ __forceinline__ void load10(const char* gs, u32 ls, const u32 (&v16)[3], LAS u32* landed, u32 k, u32 since2) {
  if constexpr (I < 10) {
    if ((I & 3) == 0) asm volatile("s_mov_b32 m0, %0" :: "s"(ls + I * 1024) : "memory");
    wait_v<INFL - 1>();
    asm volatile("global_load_lds_dwordx4 %0, %1 offset:%2 sc0 sc1" :: "v"(v16[I >> 2]), "s"(gs), "i"((I & 3) * 1024) : "memory");
    if (I + 11 <= INFL - 1 && since2 + I == INFL - 1) lds_post(landed, k - 1);
    if (I == INFL - 1) lds_post(landed, k);
    load10<I + 1>(gs, ls, v16, landed, k, since2);
  }
}
constexpr u32 FIN = 0x40000000u;
__host__ __device__ __forceinline__ u32 c_offset(int rank, u32 pc) {
  const u32 o = (pc >> 8) & 1u;
  return ((u32)(slot_row(rank ^ (o + 1), (pc >> 3) & 31u) + (pc & 1u) * 64) * NS + o * 512 + ((pc >> 2) & 1u) * 256 + ((pc >> 1) & 1u) * 128) * 2;
}

__device__ __forceinline__ void pull_role(CParams& p, LAS char* lds, int d, int j, int ncu, u32 call, int w, int lane) {
  LAS u32* landed = (LAS u32*)(lds + LDS_MAIN + 128);
  LAS u32* posted = landed + 1;
  LAS u32* started = landed + 2;
  LAS u32* done = (LAS u32*)(lds + LDS_MAIN + 192);
  LAS u32* qitem = (LAS u32*)(lds + LDS_MAIN + 256);
  LAS u32* qdst = qitem + 2 * QD;
  LAS u32* qlen = qitem + QD;
  LAS u32* list = (LAS u32*)(lds + NSLOT * SEG_STRIDE);
  static_assert(NSLOT * SEG_STRIDE + LMAX * 4 <= LDS_MAIN, "list");
  const int rank = rfl(p.rank), lt = list_type(d);
  const u32 n = rfl(LISTS.n[lt]);
  for (u32 i = threadIdx.x; i < n; i += 512) list[i] = resolve(d, rank, LISTS.e[lt][i]);
  LBAR;
  if (w == 1) {
    const u64* f1 = uni((const u64*)(p.ws[0] + F1_OFF + (call & 1) * F1_SZ) + d * NSRC);
    u32* tk = uni((u32*)(p.ws[0] + TKT_OFF + d * 128));
    const u32 tbase = call * 16384u, nin = rfl(LISTS.mark[lt][1]);
    __builtin_amdgcn_s_setprio(2);
    if (lane == 0) __hip_atomic_fetch_max(tk, tbase, __ATOMIC_RELAXED, __HIP_MEMORY_SCOPE_AGENT);
    WAIT_V(0);
#define FIDX(i) ((i) <= nin ? list[i] >> 16 : MAIL0 + ((i) < n ? (i) : n - 1) - nin - 1)
    u32 pos = j, left = 1, win = 0; u64 v = 0;
    for (u32 k = 0;; ++k) {
      if (left == 0) {
        u32 st;
        while ((st = lds_peek(started)) + LA <= k) __builtin_amdgcn_s_sleep(1);
        left = (n - pos) / (4u * ncu); left = pos >= n || left < 1u ? 1u : (left > GMAX ? GMAX : left);
        if (pos + 24u >= nin && left > GMAXC) left = GMAXC;
        if (left == 1u && GMAXC > 1u && st + LA - k >= 2u) left = 2u;
        if (k < 2) left = 1u;
        win = pos;
        const u32 wi = win + lane;
        u32 t;
        asm volatile("s_mov_b64 exec, 1\n\tglobal_atomic_add %0, %2, %3, %4 sc0\n\ts_mov_b64 exec, -1\n\tglobal_load_dwordx2 %1, %5, %6 sc0 sc1\n\ts_waitcnt vmcnt(0)"
                     : "=&v"(t), "=&v"(v) : "v"(0u), "v"(left), "s"(tk), "v"(FIDX(wi) * 8u), "s"(f1) : "memory");
        pos = rfl(t) - tbase + ncu;
      }
      u32 item = 0, len = 0;
      if (pos < n) {
        item = rfl(list[pos]);
        const u64 ok = __ballot((u32)(v >> 32) >= call);
        if (pos - win < 64u && ((ok >> (pos - win)) & 1)) len = __builtin_amdgcn_readlane((u32)v, pos - win);
        else {
          const u64 t0 = __builtin_amdgcn_s_memrealtime();
          win = pos;
          const u32 wi = win + (lane & 7);
          const u64* fp = f1 + FIDX(wi);
          for (u32 it = 1;; ++it) {
            v = __hip_atomic_load(fp, __ATOMIC_RELAXED, __HIP_MEMORY_SCOPE_SYSTEM);
            if (rfl((u32)(v >> 32)) >= call) { len = rfl((u32)v); break; }
            if ((it & 63u) == 0 && give_up(p, call, t0)) {
              for (u32 i = nin + 1 + lane; i < n; i += 64) if (list[i] & D_C) *(u32*)(p.out + c_offset(rank, list[i])) = 0x7fc07fc0u;
              pos = n; break;
            }
            __builtin_amdgcn_s_sleep(1);
          }
        }
        if (pos > nin) {
          const u32 sg_ = MAIL0 + ((len >> 16) - MAIL0) % 1024u;
          item = (sg_ << 16) | ((sg_ - MAIL0) & 511u) | (d > 3 && sg_ < 24 * 128 ? D_REL : D_C); len = (len & 0xffffu) << 4;
        }
        len = pos < n ? (len > (u32)SEG_STRIDE ? (u32)SEG_STRIDE : (len < (u32)SEG_V ? (u32)SEG_V : len & ~15u)) : 0u;
      }
      lds_post(qdst + (k % QD), item & D_C ? c_offset(rank, item) : ((item >> 7) & 31u) * (u32)PANEL_B + (item & 127u) * 16384u);
      lds_post(qitem + (k % QD), item); lds_post(qlen + (k % QD), len); lds_post(posted, k + 1);
      if (k == 0) STAMP(1);
      if (len == 0) return;
      ++pos; --left;
    }
  }
  if (w == 0) {
    const char* rsrc = uni(p.ws[d] + ENC_OFF + (call & 1) * ENC_SZ);
    __builtin_amdgcn_s_waitcnt(0x0f70);
    __builtin_amdgcn_s_setprio(3);
    const u32 v16[3] = {(u32)lane * 16, (u32)lane * 16 + 4096, (u32)lane * 16 + 8192};
    int slot = 0; u32 use = 0, k = 0, since2 = 100; int spinq = 0, spind = 0;
    static_assert(INFL <= 22, "window");
    for (;; ++k) {
      u32 item, len;
      for (u32 it = 0;; ++it) {
        u32 a, dn;
        asm volatile("ds_read_b32 %0, %4\n\tds_read_b32 %1, %5\n\tds_read_b32 %2, %6\n\tds_read_b32 %3, %6 offset:64\n\ts_waitcnt lgkmcnt(0)"
                     : "=&v"(a), "=&v"(dn), "=&v"(item), "=&v"(len)
                     : "v"((u32)(size_t)posted), "v"((u32)(size_t)(done + slot)), "v"((u32)(size_t)(qitem + (k % QD))) : "memory");
        if (rfl(a) > k && rfl(dn) >= use) break;
        if (it == DRAINIT) { WAIT_V(0); lds_post(landed, k); since2 = 100; }
        __builtin_amdgcn_s_sleep(1);
      }
      len = rfl(len);
      if (len == 0) break;
      lds_post(started, k + 1);
      const char* gs = rsrc + (size_t)(rfl(item) >> 16) * SEG_STRIDE;
      const u32 ls = (u32)(size_t)lds + slot * SEG_STRIDE;
      load10<0>(gs, ls, v16, landed, k, since2);
      for (u32 o = 10240; o < len; o += 1024) {
        wait_v<INFL - 1>();
        if (INFL > 21 && since2 + (o >> 10) == INFL - 1) lds_post(landed, k - 1);
        if (INFL > 10 && o == (INFL - 1) * 1024) lds_post(landed, k);
        if (o + lane * 16 < len) asm volatile("s_mov_b32 m0, %2\n\ts_nop 0\n\tglobal_load_lds_dwordx4 %0, %1 sc0 sc1" :: "v"(o + lane * 16), "s"(gs), "s"(ls + o) : "memory");
      }
      since2 = (len + 1023) >> 10;
      if (++slot == NSLOT) { slot = 0; use += 2; }
    }
    WAIT_V(0); lds_post(landed, k | FIN);
    STAMP(2); STAMPV(10, spinq); STAMPV(11, spind); STAMPV(12, k);
    return;
  }
  char* dec = uni(p.ws[0] + DEC_OFF);
  char* enc = uni(p.ws[0] + ENC_OFF + (call & 1) * ENC_SZ);
  char* out = uni(p.out);
  gu64* row = flag_row(p, call, lane);
  const u32 vC = (lane & 15) * (NS * 2) + ((lane & 16) << 1) + ((lane & 32) >> 1);
  u32 have = 0, pend = 0, fseg = 0, flen = 0, fmask = 0;
#define PUBLISH() do { if (pend) { \
    asm volatile("s_waitcnt vmcnt(0)\n\ts_mov_b64 exec, 3\n\tglobal_store_dword %0, %1, %2 sc0 sc1\n\ts_mov_b64 exec, -1" :: "v"(pend + lane * (NCH * 4)), "v"(call), "s"(dec) : "memory"); \
    if ((pend & 255u) == 252u) STAMP(4); \
    if (flen) post_seg(row, call, fseg, flen, fmask, lane); pend = 0; flen = 0; } } while (0)
#define PASS_ON(seg, mask) do { flen = lds_peek(qlen + (k % QD)); fseg = (seg); fmask = (mask); \
    for (u32 o = 0; o < flen; o += 6144) { u32x4 t_[6]; \
      _Pragma("unroll") for (int i = 0; i < 6; ++i) t_[i] = *(const LAS u32x4*)(ls + o + i * 1024 + lane * 16); \
      _Pragma("unroll") for (int i = 0; i < 6; ++i) if (o + i * 1024 + lane * 16 < flen) st4x1(enc, fseg * SEG_STRIDE + o + i * 1024 + lane * 16, t_[i]); } } while (0)
#define BOTH _Pragma("unroll") for (int z = 0; z < 2; ++z)
  for (;;) {
    u32 q;
    asm volatile("s_mov_b64 exec, 1\n\tds_add_rtn_u32 %0, %2, %3 offset:12\n\tds_read_b32 %1, %2\n\ts_waitcnt lgkmcnt(0)\n\ts_mov_b64 exec, -1"
                 : "=&v"(q), "=&v"(have) : "v"((u32)(size_t)landed), "v"(1u) : "memory");
    q = rfl(q); have = rfl(have);
    const u32 k = q >> 1, hf = q & 1u, slot = k % NSLOT;
    if ((have & ~FIN) <= k) {
      PUBLISH();
      while (have <= k) { __builtin_amdgcn_s_sleep(1); have = lds_peek(landed); }
      if ((have & ~FIN) <= k) break;
    }
    const LAS char* ls = lds + slot * SEG_STRIDE;
    const u32 item_v = qitem[k % QD], dst_v = qdst[k % QD];
    const u32x2 hdr_v = *(const LAS u32x2*)(ls + SEG_H + hf * 8);
    u32x2 Pp[2]; u32x4 S[2][2];
    BOTH Pp[z] = *(const LAS u32x2*)(ls + SEG_P + hf * 1024 + z * 512 + lane * 8);
    BOTH { S[z][0] = *(const LAS u32x4*)(ls + hf * 4096 + z * 2048 + lane * 16); S[z][1] = *(const LAS u32x4*)(ls + hf * 4096 + z * 2048 + 1024 + lane * 16); }
    const u32 item = rfl(item_v), dst = rfl(dst_v);
    if (item & D_REL) {
      PUBLISH();
      if (hf == 0) {
        PASS_ON(24 * 128 + (item & 511u), 1u << (d ^ (((item >> 8) & 1u) + 1)));
        lds_inc(done + slot);
        const u32 place = mail_place(dec + (MBX_OFF - DEC_OFF), fmask, lane);
        WAIT_V(0);
        post_mail(row, call, place, fseg, flen, fmask, lane); flen = 0;
      } else lds_inc(done + slot);
      continue;
    }
    u32 hdr[2], vb[2], E1[2], n1[2], N1[2], o1[2], q1[2], q0[2], Q1[2], Q0[2], E2[2], n2[2], N2[2], o2[2], r2[2], r1[2], r0[2], E[2][8]; Mv ma[2], mb[2];
    BOTH { hdr[z] = rfl(hdr_v[z]); vb[z] = ((u32)(size_t)ls + SEG_V + ((hdr[z] >> 16) & 2047u) * 4) * 8; }
    BOTH { E1[z] = ~(Pp[z][0] | Pp[z][1]); ma[z] = move_masks(E1[z]); n1[z] = __builtin_popcount(E1[z]); }
    { const u32 i = wave_scan(n1[0] | (n1[1] << 16)), t = __builtin_amdgcn_readlane(i, 63);
      N1[0] = t & 0xffffu; N1[1] = t >> 16; o1[0] = vb[0] + 2 * ((i & 0xffffu) - n1[0]); o1[1] = vb[1] + 2 * ((i >> 16) - n1[1]); }
    BOTH { q1[z] = get_bits(o1[z]); q0[z] = q1[z] >> (n1[z] & 31u); }
    if (__any((n1[0] > n1[1] ? n1[0] : n1[1]) > 16u)) BOTH q0[z] = get_bits(o1[z] + n1[z]);
    BOTH { Q1[z] = pdep(q1[z], ma[z]); Q0[z] = pdep(q0[z], ma[z]); E2[z] = BOP(E1[z], Q1[z], Q0[z], A & ~(B | C)); mb[z] = move_masks(E2[z]); n2[z] = __builtin_popcount(E2[z]); }
    { const u32 i = wave_scan(n2[0] | (n2[1] << 16)), t = __builtin_amdgcn_readlane(i, 63);
      N2[0] = t & 0xffffu; N2[1] = t >> 16; o2[0] = mad3((i & 0xffffu) - n2[0], vb[0] + 2 * N1[0]); o2[1] = mad3((i >> 16) - n2[1], vb[1] + 2 * N1[1]); }
    u32 xe[2];
    BOTH { r2[z] = get_bits(o2[z]); r1[z] = r2[z] >> (n2[z] & 31u); r0[z] = r1[z] >> (n2[z] & 31u);
           xe[z] = *(const LAS u32*)(size_t)((vb[z] >> 3) + (((2 * N1[z] + 3 * N2[z] + 31) >> 5) + lane) * 4); }
    if (__any((n2[0] > n2[1] ? n2[0] : n2[1]) > 10u)) BOTH { r1[z] = get_bits(o2[z] + n2[z]); r0[z] = get_bits(o2[z] + 2 * n2[z]); }
    BOTH {
      const u32 R2 = pdep(r2[z], mb[z]) & E2[z], R1 = pdep(r1[z], mb[z]), R0 = pdep(r0[z], mb[z]);
      const u32 t7 = BOP(R2, R1, R0, A & B & C), q7 = BOP(E1[z], Q1[z], Q0[z], A & ~B & C);
      u32 dd[4] = {BOP(Pp[z][1] | q7, R0, E2[z], A | (B & C)), BOP(Pp[z][0], q7, BOP(R1, E2[z], t7, A & B & ~C), A | B | C), BOP(Q0[z], E1[z], R2, (A & B) | C), BOP(E1[z], Q1[z], t7, ~A | B | C)};
      transpose4(dd);
      const u32 b4 = (hdr[z] & 255u) * 0x01010101u;
#pragma unroll
      for (int i = 0; i < 4; ++i) { E[z][2 * i] = (dd[i] & 0x0f0f0f0fu) + b4; E[z][2 * i + 1] = ((dd[i] >> 4) & 0x0f0f0f0fu) + b4; }
    }
    asm volatile("" : "+v"(xe[0]), "+v"(xe[1]));
    BOTH {
      const u32 NX = (hdr[z] >> 8) & 255u;
      if (NX <= XCAP) {
        for (u32 i = 0; i < NX; ++i) {
          const u32 en = __builtin_amdgcn_readlane(xe[z], i);
          const u32 dv = (u32)lane == (en & 63u) ? ((en >> 16) & 255u) << (8 * ((en >> 9) & 3u)) : 0u;
          const int hot = (int)(0x80000000u >> ((en >> 6) & 7u));
#pragma unroll
          for (int i2 = 0; i2 < 8; ++i2) E[z][i2] = BOP(E[z][i2], dv, (u32)((hot << i2) >> 31), A ^ (B & C));
        }
      } else {
        E[z][0] = Pp[z][0]; E[z][1] = Pp[z][1];
#pragma unroll
        for (int i2 = 2; i2 < 8; ++i2) E[z][i2] = *(const LAS u32*)(size_t)((vb[z] >> 3) + ((i2 - 2) * 64 + lane) * 4);
      }
    }
    u32x4 O[2][4];
    BOTH {
#pragma unroll
      for (int i = 0; i < 8; ++i) {
        const u32 sb = S[z][i >> 2][i & 3];
        const u32 a = __builtin_amdgcn_perm(E[z][i], sb, 0x05010400u), b = __builtin_amdgcn_perm(E[z][i], sb, 0x07030602u);
        O[z][i >> 1][(i & 1) * 2] = __builtin_amdgcn_alignbit(a, a, 1); O[z][i >> 1][(i & 1) * 2 + 1] = __builtin_amdgcn_alignbit(b, b, 1);
      }
      asm volatile("" : "+v"(O[z][0]), "+v"(O[z][1]), "+v"(O[z][2]), "+v"(O[z][3]));
    }
    PUBLISH();
    if (item & D_C) {
      char* o = out + (dst + hf * 128 + vC);
#pragma unroll
      for (int m = 0; m < 4; ++m) BOTH *(u32x4*)(o + m * 16 * NS * 2 + z * 64) = O[z][m];
    } else {
      BOTH st4x4(dec, dst + (hf * 2 + z) * 4096u + lane * 16, O[z][0], O[z][1], O[z][2], O[z][3]);
      pend = (u32)(F2_OFF - DEC_OFF) + ((((item >> 7) & 31u) * 8 + (item & 1u) * NB + hf * 2) * NCH + ((item >> 1) & 63u)) * 4;
      if ((item & D_FWD) && hf == 0) PASS_ON((12 + d) * 128 + (item & 127u), 1u << (d ^ 3));
    }
    lds_inc(done + slot);
    if (q < 2) STAMP(1);
  }
  STAMP(2);
}

__device__ __forceinline__ int wait_chunks(CParams& p, const u32* f, u32 o0, u32 o1, u32 o2, int need, u32 call) {
  const u64 t0 = __builtin_amdgcn_s_memrealtime();
  __builtin_amdgcn_s_waitcnt(0xc07f);
  for (u32 it = 1;; ++it) {
    const u32 a = __hip_atomic_load(f + o0, __ATOMIC_RELAXED, __HIP_MEMORY_SCOPE_SYSTEM);
    const u32 b = __hip_atomic_load(f + o1, __ATOMIC_RELAXED, __HIP_MEMORY_SCOPE_SYSTEM);
    const u32 c = __hip_atomic_load(f + o2, __ATOMIC_RELAXED, __HIP_MEMORY_SCOPE_SYSTEM);
    const u64 miss = __ballot(a < call || b < call || c < call);
    const int n = miss ? __builtin_ctzll(miss) : NCH;
    if (n >= need) return n;
    if ((it & 63u) == 0 && give_up(p, call, t0)) return NCH;
    __builtin_amdgcn_s_sleep(2);
  }
}
#define FL_ADDR(next) do { fbase = (next); const u32 c_ = fbase + (lane & 15); fvo = foff + 4 * (c_ < 63u ? c_ : 63u); } while (0)
#define FL_ISSUE() asm volatile("s_mov_b32 m0, %2\n\ts_nop 0\n\tglobal_load_lds_dword %0, %1 sc0 sc1" :: "v"(fvo), "s"(f2), "s"(lds0 + LDS_FLG + w * 256) : "memory")
#define FL_READ() fval = *(const volatile LAS u32*)(lds + LDS_FLG + w * 256 + lane * 4)
#define FL_USE() do { u64 m_ = __ballot(fval < call); m_ |= m_ >> 32; const u32 h_ = (u32)m_ | ((u32)m_ >> 16) | 0x10000u; \
    const int n_ = fbase + __builtin_ctz(h_); avail = n_ > avail ? n_ : avail; } while (0)
__device__ __forceinline__ u32 cvt_pk(float lo, float hi) { u32 r; asm("v_cvt_pk_bf16_f32 %0, %1, %2" : "=v"(r) : "v"(lo), "v"(hi)); return r; }
#define QNAN __builtin_nanf("")
typedef float f32x2 __attribute__((ext_vector_type(2)));
__device__ __forceinline__ u32 silu2(float a, float b) {
  const f32x2 t = (f32x2){__builtin_amdgcn_fmed3f(a, -60.0f, 60.0f), __builtin_amdgcn_fmed3f(b, -60.0f, 60.0f)} * -1.44269504088896341f;
  const f32x2 e = (f32x2){__builtin_amdgcn_exp2f(t.x), __builtin_amdgcn_exp2f(t.y)} + 1.0f;
  const f32x2 r = (f32x2){a, b} * (f32x2){__builtin_amdgcn_rcpf(e.x), __builtin_amdgcn_rcpf(e.y)};
  return cvt_pk(r.x, r.y);
}
__device__ __forceinline__ u32x4 finish8(const f32x4& u, const f32x4& v) {
  const auto s0 = __builtin_amdgcn_permlane16_swap(silu2(u[0], u[1]), silu2(v[0], v[1]), false, false);
  const auto s1 = __builtin_amdgcn_permlane16_swap(silu2(u[2], u[3]), silu2(v[2], v[3]), false, false);
  return (u32x4){s0[0], s1[0], s0[1], s1[1]};
}
__device__ __forceinline__ bool timed_out(CParams& p, u32 call) {
  return rfl(__hip_atomic_load((u32*)(p.ws[0] + TMO_OFF), __ATOMIC_RELAXED, __HIP_MEMORY_SCOPE_AGENT)) == call;
}
constexpr int HT = 16384;
#define A_SETUP(own) \
  const u32 sCa = (own) ? 256u : 32768u, sKa = (own) ? 128u : 2048u; \
  const u32 voffA0 = (own) ? (u32)(w * 16 + (lane >> 3)) * (KD * 2) + (((lane & 7) ^ (lane >> 4)) << 4) : (u32)(w * 4096 + lane * 16); \
  const u32 voffA1 = (own) ? (voffA0 ^ 64u) + 8 * KD * 2 - 1024 : voffA0;        \
  const u32 lds0 = (u32)(size_t)lds, ldsA = lds0 + w * 2048, ldsB = lds0 + w * 1024; \
  u32 aoff[2]; \
  aoff[0] = lds0 + wr * 8192 + ((own) ? (lane & 15) * 128 + (((lane >> 4) ^ ((lane >> 1) & 7)) << 4) : lane * 16); \
  aoff[1] = (own) ? aoff[0] ^ 64u : aoff[0] + 1024;
#define B_READ_OFF(n, t) (256 * (8 * (lane >> 4) + 4 * (t) + ((lane & 15) >> 2)) + 8 * (lane & 1) + \
    16 * ((4 * wc + 2 * (n) + ((lane & 3) >> 1)) ^ ((((lane & 15) >> 2) << 2) | ((2 * (lane >> 4) + (t)) & 3))))
#define VOFF_W ((u32)(4 * w + (lane >> 4)) * 2048u + (((lane & 15) ^ (((lane >> 4) << 2) | (w & 3))) << 4))
#define GB " sc0 sc1"
#define STA2(buf, gp, v0, v1) asm volatile("s_mov_b32 m0, %3\n\ts_nop 0\n\tglobal_load_lds_dwordx4 %0, %2" GB "\n\tglobal_load_lds_dwordx4 %1, %2 offset:1024" GB \
    :: "v"(v0), "v"(v1), "s"(gp), "s"(ldsA + (buf)) : "memory")
#define STA(buf, gp) STA2(buf, gp, voffA0, voffA1)
#define STB(buf, gp) asm volatile("s_mov_b32 m0, %3\n\ts_nop 0\n\tglobal_load_lds_dwordx4 %0, %1" GB "\n\ts_add_u32 m0, m0, 0x2000\n\ts_nop 0\n\tglobal_load_lds_dwordx4 %0, %2" GB \
    :: "v"(voffB), "s"(gp), "s"((gp) + sIb), "s"(ldsB + (buf)) : "memory", "scc")
#define LDA(dst, ad, imm) do { _Pragma("unroll") for (int m = 0; m < MF; ++m) _Pragma("unroll") for (int k = 0; k < 2; ++k) \
    dst[m][k] = *(const LAS s16x8*)(size_t)(ad[k] + (imm) + m * 2048); } while (0)
#define LDB(dst, ad, imm) do { _Pragma("unroll") for (int n = 0; n < 2; ++n) _Pragma("unroll") for (int k = 0; k < 2; ++k) { \
    const s16x4 lo_ = __builtin_amdgcn_ds_read_tr16_b64_v4i16((LAS s16x4*)(size_t)(ad[n][0] + (imm) + k * 8192)); \
    const s16x4 hi_ = __builtin_amdgcn_ds_read_tr16_b64_v4i16((LAS s16x4*)(size_t)(ad[n][1] + (imm) + k * 8192)); \
    dst[n][k] = __builtin_shufflevector(lo_, hi_, 0, 1, 2, 3, 4, 5, 6, 7); } } while (0)
#define MMAX(ac, At, Bt, EXTRA) do { __builtin_amdgcn_s_setprio(1); EXTRA; _Pragma("unroll") for (int m = 0; m < MF; ++m) _Pragma("unroll") for (int n = 0; n < 2; ++n) \
    _Pragma("unroll") for (int k = 0; k < 2; ++k) ac[m][n] = __builtin_amdgcn_mfma_f32_16x16x32_bf16(__builtin_bit_cast(bf8, Bt[n][k]), __builtin_bit_cast(bf8, At[m][k]), ac[m][n], 0, 0, 0); \
    __builtin_amdgcn_s_setprio(0); } while (0)
#define MMA(ac, At, Bt) MMAX(ac, At, Bt, (void)0)
#define WEAVE do { _Pragma("unroll") for (int i_ = 0; i_ < 4 * MF - 1; ++i_) { __builtin_amdgcn_sched_group_barrier(0x8, 1, 0); __builtin_amdgcn_sched_group_barrier(0x6, 1, 0); } } while (0)

template <int MF, bool ALLOWN> __device__ __forceinline__ void gemm_wide(CParams& p, LAS char* lds, int F0, int tn, u32 call, int w, int lane) {
  const int wr = w >> 2, wc = w & 3;
  const bool own = ALLOWN || F0 < 64;
  const char* Ab = uni(own ? p.x : p.ws[0] + DEC_OFF);
  const char* Bb = uni(p.w + tn * 512);
  A_SETUP(own)
  const int f0 = F0 + (w < 2 * MF ? w : 0);
  u32 vh[2][2];
#pragma unroll
  for (int h = 0; h < 2; ++h) {
    const int f = f0 + 2 * MF * h;
    const u32 fo = own ? (u32)(f - w) * (16 * KD * 2) : (f >> 3) * (u32)PANEL_B + (u32)((f & 7) - w) * 4096u;
    vh[h][0] = w < 2 * MF ? fo + voffA0 : rfl(fo + voffA0); vh[h][1] = w < 2 * MF ? fo + voffA1 : vh[h][0];
  }
  aoff[0] -= wr * (4 - MF) * 2048; aoff[1] -= wr * (4 - MF) * 2048;
#define STAH(buf, gp, h) STA2(buf, gp, vh[h][0], vh[h][1])
  const u32 voffB = VOFF_W; constexpr int sIb = 65536;
  const u32* flg = (const u32*)(p.ws[0] + F2_OFF) + f0 * NCH + lane;
  const char* f2 = uni(p.ws[0] + F2_OFF);
  const u32 foff = (f0 + 2 * MF * (lane >> 5)) * NCH * 4;
  int fbase = 0, npoll = 0; u32 fval = 0, fvo = 0;
  u32 boff[2][2];
#pragma unroll
  for (int n = 0; n < 2; ++n)
#pragma unroll
    for (int t = 0; t < 2; ++t) { boff[n][t] = lds0 + 4 * HT + B_READ_OFF(n, t); asm volatile("" : "+v"(boff[n][t])); }
#define SA(b, h) (((b) * 2 + (h)) * HT)
#define SB(b, h) ((4 + (b) * 2 + (h)) * HT)
#define WAIT_LA asm volatile("s_waitcnt lgkmcnt(%0)" :: "i"(2 * MF) : "memory")
  f32x4 acc[2][2][MF][2];
#pragma unroll
  for (int a = 0; a < 2; ++a)
#pragma unroll
    for (int b = 0; b < 2; ++b)
#pragma unroll
      for (int m = 0; m < MF; ++m)
#pragma unroll
        for (int n = 0; n < 2; ++n) acc[a][b][m][n] = (f32x4){0.f, 0.f, 0.f, 0.f};
  s16x8 At[MF][2], B0[2][2], B1[2][2];
  int avail = own ? NCH : wait_chunks(p, flg, 0, 2 * MF * NCH, 0, 1, call);
  STAMP(3); STAMPC(8);
  __builtin_amdgcn_s_waitcnt(0x0f70);
  if (!ALLOWN) { FL_ADDR(1); FL_ISSUE(); }
  STB(SB(0, 0), Bb); STAH(SA(0, 0), Ab, 0); STB(SB(0, 1), Bb + 256); STAH(SA(0, 1), Ab, 1);
  if (wr == 1) BAR;
  WAIT_V(4); BAR;
  STB(SB(1, 0), Bb + 131072); STAH(SA(1, 0), Ab + sKa, 0); STB(SB(1, 1), Bb + 131072 + 256);
  WAIT_V(6); BAR;
#pragma unroll 1
  for (int c = 0; c < NCH; ++c) {
    const int cn = c + 1 < NCH ? c + 1 : c;
    const char* a1 = Ab + (size_t)(c * sCa + sKa);
    const char* a2 = Ab + (size_t)(cn * sCa);
    const char* b2 = Bb + (size_t)cn * 262144;
    LDB(B0, boff, SA(0, 0)); SCHED; LDA(At, aoff, SA(0, 0)); STAH(SA(1, 1), a1, 1);
    WAIT_LA; BAR; WAIT_L(0); MMA(acc[0][0], At, B0); BAR; SCHED;
    LDB(B1, boff, SA(0, 1)); if (!ALLOWN) FL_READ(); STB(SB(0, 0), b2);
    if (ALLOWN) { BAR; WAIT_L(0); MMA(acc[0][1], At, B1); BAR; }
    else {
      BAR; WAIT_L(0); SCHED; MMAX(acc[0][1], At, B1, FL_USE(); FL_ADDR(cn + 1); WEAVE); asm volatile("" : "+s"(avail), "+v"(fvo)); BAR;
      if (avail <= cn) { avail = wait_chunks(p, flg, 0, 2 * MF * NCH, 0, cn + 1, call); ++npoll; }
    }
    LDA(At, aoff, SA(0, 1)); STAH(SA(0, 0), a2, 0);
    BAR; WAIT_L(0); MMA(acc[1][0], At, B0); BAR; SCHED;
    STB(SB(0, 1), b2 + 256);
    if (ALLOWN) WAIT_V(6); else { FL_ISSUE(); WAIT_V(7); }
    BAR; MMA(acc[1][1], At, B1); BAR;
    LDB(B0, boff, SA(1, 0)); SCHED; LDA(At, aoff, SA(1, 0)); STAH(SA(0, 1), a2, 1);
    WAIT_LA; BAR; WAIT_L(0); MMA(acc[0][0], At, B0); BAR; SCHED;
    LDB(B1, boff, SA(1, 1)); STB(SB(1, 0), b2 + 131072);
    BAR; WAIT_L(0); MMA(acc[0][1], At, B1); BAR;
    LDA(At, aoff, SA(1, 1)); STAH(SA(1, 0), a2 + sKa, 0);
    BAR; WAIT_L(0); MMA(acc[1][0], At, B0); BAR; SCHED;
    STB(SB(1, 1), b2 + 131072 + 256);
    WAIT_V(6); BAR; MMA(acc[1][1], At, B1); BAR;
  }
  if (wr == 0) BAR;
  STAMP(4); STAMPC(9); STAMPV(10, npoll);
  if (!own && timed_out(p, call)) {
#pragma unroll
    for (int i = 0; i < 8 * MF; ++i) acc[i & 1][(i >> 1) & 1][i >> 3][(i >> 2) & 1] = (f32x4){QNAN, QNAN, QNAN, QNAN};
  }
  int tz = threadIdx.x; asm volatile("" : "+v"(tz));
  char* ob = p.out + ((size_t)(tz & 15) * NS + tn * 256 + wc * 32 + (tz & 16) + ((tz & 32) >> 2)) * 2;
#pragma unroll
  for (int ai = 0; ai < 2; ++ai)
#pragma unroll
    for (int m = 0; m < MF; ++m) {
      const int f = F0 + 2 * MF * ai + MF * wr + m;
      char* oa = ob + (size_t)(slot_row(p.rank, f >> 3) + (f & 7) * 16) * (NS * 2);
#pragma unroll
      for (int bj = 0; bj < 2; ++bj) *(u32x4*)(oa + bj * 256) = finish8(acc[ai][bj][m][0], acc[ai][bj][m][1]);
    }
  STAMP(5);
}

__device__ __forceinline__ void gemm_small(CParams& p, LAS char* lds, int s, int cp, u32 call, int w, int lane) {
  const int wr = w >> 2, wc = w & 3;
  constexpr int MF = 4;
  const bool own = s < 8;
  const char* Ab = uni(own ? p.x + (size_t)s * PANEL_B : p.ws[0] + DEC_OFF + (size_t)s * PANEL_B);
  const char* Bb = uni(p.ws[0] + DEC_OFF + (size_t)cp * 2 * PANEL_B);
  A_SETUP(own)
  const u32 voffB = w * 4096 + lane * 16;
  const u32* flg = (const u32*)(p.ws[0] + F2_OFF) + w * NCH + lane;
  const u32 fb0 = cp * 2 * 8 * NCH, fb1 = fb0 + 8 * NCH, fa = own ? fb0 : s * 8 * NCH;
  const char* f2 = uni(p.ws[0] + F2_OFF);
  const u32 foff = ((lane & 32 ? fa : (lane & 16 ? fb1 : fb0)) + w * NCH) * 4;
  int fbase = 0, npoll = 0; u32 fval = 0, fvo = 0;
  constexpr u32 TB = 3 * HT;
  const u32 boff = lds0 + wc * 4096 + lane * 16;
  u32 bc = boff, bn = boff + TB, ac[2] = {aoff[0], aoff[1]};
  f32x4 acc[2][4][2];
#pragma unroll
  for (int b = 0; b < 2; ++b)
#pragma unroll
    for (int m = 0; m < 4; ++m)
#pragma unroll
      for (int n = 0; n < 2; ++n) acc[b][m][n] = (f32x4){0.f, 0.f, 0.f, 0.f};
  s16x8 At[4][2], B0[2][2], B1[2][2];
  Enc e; e.init(p, lds, call, w, lane);
  asm volatile("" : "+v"(e.row));
  int avail = wait_chunks(p, flg, fa, fb0, fb1, 2, call);
  STAMP(3); STAMPC(8);
  __builtin_amdgcn_s_waitcnt(0x0f70);
#define LD2(m0v, v0, v1, gp) asm volatile("s_mov_b32 m0, %3\n\ts_nop 0\n\tglobal_load_lds_dwordx4 %0, %2" GB "\n\tglobal_load_lds_dwordx4 %1, %2 offset:1024" GB \
    :: "v"(v0), "v"(v1), "s"(gp), "s"(m0v) : "memory")
#define LD1(m0v, v0, gp, o) asm volatile("s_mov_b32 m0, %2\n\ts_nop 0\n\tglobal_load_lds_dwordx4 %0, %1 offset:" #o GB :: "v"(v0), "s"(gp), "s"(m0v) : "memory")
#define LDF(dst, ad, imm) do { _Pragma("unroll") for (int n = 0; n < 2; ++n) _Pragma("unroll") for (int k = 0; k < 2; ++k) \
    dst[n][k] = *(const LAS s16x8*)(size_t)((ad) + (imm) + n * 2048 + k * 1024); } while (0)
  FL_ADDR(2);
  LD2(ldsA + HT, voffB, voffB, Bb); LD2(ldsA, voffA0, voffA1, Ab); LD2(ldsA + 2 * HT, voffB, voffB, Bb + PANEL_B); LD2(ldsA + TB + HT, voffB, voffB, Bb + 2048);
  LD2(ldsA + TB, voffA0, voffA1, Ab + sKa); FL_ISSUE(); LD2(ldsA + TB + 2 * HT, voffB, voffB, Bb + 2048 + PANEL_B); LD2(ldsA + 2 * TB + HT, voffB, voffB, Bb + 32768);
  WAIT_V(11); BAR;
  LDF(B0, bc, HT);
  WAIT_L(0);
  if (wr == 1) BAR;
  u32 s0 = 0, s1 = TB, s2 = 2 * TB;
  const char* gA = Ab + sCa; const char* gB1 = Bb + 32768 + PANEL_B; const char* gB0 = Bb + 32768 + 2048;
  u32 mA = ldsA + s2, mB1 = mA + 2 * HT, mB0 = ldsA + HT;
#define PIN do { asm volatile("" : "+s"(gA), "+s"(gB1), "+s"(gB0), "+s"(mA), "+s"(mB1), "+s"(mB0), "+s"(avail)); \
    asm volatile("" : "+v"(fvo), "+v"(bc), "+v"(bn), "+v"(ac[0]), "+v"(ac[1])); } while (0)
#define KTILE(dA, dB1, dB0, V1, V2, ISS, CHECK, RD, X2) do { \
    LD2(mA, voffA0, voffA1, gA); LD1(mB1, voffB, gB1, 0); ISS; LDA(At, ac, 0); \
    WAIT_V(V1); BAR; WAIT_L(0); SCHED; \
    MMAX(acc[0], At, B0, gA += (dA); mA = ldsA + s0; ac[0] = aoff[0] + s1; ac[1] = aoff[1] + s1; WEAVE); PIN; BAR; SCHED; \
    CHECK; \
    LD1(mB1, voffB, gB1, 1024); LD2(mB0, voffB, voffB, gB0); LDF(B1, bc, 2 * HT); SCHED; LDF(B0, bn, HT); \
    WAIT_V(V2); WAIT_L(4); RD; BAR; WAIT_L(0); SCHED; \
    MMAX(acc[1], At, B1, gB1 += (dB1); gB0 += (dB0); mB1 = mA + 2 * HT; mB0 = ldsA + s1 + HT; bc = bn; bn = boff + s2; \
         { const u32 s_ = s0; s0 = s1; s1 = s2; s2 = s_; } X2; WEAVE2); PIN; BAR; SCHED; } while (0)
#define WEAVE2 do { _Pragma("unroll") for (int i_ = 0; i_ < 15; ++i_) { __builtin_amdgcn_sched_group_barrier(0x8, 1, 0); __builtin_amdgcn_sched_group_barrier(0x6, 2, 0); } } while (0)
  for (int c = 0; c < NCH; ++c) {
    const int c2 = c + 2 < NCH ? c + 2 : NCH - 1;
    const int nx = c + 2 < NCH;
    KTILE((int)sKa, 2048, (nx ? 32768 : 0) - 2048, 10, 9, (void)0, (void)0, FL_READ(), FL_USE(); FL_ADDR(c2 + 1));
    KTILE((int)(nx ? sCa : 0u) - (int)sKa, (nx ? 32768 : 0) - 2048, 2048, 10, 10, FL_ISSUE(),
          if (avail <= c2) { avail = wait_chunks(p, flg, fa, fb0, fb1, c2 + 1, call); ++npoll; LASTPOLL; }, (void)0, (void)0);
  }
  if (wr == 0) BAR;
  STAMP(4); STAMPC(9); STAMPV(10, npoll);
  const u32 tmo = __hip_atomic_load((u32*)(p.ws[0] + TMO_OFF), __ATOMIC_RELAXED, __HIP_MEMORY_SCOPE_AGENT);
  int tz = threadIdx.x; asm volatile("" : "+v"(tz));
  u32x4 X[4];
#pragma unroll
  for (int m = 0; m < 4; ++m) { X[m] = finish8(acc[0][m][0], acc[0][m][1]); asm volatile("" : "+v"(X[m])); }
  if (rfl(tmo) == call) {
#pragma unroll
    for (int m = 0; m < 4; ++m) { X[m] = (u32x4){0x7fc07fc0u, 0x7fc07fc0u, 0x7fc07fc0u, 0x7fc07fc0u}; acc[1][m][0] = acc[1][m][1] = (f32x4){QNAN, QNAN, QNAN, QNAN}; }
  }
  {
    STAMP(11);
    DRAIN; LBAR;
    e.lane = tz & 63;
    const u32 code = (s * 2 + (cp & 1)) * 4 + wr, seg = 20 * 128 + (cp >> 1) * 256 + code;
#pragma unroll 1
    for (int bj = 0; bj < 2; ++bj) {
      e.round(X, seg + 2 * bj, piece_mask(cp >> 1, code + 2 * bj), [] {});
      e.draw(f2 + (MBX_OFF - F2_OFF));
      if (bj == 0) {
#pragma unroll
        for (int m = 0; m < 2; ++m) { asm volatile("" : "+v"(acc[1][m][0]), "+v"(acc[1][m][1])); X[m] = finish8(acc[1][m][0], acc[1][m][1]); asm volatile("" : "+v"(X[m])); }
      }
      e.finish_mail();
      if (bj == 0) {
        STAMP(12);
#pragma unroll
        for (int m = 2; m < 4; ++m) { asm volatile("" : "+v"(acc[1][m][0]), "+v"(acc[1][m][1])); X[m] = finish8(acc[1][m][0], acc[1][m][1]); }
      }
    }
  }
  STAMP(5);
}

__global__ __launch_bounds__(512, 2) void ag_gemm(Params) {
  CParams& p = *(CParams*)__builtin_amdgcn_kernarg_segment_ptr();
  extern __shared__ __attribute__((aligned(16))) char smem[];
  LAS char* lds = (LAS char*)smem + 1024;
  const int lane = threadIdx.x & 63, w = rfl(threadIdx.x >> 6), sg = w / NB;
  const int b = blockIdx.x;
  STAMP(0);
  const Role ro = role_of(b);
  Jobs jb; jb.init(ro.u, sg);
  const int nr = jb.nr, j0 = jb.job(0), j1 = jb.job(1);
  load_job(p, lds, j0, 0, w, lane);
  load_job(p, lds, j1, 1, w, lane);
  u32* cnt = (u32*)(p.ws[0] + CNT_OFF + b * 128);
  u32 call = *(const __attribute__((address_space(4))) u32*)cnt + 1;
  asm volatile("" : "+s"(call));
  if (threadIdx.x < 32) ((LAS u32*)(lds + LDS_MAIN + 128))[lane] = 0;
  if (threadIdx.x >= 256) ((LAS u32*)smem)[threadIdx.x - 256] = mask_entry(threadIdx.x - 256);
  LBAR;
  if (threadIdx.x == 511) *cnt = call;
  if (nr) {
    if (j1 < NJOB) WAIT_V(4); else WAIT_V(0);
    encode_jobs(p, lds, call, w, lane, jb);
  }
  if (ro.kind == 0) gemm_small(p, lds, ro.a, ro.c, call, w, lane);
  else if (ro.kind == 1) gemm_wide<3, false>(p, lds, ro.a, ro.c, call, w, lane);
  else if (ro.kind == 2) gemm_wide<2, true>(p, lds, ro.a, ro.c, call, w, lane);
  else if (ro.kind == 3) {
    pull_role(p, lds, link_of(ro.a), ro.a / 7, (NPULL - ro.a % 7 + 6) / 7, call, w, lane);
  }
}
}

extern "C" void kernel_launch(int rank, int nranks, ncclComm_t comm, hipStream_t stream,
                              void* const* d_in, const int* in_sizes, int n_in,
                              void* d_out, int out_size, const int* in_dtypes, int out_dtype,
                              void* const* d_out_all, void* const* d_ws_all, size_t ws_bytes) {
  using namespace agg;
  if (nranks != NR || (unsigned)rank >= (unsigned)NR || ws_bytes < WS_NEED || n_in < 2) { fprintf(stderr, "ag_gemm: unexpected configuration\n"); return; }
  Params p;
  for (int i = 0; i < NR; ++i) p.ws[i] = (char*)d_ws_all[rank ^ i];
  p.x = (const char*)d_in[0]; p.w = (const char*)d_in[1]; p.out = (char*)d_out; p.rank = rank; p.pad = 0;
  hipError_t e = hipFuncSetAttribute((const void*)ag_gemm, hipFuncAttributeMaxDynamicSharedMemorySize, LDS_BYTES);
  if (e != hipSuccess) { fprintf(stderr, "HIP error %s\n", hipGetErrorString(e)); return; }
  hipLaunchKernelGGL(ag_gemm, dim3(GRID), dim3(512), LDS_BYTES, stream, p);
  e = hipGetLastError();
  if (e != hipSuccess) fprintf(stderr, "HIP error %s\n", hipGetErrorString(e));
}
```

```cpp
#include <hip/hip_runtime.h>
#include <rccl/rccl.h>
#include <cstdio>

#define INFL 11
#define OF1 5
#define OF2 10
#define OF3 10
#define OF4 9
#define FDIR 0xffff000000000000ull
#define P3 19
#define O3 1024
#define LEAD 6
#define REL0 0x55u
#define REL1 0x55u
#define REL2 0x55u
#define REL3 0x55u
#define RG 20u
#define WARMUP 0
#define INFL5 13
#define DEEP 0
#define MIDR 2
#define UPR 1
#define LA 3u
#define DRAINIT 2
#define LAM 3u
#define REPOLL 0
#define NEAR 80u
#define NEAR0 32u
#define BEHIND 2u
#define GMAX 4u

namespace agg {
#define LAS __attribute__((address_space(3)))
typedef unsigned u32;
typedef unsigned long long u64;
typedef __attribute__((address_space(1))) unsigned long long gu64;
typedef unsigned u32x4 __attribute__((ext_vector_type(4)));
typedef unsigned u32x2 __attribute__((ext_vector_type(2)));
typedef short s16x4 __attribute__((ext_vector_type(4)));
typedef short s16x8 __attribute__((ext_vector_type(8)));
typedef unsigned short u16x2 __attribute__((ext_vector_type(2)));
typedef float f32x4 __attribute__((ext_vector_type(4)));
typedef __bf16 bf8 __attribute__((ext_vector_type(8)));

constexpr int NR = 8, KD = 8192, NS = 1024;
constexpr int NCH = 64;
constexpr int NB = 4;
constexpr int SEG_STRIDE = NB * 4096 + 1024;
constexpr int SEG_P = NB * 2048, SEG_H = NB * 2560, SEG_V = SEG_H + NB * 4;
#define NSLOT 8
constexpr int QD = 16;
constexpr int GRID = 256;
constexpr int LDS_MAIN = 147456, LDS_FLG = LDS_MAIN + 1024, LDS_VAR = LDS_FLG + 2048, LDS_BYTES = 1024 + LDS_VAR + 8 * 1536;
static_assert(NSLOT * SEG_STRIDE <= LDS_MAIN, "ring");
constexpr u32 XCAP = 64;
constexpr int NSRC = 28 * 128, NJOB = 16 * 128;

constexpr size_t PANEL_B = (size_t)2 << 20;
constexpr size_t DEC_OFF = 0;
constexpr size_t ENC_OFF = 32 * PANEL_B;
constexpr size_t ENC_SZ = (size_t)NSRC * SEG_STRIDE;
constexpr size_t F1_OFF = ENC_OFF + 2 * ENC_SZ;
constexpr size_t F1_SZ = (size_t)NR * NSRC * 8;
constexpr size_t F2_OFF = F1_OFF + 2 * F1_SZ;
constexpr size_t F2_SZ = (size_t)32 * 8 * NCH * 4;
constexpr size_t CNT_OFF = F2_OFF + F2_SZ;
constexpr size_t TKT_OFF = CNT_OFF + GRID * 128;
constexpr size_t MBX_OFF = TKT_OFF + NR * 128;
constexpr size_t TMO_OFF = MBX_OFF + NR * 128;
constexpr size_t DBG_OFF = TMO_OFF + 128;
constexpr size_t WS_NEED = DBG_OFF + (1 << 20);
constexpr u64 DEADLINE = 50000000ull;

struct Params { char* ws[NR];   const char* x; const char* w; char* out; int rank; int pad; };
typedef const __attribute__((address_space(4))) Params CParams;

__host__ __device__ __forceinline__ int cls(int r) { return (r ^ (r >> 1)) & 1; }
__host__ __device__ __forceinline__ int slot_row(int r, int s) {
  const int h4 = 4 * cls(r), m = r & 1;
  int owner, piece;
  if (s < 8) { owner = r; piece = s; }
  else if (s < 12) { owner = r ^ 3; piece = 2 + (s & 1) + ((s & 2) << 1); }
  else {
    const int y = r ^ (4 + (s & 3)), a = (s - 12) >> 2;
    if (a == 4) { owner = r ^ 3; piece = 4 * cls(y) + (y & 1); } else { owner = y; piece = h4 + (a == 0 ? m : (a == 3 ? m ^ 1 : a + 1)); }
  }
  return owner * 1024 + piece * 128;
}
struct Role { int kind, a, c, u; };
#define NPULL 40
#define LATE0 12
#define JM 3
#define LATE1 16
constexpr int NENC = 40 - NPULL, NPA = NENC + 8 + (32 - LATE1) * 4, NPB = NENC + 88 + (32 - LATE0) * 4, NPC = NENC + 216;
__host__ __device__ __forceinline__ Role role_of(int b) {
  if (JM >= 0) { if ((b & 1) && (b >> 5) == JM) b = (b & 30) | (7 << 5); else if (!(b & 1) && (b >> 5) == 7) b = (b & 31) | 1 | (JM << 5); }
  const int xi = (b & 7) >> 1, j = b >> 3, jj = j >> 2;
  if (b & 1) { const int s = jj < 2 ? 2 * xi + jj : 4 * jj + xi; return {0, s, j & 3, NENC + (s >= LATE1 ? 8 : 88) + (31 - s) * 4 + (j & 3)}; }
  if (j < 16) return {1, 64 + 12 * (4 * xi + jj), j & 3, NPA + 16 + j * 4 + xi};
  if (j < 20) return {1, 12 * xi, j & 3, NPA + (j - 16) * 4 + xi};
  if (j < 22) { const int i = 2 * xi + j - 20; return {2, 48 + 8 * (i >> 2), i & 3, NENC + i}; }
  const int o = (j - 22) * 4 + xi;
  return {o < NPULL ? 3 : 4, o, 0, o - NPULL};
}
__host__ __device__ __forceinline__ int link_of(int o) { return 1 + (o + 2) % 7; }
constexpr int LMAX = 768;
struct Lists { unsigned short e[3][LMAX]; unsigned short n[3]; unsigned short mark[3][4]; };
constexpr void merge(Lists& L, int t, int ns, const u64* has, const int* pace, const int* off) {
  int c[8] = {};
  for (;;) {
    int b = -1;
    for (int i = 0; i < ns; ++i) {
      while (c[i] < 64 && !((has[i] >> c[i]) & 1)) ++c[i];
      if (c[i] < 64 && (b < 0 || c[i] * pace[i] + off[i] < c[b] * pace[b] + off[b])) b = i;
    }
    if (b < 0) break;
    for (int g = 0; g < 2; ++g) L.e[t][L.n[t]++] = (unsigned short)(b << 7 | c[b] << 1 | g);
    if (++c[b] == 64 && b == 0) L.mark[t][0] = L.n[t] - 1;
  }
  L.mark[t][1] = L.n[t] - 1;
}
__host__ __device__ constexpr u32 piece_link(u32 o, u32 code) {
  const u32 s = code >> 3, set = s < 12 ? REL0 : (s < RG ? REL1 : (s < 28 ? REL2 : REL3));
  return (set >> (code & 7u)) & 1u ? 4u + (((code & 1u) + ((code >> 1) & 1u) + ((code >> 1) & 2u) + s + (s >> 2)) & 3u) : o + 1;
}
constexpr u32 mail_count(u32 d) {
  u32 n = 0;
  for (u32 pc = 0; pc < 512; ++pc) { const u32 o = pc >> 8, l = piece_link(o, pc & 255u); n += (l == d) + (l > 3 && (l ^ (o + 1)) == d); }
  return n;
}
constexpr u32 NMAIL0 = mail_count(1), NMAIL2 = mail_count(4);
static_assert(NMAIL0 <= 64 * (NPULL / 7) && NMAIL2 <= 64 * (NPULL / 7), "mailbox");
static_assert(mail_count(2) == NMAIL0 && mail_count(5) == NMAIL2 && mail_count(6) == NMAIL2 && mail_count(7) == NMAIL2, "routes");
constexpr Lists make_lists() {
  Lists L = {};
  constexpr u64 ALL = ~0ull;
  { const u64 has[4] = {ALL, ALL, ALL, ALL}; const int pace[4] = {1, 1, 1, 1}, off[4] = {0, 0, LEAD, LEAD}; merge(L, 0, 4, has, pace, off); }
  { const u64 has[8] = {ALL, ALL, ALL, ALL, FDIR, FDIR, FDIR, FDIR}; const int pace[8] = {16, 16, 16, 16, P3, P3, P3, P3}, off[8] = {0, 0, 0, 0, O3, O3, O3, O3};
    merge(L, 1, 8, has, pace, off); }
  { const u64 has[5] = {ALL, ALL, ALL, ALL, ~FDIR}; const int pace[5] = {1, 1, 1, 1, 1}, off[5] = {0, OF1, OF2, OF3, OF4}; merge(L, 2, 5, has, pace, off); }
  L.mark[0][2] = L.n[0] + NMAIL0 / 2; L.mark[0][3] = L.n[0] + NMAIL0 * 3 / 4; L.mark[2][2] = L.n[2] + NMAIL2 / 2; L.mark[2][3] = L.n[2] + NMAIL2 * 3 / 4;
  return L;
}
constexpr Lists LISTS = make_lists();
static_assert(OF4 > 0, "lists");
__host__ __device__ __forceinline__ int list_type(int d) { return d < 3 ? 0 : (d == 3 ? 1 : 2); }
constexpr u32 D_FWD = 1u << 12, D_C = 1u << 13, D_REL = 1u << 14;
__host__ __device__ __forceinline__ u32 resolve(int d, int r, u32 en) {
  const u32 a = (en >> 7) & 7u, h4 = 4 * cls(r), m = r & 1; u32 sp, dp, fl = 0;
  if (d < 3) { dp = 4u * (d - 1) + a; sp = 8 + dp; }
  else if (d == 3) { if (a < 4) { sp = 2 + (a & 1) + ((a & 2) << 1); dp = 8 + a; } else { const int y = r ^ a; sp = 4 * cls(y) + (y & 1); dp = 24 + a; } }
  else if (a < 3) { sp = h4 + (a == 0 ? m : a + 1); dp = 8 + 4 * a + d; fl = a == 0 && !((FDIR >> ((en >> 1) & 63u)) & 1) ? D_FWD : 0u; }
  else { sp = a == 4 ? 12u + (d ^ 3) : h4 + (m ^ 1); dp = 8 + 4 * a + d; }
  return ((sp * 128u + (en & 127u)) << 16) | fl | (dp << 7) | (en & 127u);
}
__host__ __device__ __forceinline__ u32 input_mask(int r, u32 pn, u32 c) {
  return pn < 8u ? (((pn >> 2) == (u32)cls(r) ? 0x90u : 0x60u) | ((pn & 2u) || ((FDIR >> c) & 1) ? 8u : 0u)) : (pn < 12u ? 2u : 4u);
}

#define STAMPV(i, v) do { (void)(v); } while (0)
#define LASTPOLL do {} while (0)
#define CLKS(c) do {} while (0)
#define STAMP(i) do {} while (0)
#define STAMPC(i) do {} while (0)

#define WAIT_V(n) asm volatile("s_waitcnt vmcnt(" #n ")" ::: "memory")
#define WAIT_L(n) asm volatile("s_waitcnt lgkmcnt(" #n ")" ::: "memory")
#define DRAIN do { WAIT_V(0); __builtin_amdgcn_s_waitcnt(0x0f70); } while (0)
#define BAR __builtin_amdgcn_s_barrier()
#define SCHED __builtin_amdgcn_sched_barrier(0)
#define LBAR do { WAIT_L(0); BAR; asm volatile("" ::: "memory"); } while (0)
template <int N> __device__ __forceinline__ void wait_v() { asm volatile("s_waitcnt vmcnt(%0)" :: "i"(N) : "memory"); }

__device__ __forceinline__ u32 rfl(u32 v) { return __builtin_amdgcn_readfirstlane(v); }
template <class T> __device__ __forceinline__ T* uni(T* q) {
  u64 v = (u64)q; v = ((u64)rfl((u32)(v >> 32)) << 32) | rfl((u32)v);
  asm volatile("s_nop 4" : "+s"(v));
  return (T*)(__attribute__((address_space(1))) T*)v;
}
__device__ __forceinline__ void st4x4(char* base, u32 off, u32x4 a, u32x4 b, u32x4 c, u32x4 d) {
  asm volatile("global_store_dwordx4 %0, %1, %5 sc0 sc1\n\tglobal_store_dwordx4 %0, %2, %5 offset:1024 sc0 sc1\n\t"
               "global_store_dwordx4 %0, %3, %5 offset:2048 sc0 sc1\n\tglobal_store_dwordx4 %0, %4, %5 offset:3072 sc0 sc1\n\ts_nop 1"
               :: "v"(off), "v"(a), "v"(b), "v"(c), "v"(d), "s"(base) : "memory");
}
__device__ __forceinline__ void st4x2(char* base, u32 off, u32x4 a, u32x4 b) {
  asm volatile("global_store_dwordx4 %0, %1, %3 sc0 sc1\n\tglobal_store_dwordx4 %0, %2, %3 offset:1024 sc0 sc1\n\ts_nop 1"
               :: "v"(off), "v"(a), "v"(b), "s"(base) : "memory");
}
__device__ __forceinline__ void st4x1(char* base, u32 off, u32x4 a) {
  asm volatile("global_store_dwordx4 %0, %1, %2 sc0 sc1\n\ts_nop 1" :: "v"(off), "v"(a), "s"(base) : "memory");
}
#define OB " sc1"
__device__ __forceinline__ void st_out(char* q, u32x4 a) { asm volatile("global_store_dwordx4 %0, %1, off" OB "\n\ts_nop 1" :: "v"(q), "v"(a) : "memory"); }
__device__ __forceinline__ bool give_up(CParams& p, u32 call, u64 t0) {
  u32* tmo = (u32*)(p.ws[0] + TMO_OFF);
  if (rfl(__hip_atomic_load(tmo, __ATOMIC_RELAXED, __HIP_MEMORY_SCOPE_AGENT)) == call) return true;
  if (__builtin_amdgcn_s_memrealtime() - t0 > DEADLINE) {
    __hip_atomic_store(tmo, call, __ATOMIC_RELAXED, __HIP_MEMORY_SCOPE_AGENT);
    WAIT_V(0);
    return true;
  }
  return false;
}
__device__ __forceinline__ gu64* flag_row(CParams& p, u32 call, int lane) {
  u32 lo = 0, hi = 0;
#pragma unroll
  for (int i = 0; i < NR; ++i) { u64 v = (u64)p.ws[i]; asm volatile("" : "+s"(v)); lo = lane == i ? (u32)v : lo; hi = lane == i ? (u32)(v >> 32) : hi; }
  return (gu64*)((((u64)hi << 32) | lo) + F1_OFF + (call & 1) * F1_SZ) + (lane & 7) * NSRC;
}
__device__ __forceinline__ void post_seg(gu64* row, u32 call, u32 seg, u32 len, u32 mask, int lane) {
  if (lane < NR && ((mask >> lane) & 1u)) __hip_atomic_store(row + seg, ((u64)call << 32) | len, __ATOMIC_RELAXED, __HIP_MEMORY_SCOPE_SYSTEM);
}
constexpr u32 MAIL0 = 20 * 128;
__device__ __forceinline__ u32 mail_place(const char* mbx, u32 mask, int lane) {
  u32 t;
  asm volatile("s_mov_b64 exec, %4\n\tglobal_atomic_add %0, %1, %2, %3 sc0\n\ts_mov_b64 exec, -1" : "=&v"(t) : "v"(lane * 128), "v"(1u), "s"(mbx), "s"((u64)(mask & 0xffu)) : "memory");
  return t;
}
__device__ __forceinline__ void post_mail(gu64* row, u32 call, u32 place, u32 seg, u32 len, u32 mask, int lane) {
  asm volatile("" : "+v"(place));
  post_seg(row, call, MAIL0 + (lane < 3 ? place % NMAIL0 : place % (NMAIL2 ? NMAIL2 : 1)), (seg << 16) | (len >> 4), mask, lane);
}
__device__ __forceinline__ u32 wave_scan(u32 v) {
  v += __builtin_amdgcn_update_dpp(0u, v, 0x111, 0xf, 0xf, false);
  v += __builtin_amdgcn_update_dpp(0u, v, 0x112, 0xf, 0xf, false);
  v += __builtin_amdgcn_update_dpp(0u, v, 0x114, 0xf, 0xf, false);
  v += __builtin_amdgcn_update_dpp(0u, v, 0x118, 0xf, 0xf, false);
  v += __builtin_amdgcn_update_dpp(0u, v, 0x142, 0xa, 0xf, false);
  v += __builtin_amdgcn_update_dpp(0u, v, 0x143, 0xc, 0xf, false);
  return v;
}
#define BOP(a, b, c, expr) (u32)__builtin_amdgcn_bitop3_b32((a), (b), (c), [] { constexpr u32 A = 0xf0, B = 0xcc, C = 0xaa; (void)A; (void)B; (void)C; return (expr) & 0xffu; }())
#define BFI(m, a, b) BOP(m, a, b, (A & B) | (~A & C))
struct Mv { u32 a0, a1, a2, c8, c16; };
__device__ __forceinline__ u32 byte_prefix_xor(u32 v) {
  v = BOP(v, v << 1, 0xfefefefeu, A ^ (B & C)); v = BOP(v, v << 2, 0xfcfcfcfcu, A ^ (B & C)); return BOP(v, v << 4, 0xf0f0f0f0u, A ^ (B & C));
}
__device__ __forceinline__ Mv calc_masks(u32 m) {
  Mv r;
  r.c16 = __builtin_popcount(m & 0xffffu);
  r.c8 = (u32)__builtin_popcount(m & 0xffu) | ((u32)__builtin_popcount(m & 0xff0000u) << 16);
  u32 mk = BOP(m << 1, 0xfefefefeu, 0u, ~A & B);
  u32 mp = byte_prefix_xor(mk);
  r.a0 = mp & m; m = BOP(m, r.a0, r.a0 >> 1, (A ^ B) | C); mk = BOP(mk, mp, 0u, A & ~B);
  mp = byte_prefix_xor(mk);
  r.a1 = mp & m; m = BOP(m, r.a1, r.a1 >> 2, (A ^ B) | C); mk = BOP(mk, mp, 0u, A & ~B);
  r.a2 = BOP(0x80808080u - mk, 0x80808080u, m, (A ^ B) & C);
  return r;
}
__device__ __forceinline__ u32 mask_entry(u32 b) { const Mv v = calc_masks(b); return v.c16 | (v.a0 << 8) | (v.a1 << 16) | (v.a2 << 24); }
__device__ __forceinline__ Mv move_masks(u32 m) {
  const u32 e0 = *(const LAS u32*)(size_t)((m & 255u) << 2), e1 = *(const LAS u32*)(size_t)(((m >> 8) & 255u) << 2);
  const u32 e2 = *(const LAS u32*)(size_t)(((m >> 16) & 255u) << 2), e3 = *(const LAS u32*)(size_t)((m >> 24) << 2);
  const u32 l02 = __builtin_amdgcn_perm(e2, e0, 0x05040100u), l13 = __builtin_amdgcn_perm(e3, e1, 0x05040100u);
  const u32 h02 = __builtin_amdgcn_perm(e2, e0, 0x07060302u), h13 = __builtin_amdgcn_perm(e3, e1, 0x07060302u);
  Mv r;
  r.a0 = __builtin_amdgcn_perm(l13, l02, 0x07030501u); r.a1 = __builtin_amdgcn_perm(h13, h02, 0x06020400u); r.a2 = __builtin_amdgcn_perm(h13, h02, 0x07030501u);
  r.c8 = l02; r.c16 = e0 + e1;
  return r;
}
__device__ __forceinline__ u32 pext(u32 x, u32 m, const Mv& v) {
  x &= m;
  u32 t = x & v.a0; x = BOP(x, t, t >> 1, (A ^ B) | C);
  t = x & v.a1; x = BOP(x, t, t >> 2, (A ^ B) | C);
  t = x & v.a2; x = BOP(x, t, t >> 4, (A ^ B) | C);
  t = __builtin_amdgcn_perm(0u, x, 0x0c030c01u);
  asm("v_pk_lshlrev_b16 %0, %1, %0" : "+v"(t) : "v"(v.c8));
  x = BOP(x, 0x00ff00ffu, t, (A & B) | C);
  return BOP(x, 0xffffu, (x >> 16) << (v.c16 & 31u), (A & B) | C);
}
__device__ __forceinline__ u32 pdep(u32 x, const Mv& v) {
  x = __builtin_amdgcn_perm(x >> (v.c16 & 31u), x, 0x05040100u);
  u32 t; asm("v_pk_lshrrev_b16 %0, %1, %2" : "=v"(t) : "v"(v.c8), "v"(x));
  x = __builtin_amdgcn_perm(t, x, 0x06020400u);
  x = BFI(v.a2, x << 4, x); x = BFI(v.a1, x << 2, x);
  return BFI(v.a0, x << 1, x);
}
__device__ __forceinline__ void transpose4(u32 (&n)[4]) {
#pragma unroll
  for (int r = 0; r < 2; ++r) { const u32 a = n[r], c = n[r + 2]; n[r] = BFI(0x33333333u, a, c << 2); n[r + 2] = BFI(0x33333333u, a >> 2, c); }
#pragma unroll
  for (int r = 0; r < 4; r += 2) { const u32 a = n[r], c = n[r + 1]; n[r] = BFI(0x55555555u, a, c << 1); n[r + 1] = BFI(0x55555555u, a >> 1, c); }
}
__device__ __forceinline__ void or_bits(LAS u32* area, u32 bit, u32 val) {
  const u64 t = (u64)val << (bit & 31);
  LAS u32* q = area + (bit >> 5);
  if ((u32)t) __hip_atomic_fetch_or(q, (u32)t, __ATOMIC_RELAXED, __HIP_MEMORY_SCOPE_WORKGROUP);
  if ((u32)(t >> 32)) __hip_atomic_fetch_or(q + 1, (u32)(t >> 32), __ATOMIC_RELAXED, __HIP_MEMORY_SCOPE_WORKGROUP);
}
__device__ __forceinline__ u32 mad3(u32 a, u32 c) { u32 r; asm("v_mad_u32_u24 %0, %1, 3, %2" : "=v"(r) : "v"(a), "s"(c)); return r; }
__device__ __forceinline__ u32 get_bits(u32 bit) {
  const LAS u32* q = (const LAS u32*)(size_t)((bit >> 3) & ~3u);
  return __builtin_amdgcn_alignbit(q[1], q[0], bit & 31);
}

struct Enc {
  char* enc; gu64* row; LAS char* lds; u32 call; int w, lane; u32 par, pseg, pmask, place; u64 h4 = 0, h5 = 0;
  __device__ __forceinline__ void init(CParams& p, LAS char* l, u32 c, int w_, int lane_) {
    enc = uni(p.ws[0] + ENC_OFF + (c & 1) * ENC_SZ); row = flag_row(p, c, lane_); lds = l; call = c; w = w_; lane = lane_; par = 0; pseg = 0; pmask = 0;
  }
  __device__ __forceinline__ LAS u32* fill(u32 q) const { return (LAS u32*)(lds + LDS_MAIN + 64) + (q & 3u) * 2 + w / NB; }
  __device__ __forceinline__ u32 take_len() { LAS u32* c = fill(par - 1); const u32 n = rfl(*c); *c = 0; return (SEG_V + n * 4 + 15) & ~15u; }
  __device__ __forceinline__ void post() { if ((w & (NB - 1)) == 0 && pmask) post_seg(row, call, pseg, take_len(), pmask, lane); pmask = 0; }
  __device__ __forceinline__ void finish() { DRAIN; LBAR; post(); }
  __device__ __forceinline__ void draw(const char* mbx) { place = mail_place(mbx, (w & (NB - 1)) == 0 ? pmask : 0u, lane); }
  __device__ __forceinline__ void finish_mail() { DRAIN; LBAR; if ((w & (NB - 1)) == 0) post_mail(row, call, place, pseg, take_len(), pmask, lane); pmask = 0; }
  template <class F> __device__ __forceinline__ void idle(F&& mid) { finish(); mid(); ++par; }
  template <bool PIPE, class F> __device__ __forceinline__ void round(const u32x4 (&X)[4], u32 seg, u32 mask, F&& mid) {
    const int rb = w % NB;
    const u32 so = seg * SEG_STRIDE;
    LAS u32* var = (LAS u32*)(lds + LDS_VAR + w * 1536);
    { const u32x4 zero = {0u, 0u, 0u, 0u}; ((LAS u32x4*)var)[lane] = zero; ((LAS u32x2*)var)[128 + lane] = (u32x2){0u, 0u}; }
    u32 R[16], S[8], sum = 0;
#pragma unroll
    for (int i = 0; i < 16; ++i) { const u32 t = X[i >> 2][i & 3]; R[i] = __builtin_amdgcn_alignbit(t, t, 31); if (i & 1) sum = __builtin_amdgcn_udot4(R[i], 0x01000100u, sum, false); }
#pragma unroll
    for (int q = 0; q < 8; ++q) S[q] = __builtin_amdgcn_perm(R[2 * q + 1], R[2 * q], 0x06040200u);
    if (!PIPE) st4x2(enc, so + rb * 2048 + lane * 16, (u32x4){S[0], S[1], S[2], S[3]}, (u32x4){S[4], S[5], S[6], S[7]});
    sum = __builtin_amdgcn_readlane(wave_scan(sum), 63);
    int basei = (int)((sum + 870u) >> 10) - 10;
    basei = basei < 0 ? 0 : (basei > 240 ? 240 : basei);
    const u32 base = (u32)basei;
    const u16x2 bsub = {(unsigned short)(base << 8), (unsigned short)(base << 8)}, top = {0x0f00, 0x0f00};
    u32 V[8], d[4];
#pragma unroll
    for (int q = 0; q < 8; ++q) {
      const u16x2 a = __builtin_elementwise_min((u16x2)(__builtin_bit_cast(u16x2, R[2 * q]) - bsub), top);
      const u16x2 b = __builtin_elementwise_min((u16x2)(__builtin_bit_cast(u16x2, R[2 * q + 1]) - bsub), top);
      V[q] = __builtin_amdgcn_perm(__builtin_bit_cast(u32, b), __builtin_bit_cast(u32, a), 0x07050301u);
    }
#pragma unroll
    for (int q = 0; q < 4; ++q) d[q] = V[2 * q] | (V[2 * q + 1] << 4);
    transpose4(d);
    u32 Xm = BOP(d[3], d[2], d[1], A & B & C);
    u32 P1 = BOP(d[3], d[2], d[1], A & ~B & C), P0 = BOP(d[3], d[2], d[0], A & ~B & C) | Xm;
    const u32 E1 = ~(P1 | P0);
    const u32 Q1 = d[3] & ~d[0], Q0 = d[2] & BOP(d[3], d[1], d[0], (A & ~C) | (~A & B & C));
    const u32 E2 = E1 & ~(Q1 | Q0);
    const Mv ma = move_masks(E1), mb = move_masks(E2);
    const u32 n1 = __builtin_popcount(E1), n2 = __builtin_popcount(E2), nx = __builtin_popcount(Xm);
    const u32 cnt = n1 | (n2 << 12) | ((nx > 3u ? 3u : nx) << 24);
    const u32 inc = wave_scan(cnt), tot = __builtin_amdgcn_readlane(inc, 63), exc = inc - cnt;
    const u32 N1 = tot & 0xfffu, NX = tot >> 24, bitw = (2 * N1 + 3 * ((tot >> 12) & 0xfffu) + 31) >> 5;
    const bool raw = (NX > XCAP) | (__ballot(nx > 3u) != 0);
    const u32 vd = raw ? 384u : bitw + NX;
    u32 voff = 0;
    if (lane == 0) voff = __hip_atomic_fetch_add(fill(par), vd, __ATOMIC_RELAXED, __HIP_MEMORY_SCOPE_WORKGROUP);
    const u32 q1 = pext(Q1, E1, ma), q0 = pext(Q0, E1, ma), r2 = pext(d[2], E2, mb), r1 = pext(d[1] | d[3], E2, mb), r0 = pext(d[0], E2, mb);
    const u32 o1 = 2 * (exc & 0xfffu), o2 = 2 * N1 + 3 * ((exc >> 12) & 0xfffu);
    if (__all(n1 <= 16u && n2 <= 10u)) { or_bits(var, o1, q1 | (q0 << n1)); or_bits(var, o2, r2 | (r1 << n2) | (r0 << (2 * n2))); }
    else { or_bits(var, o1, q1); or_bits(var, o1 + n1, q0); or_bits(var, o2, r2); or_bits(var, o2 + n2, r1); or_bits(var, o2 + 2 * n2, r0); }
    if (NX) {
      u32 E[8];
#pragma unroll
      for (int q = 0; q < 8; ++q) E[q] = __builtin_amdgcn_perm(R[2 * q + 1], R[2 * q], 0x07050301u);
      if (!raw) {
        u32 xo = bitw + (exc >> 24);
        do {
          if (Xm != 0u) {
            const u32 j = __builtin_ctz(Xm); Xm &= Xm - 1;
            const u32 vi = 2 * (j & 3) + ((j >> 2) & 1), by = j >> 3;
            const u32 k0 = 0u - (vi & 1u), k1 = 0u - ((vi >> 1) & 1u), k2 = 0u - (vi >> 2);
            const u32 vs = BFI(k2, BFI(k1, BFI(k0, E[7], E[6]), BFI(k0, E[5], E[4])), BFI(k1, BFI(k0, E[3], E[2]), BFI(k0, E[1], E[0])));
            var[xo] = (u32)lane | (vi << 6) | (by << 9) | ((((vs >> (8 * by)) & 255u) ^ (base + 9u)) << 16);
            ++xo;
          }
        } while (__any(Xm != 0u));
      } else {
        P1 = E[0]; P0 = E[1];
#pragma unroll
        for (int q = 2; q < 8; ++q) var[(q - 2) * 64 + lane] = E[q];
      }
    }
    asm volatile("" ::: "memory");
    u32 v0 = var[lane];
    voff = rfl(voff);
    if (PIPE) {
      asm volatile("" : "+v"(v0));
      finish();
      mid();
      st4x2(enc, so + rb * 2048 + lane * 16, (u32x4){S[0], S[1], S[2], S[3]}, (u32x4){S[4], S[5], S[6], S[7]});
    }
    asm volatile("global_store_dwordx2 %0, %1, %4 sc0 sc1\n\ts_mov_b64 exec, 1\n\tglobal_store_dword %2, %3, %4 sc0 sc1\n\ts_mov_b64 exec, -1"
                 :: "v"(so + SEG_P + rb * 512 + lane * 8), "v"((u32x2){P1, P0}), "v"(so + SEG_H + rb * 4), "v"(base | ((raw ? 255u : NX) << 8) | (voff << 16)), "s"(enc) : "memory");
    const u32 vo = so + SEG_V + (voff + lane) * 4;
    if ((u32)lane < vd) asm volatile("global_store_dword %0, %1, %2 sc0 sc1" :: "v"(vo), "v"(v0), "s"(enc) : "memory");
    for (u32 i = 64; i < vd; i += 64) if (i + lane < vd) asm volatile("global_store_dword %0, %1, %2 sc0 sc1" :: "v"(vo + i * 4), "v"(var[i + lane]), "s"(enc) : "memory");
    pseg = seg; pmask = mask; ++par;
  }
};
constexpr int XBUF = 65536;
__device__ __forceinline__ void load_job(CParams& p, LAS char* lds, int J, int buf, int w, int lane) {
  if (J >= NJOB) return;
  const int c = J >> 5, pn = (J >> 1) & 15, st = (J & 1) * NB + (w & (NB - 1));
  const char* q; u32 off, step;
  if (pn < 8) { q = p.x; off = ((u32)((pn * 8 + st) * 16 + (lane & 15)) * KD + c * 128 + (lane >> 4) * 8) * 2; step = 64; }
  else { q = p.w; off = (u32)(c * 128 + (lane >> 1)) * (NS * 2) + (pn - 8) * 256 + st * 32 + (lane & 1) * 16; step = 65536; }
  q = uni(q);
  const u32 dst = (u32)(size_t)lds + XBUF + buf * 32768 + w * 4096;
#pragma unroll
  for (int k = 0; k < 4; ++k) asm volatile("s_mov_b32 m0, %2\n\ts_nop 0\n\tglobal_load_lds_dwordx4 %0, %1" :: "v"(off + k * step), "s"(q), "s"(dst + k * 1024) : "memory");
}
constexpr int JOB1 = NPC + UPR * 2 * NPC, JOB2 = JOB1 + MIDR * 2 * NPB;

struct Jobs {
  int nr, u, sg;
  __device__ __forceinline__ void init(int u_, int sg_) {
    u = u_; sg = sg_;
    nr = u < 0 ? 0 : (u >= NPB ? 1 + UPR : 1 + UPR + MIDR + (u >= NPA || JOB2 + 2 * u >= NJOB ? 0 : (NJOB - JOB2 - 2 * u + 2 * NPA - 1) / (2 * NPA)));
  }
  __device__ __forceinline__ int job(int r) const {
    if (r >= nr) return NJOB;
    if (r == 0) return sg ? NJOB : u;
    return 2 * u + sg + (r <= UPR ? NPC + (r - 1) * 2 * NPC : (r <= UPR + MIDR ? JOB1 + (r - 1 - UPR) * 2 * NPB : JOB2 + (r - 1 - UPR - MIDR) * 2 * NPA));
  }
};
__device__ __forceinline__ void encode_jobs(CParams& p, LAS char* lds, u32 call, int w, int lane, const Jobs& jb) {
  Enc e; e.init(p, lds, call, w, lane);
  for (int r = 0; r < jb.nr; ++r) {
    const int J = jb.job(r), J2 = jb.job(r + 2);
    auto mid = [&] { if (r != 0) load_job(p, lds, J2, r & 1, w, lane); };
    if (J < NJOB) {
      u32x4 X[4];
      const u32 pn = (J >> 1) & 15;
      LAS char* xb = lds + XBUF + (r & 1) * 32768 + w * 4096;
      if (pn < 8u) {
#pragma unroll
        for (int k = 0; k < 4; ++k) X[k] = *(const LAS u32x4*)(xb + k * 1024 + lane * 16);
      } else {
#pragma unroll
        for (int k = 0; k < 4; ++k) {
          LAS char* q = xb + k * 1024 + (lane >> 4) * 256 + (lane & 15) * 8;
          const s16x4 lo = __builtin_amdgcn_ds_read_tr16_b64_v4i16((LAS s16x4*)q), hi = __builtin_amdgcn_ds_read_tr16_b64_v4i16((LAS s16x4*)(q + 128));
          X[k] = __builtin_bit_cast(u32x4, __builtin_shufflevector(lo, hi, 0, 1, 2, 3, 4, 5, 6, 7));
        }
      }
      e.round<true>(X, pn * 128 + (J >> 5) * 2 + (J & 1), input_mask(p.rank, pn, J >> 5), mid);
    } else e.idle(mid);
    if (r == 0) { e.finish(); load_job(p, lds, J2, 0, w, lane); STAMP(1);
    }
  }
  e.finish();
  STAMP(2);
}

__device__ __forceinline__ void lds_post(LAS u32* q, u32 v) { asm volatile("ds_write_b32 %0, %1" :: "v"((u32)(size_t)q), "v"(v) : "memory"); }
__device__ __forceinline__ u32 lds_peek(LAS u32* q) { u32 v; asm volatile("ds_read_b32 %0, %1\n\ts_waitcnt lgkmcnt(0)" : "=&v"(v) : "v"((u32)(size_t)q) : "memory"); return rfl(v); }
__device__ __forceinline__ void lds_inc(LAS u32* q) { asm volatile("s_mov_b64 exec, 1\n\tds_add_u32 %0, %1\n\ts_mov_b64 exec, -1" :: "v"((u32)(size_t)q), "v"(1u) : "memory"); }
template <int WIN, int I> __device__ __forceinline__ void load10(const char* gs, u32 ls, const u32 (&v16)[3], LAS u32* landed, u32 k, u32 since2) {
  if constexpr (I < 10) {
    if ((I & 3) == 0) asm volatile("s_mov_b32 m0, %0" :: "s"(ls + I * 1024) : "memory");
    wait_v<WIN - 1>();
    asm volatile("global_load_lds_dwordx4 %0, %1 offset:%2 sc0 sc1" :: "v"(v16[I >> 2]), "s"(gs), "i"((I & 3) * 1024) : "memory");
    if (I + 11 <= WIN - 1 && since2 + I == WIN - 1) lds_post(landed, k - 1);
    if (I == WIN - 1) lds_post(landed, k);
    load10<WIN, I + 1>(gs, ls, v16, landed, k, since2);
  }
}
constexpr u32 FIN = 0x40000000u;
__host__ __device__ __forceinline__ u32 c_offset(int rank, u32 pc) {
  const u32 o = (pc >> 8) & 1u;
  return ((u32)(slot_row(rank ^ (o + 1), (pc >> 3) & 31u) + (pc & 1u) * 64) * NS + o * 512 + ((pc >> 2) & 1u) * 256 + ((pc >> 1) & 1u) * 128) * 2;
}

template <int WIN> __device__ __forceinline__ void loader(CParams& p, LAS char* lds, int d, u32 call, int lane) {
    LAS u32* landed = (LAS u32*)(lds + LDS_MAIN + 128); LAS u32* posted = landed + 1; LAS u32* started = landed + 2;
    LAS u32* done = (LAS u32*)(lds + LDS_MAIN + 192); LAS u32* qitem = (LAS u32*)(lds + LDS_MAIN + 256);
    const char* rsrc = uni(p.ws[d] + ENC_OFF + (call & 1) * ENC_SZ);
    __builtin_amdgcn_s_waitcnt(0x0f70);
    __builtin_amdgcn_s_setprio(3);
    const u32 v16[3] = {(u32)lane * 16, (u32)lane * 16 + 4096, (u32)lane * 16 + 8192};
    int slot = 0; u32 use = 0, k = 0, since2 = 100; int spinq = 0, spind = 0;
    static_assert(WIN <= 22, "window");
    for (;; ++k) {
      u32 item, len;
      for (u32 it = 0;; ++it) {
        u32 a, dn;
        asm volatile("ds_read_b32 %0, %4\n\tds_read_b32 %1, %5\n\tds_read_b32 %2, %6\n\tds_read_b32 %3, %6 offset:64\n\ts_waitcnt lgkmcnt(0)"
                     : "=&v"(a), "=&v"(dn), "=&v"(item), "=&v"(len)
                     : "v"((u32)(size_t)posted), "v"((u32)(size_t)(done + slot)), "v"((u32)(size_t)(qitem + (k % QD))) : "memory");
        if (rfl(a) > k && rfl(dn) >= use) break;
        if (it == DRAINIT) { WAIT_V(0); lds_post(landed, k); since2 = 100; }
        __builtin_amdgcn_s_sleep(1);
      }
      len = rfl(len);
      if (len == 0) break;
      lds_post(started, k + 1);
      const char* gs = rsrc + (size_t)(rfl(item) >> 16) * SEG_STRIDE;
      const u32 ls = (u32)(size_t)lds + slot * SEG_STRIDE;
      load10<WIN, 0>(gs, ls, v16, landed, k, since2);
      for (u32 o = 10240; o < len; o += 1024) {
        wait_v<WIN - 1>();
        if (WIN > 21 && since2 + (o >> 10) == WIN - 1) lds_post(landed, k - 1);
        if (WIN > 10 && o == (WIN - 1) * 1024) lds_post(landed, k);
        if (o + lane * 16 < len) asm volatile("s_mov_b32 m0, %2\n\ts_nop 0\n\tglobal_load_lds_dwordx4 %0, %1 sc0 sc1" :: "v"(o + lane * 16), "s"(gs), "s"(ls + o) : "memory");
      }
      since2 = (len + 1023) >> 10;
      if (++slot == NSLOT) { slot = 0; use += 2; }
    }
    WAIT_V(0); lds_post(landed, k | FIN);
    STAMP(2); STAMPV(10, spinq); STAMPV(11, spind); STAMPV(12, k);
}
__device__ __forceinline__ void pull_role(CParams& p, LAS char* lds, int d, int j, int ncu, u32 call, int w, int lane) {
  LAS u32* landed = (LAS u32*)(lds + LDS_MAIN + 128);
  LAS u32* posted = landed + 1;
  LAS u32* started = landed + 2;
  LAS u32* done = (LAS u32*)(lds + LDS_MAIN + 192);
  LAS u32* qitem = (LAS u32*)(lds + LDS_MAIN + 256);
  LAS u32* qdst = qitem + 2 * QD;
  LAS u32* qlen = qitem + QD;
  LAS u32* list = (LAS u32*)(lds + NSLOT * SEG_STRIDE);
  static_assert(NSLOT * SEG_STRIDE + LMAX * 4 <= LDS_MAIN, "list");
  const int rank = rfl(p.rank), lt = list_type(d);
  const u32 nl = rfl(LISTS.n[lt]), n = nl + (lt == 0 ? NMAIL0 : (lt == 2 ? NMAIL2 : 0u));
  for (u32 i = threadIdx.x; i < nl; i += 512) list[i] = resolve(d, rank, LISTS.e[lt][i]);
  LBAR;
  if (w == 1) {
    const u64* f1 = uni((const u64*)(p.ws[0] + F1_OFF + (call & 1) * F1_SZ) + d * NSRC);
    u32* tk = uni((u32*)(p.ws[0] + TKT_OFF + d * 128));
    const u32 tbase = call * 16384u, nin = nl - 1;
    __builtin_amdgcn_s_setprio(2);
    if (lane == 0) __hip_atomic_fetch_max(tk, tbase, __ATOMIC_RELAXED, __HIP_MEMORY_SCOPE_AGENT);
    WAIT_V(0);
#define FIDX(i) (list[(i) < nl ? (i) : nin] >> 16)
#define ENQUEUE(item, len, dst) do { lds_post(qdst + (k % QD), dst); lds_post(qitem + (k % QD), item); lds_post(qlen + (k % QD), len); lds_post(posted, ++k); } while (0)
#define CLAMP_LEN(l) ((l) > (u32)SEG_STRIDE ? (u32)SEG_STRIDE : ((l) < (u32)SEG_V ? (u32)SEG_V : (l) & ~15u))
    u32 pos = j, left = 1, win = 0, k = 0; u64 v = 0; bool dead = false;
    for (;;) {
      if (left == 0) {
        u32 st;
        while ((st = lds_peek(started)) + LA <= k) __builtin_amdgcn_s_sleep(1);
        left = pos + NEAR >= nin || pos < NEAR0 ? (st + LA - k >= BEHIND && k >= 2 ? 2u : 1u) : GMAX;
        win = pos;
        u32 t;
        asm volatile("s_mov_b64 exec, 1\n\tglobal_atomic_add %0, %2, %3, %4 sc0\n\ts_mov_b64 exec, -1\n\tglobal_load_dwordx2 %1, %5, %6 sc0 sc1\n\ts_waitcnt vmcnt(0)"
                     : "=&v"(t), "=&v"(v) : "v"(0u), "v"(left), "s"(tk), "v"(FIDX(win + lane) * 8u), "s"(f1) : "memory");
        pos = rfl(t) - tbase + ncu;
      }
      if (pos >= nl) break;
      const u32 item = rfl(list[pos]);
      u32 len = 0;
      const u64 ok = __ballot((u32)(v >> 32) >= call);
      if (pos - win < 64u && ((ok >> (pos - win)) & 1)) len = __builtin_amdgcn_readlane((u32)v, pos - win);
      else {
        const u64 t0 = __builtin_amdgcn_s_memrealtime();
        win = pos;
        const u64* fp = f1 + FIDX(win + (lane & 7));
        for (u32 it = 1;; ++it) {
          v = __hip_atomic_load(fp, __ATOMIC_RELAXED, __HIP_MEMORY_SCOPE_SYSTEM);
          if (rfl((u32)(v >> 32)) >= call) { len = rfl((u32)v); break; }
          if ((it & 63u) == 0 && give_up(p, call, t0)) { dead = true; break; }
          __builtin_amdgcn_s_sleep(1);
        }
        if (dead) break;
      }
      ENQUEUE(item, CLAMP_LEN(len), ((item >> 7) & 31u) * (u32)PANEL_B + (item & 127u) * 16384u);
      if (k == 1) STAMP(1);
      ++pos; --left;
    }
    const u32 place = j + lane * ncu;
    u64 todo = dead ? 0 : __ballot(place < n - nl);
    const u64* fp = f1 + MAIL0 + (place < n - nl ? place : 0u);
    u64 t0 = __builtin_amdgcn_s_memrealtime();
    for (u32 it = 1; todo; ++it) {
      v = __hip_atomic_load(fp, __ATOMIC_RELAXED, __HIP_MEMORY_SCOPE_SYSTEM);
      u64 rdy = __ballot((u32)(v >> 32) >= call) & todo;
      if (rdy) {
        todo &= ~rdy;
        const u32 sg_ = (((u32)v >> 16) - MAIL0) & 1023u, ln = ((u32)v & 0xffffu) << 4;
        const u32 item = ((MAIL0 + sg_) << 16) | (sg_ & 511u) | (d > 3 && sg_ < 512u ? D_REL : D_C), len = CLAMP_LEN(ln), dst = c_offset(rank, sg_);
        const u64 first = __ballot(item & D_REL);
        do {
          while (lds_peek(started) + LAM <= k) __builtin_amdgcn_s_sleep(1);
          const u32 l = __builtin_ctzll(rdy & first ? rdy & first : rdy); rdy &= ~(1ull << l);
          ENQUEUE(__builtin_amdgcn_readlane(item, l), __builtin_amdgcn_readlane(len, l), __builtin_amdgcn_readlane(dst, l));
          if (REPOLL && todo && lds_peek(started) + LAM <= k) { todo |= rdy; rdy = 0; }
        } while (rdy);
        if (REPOLL) while (lds_peek(started) + LAM <= k + 1 - REPOLL) __builtin_amdgcn_s_sleep(1);
      } else {
        if ((it & 63u) == 0 && give_up(p, call, t0)) { dead = true; break; }
        __builtin_amdgcn_s_sleep(1);
      }
    }
    if (dead)
      for (u32 pc = lane; pc < 512u; pc += 64) { const u32 l = piece_link(pc >> 8, pc & 255u);
        if ((l > 3 ? l ^ ((pc >> 8) + 1) : l) == (u32)d) *(u32*)(p.out + c_offset(rank, pc)) = 0x7fc07fc0u; }
    while (lds_peek(started) + LAM <= k) __builtin_amdgcn_s_sleep(1);
    ENQUEUE(0u, 0u, 0u);
    return;
  }
  if (w == 0) {
    if (ncu * INFL >= 60) loader<INFL>(p, lds, d, call, lane); else loader<INFL5>(p, lds, d, call, lane);
    return;
  }
  char* dec = uni(p.ws[0] + DEC_OFF);
  char* enc = uni(p.ws[0] + ENC_OFF + (call & 1) * ENC_SZ);
  char* out = uni(p.out);
  gu64* row = flag_row(p, call, lane);
  const u32 vC = (lane & 15) * (NS * 2) + ((lane & 16) << 1) + ((lane & 32) >> 1);
  u32 have = 0, pend = 0, fseg = 0, flen = 0, fmask = 0;
#define PUBLISH() do { if (pend) { \
    asm volatile("s_waitcnt vmcnt(0)\n\ts_mov_b64 exec, 3\n\tglobal_store_dword %0, %1, %2 sc0 sc1\n\ts_mov_b64 exec, -1" :: "v"(pend + lane * (NCH * 4)), "v"(call), "s"(dec) : "memory"); \
    if ((pend & 255u) == 252u) STAMP(4); \
    if (flen) post_seg(row, call, fseg, flen, fmask, lane); pend = 0; flen = 0; } } while (0)
#define PASS_ON(seg, mask) do { flen = lds_peek(qlen + (k % QD)); fseg = (seg); fmask = (mask); \
    for (u32 o = 0; o < flen; o += 6144) { u32x4 t_[6]; \
      _Pragma("unroll") for (int i = 0; i < 6; ++i) t_[i] = *(const LAS u32x4*)(ls + o + i * 1024 + lane * 16); \
      _Pragma("unroll") for (int i = 0; i < 6; ++i) if (o + i * 1024 + lane * 16 < flen) st4x1(enc, fseg * SEG_STRIDE + o + i * 1024 + lane * 16, t_[i]); } } while (0)
#define BOTH _Pragma("unroll") for (int z = 0; z < 2; ++z)
  for (;;) {
    u32 q;
    asm volatile("s_mov_b64 exec, 1\n\tds_add_rtn_u32 %0, %2, %3 offset:12\n\tds_read_b32 %1, %2\n\ts_waitcnt lgkmcnt(0)\n\ts_mov_b64 exec, -1"
                 : "=&v"(q), "=&v"(have) : "v"((u32)(size_t)landed), "v"(1u) : "memory");
    q = rfl(q); have = rfl(have);
    const u32 k = q >> 1, hf = q & 1u, slot = k % NSLOT;
    if ((have & ~FIN) <= k) {
      PUBLISH();
      while (have <= k) { __builtin_amdgcn_s_sleep(1); have = lds_peek(landed); }
      if ((have & ~FIN) <= k) break;
    }
    const LAS char* ls = lds + slot * SEG_STRIDE;
    const u32 item_v = qitem[k % QD], dst_v = qdst[k % QD];
    const u32x2 hdr_v = *(const LAS u32x2*)(ls + SEG_H + hf * 8);
    u32x2 Pp[2]; u32x4 S[2][2];
    BOTH Pp[z] = *(const LAS u32x2*)(ls + SEG_P + hf * 1024 + z * 512 + lane * 8);
    BOTH { S[z][0] = *(const LAS u32x4*)(ls + hf * 4096 + z * 2048 + lane * 16); S[z][1] = *(const LAS u32x4*)(ls + hf * 4096 + z * 2048 + 1024 + lane * 16); }
    const u32 item = rfl(item_v), dst = rfl(dst_v);
    if (item & D_REL) {
      PUBLISH();
      if (hf == 0) {
        PASS_ON(24 * 128 + (item & 511u), 1u << (d ^ (((item >> 8) & 1u) + 1)));
        lds_inc(done + slot);
        const u32 place = mail_place(dec + (MBX_OFF - DEC_OFF), fmask, lane);
        WAIT_V(0);
        post_mail(row, call, place, fseg, flen, fmask, lane); flen = 0;
      } else lds_inc(done + slot);
      continue;
    }
    u32 hdr[2], vb[2], E1[2], n1[2], N1[2], o1[2], q1[2], q0[2], Q1[2], Q0[2], E2[2], n2[2], N2[2], o2[2], r2[2], r1[2], r0[2], E[2][8]; Mv ma[2], mb[2];
    BOTH { hdr[z] = rfl(hdr_v[z]); vb[z] = ((u32)(size_t)ls + SEG_V + ((hdr[z] >> 16) & 2047u) * 4) * 8; }
    BOTH { E1[z] = ~(Pp[z][0] | Pp[z][1]); ma[z] = move_masks(E1[z]); n1[z] = __builtin_popcount(E1[z]); }
    { const u32 i = wave_scan(n1[0] | (n1[1] << 16)), t = __builtin_amdgcn_readlane(i, 63);
      N1[0] = t & 0xffffu; N1[1] = t >> 16; o1[0] = vb[0] + 2 * ((i & 0xffffu) - n1[0]); o1[1] = vb[1] + 2 * ((i >> 16) - n1[1]); }
    BOTH { q1[z] = get_bits(o1[z]); q0[z] = q1[z] >> (n1[z] & 31u); }
    if (__any((n1[0] > n1[1] ? n1[0] : n1[1]) > 16u)) BOTH q0[z] = get_bits(o1[z] + n1[z]);
    BOTH { Q1[z] = pdep(q1[z], ma[z]); Q0[z] = pdep(q0[z], ma[z]); E2[z] = BOP(E1[z], Q1[z], Q0[z], A & ~(B | C)); mb[z] = move_masks(E2[z]); n2[z] = __builtin_popcount(E2[z]); }
    { const u32 i = wave_scan(n2[0] | (n2[1] << 16)), t = __builtin_amdgcn_readlane(i, 63);
      N2[0] = t & 0xffffu; N2[1] = t >> 16; o2[0] = mad3((i & 0xffffu) - n2[0], vb[0] + 2 * N1[0]); o2[1] = mad3((i >> 16) - n2[1], vb[1] + 2 * N1[1]); }
    u32 xe[2];
    BOTH { r2[z] = get_bits(o2[z]); r1[z] = r2[z] >> (n2[z] & 31u); r0[z] = r1[z] >> (n2[z] & 31u);
           xe[z] = *(const LAS u32*)(size_t)((vb[z] >> 3) + (((2 * N1[z] + 3 * N2[z] + 31) >> 5) + lane) * 4); }
    if (__any((n2[0] > n2[1] ? n2[0] : n2[1]) > 10u)) BOTH { r1[z] = get_bits(o2[z] + n2[z]); r0[z] = get_bits(o2[z] + 2 * n2[z]); }
    BOTH {
      const u32 R2 = pdep(r2[z], mb[z]) & E2[z], R1 = pdep(r1[z], mb[z]), R0 = pdep(r0[z], mb[z]);
      const u32 t7 = BOP(R2, R1, R0, A & B & C), q7 = BOP(E1[z], Q1[z], Q0[z], A & ~B & C);
      u32 dd[4] = {BOP(Pp[z][1] | q7, R0, E2[z], A | (B & C)), BOP(Pp[z][0], q7, BOP(R1, E2[z], t7, A & B & ~C), A | B | C), BOP(Q0[z], E1[z], R2, (A & B) | C), BOP(E1[z], Q1[z], t7, ~A | B | C)};
      transpose4(dd);
      const u32 b4 = (hdr[z] & 255u) * 0x01010101u;
#pragma unroll
      for (int i = 0; i < 4; ++i) { E[z][2 * i] = (dd[i] & 0x0f0f0f0fu) + b4; E[z][2 * i + 1] = ((dd[i] >> 4) & 0x0f0f0f0fu) + b4; }
    }
    asm volatile("" : "+v"(xe[0]), "+v"(xe[1]));
    BOTH {
      const u32 NX = (hdr[z] >> 8) & 255u;
      if (NX <= XCAP) {
        for (u32 i = 0; i < NX; ++i) {
          const u32 en = __builtin_amdgcn_readlane(xe[z], i);
          const u32 dv = (u32)lane == (en & 63u) ? ((en >> 16) & 255u) << (8 * ((en >> 9) & 3u)) : 0u;
          const int hot = (int)(0x80000000u >> ((en >> 6) & 7u));
#pragma unroll
          for (int i2 = 0; i2 < 8; ++i2) E[z][i2] = BOP(E[z][i2], dv, (u32)((hot << i2) >> 31), A ^ (B & C));
        }
      } else {
        E[z][0] = Pp[z][0]; E[z][1] = Pp[z][1];
#pragma unroll
        for (int i2 = 2; i2 < 8; ++i2) E[z][i2] = *(const LAS u32*)(size_t)((vb[z] >> 3) + ((i2 - 2) * 64 + lane) * 4);
      }
    }
    u32x4 O[2][4];
    BOTH {
#pragma unroll
      for (int i = 0; i < 8; ++i) {
        const u32 sb = S[z][i >> 2][i & 3];
        const u32 a = __builtin_amdgcn_perm(E[z][i], sb, 0x05010400u), b = __builtin_amdgcn_perm(E[z][i], sb, 0x07030602u);
        O[z][i >> 1][(i & 1) * 2] = __builtin_amdgcn_alignbit(a, a, 1); O[z][i >> 1][(i & 1) * 2 + 1] = __builtin_amdgcn_alignbit(b, b, 1);
      }
      asm volatile("" : "+v"(O[z][0]), "+v"(O[z][1]), "+v"(O[z][2]), "+v"(O[z][3]));
    }
    PUBLISH();
    if (item & D_C) {
      char* o = out + (dst + hf * 128 + vC);
#pragma unroll
      for (int m = 0; m < 4; ++m) BOTH st_out(o + m * 16 * NS * 2 + z * 64, O[z][m]);
    } else {
      BOTH st4x4(dec, dst + (hf * 2 + z) * 4096u + lane * 16, O[z][0], O[z][1], O[z][2], O[z][3]);
      pend = (u32)(F2_OFF - DEC_OFF) + ((((item >> 7) & 31u) * 8 + (item & 1u) * NB + hf * 2) * NCH + ((item >> 1) & 63u)) * 4;
      if ((item & D_FWD) && hf == 0) PASS_ON((12 + d) * 128 + (item & 127u), 1u << (d ^ 3));
    }
    lds_inc(done + slot);
    if (q < 2) STAMP(1);
  }
  STAMP(2);
}

__device__ __forceinline__ int wait_chunks(CParams& p, const u32* f, u32 o0, u32 o1, u32 o2, int need, u32 call) {
  const u64 t0 = __builtin_amdgcn_s_memrealtime();
  __builtin_amdgcn_s_waitcnt(0xc07f);
  for (u32 it = 1;; ++it) {
    const u32 a = __hip_atomic_load(f + o0, __ATOMIC_RELAXED, __HIP_MEMORY_SCOPE_SYSTEM);
    const u32 b = __hip_atomic_load(f + o1, __ATOMIC_RELAXED, __HIP_MEMORY_SCOPE_SYSTEM);
    const u32 c = __hip_atomic_load(f + o2, __ATOMIC_RELAXED, __HIP_MEMORY_SCOPE_SYSTEM);
    const u64 miss = __ballot(a < call || b < call || c < call);
    const int n = miss ? __builtin_ctzll(miss) : NCH;
    if (n >= need) return n;
    if ((it & 63u) == 0 && give_up(p, call, t0)) return NCH;
    __builtin_amdgcn_s_sleep(2);
  }
}
#define FL_ADDR(next) do { fbase = (next); const u32 c_ = fbase + (lane & 15); fvo = foff + 4 * (c_ < 63u ? c_ : 63u); } while (0)
#define FL_ISSUE() asm volatile("s_mov_b32 m0, %2\n\ts_nop 0\n\tglobal_load_lds_dword %0, %1 sc0 sc1" :: "v"(fvo), "s"(f2), "s"(lds0 + LDS_FLG + w * 256) : "memory")
#define FL_READ() fval = *(const volatile LAS u32*)(lds + LDS_FLG + w * 256 + lane * 4)
#define FL_USE() do { u64 m_ = __ballot(fval < call); m_ |= m_ >> 32; const u32 h_ = (u32)m_ | ((u32)m_ >> 16) | 0x10000u; \
    const int n_ = fbase + __builtin_ctz(h_); avail = n_ > avail ? n_ : avail; } while (0)
__device__ __forceinline__ u32 cvt_pk(float lo, float hi) { u32 r; asm("v_cvt_pk_bf16_f32 %0, %1, %2" : "=v"(r) : "v"(lo), "v"(hi)); return r; }
#define QNAN __builtin_nanf("")
typedef float f32x2 __attribute__((ext_vector_type(2)));
__device__ __forceinline__ u32 silu2(float a, float b) {
  const f32x2 t = (f32x2){__builtin_amdgcn_fmed3f(a, -60.0f, 60.0f), __builtin_amdgcn_fmed3f(b, -60.0f, 60.0f)} * -1.44269504088896341f;
  const f32x2 e = (f32x2){__builtin_amdgcn_exp2f(t.x), __builtin_amdgcn_exp2f(t.y)} + 1.0f;
  const f32x2 r = (f32x2){a, b} * (f32x2){__builtin_amdgcn_rcpf(e.x), __builtin_amdgcn_rcpf(e.y)};
  return cvt_pk(r.x, r.y);
}
__device__ __forceinline__ u32x4 finish8(const f32x4& u, const f32x4& v) {
  const auto s0 = __builtin_amdgcn_permlane16_swap(silu2(u[0], u[1]), silu2(v[0], v[1]), false, false);
  const auto s1 = __builtin_amdgcn_permlane16_swap(silu2(u[2], u[3]), silu2(v[2], v[3]), false, false);
  return (u32x4){s0[0], s1[0], s0[1], s1[1]};
}
__device__ __forceinline__ bool timed_out(CParams& p, u32 call) {
  return rfl(__hip_atomic_load((u32*)(p.ws[0] + TMO_OFF), __ATOMIC_RELAXED, __HIP_MEMORY_SCOPE_AGENT)) == call;
}
constexpr int HT = 16384;
#define A_SETUP(own) \
  const u32 sCa = (own) ? 256u : 32768u, sKa = (own) ? 128u : 2048u; \
  const u32 voffA0 = (own) ? (u32)(w * 16 + (lane >> 3)) * (KD * 2) + (((lane & 7) ^ (lane >> 4)) << 4) : (u32)(w * 4096 + lane * 16); \
  const u32 voffA1 = (own) ? (voffA0 ^ 64u) + 8 * KD * 2 - 1024 : voffA0;        \
  const u32 lds0 = (u32)(size_t)lds, ldsA = lds0 + w * 2048, ldsB = lds0 + w * 1024; \
  u32 aoff[2]; \
  aoff[0] = lds0 + wr * 8192 + ((own) ? (lane & 15) * 128 + (((lane >> 4) ^ ((lane >> 1) & 7)) << 4) : lane * 16); \
  aoff[1] = (own) ? aoff[0] ^ 64u : aoff[0] + 1024;
#define B_READ_OFF(n, t) (256 * (8 * (lane >> 4) + 4 * (t) + ((lane & 15) >> 2)) + 8 * (lane & 1) + \
    16 * ((4 * wc + 2 * (n) + ((lane & 3) >> 1)) ^ ((((lane & 15) >> 2) << 2) | ((2 * (lane >> 4) + (t)) & 3))))
#define VOFF_W ((u32)(4 * w + (lane >> 4)) * 2048u + (((lane & 15) ^ (((lane >> 4) << 2) | (w & 3))) << 4))
#define GB " sc0 sc1"
#define STA2(buf, gp, v0, v1) asm volatile("s_mov_b32 m0, %3\n\ts_nop 0\n\tglobal_load_lds_dwordx4 %0, %2" GB "\n\tglobal_load_lds_dwordx4 %1, %2 offset:1024" GB \
    :: "v"(v0), "v"(v1), "s"(gp), "s"(ldsA + (buf)) : "memory")
#define STA(buf, gp) STA2(buf, gp, voffA0, voffA1)
#define STB(buf, gp) asm volatile("s_mov_b32 m0, %3\n\ts_nop 0\n\tglobal_load_lds_dwordx4 %0, %1" GB "\n\ts_add_u32 m0, m0, 0x2000\n\ts_nop 0\n\tglobal_load_lds_dwordx4 %0, %2" GB \
    :: "v"(voffB), "s"(gp), "s"((gp) + sIb), "s"(ldsB + (buf)) : "memory", "scc")
#define LDA(dst, ad, imm) do { _Pragma("unroll") for (int m = 0; m < MF; ++m) _Pragma("unroll") for (int k = 0; k < 2; ++k) \
    dst[m][k] = *(const LAS s16x8*)(size_t)(ad[k] + (imm) + m * 2048); } while (0)
#define LDB(dst, ad, imm) do { _Pragma("unroll") for (int n = 0; n < 2; ++n) _Pragma("unroll") for (int k = 0; k < 2; ++k) { \
    const s16x4 lo_ = __builtin_amdgcn_ds_read_tr16_b64_v4i16((LAS s16x4*)(size_t)(ad[n][0] + (imm) + k * 8192)); \
    const s16x4 hi_ = __builtin_amdgcn_ds_read_tr16_b64_v4i16((LAS s16x4*)(size_t)(ad[n][1] + (imm) + k * 8192)); \
    dst[n][k] = __builtin_shufflevector(lo_, hi_, 0, 1, 2, 3, 4, 5, 6, 7); } } while (0)
#define MMAX(ac, At, Bt, EXTRA) do { __builtin_amdgcn_s_setprio(1); EXTRA; _Pragma("unroll") for (int m = 0; m < MF; ++m) _Pragma("unroll") for (int n = 0; n < 2; ++n) \
    _Pragma("unroll") for (int k = 0; k < 2; ++k) ac[m][n] = __builtin_amdgcn_mfma_f32_16x16x32_bf16(__builtin_bit_cast(bf8, Bt[n][k]), __builtin_bit_cast(bf8, At[m][k]), ac[m][n], 0, 0, 0); \
    __builtin_amdgcn_s_setprio(0); } while (0)
#define MMA(ac, At, Bt) MMAX(ac, At, Bt, (void)0)
#define WEAVE do { _Pragma("unroll") for (int i_ = 0; i_ < 4 * MF - 1; ++i_) { __builtin_amdgcn_sched_group_barrier(0x8, 1, 0); __builtin_amdgcn_sched_group_barrier(0x6, 1, 0); } } while (0)

#define WEAVE2 do { _Pragma("unroll") for (int i_ = 0; i_ < 4 * MF - 1; ++i_) { __builtin_amdgcn_sched_group_barrier(0x8, 1, 0); __builtin_amdgcn_sched_group_barrier(0x6, 2, 0); } } while (0)
template <int MF, bool ALLOWN> __device__ __forceinline__ void gemm_wide(CParams& p, LAS char* lds, int F0, int tn, u32 call, int w, int lane) {
  const int wr = w >> 2, wc = w & 3;
  const bool own = ALLOWN || F0 < 64;
  const char* Ab = uni(own ? p.x : p.ws[0] + DEC_OFF);
  const char* Bb = uni(p.w + tn * 512);
  A_SETUP(own)
  const int f0 = F0 + (w < 2 * MF ? w : 0);
  u32 vh[2][2];
#pragma unroll
  for (int h = 0; h < 2; ++h) {
    const int f = f0 + 2 * MF * h;
    const u32 fo = own ? (u32)(f - w) * (16 * KD * 2) : (f >> 3) * (u32)PANEL_B + (u32)((f & 7) - w) * 4096u;
    vh[h][0] = w < 2 * MF ? fo + voffA0 : rfl(fo + voffA0); vh[h][1] = w < 2 * MF ? fo + voffA1 : vh[h][0];
  }
  aoff[0] -= wr * (4 - MF) * 2048; aoff[1] -= wr * (4 - MF) * 2048;
#define STAH(buf, gp, h) STA2(buf, gp, vh[h][0], vh[h][1])
  const u32 voffB = VOFF_W; constexpr int sIb = 65536;
  const u32* flg = (const u32*)(p.ws[0] + F2_OFF) + f0 * NCH + lane;
  const char* f2 = uni(p.ws[0] + F2_OFF);
  const u32 foff = (f0 + 2 * MF * (lane >> 5)) * NCH * 4;
  int fbase = 0, npoll = 0; u32 fval = 0, fvo = 0;
  u32 boff[2][2];
#pragma unroll
  for (int n = 0; n < 2; ++n)
#pragma unroll
    for (int t = 0; t < 2; ++t) { boff[n][t] = lds0 + 4 * HT + B_READ_OFF(n, t); asm volatile("" : "+v"(boff[n][t])); }
#define SA(b, h) (((b) * 2 + (h)) * HT)
#define SB(b, h) ((4 + (b) * 2 + (h)) * HT)
#define WAIT_LA asm volatile("s_waitcnt lgkmcnt(%0)" :: "i"(2 * MF) : "memory")
  f32x4 acc[2][2][MF][2];
#pragma unroll
  for (int a = 0; a < 2; ++a)
#pragma unroll
    for (int b = 0; b < 2; ++b)
#pragma unroll
      for (int m = 0; m < MF; ++m)
#pragma unroll
        for (int n = 0; n < 2; ++n) acc[a][b][m][n] = (f32x4){0.f, 0.f, 0.f, 0.f};
  s16x8 At[MF][2], B0[2][2], B1[2][2];
  int avail = own ? NCH : wait_chunks(p, flg, 0, 2 * MF * NCH, 0, 1, call);
  STAMP(3); STAMPC(8);
  __builtin_amdgcn_s_waitcnt(0x0f70);
  if (!ALLOWN) { FL_ADDR(1); FL_ISSUE(); }
  STB(SB(0, 0), Bb); STAH(SA(0, 0), Ab, 0); STB(SB(0, 1), Bb + 256); STAH(SA(0, 1), Ab, 1);
  if (wr == 1) BAR;
  WAIT_V(4); BAR;
  STB(SB(1, 0), Bb + 131072); STAH(SA(1, 0), Ab + sKa, 0); STB(SB(1, 1), Bb + 131072 + 256);
  WAIT_V(6); BAR;
  const u32 voffB2 = voffB + 65536;
  const char* g = Ab + sKa;
#define STBW(buf) asm volatile("s_mov_b32 m0, %3\n\ts_nop 0\n\tglobal_load_lds_dwordx4 %0, %2\n\ts_add_u32 m0, m0, 0x2000\n\ts_nop 0\n\tglobal_load_lds_dwordx4 %1, %2" \
    :: "v"(voffB), "v"(voffB2), "s"(g), "s"(ldsB + (buf)) : "memory", "scc")
#define NEXT(ac, Bt, expr) do { SCHED; MMAX(ac, At, Bt, g = (expr); WEAVE); asm volatile("" : "+s"(g)); BAR; SCHED; } while (0)
#pragma unroll 1
  for (int c = 0; c < NCH; ++c) {
    const int cn = c + 1 < NCH ? c + 1 : c;
    LDB(B0, boff, SA(0, 0)); SCHED; LDA(At, aoff, SA(0, 0)); STAH(SA(1, 1), g, 1);
    WAIT_LA; BAR; WAIT_L(0); NEXT(acc[0][0], B0, Bb + (size_t)cn * 262144);
    const char* b2 = g;
    LDB(B1, boff, SA(0, 1)); if (!ALLOWN) FL_READ(); STBW(SB(0, 0));
    BAR; WAIT_L(0);
    if (ALLOWN) NEXT(acc[0][1], B1, Ab + (size_t)(cn * sCa));
    else {
      SCHED; MMAX(acc[0][1], At, B1, g = Ab + (size_t)(cn * sCa); FL_USE(); FL_ADDR(cn + 1); WEAVE2); asm volatile("" : "+s"(avail), "+s"(g)); asm volatile("" : "+v"(fvo)); BAR; SCHED;
      if (avail <= cn) { avail = wait_chunks(p, flg, 0, 2 * MF * NCH, 0, cn + 1, call); ++npoll; }
    }
    const char* a2 = g;
    LDA(At, aoff, SA(0, 1)); STAH(SA(0, 0), g, 0);
    BAR; WAIT_L(0); NEXT(acc[1][0], B0, b2 + 256);
    STBW(SB(0, 1));
    if (ALLOWN) WAIT_V(6); else { FL_ISSUE(); WAIT_V(7); }
    BAR; MMA(acc[1][1], At, B1); BAR;
    LDB(B0, boff, SA(1, 0)); SCHED; LDA(At, aoff, SA(1, 0)); STAH(SA(0, 1), a2, 1);
    WAIT_LA; BAR; WAIT_L(0); NEXT(acc[0][0], B0, b2 + 131072);
    LDB(B1, boff, SA(1, 1)); STBW(SB(1, 0));
    BAR; WAIT_L(0); NEXT(acc[0][1], B1, a2 + sKa);
    const char* a3 = g;
    LDA(At, aoff, SA(1, 1)); STAH(SA(1, 0), g, 0);
    BAR; WAIT_L(0); NEXT(acc[1][0], B0, b2 + 131072 + 256);
    STBW(SB(1, 1));
    WAIT_V(6); BAR; MMA(acc[1][1], At, B1); BAR;
    g = a3;
  }
  if (wr == 0) BAR;
  STAMP(4); STAMPC(9); STAMPV(10, npoll);
  if (!own && timed_out(p, call)) {
#pragma unroll
    for (int i = 0; i < 8 * MF; ++i) acc[i & 1][(i >> 1) & 1][i >> 3][(i >> 2) & 1] = (f32x4){QNAN, QNAN, QNAN, QNAN};
  }
  int tz = threadIdx.x; asm volatile("" : "+v"(tz));
  char* ob = p.out + ((size_t)(tz & 15) * NS + tn * 256 + wc * 32 + (tz & 16) + ((tz & 32) >> 2)) * 2;
#pragma unroll
  for (int ai = 0; ai < 2; ++ai)
#pragma unroll
    for (int m = 0; m < MF; ++m) {
      const int f = F0 + 2 * MF * ai + MF * wr + m;
      char* oa = ob + (size_t)(slot_row(p.rank, f >> 3) + (f & 7) * 16) * (NS * 2);
#pragma unroll
      for (int bj = 0; bj < 2; ++bj) st_out(oa + bj * 256, finish8(acc[ai][bj][m][0], acc[ai][bj][m][1]));
    }
  STAMP(5);
}

__device__ __forceinline__ void gemm_small(CParams& p, LAS char* lds, int s, int cp, u32 call, int w, int lane) {
  const int wr = w >> 2, wc = w & 3;
  constexpr int MF = 4;
  const bool own = s < 8;
  const char* Ab = uni(own ? p.x + (size_t)s * PANEL_B : p.ws[0] + DEC_OFF + (size_t)s * PANEL_B);
  const char* Bb = uni(p.ws[0] + DEC_OFF + (size_t)cp * 2 * PANEL_B);
  A_SETUP(own)
  const u32 voffB = w * 4096 + lane * 16;
  const u32* flg = (const u32*)(p.ws[0] + F2_OFF) + w * NCH + lane;
  const u32 fb0 = cp * 2 * 8 * NCH, fb1 = fb0 + 8 * NCH, fa = own ? fb0 : s * 8 * NCH;
  const char* f2 = uni(p.ws[0] + F2_OFF);
  const u32 foff = ((lane & 32 ? fa : (lane & 16 ? fb1 : fb0)) + w * NCH) * 4;
  int fbase = 0, npoll = 0; u32 fval = 0, fvo = 0;
  constexpr u32 TB = 3 * HT;
  const u32 boff = lds0 + wc * 4096 + lane * 16;
  u32 bc = boff, bn = boff + TB, ac[2] = {aoff[0], aoff[1]};
  s16x8 At[4][2], B0[2][2], B1[2][2];
  Enc e; e.init(p, lds, call, w, lane);
  asm volatile("" : "+v"(e.row));
  constexpr int trip = 1;
  {
  f32x4 acc[2][4][2];
#pragma unroll
  for (int b = 0; b < 2; ++b)
#pragma unroll
    for (int m = 0; m < 4; ++m)
#pragma unroll
      for (int n = 0; n < 2; ++n) acc[b][m][n] = (f32x4){0.f, 0.f, 0.f, 0.f};
  if (trip) {
  int avail = wait_chunks(p, flg, fa, fb0, fb1, 2, call);
  STAMP(3); STAMPC(8);
  __builtin_amdgcn_s_waitcnt(0x0f70);
#define LD2(m0v, v0, v1, gp) asm volatile("s_mov_b32 m0, %3\n\ts_nop 0\n\tglobal_load_lds_dwordx4 %0, %2" GB "\n\tglobal_load_lds_dwordx4 %1, %2 offset:1024" GB \
    :: "v"(v0), "v"(v1), "s"(gp), "s"(m0v) : "memory")
#define LD1(m0v, v0, gp, o) asm volatile("s_mov_b32 m0, %2\n\ts_nop 0\n\tglobal_load_lds_dwordx4 %0, %1 offset:" #o GB :: "v"(v0), "s"(gp), "s"(m0v) : "memory")
#define LDF(dst, ad, imm) do { _Pragma("unroll") for (int n = 0; n < 2; ++n) _Pragma("unroll") for (int k = 0; k < 2; ++k) \
    dst[n][k] = *(const LAS s16x8*)(size_t)((ad) + (imm) + n * 2048 + k * 1024); } while (0)
  FL_ADDR(2);
  LD2(ldsA + HT, voffB, voffB, Bb); LD2(ldsA, voffA0, voffA1, Ab); LD2(ldsA + 2 * HT, voffB, voffB, Bb + PANEL_B); LD2(ldsA + TB + HT, voffB, voffB, Bb + 2048);
  LD2(ldsA + TB, voffA0, voffA1, Ab + sKa); FL_ISSUE(); LD2(ldsA + TB + 2 * HT, voffB, voffB, Bb + 2048 + PANEL_B); LD2(ldsA + 2 * TB + HT, voffB, voffB, Bb + 32768);
  WAIT_V(11); BAR;
  LDF(B0, bc, HT);
  WAIT_L(0);
  if (wr == 1) BAR;
  u32 s0 = 0, s1 = TB, s2 = 2 * TB;
  const char* gA = Ab + sCa; const char* gB1 = Bb + 32768 + PANEL_B; const char* gB0 = Bb + 32768 + 2048;
  u32 mA = ldsA + s2, mB1 = mA + 2 * HT, mB0 = ldsA + HT;
#define PIN do { asm volatile("" : "+s"(gA), "+s"(gB1), "+s"(gB0), "+s"(mA), "+s"(mB1), "+s"(mB0), "+s"(avail)); \
    asm volatile("" : "+v"(fvo), "+v"(bc), "+v"(bn), "+v"(ac[0]), "+v"(ac[1])); } while (0)
#define KTILE(dA, dB1, dB0, V1, V2, ISS, CHECK, RD, X2) do { \
    LD2(mA, voffA0, voffA1, gA); LD1(mB1, voffB, gB1, 0); ISS; LDA(At, ac, 0); \
    WAIT_V(V1); BAR; WAIT_L(0); SCHED; \
    MMAX(acc[0], At, B0, gA += (dA); mA = ldsA + s0; ac[0] = aoff[0] + s1; ac[1] = aoff[1] + s1; WEAVE); PIN; BAR; SCHED; \
    CHECK; \
    LD1(mB1, voffB, gB1, 1024); LD2(mB0, voffB, voffB, gB0); LDF(B1, bc, 2 * HT); SCHED; LDF(B0, bn, HT); \
    WAIT_V(V2); WAIT_L(4); RD; BAR; WAIT_L(0); SCHED; \
    MMAX(acc[1], At, B1, gB1 += (dB1); gB0 += (dB0); mB1 = mA + 2 * HT; mB0 = ldsA + s1 + HT; bc = bn; bn = boff + s2; \
         { const u32 s_ = s0; s0 = s1; s1 = s2; s2 = s_; } X2; WEAVE2); PIN; BAR; SCHED; } while (0)
  for (int c = 0; c < NCH; ++c) {
    const int c2 = c + 2 < NCH ? c + 2 : NCH - 1;
    const int nx = c + 2 < NCH;
    KTILE((int)sKa, 2048, (nx ? 32768 : 0) - 2048, 10, 9, (void)0, (void)0, FL_READ(), FL_USE(); FL_ADDR(c2 + 1));
    KTILE((int)(nx ? sCa : 0u) - (int)sKa, (nx ? 32768 : 0) - 2048, 2048, 10, 10, FL_ISSUE(),
          if (avail <= c2) { avail = wait_chunks(p, flg, fa, fb0, fb1, c2 + 1, call); ++npoll; LASTPOLL; }, (void)0, (void)0);
  }
  if (wr == 0) BAR;
  }
  STAMP(4); STAMPC(9); STAMPV(10, npoll);
  const u32 tmo = __hip_atomic_load((u32*)(p.ws[0] + TMO_OFF), __ATOMIC_RELAXED, __HIP_MEMORY_SCOPE_AGENT);
  int tz = threadIdx.x; asm volatile("" : "+v"(tz));
  u32x4 X[4];
#pragma unroll
  for (int m = 0; m < 4; ++m) { X[m] = finish8(acc[0][m][0], acc[0][m][1]); asm volatile("" : "+v"(X[m])); }
  if (rfl(tmo) == call) {
#pragma unroll
    for (int m = 0; m < 4; ++m) { X[m] = (u32x4){0x7fc07fc0u, 0x7fc07fc0u, 0x7fc07fc0u, 0x7fc07fc0u}; acc[1][m][0] = acc[1][m][1] = (f32x4){QNAN, QNAN, QNAN, QNAN}; }
  }
  {
    STAMP(11);
    WAIT_V(0);
    e.lane = tz & 63;
    const u32 code = (s * 2 + (cp & 1)) * 4 + wr, seg = 20 * 128 + (cp >> 1) * 256 + code;
#pragma unroll 1
    for (int bj = 0; bj <= trip; ++bj) {
      e.round<false>(X, seg + 2 * bj, trip ? 1u << piece_link(cp >> 1, code + 2 * bj) : 0u, [] {});
      e.draw(f2 + (MBX_OFF - F2_OFF));
      if (bj == 0) {
#pragma unroll
        for (int m = 0; m < 2; ++m) { asm volatile("" : "+v"(acc[1][m][0]), "+v"(acc[1][m][1])); X[m] = finish8(acc[1][m][0], acc[1][m][1]); asm volatile("" : "+v"(X[m])); }
      }
      e.finish_mail();
      if (bj == 0) {
        STAMP(12);
#pragma unroll
        for (int m = 2; m < 4; ++m) { asm volatile("" : "+v"(acc[1][m][0]), "+v"(acc[1][m][1])); X[m] = finish8(acc[1][m][0], acc[1][m][1]); }
      }
    }
  }
  }
  STAMP(5);
}

__global__ __launch_bounds__(512, 2) void ag_gemm(Params) {
  CParams& p = *(CParams*)__builtin_amdgcn_kernarg_segment_ptr();
  extern __shared__ __attribute__((aligned(16))) char smem[];
  LAS char* lds = (LAS char*)smem + 1024;
  const int lane = threadIdx.x & 63, w = rfl(threadIdx.x >> 6), sg = w / NB;
  const int b = blockIdx.x;
  STAMP(0);
  const Role ro = role_of(b);
  Jobs jb; jb.init(ro.u, sg);
  const int nr = jb.nr, j0 = jb.job(0), j1 = jb.job(1);
  load_job(p, lds, j0, 0, w, lane);
  load_job(p, lds, j1, 1, w, lane);
  u32* cnt = (u32*)(p.ws[0] + CNT_OFF + b * 128);
  u32 call = *(const __attribute__((address_space(4))) u32*)cnt + 1;
  asm volatile("" : "+s"(call));
  if (threadIdx.x < 64) ((LAS u32*)(lds + LDS_MAIN))[lane] = 0;
  if (threadIdx.x >= 256) ((LAS u32*)smem)[threadIdx.x - 256] = mask_entry(threadIdx.x - 256);
  LBAR;
  if (threadIdx.x == 511) *cnt = call;
  if (nr) {
    if (j1 < NJOB) WAIT_V(4); else WAIT_V(0);
    encode_jobs(p, lds, call, w, lane, jb);
  }
  if (ro.kind == 0) gemm_small(p, lds, ro.a, ro.c, call, w, lane);
  else if (ro.kind == 1) gemm_wide<3, false>(p, lds, ro.a, ro.c, call, w, lane);
  else if (ro.kind == 2) gemm_wide<2, true>(p, lds, ro.a, ro.c, call, w, lane);
  else if (ro.kind == 3) {
    pull_role(p, lds, link_of(ro.a), ro.a / 7, (NPULL - ro.a % 7 + 6) / 7, call, w, lane);
  }
}
}

extern "C" void kernel_launch(int rank, int nranks, ncclComm_t comm, hipStream_t stream,
                              void* const* d_in, const int* in_sizes, int n_in,
                              void* d_out, int out_size, const int* in_dtypes, int out_dtype,
                              void* const* d_out_all, void* const* d_ws_all, size_t ws_bytes) {
  using namespace agg;
  if (nranks != NR || (unsigned)rank >= (unsigned)NR || ws_bytes < WS_NEED || n_in < 2) { fprintf(stderr, "ag_gemm: unexpected configuration\n"); return; }
  Params p;
  for (int i = 0; i < NR; ++i) p.ws[i] = (char*)d_ws_all[rank ^ i];
  p.x = (const char*)d_in[0]; p.w = (const char*)d_in[1]; p.out = (char*)d_out; p.rank = rank; p.pad = 0;
  hipError_t e = hipFuncSetAttribute((const void*)ag_gemm, hipFuncAttributeMaxDynamicSharedMemorySize, LDS_BYTES);
  if (e != hipSuccess) { fprintf(stderr, "HIP error %s\n", hipGetErrorString(e)); return; }
  hipLaunchKernelGGL(ag_gemm, dim3(GRID), dim3(512), LDS_BYTES, stream, p);
  e = hipGetLastError();
  if (e != hipSuccess) fprintf(stderr, "HIP error %s\n", hipGetErrorString(e));
}
```

```cpp
#include <hip/hip_runtime.h>
#include <rccl/rccl.h>
#include <cstdio>

#define INFL 11
#define OF1 5
#define OF2 9
#define OF3 9
#define OF4 8
#define RAMP 44
#define OS1 4
#define OS4 4
#define FDIR 0xffff000000000000ull
#define P3 19
#define O3 1024
#define LEAD 6
#define REL0 0x55u
#define REL1 0x55u
#define REL2 0x55u
#define REL3 0x55u
#define RG 20u
#define WARMUP 0
#define INFL5 13
#define DEEP 0
#define PUBAT 2
#define LAP 3u
#define WREF 24u
#define NJ6 1
#define NJ5 2
#define MIDR 2
#define UPR 1
#define DRAINIT 2
#define LAM 3u
#define REPOLL 0

namespace agg {
#define LAS __attribute__((address_space(3)))
typedef unsigned u32;
typedef unsigned long long u64;
typedef __attribute__((address_space(1))) unsigned long long gu64;
typedef unsigned u32x4 __attribute__((ext_vector_type(4)));
typedef unsigned u32x2 __attribute__((ext_vector_type(2)));
typedef short s16x4 __attribute__((ext_vector_type(4)));
typedef short s16x8 __attribute__((ext_vector_type(8)));
typedef unsigned short u16x2 __attribute__((ext_vector_type(2)));
typedef float f32x4 __attribute__((ext_vector_type(4)));
typedef __bf16 bf8 __attribute__((ext_vector_type(8)));

constexpr int NR = 8, KD = 8192, NS = 1024;
constexpr int NCH = 64;
constexpr int NB = 4;
constexpr int SEG_STRIDE = NB * 4096 + 1024;
constexpr int SEG_P = NB * 2048, SEG_H = NB * 2560, SEG_V = SEG_H + NB * 4;
#define NSLOT 8
constexpr int QD = 16;
constexpr int GRID = 256;
constexpr int LDS_MAIN = 147456, LDS_FLG = LDS_MAIN + 1024, LDS_VAR = LDS_FLG + 2048, LDS_BYTES = 1024 + LDS_VAR + 8 * 1536;
static_assert(NSLOT * SEG_STRIDE <= LDS_MAIN, "ring");
constexpr u32 XCAP = 64;
constexpr int NSRC = 28 * 128, NJOB = 16 * 128;

constexpr size_t PANEL_B = (size_t)2 << 20;
constexpr size_t DEC_OFF = 0;
constexpr size_t ENC_OFF = 32 * PANEL_B;
constexpr size_t ENC_SZ = (size_t)NSRC * SEG_STRIDE;
constexpr size_t F1_OFF = ENC_OFF + 2 * ENC_SZ;
constexpr size_t F1_SZ = (size_t)NR * NSRC * 8;
constexpr size_t F2_OFF = F1_OFF + 2 * F1_SZ;
constexpr size_t F2_SZ = (size_t)32 * 8 * NCH * 4;
constexpr size_t CNT_OFF = F2_OFF + F2_SZ;
constexpr size_t TKT_OFF = CNT_OFF + GRID * 128;
constexpr size_t MBX_OFF = TKT_OFF + NR * 128;
constexpr size_t TMO_OFF = MBX_OFF + NR * 128;
constexpr size_t DBG_OFF = TMO_OFF + 128;
constexpr size_t WS_NEED = DBG_OFF + (1 << 20);
constexpr u64 DEADLINE = 50000000ull;

struct Params { char* ws[NR];   const char* x; const char* w; char* out; int rank; int pad; };
typedef const __attribute__((address_space(4))) Params CParams;

__host__ __device__ __forceinline__ int cls(int r) { return (r ^ (r >> 1)) & 1; }
__host__ __device__ __forceinline__ int slot_row(int r, int s) {
  const int h4 = 4 * cls(r), m = r & 1;
  int owner, piece;
  if (s < 8) { owner = r; piece = s; }
  else if (s < 12) { owner = r ^ 3; piece = 2 + (s & 1) + ((s & 2) << 1); }
  else {
    const int y = r ^ (4 + (s & 3)), a = (s - 12) >> 2;
    if (a == 4) { owner = r ^ 3; piece = 4 * cls(y) + (y & 1); } else { owner = y; piece = h4 + (a == 0 ? m : (a == 3 ? m ^ 1 : a + 1)); }
  }
  return owner * 1024 + piece * 128;
}
struct Role { int kind, a, c, u; };
#define NPULL 40
#define LATE0 12
#define JM 3
#define LATE1 16
constexpr int NENC = 40 - NPULL, NPA = NENC + 8 + (32 - LATE1) * 4, NPB = NENC + 88 + (32 - LATE0) * 4, NPC = NENC + 216;
__host__ __device__ __forceinline__ Role role_of(int b) {
  if (JM >= 0) { if ((b & 1) && (b >> 5) == JM) b = (b & 30) | (7 << 5); else if (!(b & 1) && (b >> 5) == 7) b = (b & 31) | 1 | (JM << 5); }
  const int xi = (b & 7) >> 1, j = b >> 3, jj = j >> 2;
  if (b & 1) { const int s = jj < 2 ? 2 * xi + jj : 4 * jj + xi; return {0, s, j & 3, NENC + (s >= LATE1 ? 8 : 88) + (31 - s) * 4 + (j & 3)}; }
  if (j < 16) return {1, 64 + 12 * (4 * xi + jj), j & 3, NPA + 16 + j * 4 + xi};
  if (j < 20) return {1, 12 * xi, j & 3, NPA + (j - 16) * 4 + xi};
  if (j < 22) { const int i = 2 * xi + j - 20; return {2, 48 + 8 * (i >> 2), i & 3, NENC + i}; }
  const int o = (j - 22) * 4 + xi;
  return {o < NPULL ? 3 : 4, o, 0, o - NPULL};
}
__host__ __device__ __forceinline__ int link_of(int o) { return 1 + (o + 2) % 7; }
constexpr int LMAX = 768;
struct Lists { unsigned short e[3][LMAX]; unsigned short n[3]; unsigned short mark[3][4]; };
constexpr int key(int c, int i, const int* pace, const int* off, const int* cap) {
  const int k = c * pace[i] + off[i];
  return cap && k > c * pace[0] + cap[i] ? c * pace[0] + cap[i] : k;
}
constexpr void merge(Lists& L, int t, int ns, const u64* has, const int* pace, const int* off, const int* cap = nullptr) {
  int c[8] = {};
  for (;;) {
    int b = -1;
    for (int i = 0; i < ns; ++i) {
      while (c[i] < 64 && !((has[i] >> c[i]) & 1)) ++c[i];
      if (c[i] < 64 && (b < 0 || key(c[i], i, pace, off, cap) < key(c[b], b, pace, off, cap))) b = i;
    }
    if (b < 0) break;
    for (int g = 0; g < 2; ++g) L.e[t][L.n[t]++] = (unsigned short)(b << 7 | c[b] << 1 | g);
    if (++c[b] == 64 && b == 0) L.mark[t][0] = L.n[t] - 1;
  }
  L.mark[t][1] = L.n[t] - 1;
}
__host__ __device__ constexpr u32 piece_link(u32 o, u32 code) {
  const u32 s = code >> 3, set = s < 12 ? REL0 : (s < RG ? REL1 : (s < 28 ? REL2 : REL3));
  return (set >> (code & 7u)) & 1u ? 4u + (((code & 1u) + ((code >> 1) & 1u) + ((code >> 1) & 2u) + s + (s >> 2)) & 3u) : o + 1;
}
constexpr u32 mail_count(u32 d) {
  u32 n = 0;
  for (u32 pc = 0; pc < 512; ++pc) { const u32 o = pc >> 8, l = piece_link(o, pc & 255u); n += (l == d) + (l > 3 && (l ^ (o + 1)) == d); }
  return n;
}
constexpr u32 NMAIL0 = mail_count(1), NMAIL2 = mail_count(4);
static_assert(NMAIL0 <= 64 * (NPULL / 7) && NMAIL2 <= 64 * (NPULL / 7), "mailbox");
static_assert(mail_count(2) == NMAIL0 && mail_count(5) == NMAIL2 && mail_count(6) == NMAIL2 && mail_count(7) == NMAIL2, "routes");
constexpr Lists make_lists() {
  Lists L = {};
  constexpr u64 ALL = ~0ull;
  { const u64 has[4] = {ALL, ALL, ALL, ALL}; const int pace[4] = {1, 1, 1, 1}, off[4] = {0, 0, LEAD, LEAD}; merge(L, 0, 4, has, pace, off); }
  { const u64 has[8] = {ALL, ALL, ALL, ALL, FDIR, FDIR, FDIR, FDIR}; const int pace[8] = {16, 16, 16, 16, P3, P3, P3, P3}, off[8] = {0, 0, 0, 0, O3, O3, O3, O3};
    merge(L, 1, 8, has, pace, off); }
  { const u64 has[5] = {ALL, ALL, ALL, ALL, ~FDIR}; const int pace[5] = {RAMP, RAMP + OF1 - OS1, RAMP + OF2 - OS1, RAMP + OF3 - OS1, RAMP + OF4 - OS4}, off[5] = {0, OS1 * RAMP, OS1 * RAMP, OS1 * RAMP, OS4 * RAMP},
      cap[5] = {0, OF1 * RAMP, OF2 * RAMP, OF3 * RAMP, OF4 * RAMP};
    merge(L, 2, 5, has, pace, off, cap); }
  L.mark[0][2] = L.n[0] + NMAIL0 / 2; L.mark[0][3] = L.n[0] + NMAIL0 * 3 / 4; L.mark[2][2] = L.n[2] + NMAIL2 / 2; L.mark[2][3] = L.n[2] + NMAIL2 * 3 / 4;
  return L;
}
constexpr Lists LISTS = make_lists();
static_assert(OF4 > 0, "lists");
__host__ __device__ __forceinline__ int list_type(int d) { return d < 3 ? 0 : (d == 3 ? 1 : 2); }
constexpr u32 D_FWD = 1u << 12, D_C = 1u << 13, D_REL = 1u << 14;
__host__ __device__ __forceinline__ u32 resolve(int d, int r, u32 en) {
  const u32 a = (en >> 7) & 7u, h4 = 4 * cls(r), m = r & 1; u32 sp, dp, fl = 0;
  if (d < 3) { dp = 4u * (d - 1) + a; sp = 8 + dp; }
  else if (d == 3) { if (a < 4) { sp = 2 + (a & 1) + ((a & 2) << 1); dp = 8 + a; } else { const int y = r ^ a; sp = 4 * cls(y) + (y & 1); dp = 24 + a; } }
  else if (a < 3) { sp = h4 + (a == 0 ? m : a + 1); dp = 8 + 4 * a + d; fl = a == 0 && !((FDIR >> ((en >> 1) & 63u)) & 1) ? D_FWD : 0u; }
  else { sp = a == 4 ? 12u + (d ^ 3) : h4 + (m ^ 1); dp = 8 + 4 * a + d; }
  return ((sp * 128u + (en & 127u)) << 16) | fl | (dp << 7) | (en & 127u);
}
__host__ __device__ __forceinline__ u32 input_mask(int r, u32 pn, u32 c) {
  return pn < 8u ? (((pn >> 2) == (u32)cls(r) ? 0x90u : 0x60u) | ((pn & 2u) || ((FDIR >> c) & 1) ? 8u : 0u)) : (pn < 12u ? 2u : 4u);
}

#define STAMPV(i, v) do { (void)(v); } while (0)
#define LASTPOLL do {} while (0)
#define CLKS(c) do {} while (0)
#define STAMP(i) do {} while (0)
#define STAMPC(i) do {} while (0)

#define WAIT_V(n) asm volatile("s_waitcnt vmcnt(" #n ")" ::: "memory")
#define WAIT_L(n) asm volatile("s_waitcnt lgkmcnt(" #n ")" ::: "memory")
#define DRAIN do { WAIT_V(0); __builtin_amdgcn_s_waitcnt(0x0f70); } while (0)
#define BAR __builtin_amdgcn_s_barrier()
#define SCHED __builtin_amdgcn_sched_barrier(0)
#define LBAR do { WAIT_L(0); BAR; asm volatile("" ::: "memory"); } while (0)
template <int N> __device__ __forceinline__ void wait_v() { asm volatile("s_waitcnt vmcnt(%0)" :: "i"(N) : "memory"); }

__device__ __forceinline__ u32 rfl(u32 v) { return __builtin_amdgcn_readfirstlane(v); }
template <class T> __device__ __forceinline__ T* uni(T* q) {
  u64 v = (u64)q; v = ((u64)rfl((u32)(v >> 32)) << 32) | rfl((u32)v);
  asm volatile("s_nop 4" : "+s"(v));
  return (T*)(__attribute__((address_space(1))) T*)v;
}
__device__ __forceinline__ void st4x4(char* base, u32 off, u32x4 a, u32x4 b, u32x4 c, u32x4 d) {
  asm volatile("global_store_dwordx4 %0, %1, %5 sc0 sc1\n\tglobal_store_dwordx4 %0, %2, %5 offset:1024 sc0 sc1\n\t"
               "global_store_dwordx4 %0, %3, %5 offset:2048 sc0 sc1\n\tglobal_store_dwordx4 %0, %4, %5 offset:3072 sc0 sc1\n\ts_nop 1"
               :: "v"(off), "v"(a), "v"(b), "v"(c), "v"(d), "s"(base) : "memory");
}
__device__ __forceinline__ void st4x2(char* base, u32 off, u32x4 a, u32x4 b) {
  asm volatile("global_store_dwordx4 %0, %1, %3 sc0 sc1\n\tglobal_store_dwordx4 %0, %2, %3 offset:1024 sc0 sc1\n\ts_nop 1"
               :: "v"(off), "v"(a), "v"(b), "s"(base) : "memory");
}
__device__ __forceinline__ void st4x1(char* base, u32 off, u32x4 a) {
  asm volatile("global_store_dwordx4 %0, %1, %2 sc0 sc1\n\ts_nop 1" :: "v"(off), "v"(a), "s"(base) : "memory");
}
#define OB " sc1"
__device__ __forceinline__ void st_out(char* q, u32x4 a) { asm volatile("global_store_dwordx4 %0, %1, off" OB "\n\ts_nop 1" :: "v"(q), "v"(a) : "memory"); }
__device__ __forceinline__ bool give_up(CParams& p, u32 call, u64 t0) {
  u32* tmo = (u32*)(p.ws[0] + TMO_OFF);
  if (rfl(__hip_atomic_load(tmo, __ATOMIC_RELAXED, __HIP_MEMORY_SCOPE_AGENT)) == call) return true;
  if (__builtin_amdgcn_s_memrealtime() - t0 > DEADLINE) {
    __hip_atomic_store(tmo, call, __ATOMIC_RELAXED, __HIP_MEMORY_SCOPE_AGENT);
    WAIT_V(0);
    return true;
  }
  return false;
}
__device__ __forceinline__ gu64* flag_row(CParams& p, u32 call, int lane) {
  u32 lo = 0, hi = 0;
#pragma unroll
  for (int i = 0; i < NR; ++i) { u64 v = (u64)p.ws[i]; asm volatile("" : "+s"(v)); lo = lane == i ? (u32)v : lo; hi = lane == i ? (u32)(v >> 32) : hi; }
  return (gu64*)((((u64)hi << 32) | lo) + F1_OFF + (call & 1) * F1_SZ) + (lane & 7) * NSRC;
}
__device__ __forceinline__ void post_seg(gu64* row, u32 call, u32 seg, u32 len, u32 mask, int lane) {
  if (lane < NR && ((mask >> lane) & 1u)) __hip_atomic_store(row + seg, ((u64)call << 32) | len, __ATOMIC_RELAXED, __HIP_MEMORY_SCOPE_SYSTEM);
}
constexpr u32 MAIL0 = 20 * 128;
__device__ __forceinline__ u32 mail_place(const char* mbx, u32 mask, int lane) {
  u32 t;
  asm volatile("s_mov_b64 exec, %4\n\tglobal_atomic_add %0, %1, %2, %3 sc0\n\ts_mov_b64 exec, -1" : "=&v"(t) : "v"(lane * 128), "v"(1u), "s"(mbx), "s"((u64)(mask & 0xffu)) : "memory");
  return t;
}
__device__ __forceinline__ void post_mail(gu64* row, u32 call, u32 place, u32 seg, u32 len, u32 mask, int lane) {
  asm volatile("" : "+v"(place));
  post_seg(row, call, MAIL0 + (lane < 3 ? place % NMAIL0 : place % (NMAIL2 ? NMAIL2 : 1)), (seg << 16) | (len >> 4), mask, lane);
}
__device__ __forceinline__ u32 wave_scan(u32 v) {
  v += __builtin_amdgcn_update_dpp(0u, v, 0x111, 0xf, 0xf, false);
  v += __builtin_amdgcn_update_dpp(0u, v, 0x112, 0xf, 0xf, false);
  v += __builtin_amdgcn_update_dpp(0u, v, 0x114, 0xf, 0xf, false);
  v += __builtin_amdgcn_update_dpp(0u, v, 0x118, 0xf, 0xf, false);
  v += __builtin_amdgcn_update_dpp(0u, v, 0x142, 0xa, 0xf, false);
  v += __builtin_amdgcn_update_dpp(0u, v, 0x143, 0xc, 0xf, false);
  return v;
}
#define BOP(a, b, c, expr) (u32)__builtin_amdgcn_bitop3_b32((a), (b), (c), [] { constexpr u32 A = 0xf0, B = 0xcc, C = 0xaa; (void)A; (void)B; (void)C; return (expr) & 0xffu; }())
#define BFI(m, a, b) BOP(m, a, b, (A & B) | (~A & C))
struct Mv { u32 a0, a1, a2, c8, c16; };
__device__ __forceinline__ u32 byte_prefix_xor(u32 v) {
  v = BOP(v, v << 1, 0xfefefefeu, A ^ (B & C)); v = BOP(v, v << 2, 0xfcfcfcfcu, A ^ (B & C)); return BOP(v, v << 4, 0xf0f0f0f0u, A ^ (B & C));
}
__device__ __forceinline__ Mv calc_masks(u32 m) {
  Mv r;
  r.c16 = __builtin_popcount(m & 0xffffu);
  r.c8 = (u32)__builtin_popcount(m & 0xffu) | ((u32)__builtin_popcount(m & 0xff0000u) << 16);
  u32 mk = BOP(m << 1, 0xfefefefeu, 0u, ~A & B);
  u32 mp = byte_prefix_xor(mk);
  r.a0 = mp & m; m = BOP(m, r.a0, r.a0 >> 1, (A ^ B) | C); mk = BOP(mk, mp, 0u, A & ~B);
  mp = byte_prefix_xor(mk);
  r.a1 = mp & m; m = BOP(m, r.a1, r.a1 >> 2, (A ^ B) | C); mk = BOP(mk, mp, 0u, A & ~B);
  r.a2 = BOP(0x80808080u - mk, 0x80808080u, m, (A ^ B) & C);
  return r;
}
__device__ __forceinline__ u32 mask_entry(u32 b) { const Mv v = calc_masks(b); return v.c16 | (v.a0 << 8) | (v.a1 << 16) | (v.a2 << 24); }
__device__ __forceinline__ Mv move_masks(u32 m) {
  const u32 e0 = *(const LAS u32*)(size_t)((m & 255u) << 2), e1 = *(const LAS u32*)(size_t)(((m >> 8) & 255u) << 2);
  const u32 e2 = *(const LAS u32*)(size_t)(((m >> 16) & 255u) << 2), e3 = *(const LAS u32*)(size_t)((m >> 24) << 2);
  const u32 l02 = __builtin_amdgcn_perm(e2, e0, 0x05040100u), l13 = __builtin_amdgcn_perm(e3, e1, 0x05040100u);
  const u32 h02 = __builtin_amdgcn_perm(e2, e0, 0x07060302u), h13 = __builtin_amdgcn_perm(e3, e1, 0x07060302u);
  Mv r;
  r.a0 = __builtin_amdgcn_perm(l13, l02, 0x07030501u); r.a1 = __builtin_amdgcn_perm(h13, h02, 0x06020400u); r.a2 = __builtin_amdgcn_perm(h13, h02, 0x07030501u);
  r.c8 = l02; r.c16 = e0 + e1;
  return r;
}
__device__ __forceinline__ u32 pext(u32 x, u32 m, const Mv& v) {
  x &= m;
  u32 t = x & v.a0; x = BOP(x, t, t >> 1, (A ^ B) | C);
  t = x & v.a1; x = BOP(x, t, t >> 2, (A ^ B) | C);
  t = x & v.a2; x = BOP(x, t, t >> 4, (A ^ B) | C);
  t = __builtin_amdgcn_perm(0u, x, 0x0c030c01u);
  asm("v_pk_lshlrev_b16 %0, %1, %0" : "+v"(t) : "v"(v.c8));
  x = BOP(x, 0x00ff00ffu, t, (A & B) | C);
  return BOP(x, 0xffffu, (x >> 16) << (v.c16 & 31u), (A & B) | C);
}
__device__ __forceinline__ u32 pdep(u32 x, const Mv& v) {
  x = __builtin_amdgcn_perm(x >> (v.c16 & 31u), x, 0x05040100u);
  u32 t; asm("v_pk_lshrrev_b16 %0, %1, %2" : "=v"(t) : "v"(v.c8), "v"(x));
  x = __builtin_amdgcn_perm(t, x, 0x06020400u);
  x = BFI(v.a2, x << 4, x); x = BFI(v.a1, x << 2, x);
  return BFI(v.a0, x << 1, x);
}
__device__ __forceinline__ void transpose4(u32 (&n)[4]) {
#pragma unroll
  for (int r = 0; r < 2; ++r) { const u32 a = n[r], c = n[r + 2]; n[r] = BFI(0x33333333u, a, c << 2); n[r + 2] = BFI(0x33333333u, a >> 2, c); }
#pragma unroll
  for (int r = 0; r < 4; r += 2) { const u32 a = n[r], c = n[r + 1]; n[r] = BFI(0x55555555u, a, c << 1); n[r + 1] = BFI(0x55555555u, a >> 1, c); }
}
__device__ __forceinline__ void or_bits(LAS u32* area, u32 bit, u32 val) {
  const u64 t = (u64)val << (bit & 31);
  LAS u32* q = area + (bit >> 5);
  if ((u32)t) __hip_atomic_fetch_or(q, (u32)t, __ATOMIC_RELAXED, __HIP_MEMORY_SCOPE_WORKGROUP);
  if ((u32)(t >> 32)) __hip_atomic_fetch_or(q + 1, (u32)(t >> 32), __ATOMIC_RELAXED, __HIP_MEMORY_SCOPE_WORKGROUP);
}
__device__ __forceinline__ u32 mad3(u32 a, u32 c) { u32 r; asm("v_mad_u32_u24 %0, %1, 3, %2" : "=v"(r) : "v"(a), "s"(c)); return r; }
__device__ __forceinline__ u32 get_bits(u32 bit) {
  const LAS u32* q = (const LAS u32*)(size_t)((bit >> 3) & ~3u);
  return __builtin_amdgcn_alignbit(q[1], q[0], bit & 31);
}

struct Enc {
  char* enc; gu64* row; LAS char* lds; u32 call; int w, lane; u32 par, pseg, pmask, place; u64 h4 = 0, h5 = 0;
  __device__ __forceinline__ void init(CParams& p, LAS char* l, u32 c, int w_, int lane_) {
    enc = uni(p.ws[0] + ENC_OFF + (c & 1) * ENC_SZ); row = flag_row(p, c, lane_); lds = l; call = c; w = w_; lane = lane_; par = 0; pseg = 0; pmask = 0;
  }
  __device__ __forceinline__ LAS u32* fill(u32 q) const { return (LAS u32*)(lds + LDS_MAIN + 64) + (q & 3u) * 2 + w / NB; }
  __device__ __forceinline__ u32 take_len() { LAS u32* c = fill(par - 1); const u32 n = rfl(*c); *c = 0; return (SEG_V + n * 4 + 15) & ~15u; }
  __device__ __forceinline__ void post() { if ((w & (NB - 1)) == 0 && pmask) post_seg(row, call, pseg, take_len(), pmask, lane); pmask = 0; }
  __device__ __forceinline__ void finish() { DRAIN; LBAR; post(); }
  __device__ __forceinline__ void draw(const char* mbx) { place = mail_place(mbx, (w & (NB - 1)) == 0 ? pmask : 0u, lane); }
  __device__ __forceinline__ void finish_mail() { DRAIN; LBAR; if ((w & (NB - 1)) == 0) post_mail(row, call, place, pseg, take_len(), pmask, lane); pmask = 0; }
  template <class F> __device__ __forceinline__ void idle(F&& mid) { if (par) finish(); mid(); ++par; }
  template <bool PIPE, class F> __device__ __forceinline__ void round(const u32x4 (&X)[4], u32 seg, u32 mask, F&& mid) {
    const int rb = w % NB;
    const u32 so = seg * SEG_STRIDE;
    LAS u32* var = (LAS u32*)(lds + LDS_VAR + w * 1536);
    { const u32x4 zero = {0u, 0u, 0u, 0u}; ((LAS u32x4*)var)[lane] = zero; ((LAS u32x2*)var)[128 + lane] = (u32x2){0u, 0u}; }
    u32 R[16], S[8], sum = 0;
#pragma unroll
    for (int i = 0; i < 16; ++i) { const u32 t = X[i >> 2][i & 3]; R[i] = __builtin_amdgcn_alignbit(t, t, 31); if (i & 1) sum = __builtin_amdgcn_udot4(R[i], 0x01000100u, sum, false); }
#pragma unroll
    for (int q = 0; q < 8; ++q) S[q] = __builtin_amdgcn_perm(R[2 * q + 1], R[2 * q], 0x06040200u);
    if (!PIPE || par == 0) st4x2(enc, so + rb * 2048 + lane * 16, (u32x4){S[0], S[1], S[2], S[3]}, (u32x4){S[4], S[5], S[6], S[7]});
    sum = __builtin_amdgcn_readlane(wave_scan(sum), 63);
    int basei = (int)((sum + 870u) >> 10) - 10;
    basei = basei < 0 ? 0 : (basei > 240 ? 240 : basei);
    const u32 base = (u32)basei;
    const u16x2 bsub = {(unsigned short)(base << 8), (unsigned short)(base << 8)}, top = {0x0f00, 0x0f00};
    u32 V[8], d[4];
#pragma unroll
    for (int q = 0; q < 8; ++q) {
      const u16x2 a = __builtin_elementwise_min((u16x2)(__builtin_bit_cast(u16x2, R[2 * q]) - bsub), top);
      const u16x2 b = __builtin_elementwise_min((u16x2)(__builtin_bit_cast(u16x2, R[2 * q + 1]) - bsub), top);
      V[q] = __builtin_amdgcn_perm(__builtin_bit_cast(u32, b), __builtin_bit_cast(u32, a), 0x07050301u);
    }
#pragma unroll
    for (int q = 0; q < 4; ++q) d[q] = V[2 * q] | (V[2 * q + 1] << 4);
    transpose4(d);
    u32 Xm = BOP(d[3], d[2], d[1], A & B & C);
    u32 P1 = BOP(d[3], d[2], d[1], A & ~B & C), P0 = BOP(d[3], d[2], d[0], A & ~B & C) | Xm;
    const u32 E1 = ~(P1 | P0);
    const u32 Q1 = d[3] & ~d[0], Q0 = d[2] & BOP(d[3], d[1], d[0], (A & ~C) | (~A & B & C));
    const u32 E2 = E1 & ~(Q1 | Q0);
    const Mv ma = move_masks(E1), mb = move_masks(E2);
    const u32 n1 = __builtin_popcount(E1), n2 = __builtin_popcount(E2), nx = __builtin_popcount(Xm);
    const u32 cnt = n1 | (n2 << 12) | ((nx > 3u ? 3u : nx) << 24);
    const u32 inc = wave_scan(cnt), tot = __builtin_amdgcn_readlane(inc, 63), exc = inc - cnt;
    const u32 N1 = tot & 0xfffu, NX = tot >> 24, bitw = (2 * N1 + 3 * ((tot >> 12) & 0xfffu) + 31) >> 5;
    const bool raw = (NX > XCAP) | (__ballot(nx > 3u) != 0);
    const u32 vd = raw ? 384u : bitw + NX;
    u32 voff = 0;
    if (lane == 0) voff = __hip_atomic_fetch_add(fill(par), vd, __ATOMIC_RELAXED, __HIP_MEMORY_SCOPE_WORKGROUP);
    const u32 q1 = pext(Q1, E1, ma), q0 = pext(Q0, E1, ma), r2 = pext(d[2], E2, mb), r1 = pext(d[1] | d[3], E2, mb), r0 = pext(d[0], E2, mb);
    const u32 o1 = 2 * (exc & 0xfffu), o2 = 2 * N1 + 3 * ((exc >> 12) & 0xfffu);
    if (__all(n1 <= 16u && n2 <= 10u)) { or_bits(var, o1, q1 | (q0 << n1)); or_bits(var, o2, r2 | (r1 << n2) | (r0 << (2 * n2))); }
    else { or_bits(var, o1, q1); or_bits(var, o1 + n1, q0); or_bits(var, o2, r2); or_bits(var, o2 + n2, r1); or_bits(var, o2 + 2 * n2, r0); }
    if (NX) {
      u32 E[8];
#pragma unroll
      for (int q = 0; q < 8; ++q) E[q] = __builtin_amdgcn_perm(R[2 * q + 1], R[2 * q], 0x07050301u);
      if (!raw) {
        u32 xo = bitw + (exc >> 24);
        do {
          if (Xm != 0u) {
            const u32 j = __builtin_ctz(Xm); Xm &= Xm - 1;
            const u32 vi = 2 * (j & 3) + ((j >> 2) & 1), by = j >> 3;
            const u32 k0 = 0u - (vi & 1u), k1 = 0u - ((vi >> 1) & 1u), k2 = 0u - (vi >> 2);
            const u32 vs = BFI(k2, BFI(k1, BFI(k0, E[7], E[6]), BFI(k0, E[5], E[4])), BFI(k1, BFI(k0, E[3], E[2]), BFI(k0, E[1], E[0])));
            var[xo] = (u32)lane | (vi << 6) | (by << 9) | ((((vs >> (8 * by)) & 255u) ^ (base + 9u)) << 16);
            ++xo;
          }
        } while (__any(Xm != 0u));
      } else {
        P1 = E[0]; P0 = E[1];
#pragma unroll
        for (int q = 2; q < 8; ++q) var[(q - 2) * 64 + lane] = E[q];
      }
    }
    asm volatile("" ::: "memory");
    u32 v0 = var[lane];
    voff = rfl(voff);
    if (PIPE) {
      asm volatile("" : "+v"(v0));
      if (par) finish();
      mid();
      if (par) st4x2(enc, so + rb * 2048 + lane * 16, (u32x4){S[0], S[1], S[2], S[3]}, (u32x4){S[4], S[5], S[6], S[7]});
    }
    asm volatile("global_store_dwordx2 %0, %1, %4 sc0 sc1\n\ts_mov_b64 exec, 1\n\tglobal_store_dword %2, %3, %4 sc0 sc1\n\ts_mov_b64 exec, -1"
                 :: "v"(so + SEG_P + rb * 512 + lane * 8), "v"((u32x2){P1, P0}), "v"(so + SEG_H + rb * 4), "v"(base | ((raw ? 255u : NX) << 8) | (voff << 16)), "s"(enc) : "memory");
    const u32 vo = so + SEG_V + (voff + lane) * 4;
    if ((u32)lane < vd) asm volatile("global_store_dword %0, %1, %2 sc0 sc1" :: "v"(vo), "v"(v0), "s"(enc) : "memory");
    for (u32 i = 64; i < vd; i += 64) if (i + lane < vd) asm volatile("global_store_dword %0, %1, %2 sc0 sc1" :: "v"(vo + i * 4), "v"(var[i + lane]), "s"(enc) : "memory");
    pseg = seg; pmask = mask; ++par;
  }
};
constexpr int XBUF = 65536;
__device__ __forceinline__ void load_job(const char* px, const char* pw, LAS char* lds, int J, int buf, int w, int lane) {
  if (J >= NJOB) return;
  const int c = J >> 5, pn = (J >> 1) & 15, st = (J & 1) * NB + (w & (NB - 1));
  const char* q; u32 off, step;
  if (pn < 8) { q = px; off = ((u32)((pn * 8 + st) * 16 + (lane & 15)) * KD + c * 128 + (lane >> 4) * 8) * 2; step = 64; }
  else { q = pw; off = (u32)(c * 128 + (lane >> 1)) * (NS * 2) + (pn - 8) * 256 + st * 32 + (lane & 1) * 16; step = 65536; }
  q = uni(q);
  const u32 dst = (u32)(size_t)lds + XBUF + buf * 32768 + w * 4096;
#pragma unroll
  for (int k = 0; k < 4; ++k) asm volatile("s_mov_b32 m0, %2\n\ts_nop 0\n\tglobal_load_lds_dwordx4 %0, %1" :: "v"(off + k * step), "s"(q), "s"(dst + k * 1024) : "memory");
}
constexpr int JOB1 = NPC + UPR * 2 * NPC, JOB2 = JOB1 + MIDR * 2 * NPB;

struct Jobs {
  int nr, u, sg;
  __device__ __forceinline__ void init(int u_, int sg_) {
    u = u_; sg = sg_;
    nr = u < 0 ? 0 : (u >= NPB ? 1 + UPR : 1 + UPR + MIDR + (u >= NPA || JOB2 + 2 * u >= NJOB ? 0 : (NJOB - JOB2 - 2 * u + 2 * NPA - 1) / (2 * NPA)));
  }
  __device__ __forceinline__ int job(int r) const {
    if (r >= nr) return NJOB;
    if (r == 0) return sg ? NJOB : u;
    return 2 * u + sg + (r <= UPR ? NPC + (r - 1) * 2 * NPC : (r <= UPR + MIDR ? JOB1 + (r - 1 - UPR) * 2 * NPB : JOB2 + (r - 1 - UPR - MIDR) * 2 * NPA));
  }
};
__device__ __forceinline__ void encode_jobs(CParams& p, LAS char* lds, u32 call, int w, int lane, const Jobs& jb) {
  Enc e; e.init(p, lds, call, w, lane);
  const int rank = p.rank;
  asm volatile("" : "+v"(e.row));
  if (jb.job(1) < NJOB) WAIT_V(4); else WAIT_V(0);
  for (int r = 0; r < jb.nr; ++r) {
    const int J = jb.job(r), J2 = jb.job(r + 2);
    auto mid = [&] { if (r != 0) load_job(p.x, p.w, lds, J2, r & 1, w, lane); };
    if (J < NJOB) {
      u32x4 X[4];
      const u32 pn = (J >> 1) & 15;
      LAS char* xb = lds + XBUF + (r & 1) * 32768 + w * 4096;
      if (pn < 8u) {
#pragma unroll
        for (int k = 0; k < 4; ++k) X[k] = *(const LAS u32x4*)(xb + k * 1024 + lane * 16);
      } else {
#pragma unroll
        for (int k = 0; k < 4; ++k) {
          LAS char* q = xb + k * 1024 + (lane >> 4) * 256 + (lane & 15) * 8;
          const s16x4 lo = __builtin_amdgcn_ds_read_tr16_b64_v4i16((LAS s16x4*)q), hi = __builtin_amdgcn_ds_read_tr16_b64_v4i16((LAS s16x4*)(q + 128));
          X[k] = __builtin_bit_cast(u32x4, __builtin_shufflevector(lo, hi, 0, 1, 2, 3, 4, 5, 6, 7));
        }
      }
      e.round<true>(X, pn * 128 + (J >> 5) * 2 + (J & 1), input_mask(rank, pn, J >> 5), mid);
    } else e.idle(mid);
    if (r == 0) { e.finish(); load_job(p.x, p.w, lds, J2, 0, w, lane); STAMP(1);
    }
  }
  e.finish();
  STAMP(2);
}

__device__ __forceinline__ void lds_post(LAS u32* q, u32 v) { asm volatile("ds_write_b32 %0, %1" :: "v"((u32)(size_t)q), "v"(v) : "memory"); }
__device__ __forceinline__ u32 lds_peek(LAS u32* q) { u32 v; asm volatile("ds_read_b32 %0, %1\n\ts_waitcnt lgkmcnt(0)" : "=&v"(v) : "v"((u32)(size_t)q) : "memory"); return rfl(v); }
__device__ __forceinline__ void lds_inc(LAS u32* q) { asm volatile("s_mov_b64 exec, 1\n\tds_add_u32 %0, %1\n\ts_mov_b64 exec, -1" :: "v"((u32)(size_t)q), "v"(1u) : "memory"); }
template <int WIN, int I> __device__ __forceinline__ void load10(const char* gs, u32 ls, const u32 (&v16)[3], LAS u32* landed, u32 k, u32 since2) {
  if constexpr (I < 10) {
    if ((I & 3) == 0) asm volatile("s_mov_b32 m0, %0" :: "s"(ls + I * 1024) : "memory");
    wait_v<WIN - 1>();
    asm volatile("global_load_lds_dwordx4 %0, %1 offset:%2 sc0 sc1" :: "v"(v16[I >> 2]), "s"(gs), "i"((I & 3) * 1024) : "memory");
    if (I + 11 <= WIN - 1 && since2 + I == WIN - 1) lds_post(landed, k - 1);
    if (I == WIN - 1) lds_post(landed, k);
    load10<WIN, I + 1>(gs, ls, v16, landed, k, since2);
  }
}
constexpr u32 FIN = 0x40000000u;
__host__ __device__ __forceinline__ u32 c_offset(int rank, u32 pc) {
  const u32 o = (pc >> 8) & 1u;
  return ((u32)(slot_row(rank ^ (o + 1), (pc >> 3) & 31u) + (pc & 1u) * 64) * NS + o * 512 + ((pc >> 2) & 1u) * 256 + ((pc >> 1) & 1u) * 128) * 2;
}

template <int WIN, int NJ> __device__ __forceinline__ void loader(CParams& p, LAS char* lds, int d, u32 call, int lane) {
    LAS u32* landed = (LAS u32*)(lds + LDS_MAIN + 128); LAS u32* posted = landed + 1; LAS u32* started = landed + 2;
    LAS u32* done = (LAS u32*)(lds + LDS_MAIN + 192); LAS u32* qitem = (LAS u32*)(lds + LDS_MAIN + 256);
    const char* rsrc = uni(p.ws[d] + ENC_OFF + (call & 1) * ENC_SZ);
    __builtin_amdgcn_s_waitcnt(0x0f70);
    __builtin_amdgcn_s_setprio(3);
    const u32 v16[3] = {(u32)lane * 16, (u32)lane * 16 + 4096, (u32)lane * 16 + 8192};
    int slot = 0; u32 use = 0, k = 0, since2 = 100; int spinq = 0, spind = 0;
    static_assert(WIN <= 22, "window");
    for (;; ++k) {
      u32 item, len;
      for (u32 it = 0;; ++it) {
        u32 a, dn;
        asm volatile("ds_read_b32 %0, %4\n\tds_read_b32 %1, %5\n\tds_read_b32 %2, %6\n\tds_read_b32 %3, %6 offset:64\n\ts_waitcnt lgkmcnt(0)"
                     : "=&v"(a), "=&v"(dn), "=&v"(item), "=&v"(len)
                     : "v"((u32)(size_t)posted), "v"((u32)(size_t)(done + slot)), "v"((u32)(size_t)(qitem + (k % QD))) : "memory");
        if (rfl(a) > k && rfl(dn) >= use) break;
        if (it == DRAINIT) { WAIT_V(0); lds_post(landed, k); since2 = 100; }
        __builtin_amdgcn_s_sleep(1);
      }
      len = rfl(len);
      if (len == 0) break;
      lds_post(started, k + 1);
      const char* gs = rsrc + (size_t)(rfl(item) >> 16) * SEG_STRIDE;
      const u32 ls = (u32)(size_t)lds + slot * SEG_STRIDE;
      load10<WIN, 0>(gs, ls, v16, landed, k, since2);
      for (u32 o = 10240; o < len; o += 1024) {
        wait_v<WIN - 1>();
        if (WIN > 21 && since2 + (o >> 10) == WIN - 1) lds_post(landed, k - 1);
        if (WIN > 10 && o == (WIN - 1) * 1024) lds_post(landed, k);
        if (o + lane * 16 < len) asm volatile("s_mov_b32 m0, %2\n\ts_nop 0\n\tglobal_load_lds_dwordx4 %0, %1 sc0 sc1" :: "v"(o + lane * 16), "s"(gs), "s"(ls + o) : "memory");
      }
      since2 = (len + 1023) >> 10;
      if (++slot == NSLOT) { slot = 0; use += NB / NJ; }
    }
    WAIT_V(0); lds_post(landed, k | FIN);
    STAMP(2); STAMPV(10, spinq); STAMPV(11, spind); STAMPV(12, k);
}
template <int NJ> __device__ __forceinline__ void decoder(CParams& p, LAS char* lds, int d, u32 call, int lane) {
  LAS u32* landed = (LAS u32*)(lds + LDS_MAIN + 128); LAS u32* done = (LAS u32*)(lds + LDS_MAIN + 192);
  LAS u32* qitem = (LAS u32*)(lds + LDS_MAIN + 256); LAS u32* qlen = qitem + QD; LAS u32* qdst = qitem + 2 * QD;
  char* dec = uni(p.ws[0] + DEC_OFF);
  char* enc = uni(p.ws[0] + ENC_OFF + (call & 1) * ENC_SZ);
  char* out = uni(p.out);
  gu64* row = flag_row(p, call, lane);
  const u32 vC = (lane & 15) * (NS * 2) + ((lane & 16) << 1) + ((lane & 32) >> 1);
  u32 have = 0, pend = 0, fseg = 0, flen = 0, fmask = 0;
#define TRACEP do {} while (0)
#define PUBLISH() do { if (pend) { \
    asm volatile("s_waitcnt vmcnt(0)\n\ts_mov_b64 exec, %3\n\tglobal_store_dword %0, %1, %2 sc0 sc1\n\ts_mov_b64 exec, -1" :: "v"(pend + lane * (NCH * 4)), "v"(call), "s"(dec), "i"(2 * NJ - 1) : "memory"); \
    if ((pend & 255u) == 252u) STAMP(4); \
    TRACEP; \
    if (flen) post_seg(row, call, fseg, flen, fmask, lane); pend = 0; flen = 0; } } while (0)
#define PASS_ON(seg, mask) do { flen = lds_peek(qlen + (k % QD)); fseg = (seg); fmask = (mask); \
    for (u32 o = 0; o < flen; o += 6144) { u32x4 t_[6]; \
      _Pragma("unroll") for (int i = 0; i < 6; ++i) t_[i] = *(const LAS u32x4*)(ls + o + i * 1024 + lane * 16); \
      _Pragma("unroll") for (int i = 0; i < 6; ++i) if (o + i * 1024 + lane * 16 < flen) st4x1(enc, fseg * SEG_STRIDE + o + i * 1024 + lane * 16, t_[i]); } } while (0)
#define BOTH _Pragma("unroll") for (int z = 0; z < NJ; ++z)
  for (;;) {
    u32 q;
    asm volatile("s_mov_b64 exec, 1\n\tds_add_rtn_u32 %0, %2, %3 offset:12\n\tds_read_b32 %1, %2\n\ts_waitcnt lgkmcnt(0)\n\ts_mov_b64 exec, -1"
                 : "=&v"(q), "=&v"(have) : "v"((u32)(size_t)landed), "v"(1u) : "memory");
    q = rfl(q); have = rfl(have);
    const u32 k = q / (NB / NJ), hf = q % (NB / NJ) * NJ, slot = k % NSLOT;
    if ((have & ~FIN) <= k) {
      PUBLISH();
      while (have <= k) { __builtin_amdgcn_s_sleep(1); have = lds_peek(landed); }
      if ((have & ~FIN) <= k) break;
    }
    const LAS char* ls = lds + slot * SEG_STRIDE;
    const u32 item_v = qitem[k % QD], dst_v = qdst[k % QD];
    u32 hdr_v[2]; u32x2 Pp[2]; u32x4 S[2][2];
    BOTH hdr_v[z] = *(const LAS u32*)(ls + SEG_H + (hf + z) * 4);
    BOTH Pp[z] = *(const LAS u32x2*)(ls + SEG_P + hf * 512 + z * 512 + lane * 8);
    BOTH { S[z][0] = *(const LAS u32x4*)(ls + hf * 2048 + z * 2048 + lane * 16); S[z][1] = *(const LAS u32x4*)(ls + hf * 2048 + z * 2048 + 1024 + lane * 16); }
    const u32 item = rfl(item_v), dst = rfl(dst_v);
    if (item & D_REL) {
      PUBLISH();
      if (hf == 0) {
        PASS_ON(24 * 128 + (item & 511u), 1u << (d ^ (((item >> 8) & 1u) + 1)));
        lds_inc(done + slot);
        const u32 place = mail_place(dec + (MBX_OFF - DEC_OFF), fmask, lane);
        WAIT_V(0);
        post_mail(row, call, place, fseg, flen, fmask, lane); flen = 0;
      } else lds_inc(done + slot);
      continue;
    }
    u32 hdr[2], vb[2], E1[2], n1[2], N1[2], o1[2], q1[2], q0[2], Q1[2], Q0[2], E2[2], n2[2], N2[2], o2[2], r2[2], r1[2], r0[2], E[2][8]; Mv ma[2], mb[2];
    BOTH { hdr[z] = rfl(hdr_v[z]); vb[z] = ((u32)(size_t)ls + SEG_V + ((hdr[z] >> 16) & 2047u) * 4) * 8; }
    BOTH { E1[z] = ~(Pp[z][0] | Pp[z][1]); ma[z] = move_masks(E1[z]); n1[z] = __builtin_popcount(E1[z]); }
    if (NJ == 1) { const u32 i = wave_scan(n1[0]); N1[0] = __builtin_amdgcn_readlane(i, 63); o1[0] = vb[0] + 2 * (i - n1[0]); n1[1] = 0; }
    else { const u32 i = wave_scan(n1[0] | (n1[1] << 16)), t = __builtin_amdgcn_readlane(i, 63);
      N1[0] = t & 0xffffu; N1[1] = t >> 16; o1[0] = vb[0] + 2 * ((i & 0xffffu) - n1[0]); o1[1] = vb[1] + 2 * ((i >> 16) - n1[1]); }
    BOTH { q1[z] = get_bits(o1[z]); q0[z] = q1[z] >> (n1[z] & 31u); }
    if (__any((n1[0] > n1[1] ? n1[0] : n1[1]) > 16u)) BOTH q0[z] = get_bits(o1[z] + n1[z]);
    if (NJ == 2 && PUBAT == 1) PUBLISH();
    BOTH { Q1[z] = pdep(q1[z], ma[z]); Q0[z] = pdep(q0[z], ma[z]); E2[z] = BOP(E1[z], Q1[z], Q0[z], A & ~(B | C)); mb[z] = move_masks(E2[z]); n2[z] = __builtin_popcount(E2[z]); }
    if (NJ == 1) { const u32 i = wave_scan(n2[0]); N2[0] = __builtin_amdgcn_readlane(i, 63); o2[0] = mad3(i - n2[0], vb[0] + 2 * N1[0]); n2[1] = 0; }
    else { const u32 i = wave_scan(n2[0] | (n2[1] << 16)), t = __builtin_amdgcn_readlane(i, 63);
      N2[0] = t & 0xffffu; N2[1] = t >> 16; o2[0] = mad3((i & 0xffffu) - n2[0], vb[0] + 2 * N1[0]); o2[1] = mad3((i >> 16) - n2[1], vb[1] + 2 * N1[1]); }
    u32 xe[2];
    BOTH { r2[z] = get_bits(o2[z]); r1[z] = r2[z] >> (n2[z] & 31u); r0[z] = r1[z] >> (n2[z] & 31u);
           xe[z] = *(const LAS u32*)(size_t)((vb[z] >> 3) + (((2 * N1[z] + 3 * N2[z] + 31) >> 5) + lane) * 4); }
    if (__any((n2[0] > n2[1] ? n2[0] : n2[1]) > 10u)) BOTH { r1[z] = get_bits(o2[z] + n2[z]); r0[z] = get_bits(o2[z] + 2 * n2[z]); }
    if (NJ == 2 && PUBAT == 2) PUBLISH();
    BOTH {
      const u32 R2 = pdep(r2[z], mb[z]) & E2[z], R1 = pdep(r1[z], mb[z]), R0 = pdep(r0[z], mb[z]);
      const u32 t7 = BOP(R2, R1, R0, A & B & C), q7 = BOP(E1[z], Q1[z], Q0[z], A & ~B & C);
      u32 dd[4] = {BOP(Pp[z][1] | q7, R0, E2[z], A | (B & C)), BOP(Pp[z][0], q7, BOP(R1, E2[z], t7, A & B & ~C), A | B | C), BOP(Q0[z], E1[z], R2, (A & B) | C), BOP(E1[z], Q1[z], t7, ~A | B | C)};
      transpose4(dd);
      const u32 b4 = (hdr[z] & 255u) * 0x01010101u;
#pragma unroll
      for (int i = 0; i < 4; ++i) { E[z][2 * i] = (dd[i] & 0x0f0f0f0fu) + b4; E[z][2 * i + 1] = ((dd[i] >> 4) & 0x0f0f0f0fu) + b4; }
    }
    BOTH asm volatile("" : "+v"(xe[z]));
    BOTH {
      const u32 NX = (hdr[z] >> 8) & 255u;
      if (NX <= XCAP) {
        for (u32 i = 0; i < NX; ++i) {
          const u32 en = __builtin_amdgcn_readlane(xe[z], i);
          const u32 dv = (u32)lane == (en & 63u) ? ((en >> 16) & 255u) << (8 * ((en >> 9) & 3u)) : 0u;
          const int hot = (int)(0x80000000u >> ((en >> 6) & 7u));
#pragma unroll
          for (int i2 = 0; i2 < 8; ++i2) E[z][i2] = BOP(E[z][i2], dv, (u32)((hot << i2) >> 31), A ^ (B & C));
        }
      } else {
        E[z][0] = Pp[z][0]; E[z][1] = Pp[z][1];
#pragma unroll
        for (int i2 = 2; i2 < 8; ++i2) E[z][i2] = *(const LAS u32*)(size_t)((vb[z] >> 3) + ((i2 - 2) * 64 + lane) * 4);
      }
    }
    u32x4 O[2][4];
    BOTH {
#pragma unroll
      for (int i = 0; i < 8; ++i) {
        const u32 sb = S[z][i >> 2][i & 3];
        const u32 a = __builtin_amdgcn_perm(E[z][i], sb, 0x05010400u), b = __builtin_amdgcn_perm(E[z][i], sb, 0x07030602u);
        O[z][i >> 1][(i & 1) * 2] = __builtin_amdgcn_alignbit(a, a, 1); O[z][i >> 1][(i & 1) * 2 + 1] = __builtin_amdgcn_alignbit(b, b, 1);
      }
      asm volatile("" : "+v"(O[z][0]), "+v"(O[z][1]), "+v"(O[z][2]), "+v"(O[z][3]));
    }
    if (NJ == 1 || PUBAT == 3) PUBLISH();
    if (item & D_C) {
      char* o = out + (dst + hf * 64 + vC);
#pragma unroll
      for (int m = 0; m < 4; ++m) BOTH st_out(o + m * 16 * NS * 2 + z * 64, O[z][m]);
    } else {
      BOTH st4x4(dec, dst + (hf + z) * 4096u + lane * 16, O[z][0], O[z][1], O[z][2], O[z][3]);
      pend = (u32)(F2_OFF - DEC_OFF) + ((((item >> 7) & 31u) * 8 + (item & 1u) * NB + hf) * NCH + ((item >> 1) & 63u)) * 4;
      if ((item & D_FWD) && hf == 0) PASS_ON((12 + d) * 128 + (item & 127u), 1u << (d ^ 3));
    }
    lds_inc(done + slot);
    if (q < 2) STAMP(1);
  }
  STAMP(2);
}

__device__ __forceinline__ void pull_role(CParams& p, LAS char* lds, int d, int j, int ncu, u32 call, int w, int lane) {
  LAS u32* landed = (LAS u32*)(lds + LDS_MAIN + 128);
  LAS u32* posted = landed + 1;
  LAS u32* started = landed + 2;
  LAS u32* done = (LAS u32*)(lds + LDS_MAIN + 192);
  LAS u32* qitem = (LAS u32*)(lds + LDS_MAIN + 256);
  LAS u32* qdst = qitem + 2 * QD;
  LAS u32* qlen = qitem + QD;
  LAS u32* list = (LAS u32*)(lds + NSLOT * SEG_STRIDE);
  static_assert(NSLOT * SEG_STRIDE + LMAX * 4 <= LDS_MAIN, "list");
  const int rank = rfl(p.rank), lt = list_type(d);
  const u32 nl = rfl(LISTS.n[lt]), n = nl + (lt == 0 ? NMAIL0 : (lt == 2 ? NMAIL2 : 0u));
  for (u32 i = threadIdx.x; i < nl; i += 512) list[i] = resolve(d, rank, LISTS.e[lt][i]);
  LBAR;
  if (w == 1) {
    const u64* f1 = uni((const u64*)(p.ws[0] + F1_OFF + (call & 1) * F1_SZ) + d * NSRC);
    u32* tk = uni((u32*)(p.ws[0] + TKT_OFF + d * 128));
    const u32 tbase = call * 1024u, nin = nl - 1;
    __builtin_amdgcn_s_setprio(2);
    if (lane == 0) __hip_atomic_fetch_max(tk, tbase, __ATOMIC_RELAXED, __HIP_MEMORY_SCOPE_AGENT);
    WAIT_V(0);
#define FIDX(i) (list[(i) < nl ? (i) : nin] >> 16)
#define ENQUEUE(item, len, dst) do { lds_post(qdst + (k % QD), dst); lds_post(qitem + (k % QD), item); lds_post(qlen + (k % QD), len); lds_post(posted, ++k); } while (0)
#define CLAMP_LEN(l) ((l) > (u32)SEG_STRIDE ? (u32)SEG_STRIDE : ((l) < (u32)SEG_V ? (u32)SEG_V : (l) & ~15u))
    u32 pos = j, k = 0, t, wa = ~0u, wb, win = ~63u, asked = ~WREF; u64 v = 0, vn; bool dead = false, more = true;
    asm volatile("v_mov_b32 v248, %0" :: "s"(tbase + j - ncu) : "v248");
#define DRAW(ww, wo, TN, VN, TO, VO) do { while (more && lds_peek(started) + LAP <= k + 1) __builtin_amdgcn_s_sleep(1); \
      ww = pos - asked >= WREF ? pos : ~0u; if (ww != ~0u) asked = pos;        \
      asm volatile("s_mov_b64 exec, 1\n\tglobal_atomic_add " TN ", %2, %3, %4 sc0\n\ts_mov_b64 exec, %7\n\tglobal_load_dwordx2 " VN ", %5, %6 sc0 sc1\n\ts_mov_b64 exec, -1\n\t" \
                   "s_waitcnt vmcnt(2)\n\tv_mov_b32 %0, " TO "\n\tv_mov_b64 %1, " VO \
                   : "=&v"(t), "=&v"(vn) : "v"(0u), "v"(more ? 1u : 0u), "s"(tk), "v"(FIDX(pos + lane) * 8u), "s"(f1), "s"(ww != ~0u ? ~0ull : 1ull) \
                   : "memory", "v248", "v249", "v250", "v251", "v252", "v253"); \
      if (wo != ~0u) { win = wo; v = vn; } } while (0)
#define SETTLE(t, vv, ww) do { const u32 q_ = rfl(t) - tbase + ncu; \
      if (q_ >= nl) more = false; \
      else { pos = q_; const u32 item = rfl(list[pos]); u32 len = 0; \
        if (pos - ww < 64u && ((__ballot((u32)(vv >> 32) >= call) >> (pos - ww)) & 1)) len = __builtin_amdgcn_readlane((u32)vv, pos - ww); \
        else { const u64 t0 = __builtin_amdgcn_s_memrealtime(); asked = pos - WREF; \
          for (u32 it = 1;; ++it) { const u64 u = __hip_atomic_load(f1 + (item >> 16), __ATOMIC_RELAXED, __HIP_MEMORY_SCOPE_SYSTEM); \
            if (rfl((u32)(u >> 32)) >= call) { len = rfl((u32)u); break; } \
            if ((it & 63u) == 0 && give_up(p, call, t0)) { dead = true; more = false; break; } \
            __builtin_amdgcn_s_sleep(1); } } \
        if (!dead) { ENQUEUE(item, CLAMP_LEN(len), ((item >> 7) & 31u) * (u32)PANEL_B + (item & 127u) * 16384u); MARKS; } } } while (0)
#define MARKS do {} while (0)
    for (;;) {
      bool last = !more;
      DRAW(wb, wa, "v249", "v[252:253]", "v248", "v[250:251]"); SETTLE(t, v, win);
      if (last || dead) break;
      last = !more;
      DRAW(wa, wb, "v248", "v[250:251]", "v249", "v[252:253]"); SETTLE(t, v, win);
      if (last || dead) break;
    }
    const u32 place = j + lane * ncu;
    u64 todo = dead ? 0 : __ballot(place < n - nl);
    const u64* fp = f1 + MAIL0 + (place < n - nl ? place : 0u);
    u64 t0 = __builtin_amdgcn_s_memrealtime();
    for (u32 it = 1; todo; ++it) {
      v = __hip_atomic_load(fp, __ATOMIC_RELAXED, __HIP_MEMORY_SCOPE_SYSTEM);
      u64 rdy = __ballot((u32)(v >> 32) >= call) & todo;
      if (rdy) {
        todo &= ~rdy;
        const u32 sg_ = (((u32)v >> 16) - MAIL0) & 1023u, ln = ((u32)v & 0xffffu) << 4;
        const u32 item = ((MAIL0 + sg_) << 16) | (sg_ & 511u) | (d > 3 && sg_ < 512u ? D_REL : D_C), len = CLAMP_LEN(ln), dst = c_offset(rank, sg_);
        const u64 first = __ballot(item & D_REL);
        do {
          while (lds_peek(started) + LAM <= k) __builtin_amdgcn_s_sleep(1);
          const u32 l = __builtin_ctzll(rdy & first ? rdy & first : rdy); rdy &= ~(1ull << l);
          ENQUEUE(__builtin_amdgcn_readlane(item, l), __builtin_amdgcn_readlane(len, l), __builtin_amdgcn_readlane(dst, l));
          if (REPOLL && todo && lds_peek(started) + LAM <= k) { todo |= rdy; rdy = 0; }
        } while (rdy);
        if (REPOLL) while (lds_peek(started) + LAM <= k + 1 - REPOLL) __builtin_amdgcn_s_sleep(1);
      } else {
        if ((it & 63u) == 0 && give_up(p, call, t0)) { dead = true; break; }
        __builtin_amdgcn_s_sleep(1);
      }
    }
    if (dead)
      for (u32 pc = lane; pc < 512u; pc += 64) { const u32 l = piece_link(pc >> 8, pc & 255u);
        if ((l > 3 ? l ^ ((pc >> 8) + 1) : l) == (u32)d) *(u32*)(p.out + c_offset(rank, pc)) = 0x7fc07fc0u; }
    while (lds_peek(started) + LAM <= k) __builtin_amdgcn_s_sleep(1);
    ENQUEUE(0u, 0u, 0u);
    return;
  }
  if (w == 0) {
    if (ncu > NPULL / 7) loader<INFL, NJ6>(p, lds, d, call, lane); else loader<INFL5, NJ5>(p, lds, d, call, lane);
    return;
  }
  if (ncu > NPULL / 7) decoder<NJ6>(p, lds, d, call, lane); else decoder<NJ5>(p, lds, d, call, lane);
}

__device__ __forceinline__ int wait_chunks(CParams& p, const u32* f, u32 o0, u32 o1, u32 o2, int need, u32 call) {
  const u64 t0 = __builtin_amdgcn_s_memrealtime();
  __builtin_amdgcn_s_waitcnt(0xc07f);
  for (u32 it = 1;; ++it) {
    const u32 a = __hip_atomic_load(f + o0, __ATOMIC_RELAXED, __HIP_MEMORY_SCOPE_SYSTEM);
    const u32 b = __hip_atomic_load(f + o1, __ATOMIC_RELAXED, __HIP_MEMORY_SCOPE_SYSTEM);
    const u32 c = __hip_atomic_load(f + o2, __ATOMIC_RELAXED, __HIP_MEMORY_SCOPE_SYSTEM);
    const u64 miss = __ballot(a < call || b < call || c < call);
    const int n = miss ? __builtin_ctzll(miss) : NCH;
    if (n >= need) return n;
    if ((it & 63u) == 0 && give_up(p, call, t0)) return NCH;
    __builtin_amdgcn_s_sleep(2);
  }
}
#define FL_ADDR(next) do { fbase = (next); const u32 c_ = fbase + (lane & 15); fvo = foff + 4 * (c_ < 63u ? c_ : 63u); } while (0)
#define FL_ISSUE() asm volatile("s_mov_b32 m0, %2\n\ts_nop 0\n\tglobal_load_lds_dword %0, %1 sc0 sc1" :: "v"(fvo), "s"(f2), "s"(lds0 + LDS_FLG + w * 256) : "memory")
#define FL_READ() fval = *(const volatile LAS u32*)(lds + LDS_FLG + w * 256 + lane * 4)
#define FL_USE() do { u64 m_ = __ballot(fval < call); m_ |= m_ >> 32; const u32 h_ = (u32)m_ | ((u32)m_ >> 16) | 0x10000u; \
    const int n_ = fbase + __builtin_ctz(h_); avail = n_ > avail ? n_ : avail; } while (0)
__device__ __forceinline__ u32 cvt_pk(float lo, float hi) { u32 r; asm("v_cvt_pk_bf16_f32 %0, %1, %2" : "=v"(r) : "v"(lo), "v"(hi)); return r; }
#define QNAN __builtin_nanf("")
typedef float f32x2 __attribute__((ext_vector_type(2)));
__device__ __forceinline__ u32 silu2(float a, float b) {
  const f32x2 t = (f32x2){__builtin_amdgcn_fmed3f(a, -60.0f, 60.0f), __builtin_amdgcn_fmed3f(b, -60.0f, 60.0f)} * -1.44269504088896341f;
  const f32x2 e = (f32x2){__builtin_amdgcn_exp2f(t.x), __builtin_amdgcn_exp2f(t.y)} + 1.0f;
  const f32x2 r = (f32x2){a, b} * (f32x2){__builtin_amdgcn_rcpf(e.x), __builtin_amdgcn_rcpf(e.y)};
  return cvt_pk(r.x, r.y);
}
__device__ __forceinline__ u32x4 finish8(const f32x4& u, const f32x4& v) {
  const auto s0 = __builtin_amdgcn_permlane16_swap(silu2(u[0], u[1]), silu2(v[0], v[1]), false, false);
  const auto s1 = __builtin_amdgcn_permlane16_swap(silu2(u[2], u[3]), silu2(v[2], v[3]), false, false);
  return (u32x4){s0[0], s1[0], s0[1], s1[1]};
}
__device__ __forceinline__ bool timed_out(CParams& p, u32 call) {
  return rfl(__hip_atomic_load((u32*)(p.ws[0] + TMO_OFF), __ATOMIC_RELAXED, __HIP_MEMORY_SCOPE_AGENT)) == call;
}
constexpr int HT = 16384;
#define A_SETUP(own) \
  const u32 sCa = (own) ? 256u : 32768u, sKa = (own) ? 128u : 2048u; \
  const u32 voffA0 = (own) ? (u32)(w * 16 + (lane >> 3)) * (KD * 2) + (((lane & 7) ^ (lane >> 4)) << 4) : (u32)(w * 4096 + lane * 16); \
  const u32 voffA1 = (own) ? (voffA0 ^ 64u) + 8 * KD * 2 - 1024 : voffA0;        \
  const u32 lds0 = (u32)(size_t)lds, ldsA = lds0 + w * 2048, ldsB = lds0 + w * 1024; \
  u32 aoff[2]; \
  aoff[0] = lds0 + wr * 8192 + ((own) ? (lane & 15) * 128 + (((lane >> 4) ^ ((lane >> 1) & 7)) << 4) : lane * 16); \
  aoff[1] = (own) ? aoff[0] ^ 64u : aoff[0] + 1024;
#define B_READ_OFF(n, t) (256 * (8 * (lane >> 4) + 4 * (t) + ((lane & 15) >> 2)) + 8 * (lane & 1) + \
    16 * ((4 * wc + 2 * (n) + ((lane & 3) >> 1)) ^ ((((lane & 15) >> 2) << 2) | ((2 * (lane >> 4) + (t)) & 3))))
#define VOFF_W ((u32)(4 * w + (lane >> 4)) * 2048u + (((lane & 15) ^ (((lane >> 4) << 2) | (w & 3))) << 4))
#define GB " sc0 sc1"
#define STA2(buf, gp, v0, v1) asm volatile("s_mov_b32 m0, %3\n\ts_nop 0\n\tglobal_load_lds_dwordx4 %0, %2" GB "\n\tglobal_load_lds_dwordx4 %1, %2 offset:1024" GB \
    :: "v"(v0), "v"(v1), "s"(gp), "s"(ldsA + (buf)) : "memory")
#define STA(buf, gp) STA2(buf, gp, voffA0, voffA1)
#define STB(buf, gp) asm volatile("s_mov_b32 m0, %3\n\ts_nop 0\n\tglobal_load_lds_dwordx4 %0, %1" GB "\n\ts_add_u32 m0, m0, 0x2000\n\ts_nop 0\n\tglobal_load_lds_dwordx4 %0, %2" GB \
    :: "v"(voffB), "s"(gp), "s"((gp) + sIb), "s"(ldsB + (buf)) : "memory", "scc")
#define LDA(dst, ad, imm) do { _Pragma("unroll") for (int m = 0; m < MF; ++m) _Pragma("unroll") for (int k = 0; k < 2; ++k) \
    dst[m][k] = *(const LAS s16x8*)(size_t)(ad[k] + (imm) + m * 2048); } while (0)
#define LDB(dst, ad, imm) do { _Pragma("unroll") for (int n = 0; n < 2; ++n) _Pragma("unroll") for (int k = 0; k < 2; ++k) { \
    const s16x4 lo_ = __builtin_amdgcn_ds_read_tr16_b64_v4i16((LAS s16x4*)(size_t)(ad[n][0] + (imm) + k * 8192)); \
    const s16x4 hi_ = __builtin_amdgcn_ds_read_tr16_b64_v4i16((LAS s16x4*)(size_t)(ad[n][1] + (imm) + k * 8192)); \
    dst[n][k] = __builtin_shufflevector(lo_, hi_, 0, 1, 2, 3, 4, 5, 6, 7); } } while (0)
#define MMAX(ac, At, Bt, EXTRA) do { __builtin_amdgcn_s_setprio(1); EXTRA; _Pragma("unroll") for (int m = 0; m < MF; ++m) _Pragma("unroll") for (int n = 0; n < 2; ++n) \
    _Pragma("unroll") for (int k = 0; k < 2; ++k) ac[m][n] = __builtin_amdgcn_mfma_f32_16x16x32_bf16(__builtin_bit_cast(bf8, Bt[n][k]), __builtin_bit_cast(bf8, At[m][k]), ac[m][n], 0, 0, 0); \
    __builtin_amdgcn_s_setprio(0); } while (0)
#define MMA(ac, At, Bt) MMAX(ac, At, Bt, (void)0)
#define WEAVE do { _Pragma("unroll") for (int i_ = 0; i_ < 4 * MF - 1; ++i_) { __builtin_amdgcn_sched_group_barrier(0x8, 1, 0); __builtin_amdgcn_sched_group_barrier(0x6, 1, 0); } } while (0)

#define WEAVE2 do { _Pragma("unroll") for (int i_ = 0; i_ < 4 * MF - 1; ++i_) { __builtin_amdgcn_sched_group_barrier(0x8, 1, 0); __builtin_amdgcn_sched_group_barrier(0x6, 2, 0); } } while (0)
template <int MF, bool ALLOWN> __device__ __forceinline__ void gemm_wide(CParams& p, LAS char* lds, int F0, int tn, u32 call, int w, int lane) {
  const int wr = w >> 2, wc = w & 3;
  const bool own = ALLOWN || F0 < 64;
  const char* Ab = uni(own ? p.x : p.ws[0] + DEC_OFF);
  const char* Bb = uni(p.w + tn * 512);
  A_SETUP(own)
  const int f0 = F0 + (w < 2 * MF ? w : 0);
  u32 vh[2][2];
#pragma unroll
  for (int h = 0; h < 2; ++h) {
    const int f = f0 + 2 * MF * h;
    const u32 fo = own ? (u32)(f - w) * (16 * KD * 2) : (f >> 3) * (u32)PANEL_B + (u32)((f & 7) - w) * 4096u;
    vh[h][0] = w < 2 * MF ? fo + voffA0 : rfl(fo + voffA0); vh[h][1] = w < 2 * MF ? fo + voffA1 : vh[h][0];
  }
  aoff[0] -= wr * (4 - MF) * 2048; aoff[1] -= wr * (4 - MF) * 2048;
#define STAH(buf, gp, h) STA2(buf, gp, vh[h][0], vh[h][1])
  const u32 voffB = VOFF_W; constexpr int sIb = 65536;
  const u32* flg = (const u32*)(p.ws[0] + F2_OFF) + f0 * NCH + lane;
  const char* f2 = uni(p.ws[0] + F2_OFF);
  const u32 foff = (f0 + 2 * MF * (lane >> 5)) * NCH * 4;
  int fbase = 0, npoll = 0; u32 fval = 0, fvo = 0;
  u32 boff[2][2];
#pragma unroll
  for (int n = 0; n < 2; ++n)
#pragma unroll
    for (int t = 0; t < 2; ++t) { boff[n][t] = lds0 + 4 * HT + B_READ_OFF(n, t); asm volatile("" : "+v"(boff[n][t])); }
#define SA(b, h) (((b) * 2 + (h)) * HT)
#define SB(b, h) ((4 + (b) * 2 + (h)) * HT)
#define WAIT_LA asm volatile("s_waitcnt lgkmcnt(%0)" :: "i"(2 * MF) : "memory")
  f32x4 acc[2][2][MF][2];
#pragma unroll
  for (int a = 0; a < 2; ++a)
#pragma unroll
    for (int b = 0; b < 2; ++b)
#pragma unroll
      for (int m = 0; m < MF; ++m)
#pragma unroll
        for (int n = 0; n < 2; ++n) acc[a][b][m][n] = (f32x4){0.f, 0.f, 0.f, 0.f};
  s16x8 At[MF][2], B0[2][2], B1[2][2];
  int avail = own ? NCH : wait_chunks(p, flg, 0, 2 * MF * NCH, 0, 1, call);
  STAMP(3); STAMPC(8);
  __builtin_amdgcn_s_waitcnt(0x0f70);
  if (!ALLOWN) { FL_ADDR(1); FL_ISSUE(); }
  STB(SB(0, 0), Bb); STAH(SA(0, 0), Ab, 0); STB(SB(0, 1), Bb + 256); STAH(SA(0, 1), Ab, 1);
  if (wr == 1) BAR;
  WAIT_V(4); BAR;
  STB(SB(1, 0), Bb + 131072); STAH(SA(1, 0), Ab + sKa, 0); STB(SB(1, 1), Bb + 131072 + 256);
  WAIT_V(6); BAR;
  const u32 voffB2 = voffB + 65536;
  const char* g = Ab + sKa;
#define STBW(buf) asm volatile("s_mov_b32 m0, %3\n\ts_nop 0\n\tglobal_load_lds_dwordx4 %0, %2\n\ts_add_u32 m0, m0, 0x2000\n\ts_nop 0\n\tglobal_load_lds_dwordx4 %1, %2" \
    :: "v"(voffB), "v"(voffB2), "s"(g), "s"(ldsB + (buf)) : "memory", "scc")
#define NEXT(ac, Bt, expr) do { SCHED; MMAX(ac, At, Bt, g = (expr); WEAVE); asm volatile("" : "+s"(g)); BAR; SCHED; } while (0)
#pragma unroll 1
  for (int c = 0; c < NCH; ++c) {
    const int cn = c + 1 < NCH ? c + 1 : c;
    LDB(B0, boff, SA(0, 0)); SCHED; LDA(At, aoff, SA(0, 0)); STAH(SA(1, 1), g, 1);
    WAIT_LA; BAR; WAIT_L(0); NEXT(acc[0][0], B0, Bb + (size_t)cn * 262144);
    const char* b2 = g;
    LDB(B1, boff, SA(0, 1)); if (!ALLOWN) FL_READ(); STBW(SB(0, 0));
    BAR; WAIT_L(0);
    if (ALLOWN) NEXT(acc[0][1], B1, Ab + (size_t)(cn * sCa));
    else {
      SCHED; MMAX(acc[0][1], At, B1, g = Ab + (size_t)(cn * sCa); FL_USE(); FL_ADDR(cn + 1); WEAVE2); asm volatile("" : "+s"(avail), "+s"(g)); asm volatile("" : "+v"(fvo)); BAR; SCHED;
      if (avail <= cn) { avail = wait_chunks(p, flg, 0, 2 * MF * NCH, 0, cn + 1, call); ++npoll; }
    }
    const char* a2 = g;
    LDA(At, aoff, SA(0, 1)); STAH(SA(0, 0), g, 0);
    BAR; WAIT_L(0); NEXT(acc[1][0], B0, b2 + 256);
    STBW(SB(0, 1));
    if (ALLOWN) WAIT_V(6); else { FL_ISSUE(); WAIT_V(7); }
    BAR; MMA(acc[1][1], At, B1); BAR;
    LDB(B0, boff, SA(1, 0)); SCHED; LDA(At, aoff, SA(1, 0)); STAH(SA(0, 1), a2, 1);
    WAIT_LA; BAR; WAIT_L(0); NEXT(acc[0][0], B0, b2 + 131072);
    LDB(B1, boff, SA(1, 1)); STBW(SB(1, 0));
    BAR; WAIT_L(0); NEXT(acc[0][1], B1, a2 + sKa);
    const char* a3 = g;
    LDA(At, aoff, SA(1, 1)); STAH(SA(1, 0), g, 0);
    BAR; WAIT_L(0); NEXT(acc[1][0], B0, b2 + 131072 + 256);
    STBW(SB(1, 1));
    WAIT_V(6); BAR; MMA(acc[1][1], At, B1); BAR;
    g = a3;
  }
  if (wr == 0) BAR;
  STAMP(4); STAMPC(9); STAMPV(10, npoll);
  if (!own && timed_out(p, call)) {
#pragma unroll
    for (int i = 0; i < 8 * MF; ++i) acc[i & 1][(i >> 1) & 1][i >> 3][(i >> 2) & 1] = (f32x4){QNAN, QNAN, QNAN, QNAN};
  }
  int tz = threadIdx.x; asm volatile("" : "+v"(tz));
  char* ob = p.out + ((size_t)(tz & 15) * NS + tn * 256 + wc * 32 + (tz & 16) + ((tz & 32) >> 2)) * 2;
#pragma unroll
  for (int ai = 0; ai < 2; ++ai)
#pragma unroll
    for (int m = 0; m < MF; ++m) {
      const int f = F0 + 2 * MF * ai + MF * wr + m;
      char* oa = ob + (size_t)(slot_row(p.rank, f >> 3) + (f & 7) * 16) * (NS * 2);
#pragma unroll
      for (int bj = 0; bj < 2; ++bj) st_out(oa + bj * 256, finish8(acc[ai][bj][m][0], acc[ai][bj][m][1]));
    }
  STAMP(5);
}

__device__ __forceinline__ void gemm_small(CParams& p, LAS char* lds, int s, int cp, u32 call, int w, int lane) {
  const int wr = w >> 2, wc = w & 3;
  constexpr int MF = 4;
  const bool own = s < 8;
  const char* Ab = uni(own ? p.x + (size_t)s * PANEL_B : p.ws[0] + DEC_OFF + (size_t)s * PANEL_B);
  const char* Bb = uni(p.ws[0] + DEC_OFF + (size_t)cp * 2 * PANEL_B);
  A_SETUP(own)
  const u32 voffB = w * 4096 + lane * 16;
  const u32* flg = (const u32*)(p.ws[0] + F2_OFF) + w * NCH + lane;
  const u32 fb0 = cp * 2 * 8 * NCH, fb1 = fb0 + 8 * NCH, fa = own ? fb0 : s * 8 * NCH;
  const char* f2 = uni(p.ws[0] + F2_OFF);
  const u32 foff = ((lane & 32 ? fa : (lane & 16 ? fb1 : fb0)) + w * NCH) * 4;
  int fbase = 0, npoll = 0; u32 fval = 0, fvo = 0;
  constexpr u32 TB = 3 * HT;
  const u32 boff = lds0 + wc * 4096 + lane * 16;
  u32 bc = boff, bn = boff + TB, ac[2] = {aoff[0], aoff[1]};
  s16x8 At[4][2], B0[2][2], B1[2][2];
  Enc e; e.init(p, lds, call, w, lane);
  asm volatile("" : "+v"(e.row));
  constexpr int trip = 1;
  {
  f32x4 acc[2][4][2];
#pragma unroll
  for (int b = 0; b < 2; ++b)
#pragma unroll
    for (int m = 0; m < 4; ++m)
#pragma unroll
      for (int n = 0; n < 2; ++n) acc[b][m][n] = (f32x4){0.f, 0.f, 0.f, 0.f};
  if (trip) {
  int avail = wait_chunks(p, flg, fa, fb0, fb1, 2, call);
  STAMP(3); STAMPC(8);
  __builtin_amdgcn_s_waitcnt(0x0f70);
#define LD2(m0v, v0, v1, gp) asm volatile("s_mov_b32 m0, %3\n\ts_nop 0\n\tglobal_load_lds_dwordx4 %0, %2" GB "\n\tglobal_load_lds_dwordx4 %1, %2 offset:1024" GB \
    :: "v"(v0), "v"(v1), "s"(gp), "s"(m0v) : "memory")
#define LD1(m0v, v0, gp, o) asm volatile("s_mov_b32 m0, %2\n\ts_nop 0\n\tglobal_load_lds_dwordx4 %0, %1 offset:" #o GB :: "v"(v0), "s"(gp), "s"(m0v) : "memory")
#define LDF(dst, ad, imm) do { _Pragma("unroll") for (int n = 0; n < 2; ++n) _Pragma("unroll") for (int k = 0; k < 2; ++k) \
    dst[n][k] = *(const LAS s16x8*)(size_t)((ad) + (imm) + n * 2048 + k * 1024); } while (0)
  FL_ADDR(2);
  LD2(ldsA + HT, voffB, voffB, Bb); LD2(ldsA, voffA0, voffA1, Ab); LD2(ldsA + 2 * HT, voffB, voffB, Bb + PANEL_B); LD2(ldsA + TB + HT, voffB, voffB, Bb + 2048);
  LD2(ldsA + TB, voffA0, voffA1, Ab + sKa); FL_ISSUE(); LD2(ldsA + TB + 2 * HT, voffB, voffB, Bb + 2048 + PANEL_B); LD2(ldsA + 2 * TB + HT, voffB, voffB, Bb + 32768);
  WAIT_V(11); BAR;
  LDF(B0, bc, HT);
  WAIT_L(0);
  if (wr == 1) BAR;
  u32 s0 = 0, s1 = TB, s2 = 2 * TB;
  const char* gA = Ab + sCa; const char* gB1 = Bb + 32768 + PANEL_B; const char* gB0 = Bb + 32768 + 2048;
  u32 mA = ldsA + s2, mB1 = mA + 2 * HT, mB0 = ldsA + HT;
#define PIN do { asm volatile("" : "+s"(gA), "+s"(gB1), "+s"(gB0), "+s"(mA), "+s"(mB1), "+s"(mB0), "+s"(avail)); \
    asm volatile("" : "+v"(fvo), "+v"(bc), "+v"(bn), "+v"(ac[0]), "+v"(ac[1])); } while (0)
#define KTILE(dA, dB1, dB0, V1, V2, ISS, CHECK, RD, X2) do { \
    LD2(mA, voffA0, voffA1, gA); LD1(mB1, voffB, gB1, 0); ISS; LDA(At, ac, 0); \
    WAIT_V(V1); BAR; WAIT_L(0); SCHED; \
    MMAX(acc[0], At, B0, gA += (dA); mA = ldsA + s0; ac[0] = aoff[0] + s1; ac[1] = aoff[1] + s1; WEAVE); PIN; BAR; SCHED; \
    CHECK; \
    LD1(mB1, voffB, gB1, 1024); LD2(mB0, voffB, voffB, gB0); LDF(B1, bc, 2 * HT); SCHED; LDF(B0, bn, HT); \
    WAIT_V(V2); WAIT_L(4); RD; BAR; WAIT_L(0); SCHED; \
    MMAX(acc[1], At, B1, gB1 += (dB1); gB0 += (dB0); mB1 = mA + 2 * HT; mB0 = ldsA + s1 + HT; bc = bn; bn = boff + s2; \
         { const u32 s_ = s0; s0 = s1; s1 = s2; s2 = s_; } X2; WEAVE2); PIN; BAR; SCHED; } while (0)
  for (int c = 0; c < NCH; ++c) {
    const int c2 = c + 2 < NCH ? c + 2 : NCH - 1;
    const int nx = c + 2 < NCH;
    KTILE((int)sKa, 2048, (nx ? 32768 : 0) - 2048, 10, 9, (void)0, (void)0, FL_READ(), FL_USE(); FL_ADDR(c2 + 1));
    KTILE((int)(nx ? sCa : 0u) - (int)sKa, (nx ? 32768 : 0) - 2048, 2048, 10, 10, FL_ISSUE(),
          if (avail <= c2) { avail = wait_chunks(p, flg, fa, fb0, fb1, c2 + 1, call); ++npoll; LASTPOLL; }, (void)0, (void)0);
  }
  if (wr == 0) BAR;
  }
  STAMP(4); STAMPC(9); STAMPV(10, npoll);
  const u32 tmo = __hip_atomic_load((u32*)(p.ws[0] + TMO_OFF), __ATOMIC_RELAXED, __HIP_MEMORY_SCOPE_AGENT);
  int tz = threadIdx.x; asm volatile("" : "+v"(tz));
  u32x4 X[4];
#pragma unroll
  for (int m = 0; m < 4; ++m) { X[m] = finish8(acc[0][m][0], acc[0][m][1]); asm volatile("" : "+v"(X[m])); }
  if (rfl(tmo) == call) {
#pragma unroll
    for (int m = 0; m < 4; ++m) { X[m] = (u32x4){0x7fc07fc0u, 0x7fc07fc0u, 0x7fc07fc0u, 0x7fc07fc0u}; acc[1][m][0] = acc[1][m][1] = (f32x4){QNAN, QNAN, QNAN, QNAN}; }
  }
  {
    STAMP(11);
    WAIT_V(0);
    e.lane = tz & 63;
    const u32 code = (s * 2 + (cp & 1)) * 4 + wr, seg = 20 * 128 + (cp >> 1) * 256 + code;
#pragma unroll 1
    for (int bj = 0; bj <= trip; ++bj) {
      e.round<false>(X, seg + 2 * bj, trip ? 1u << piece_link(cp >> 1, code + 2 * bj) : 0u, [] {});
      e.draw(f2 + (MBX_OFF - F2_OFF));
      if (bj == 0) {
#pragma unroll
        for (int m = 0; m < 2; ++m) { asm volatile("" : "+v"(acc[1][m][0]), "+v"(acc[1][m][1])); X[m] = finish8(acc[1][m][0], acc[1][m][1]); asm volatile("" : "+v"(X[m])); }
      }
      e.finish_mail();
      if (bj == 0) {
        STAMP(12);
#pragma unroll
        for (int m = 2; m < 4; ++m) { asm volatile("" : "+v"(acc[1][m][0]), "+v"(acc[1][m][1])); X[m] = finish8(acc[1][m][0], acc[1][m][1]); }
      }
    }
  }
  }
  STAMP(5);
}

__global__ __launch_bounds__(512, 2) void ag_gemm(Params) {
  CParams& p = *(CParams*)__builtin_amdgcn_kernarg_segment_ptr();
  extern __shared__ __attribute__((aligned(16))) char smem[];
  LAS char* lds = (LAS char*)smem + 1024;
  const int lane = threadIdx.x & 63, w = rfl(threadIdx.x >> 6), sg = w / NB;
  const int b = blockIdx.x;
  u64 px, pw, ws0;
  asm volatile("s_load_dwordx2 %0, %3, 0x40\n\ts_load_dwordx2 %1, %3, 0x48\n\ts_load_dwordx2 %2, %3, 0x0" : "=&s"(px), "=&s"(pw), "=&s"(ws0) : "s"(&p));
  static_assert(__builtin_offsetof(Params, x) == 0x40 && __builtin_offsetof(Params, w) == 0x48, "arguments");
  STAMP(0);
  const Role ro = role_of(b);
  Jobs jb; jb.init(ro.u, sg);
  const int nr = jb.nr; int j0 = jb.job(0), j1 = jb.job(1);
  asm volatile("s_waitcnt lgkmcnt(0)" : "+s"(px), "+s"(pw), "+s"(ws0), "+s"(j0), "+s"(j1));
  u32* cnt = (u32*)(__attribute__((address_space(1))) u32*)(ws0 + CNT_OFF + b * 128);
  u32 call;
  asm volatile("s_load_dword %0, %1, 0x0" : "=&s"(call) : "s"(cnt));
  load_job((const char*)px, (const char*)pw, lds, j0, 0, w, lane);
  if (w == 4) ((LAS u32*)(lds + LDS_MAIN))[lane] = 0;
  if (w < 4) ((LAS u32*)smem)[threadIdx.x] = mask_entry(threadIdx.x);
  asm volatile("s_waitcnt lgkmcnt(0)" : "+s"(call) :: "memory");
  call += 1;
  LBAR;
  load_job((const char*)px, (const char*)pw, lds, j1, 1, w, lane);
  if (threadIdx.x == 511) *cnt = call;
  if (nr) {
    encode_jobs(p, lds, call, w, lane, jb);
  }
  if (ro.kind == 0) gemm_small(p, lds, ro.a, ro.c, call, w, lane);
  else if (ro.kind == 1) gemm_wide<3, false>(p, lds, ro.a, ro.c, call, w, lane);
  else if (ro.kind == 2) gemm_wide<2, true>(p, lds, ro.a, ro.c, call, w, lane);
  else if (ro.kind == 3) {
    pull_role(p, lds, link_of(ro.a), ro.a / 7, (NPULL - ro.a % 7 + 6) / 7, call, w, lane);
  }
}
}

extern "C" void kernel_launch(int rank, int nranks, ncclComm_t comm, hipStream_t stream,
                              void* const* d_in, const int* in_sizes, int n_in,
                              void* d_out, int out_size, const int* in_dtypes, int out_dtype,
                              void* const* d_out_all, void* const* d_ws_all, size_t ws_bytes) {
  using namespace agg;
  if (nranks != NR || (unsigned)rank >= (unsigned)NR || ws_bytes < WS_NEED || n_in < 2) { fprintf(stderr, "ag_gemm: unexpected configuration\n"); return; }
  Params p;
  for (int i = 0; i < NR; ++i) p.ws[i] = (char*)d_ws_all[rank ^ i];
  p.x = (const char*)d_in[0]; p.w = (const char*)d_in[1]; p.out = (char*)d_out; p.rank = rank; p.pad = 0;
  hipError_t e = hipFuncSetAttribute((const void*)ag_gemm, hipFuncAttributeMaxDynamicSharedMemorySize, LDS_BYTES);
  if (e != hipSuccess) { fprintf(stderr, "HIP error %s\n", hipGetErrorString(e)); return; }
  hipLaunchKernelGGL(ag_gemm, dim3(GRID), dim3(512), LDS_BYTES, stream, p);
  e = hipGetLastError();
  if (e != hipSuccess) fprintf(stderr, "HIP error %s\n", hipGetErrorString(e));
}
```

```cpp
#include <hip/hip_runtime.h>
#include <rccl/rccl.h>
#include <cstdio>

namespace agg {
#define LAS __attribute__((address_space(3)))
typedef unsigned u32;
typedef unsigned long long u64;
typedef __attribute__((address_space(1))) unsigned long long gu64;
typedef unsigned u32x4 __attribute__((ext_vector_type(4)));
typedef unsigned u32x2 __attribute__((ext_vector_type(2)));
typedef short s16x4 __attribute__((ext_vector_type(4)));
typedef short s16x8 __attribute__((ext_vector_type(8)));
typedef unsigned short u16x2 __attribute__((ext_vector_type(2)));
typedef float f32x4 __attribute__((ext_vector_type(4)));
typedef __bf16 bf8 __attribute__((ext_vector_type(8)));

constexpr int INFL = 9, INFL5 = 11;
constexpr u32 LAP = 3, LAM = 3;
constexpr u32 WREF = 24;
constexpr int LEAD = 5;
constexpr int OF1 = 5, OF2 = 9, OF3 = 9, OF4 = 8, OS1 = 2, OS4 = 7, RAMP = 44;
constexpr u64 FDIR = 0xffff000000000000ull;
constexpr u32 REL0 = 0xa4, REL1 = 0xa5, REL2 = 0x35, RG = 20;
constexpr int UPR = 1, MIDR = 2, LATE0 = 12, LATE1 = 16;

constexpr int NR = 8, KD = 8192, NS = 1024;
constexpr int NCH = 64;
constexpr int NB = 4;
constexpr int SEG_STRIDE = NB * 4096 + 1024;
constexpr int SEG_P = NB * 2048, SEG_H = NB * 2560, SEG_V = SEG_H + NB * 4;
constexpr int NSLOT = 8, QD = 16;
constexpr int GRID = 256, NPULL = 40;
constexpr int LDS_MAIN = 147456, LDS_FLG = LDS_MAIN + 1024, LDS_VAR = LDS_FLG + 2048, LDS_BYTES = 1024 + LDS_VAR + 8 * 1536;
static_assert(NSLOT * SEG_STRIDE <= LDS_MAIN, "ring");
constexpr u32 XCAP = 64;
constexpr int NSRC = 28 * 128, NJOB = 16 * 128;

constexpr size_t PANEL_B = (size_t)2 << 20;
constexpr size_t DEC_OFF = 0;
constexpr size_t ENC_OFF = 32 * PANEL_B;
constexpr size_t ENC_SZ = (size_t)NSRC * SEG_STRIDE;
constexpr size_t F1_OFF = ENC_OFF + 2 * ENC_SZ;
constexpr size_t F1_SZ = (size_t)NR * NSRC * 8;
constexpr size_t F2_OFF = F1_OFF + 2 * F1_SZ;
constexpr size_t F2_SZ = (size_t)32 * 8 * NCH * 4;
constexpr size_t CNT_OFF = F2_OFF + F2_SZ;
constexpr size_t TKT_OFF = CNT_OFF + GRID * 128;
constexpr size_t MBX_OFF = TKT_OFF + NR * 128;
constexpr size_t TMO_OFF = MBX_OFF + NR * 128;
constexpr size_t WS_NEED = TMO_OFF + 128;
constexpr u64 DEADLINE = 50000000ull;

struct Params { char* ws[NR];   const char* x; const char* w; char* out; int rank; int pad; };
typedef const __attribute__((address_space(4))) Params CParams;

__host__ __device__ __forceinline__ int cls(int r) { return (r ^ (r >> 1)) & 1; }
__host__ __device__ __forceinline__ int slot_row(int r, int s) {
  const int h4 = 4 * cls(r), m = r & 1;
  int owner, piece;
  if (s < 8) { owner = r; piece = s; }
  else if (s < 12) { owner = r ^ 3; piece = 2 + (s & 1) + ((s & 2) << 1); }
  else {
    const int y = r ^ (4 + (s & 3)), a = (s - 12) >> 2;
    if (a == 4) { owner = r ^ 3; piece = 4 * cls(y) + (y & 1); } else { owner = y; piece = h4 + (a == 0 ? m : (a == 3 ? m ^ 1 : a + 1)); }
  }
  return owner * 1024 + piece * 128;
}
struct Role { int kind, a, c, u; };
constexpr int NPA = 8 + (32 - LATE1) * 4, NPB = 88 + (32 - LATE0) * 4, NPC = 216;
__host__ __device__ __forceinline__ Role role_of(int b) {
  if ((b & 1) && (b >> 5) == 3) b = (b & 30) | (7 << 5); else if (!(b & 1) && (b >> 5) == 7) b = (b & 31) | 1 | (3 << 5);
  const int xi = (b & 7) >> 1, j = b >> 3, jj = j >> 2;
  if (b & 1) { const int s = jj < 2 ? 2 * xi + jj : 4 * jj + xi; return {0, s, j & 3, (s >= LATE1 ? 8 : 88) + (31 - s) * 4 + (j & 3)}; }
  if (j < 16) return {1, 64 + 12 * (4 * xi + jj), j & 3, NPA + 16 + j * 4 + xi};
  if (j < 20) return {1, 12 * xi, j & 3, NPA + (j - 16) * 4 + xi};
  if (j < 22) { const int i = 2 * xi + j - 20; return {2, 48 + 8 * (i >> 2), i & 3, i}; }
  return {3, (j - 22) * 4 + xi, 0, -1};
}
__host__ __device__ __forceinline__ int link_of(int o) { return 1 + (o + 2) % 7; }
constexpr int LMAX = 768;
struct Lists { unsigned short e[3][LMAX]; unsigned short n[3]; };
constexpr int key(int c, int i, const int* pace, const int* off, const int* cap) {
  const int k = c * pace[i] + off[i];
  return cap && k > c * pace[0] + cap[i] ? c * pace[0] + cap[i] : k;
}
constexpr void merge(Lists& L, int t, int ns, const u64* has, const int* pace, const int* off, const int* cap = nullptr) {
  int c[8] = {};
  for (;;) {
    int b = -1;
    for (int i = 0; i < ns; ++i) {
      while (c[i] < 64 && !((has[i] >> c[i]) & 1)) ++c[i];
      if (c[i] < 64 && (b < 0 || key(c[i], i, pace, off, cap) < key(c[b], b, pace, off, cap))) b = i;
    }
    if (b < 0) break;
    for (int g = 0; g < 2; ++g) L.e[t][L.n[t]++] = (unsigned short)(b << 7 | c[b] << 1 | g);
    ++c[b];
  }
}
__host__ __device__ constexpr u32 piece_link(u32 o, u32 code) {
  const u32 s = code >> 3, set = s < 12 ? REL0 : (s < RG ? REL1 : REL2);
  return (set >> (code & 7u)) & 1u ? 4u + (((code & 1u) + ((code >> 1) & 1u) + ((code >> 1) & 2u) + s + (s >> 2)) & 3u) : o + 1;
}
constexpr u32 mail_count(u32 d) {
  u32 n = 0;
  for (u32 pc = 0; pc < 512; ++pc) { const u32 o = pc >> 8, l = piece_link(o, pc & 255u); n += (l == d) + (l > 3 && (l ^ (o + 1)) == d); }
  return n;
}
constexpr u32 NMAIL0 = mail_count(1), NMAIL2 = mail_count(4);
static_assert(NMAIL0 <= 64 * (NPULL / 7) && NMAIL2 <= 64 * (NPULL / 7), "mailbox");
static_assert(mail_count(2) == NMAIL0 && mail_count(5) == NMAIL2 && mail_count(6) == NMAIL2 && mail_count(7) == NMAIL2, "routes");
constexpr Lists make_lists() {
  Lists L = {};
  constexpr u64 ALL = ~0ull;
  { const u64 has[4] = {ALL, ALL, ALL, ALL}; const int pace[4] = {1, 1, 1, 1}, off[4] = {0, 0, LEAD, LEAD}; merge(L, 0, 4, has, pace, off); }
  { const u64 has[8] = {ALL, ALL, ALL, ALL, FDIR, FDIR, FDIR, FDIR}; const int pace[8] = {1, 1, 1, 1, 1, 1, 1, 1}, off[8] = {0, 0, 0, 0, 64, 64, 64, 64};
    merge(L, 1, 8, has, pace, off); }
  { const u64 has[5] = {ALL, ALL, ALL, ALL, ~FDIR}; const int pace[5] = {RAMP, RAMP + OF1 - OS1, RAMP + OF2 - OS1, RAMP + OF3 - OS1, RAMP + OF4 - OS4}, off[5] = {0, OS1 * RAMP, OS1 * RAMP, OS1 * RAMP, OS4 * RAMP},
      cap[5] = {0, OF1 * RAMP, OF2 * RAMP, OF3 * RAMP, OF4 * RAMP};
    merge(L, 2, 5, has, pace, off, cap); }
  return L;
}
constexpr Lists LISTS = make_lists();
static_assert(OF4 > 0 && OS4 > 0, "a forwarded segment is listed behind its first hop");
__host__ __device__ __forceinline__ int list_type(int d) { return d < 3 ? 0 : (d == 3 ? 1 : 2); }
constexpr u32 D_FWD = 1u << 12, D_REL = 1u << 14;
__host__ __device__ __forceinline__ u32 resolve(int d, int r, u32 en) {
  const u32 a = (en >> 7) & 7u, h4 = 4 * cls(r), m = r & 1; u32 sp, dp, fl = 0;
  if (d < 3) { dp = 4u * (d - 1) + a; sp = 8 + dp; }
  else if (d == 3) { if (a < 4) { sp = 2 + (a & 1) + ((a & 2) << 1); dp = 8 + a; } else { const int y = r ^ a; sp = 4 * cls(y) + (y & 1); dp = 24 + a; } }
  else if (a < 3) { sp = h4 + (a == 0 ? m : a + 1); dp = 8 + 4 * a + d; fl = a == 0 && !((FDIR >> ((en >> 1) & 63u)) & 1) ? D_FWD : 0u; }
  else { sp = a == 4 ? 12u + (d ^ 3) : h4 + (m ^ 1); dp = 8 + 4 * a + d; }
  return ((sp * 128u + (en & 127u)) << 16) | fl | (dp << 7) | (en & 127u);
}
__host__ __device__ __forceinline__ u32 input_mask(int r, u32 pn, u32 c) {
  return pn < 8u ? (((pn >> 2) == (u32)cls(r) ? 0x90u : 0x60u) | ((pn & 2u) || ((FDIR >> c) & 1) ? 8u : 0u)) : (pn < 12u ? 2u : 4u);
}


#define WAIT_V(n) asm volatile("s_waitcnt vmcnt(" #n ")" ::: "memory")
#define WAIT_L(n) asm volatile("s_waitcnt lgkmcnt(" #n ")" ::: "memory")
#define DRAIN do { WAIT_V(0); __builtin_amdgcn_s_waitcnt(0x0f70); } while (0)
#define BAR do { __builtin_amdgcn_s_barrier(); asm volatile("" ::: "memory"); } while (0)
#define SCHED __builtin_amdgcn_sched_barrier(0)
#define LBAR do { WAIT_L(0); BAR; } while (0)
template <int N> __device__ __forceinline__ void wait_v() { asm volatile("s_waitcnt vmcnt(%0)" :: "i"(N) : "memory"); }

__device__ __forceinline__ u32 rfl(u32 v) { return __builtin_amdgcn_readfirstlane(v); }
template <class T> __device__ __forceinline__ T* uni(T* q) {
  u64 v = (u64)q; v = ((u64)rfl((u32)(v >> 32)) << 32) | rfl((u32)v);
  asm volatile("s_nop 4" : "+s"(v));
  return (T*)(__attribute__((address_space(1))) T*)v;
}
__device__ __forceinline__ void st4x4(char* base, u32 off, u32x4 a, u32x4 b, u32x4 c, u32x4 d) {
  asm volatile("global_store_dwordx4 %0, %1, %5 sc0 sc1\n\tglobal_store_dwordx4 %0, %2, %5 offset:1024 sc0 sc1\n\t"
               "global_store_dwordx4 %0, %3, %5 offset:2048 sc0 sc1\n\tglobal_store_dwordx4 %0, %4, %5 offset:3072 sc0 sc1\n\ts_nop 1"
               :: "v"(off), "v"(a), "v"(b), "v"(c), "v"(d), "s"(base) : "memory");
}
__device__ __forceinline__ void st4x2(char* base, u32 off, u32x4 a, u32x4 b) {
  asm volatile("global_store_dwordx4 %0, %1, %3 sc0 sc1\n\tglobal_store_dwordx4 %0, %2, %3 offset:1024 sc0 sc1\n\ts_nop 1"
               :: "v"(off), "v"(a), "v"(b), "s"(base) : "memory");
}
__device__ __forceinline__ void st4x1(char* base, u32 off, u32x4 a) {
  asm volatile("global_store_dwordx4 %0, %1, %2 sc0 sc1\n\ts_nop 1" :: "v"(off), "v"(a), "s"(base) : "memory");
}
__device__ __forceinline__ void st_out(char* q, u32x4 a) { asm volatile("global_store_dwordx4 %0, %1, off sc1\n\ts_nop 1" :: "v"(q), "v"(a) : "memory"); }
__device__ __forceinline__ bool give_up(CParams& p, u32 call, u64 t0) {
  u32* tmo = (u32*)(p.ws[0] + TMO_OFF);
  if (rfl(__hip_atomic_load(tmo, __ATOMIC_RELAXED, __HIP_MEMORY_SCOPE_AGENT)) == call) return true;
  if (__builtin_amdgcn_s_memrealtime() - t0 > DEADLINE) {
    __hip_atomic_store(tmo, call, __ATOMIC_RELAXED, __HIP_MEMORY_SCOPE_AGENT);
    WAIT_V(0);
    return true;
  }
  return false;
}
__device__ __forceinline__ gu64* flag_row(CParams& p, u32 call, int lane) {
  u32 lo = 0, hi = 0;
#pragma unroll
  for (int i = 0; i < NR; ++i) { u64 v = (u64)p.ws[i]; asm volatile("" : "+s"(v)); lo = lane == i ? (u32)v : lo; hi = lane == i ? (u32)(v >> 32) : hi; }
  return (gu64*)((((u64)hi << 32) | lo) + F1_OFF + (call & 1) * F1_SZ) + (lane & 7) * NSRC;
}
__device__ __forceinline__ void post_seg(gu64* row, u32 call, u32 seg, u32 len, u32 mask, int lane) {
  if (lane < NR && ((mask >> lane) & 1u)) __hip_atomic_store(row + seg, ((u64)call << 32) | len, __ATOMIC_RELAXED, __HIP_MEMORY_SCOPE_SYSTEM);
}
constexpr u32 MAIL0 = 20 * 128;
__device__ __forceinline__ u32 mail_place(const char* mbx, u32 mask, int lane) {
  u32 t;
  asm volatile("s_mov_b64 exec, %4\n\tglobal_atomic_add %0, %1, %2, %3 sc0\n\ts_mov_b64 exec, -1\n\ts_waitcnt vmcnt(0)" : "=&v"(t) : "v"(lane * 128), "v"(1u), "s"(mbx), "s"((u64)(mask & 0xffu)) : "memory");
  return t;
}
__device__ __forceinline__ void post_mail(gu64* row, u32 call, u32 place, u32 seg, u32 len, u32 mask, int lane) {
  post_seg(row, call, MAIL0 + (lane < 3 ? place % NMAIL0 : place % NMAIL2), (seg << 16) | (len >> 4), mask, lane);
}
__device__ __forceinline__ u32 wave_scan(u32 v) {
  v += __builtin_amdgcn_update_dpp(0u, v, 0x111, 0xf, 0xf, false);
  v += __builtin_amdgcn_update_dpp(0u, v, 0x112, 0xf, 0xf, false);
  v += __builtin_amdgcn_update_dpp(0u, v, 0x114, 0xf, 0xf, false);
  v += __builtin_amdgcn_update_dpp(0u, v, 0x118, 0xf, 0xf, false);
  v += __builtin_amdgcn_update_dpp(0u, v, 0x142, 0xa, 0xf, false);
  v += __builtin_amdgcn_update_dpp(0u, v, 0x143, 0xc, 0xf, false);
  return v;
}
#define BOP(a, b, c, expr) (u32)__builtin_amdgcn_bitop3_b32((a), (b), (c), [] { constexpr u32 A = 0xf0, B = 0xcc, C = 0xaa; (void)A; (void)B; (void)C; return (expr) & 0xffu; }())
#define BFI(m, a, b) BOP(m, a, b, (A & B) | (~A & C))
struct Mv { u32 a0, a1, a2, c8, c16; };
__device__ __forceinline__ u32 byte_prefix_xor(u32 v) {
  v = BOP(v, v << 1, 0xfefefefeu, A ^ (B & C)); v = BOP(v, v << 2, 0xfcfcfcfcu, A ^ (B & C)); return BOP(v, v << 4, 0xf0f0f0f0u, A ^ (B & C));
}
__device__ __forceinline__ Mv calc_masks(u32 m) {
  Mv r;
  r.c16 = __builtin_popcount(m & 0xffffu);
  r.c8 = (u32)__builtin_popcount(m & 0xffu) | ((u32)__builtin_popcount(m & 0xff0000u) << 16);
  u32 mk = BOP(m << 1, 0xfefefefeu, 0u, ~A & B);
  u32 mp = byte_prefix_xor(mk);
  r.a0 = mp & m; m = BOP(m, r.a0, r.a0 >> 1, (A ^ B) | C); mk = BOP(mk, mp, 0u, A & ~B);
  mp = byte_prefix_xor(mk);
  r.a1 = mp & m; m = BOP(m, r.a1, r.a1 >> 2, (A ^ B) | C); mk = BOP(mk, mp, 0u, A & ~B);
  r.a2 = BOP(0x80808080u - mk, 0x80808080u, m, (A ^ B) & C);
  return r;
}
__device__ __forceinline__ u32 mask_entry(u32 b) { const Mv v = calc_masks(b); return v.c16 | (v.a0 << 8) | (v.a1 << 16) | (v.a2 << 24); }
__device__ __forceinline__ Mv move_masks(u32 m) {
  const u32 e0 = *(const LAS u32*)(size_t)((m & 255u) << 2), e1 = *(const LAS u32*)(size_t)(((m >> 8) & 255u) << 2);
  const u32 e2 = *(const LAS u32*)(size_t)(((m >> 16) & 255u) << 2), e3 = *(const LAS u32*)(size_t)((m >> 24) << 2);
  const u32 l02 = __builtin_amdgcn_perm(e2, e0, 0x05040100u), l13 = __builtin_amdgcn_perm(e3, e1, 0x05040100u);
  const u32 h02 = __builtin_amdgcn_perm(e2, e0, 0x07060302u), h13 = __builtin_amdgcn_perm(e3, e1, 0x07060302u);
  Mv r;
  r.a0 = __builtin_amdgcn_perm(l13, l02, 0x07030501u); r.a1 = __builtin_amdgcn_perm(h13, h02, 0x06020400u); r.a2 = __builtin_amdgcn_perm(h13, h02, 0x07030501u);
  r.c8 = l02; r.c16 = e0 + e1;
  return r;
}
__device__ __forceinline__ u32 pext(u32 x, u32 m, const Mv& v) {
  x &= m;
  u32 t = x & v.a0; x = BOP(x, t, t >> 1, (A ^ B) | C);
  t = x & v.a1; x = BOP(x, t, t >> 2, (A ^ B) | C);
  t = x & v.a2; x = BOP(x, t, t >> 4, (A ^ B) | C);
  t = __builtin_amdgcn_perm(0u, x, 0x0c030c01u);
  asm("v_pk_lshlrev_b16 %0, %1, %0" : "+v"(t) : "v"(v.c8));
  x = BOP(x, 0x00ff00ffu, t, (A & B) | C);
  return BOP(x, 0xffffu, (x >> 16) << (v.c16 & 31u), (A & B) | C);
}
__device__ __forceinline__ u32 pdep(u32 x, const Mv& v) {
  x = __builtin_amdgcn_perm(x >> (v.c16 & 31u), x, 0x05040100u);
  u32 t; asm("v_pk_lshrrev_b16 %0, %1, %2" : "=v"(t) : "v"(v.c8), "v"(x));
  x = __builtin_amdgcn_perm(t, x, 0x06020400u);
  x = BFI(v.a2, x << 4, x); x = BFI(v.a1, x << 2, x);
  return BFI(v.a0, x << 1, x);
}
__device__ __forceinline__ void transpose4(u32 (&n)[4]) {
#pragma unroll
  for (int r = 0; r < 2; ++r) { const u32 a = n[r], c = n[r + 2]; n[r] = BFI(0x33333333u, a, c << 2); n[r + 2] = BFI(0x33333333u, a >> 2, c); }
#pragma unroll
  for (int r = 0; r < 4; r += 2) { const u32 a = n[r], c = n[r + 1]; n[r] = BFI(0x55555555u, a, c << 1); n[r + 1] = BFI(0x55555555u, a >> 1, c); }
}
__device__ __forceinline__ void or_bits(LAS u32* area, u32 bit, u32 val) {
  const u64 t = (u64)val << (bit & 31);
  LAS u32* q = area + (bit >> 5);
  if ((u32)t) __hip_atomic_fetch_or(q, (u32)t, __ATOMIC_RELAXED, __HIP_MEMORY_SCOPE_WORKGROUP);
  if ((u32)(t >> 32)) __hip_atomic_fetch_or(q + 1, (u32)(t >> 32), __ATOMIC_RELAXED, __HIP_MEMORY_SCOPE_WORKGROUP);
}
__device__ __forceinline__ u32 mad3(u32 a, u32 c) { u32 r; asm("v_mad_u32_u24 %0, %1, 3, %2" : "=v"(r) : "v"(a), "s"(c)); return r; }
__device__ __forceinline__ u32 get_bits(u32 bit) {
  const LAS u32* q = (const LAS u32*)(size_t)((bit >> 3) & ~3u);
  return __builtin_amdgcn_alignbit(q[1], q[0], bit & 31);
}

struct Enc {
  char* enc; gu64* row; LAS char* lds; u32 call; int w, lane; u32 par, pseg, pmask;
  __device__ __forceinline__ void init(CParams& p, LAS char* l, u32 c, int w_, int lane_) {
    enc = uni(p.ws[0] + ENC_OFF + (c & 1) * ENC_SZ); row = flag_row(p, c, lane_); lds = l; call = c; w = w_; lane = lane_; par = 0; pseg = 0; pmask = 0;
  }
  __device__ __forceinline__ LAS u32* fill(u32 q) const { return (LAS u32*)(lds + LDS_MAIN + 64) + (q & 3u) * 2 + w / NB; }
  __device__ __forceinline__ u32 take_len() { LAS u32* c = fill(par - 1); const u32 n = rfl(*c); *c = 0; return (SEG_V + n * 4 + 15) & ~15u; }
  __device__ __forceinline__ void post() { if ((w & (NB - 1)) == 0 && pmask) post_seg(row, call, pseg, take_len(), pmask, lane); pmask = 0; }
  __device__ __forceinline__ void finish() { DRAIN; LBAR; post(); }
  __device__ __forceinline__ void finish_mail(const char* mbx) {
    LAS u32* meet = fill(par - 1) + 8;
    const u32 place = mail_place(mbx, (w & (NB - 1)) == 0 ? pmask : 0u, lane);
    if ((w & (NB - 1)) == 0) meet[64] = __builtin_amdgcn_readlane(place, __builtin_ctz(pmask));
    u32 n;
    asm volatile("s_mov_b64 exec, 1\n\tds_add_rtn_u32 %0, %1, %2\n\ts_waitcnt lgkmcnt(0)\n\ts_mov_b64 exec, -1" : "=&v"(n) : "v"((u32)(size_t)meet), "v"(1u) : "memory");
    if (rfl(n) == NB - 1) post_mail(row, call, meet[64], pseg, take_len(), pmask, lane);
    pmask = 0;
  }
  template <class F> __device__ __forceinline__ void idle(F&& mid) { if (par) finish(); mid(); ++par; }
  template <bool PIPE, class F> __device__ __forceinline__ void round(const u32x4 (&X)[4], u32 seg, u32 mask, F&& mid) {
    const int rb = w % NB;
    const u32 so = seg * SEG_STRIDE;
    LAS u32* var = (LAS u32*)(lds + LDS_VAR + w * 1536);
    { const u32x4 zero = {0u, 0u, 0u, 0u}; ((LAS u32x4*)var)[lane] = zero; ((LAS u32x2*)var)[128 + lane] = (u32x2){0u, 0u}; }
    asm volatile("" ::: "memory");
    u32 R[16], S[8], sum = 0;
#pragma unroll
    for (int i = 0; i < 16; ++i) { const u32 t = X[i >> 2][i & 3]; R[i] = __builtin_amdgcn_alignbit(t, t, 31); if (i & 1) sum = __builtin_amdgcn_udot4(R[i], 0x01000100u, sum, false); }
#pragma unroll
    for (int q = 0; q < 8; ++q) S[q] = __builtin_amdgcn_perm(R[2 * q + 1], R[2 * q], 0x06040200u);
    if (!PIPE || par == 0) st4x2(enc, so + rb * 2048 + lane * 16, (u32x4){S[0], S[1], S[2], S[3]}, (u32x4){S[4], S[5], S[6], S[7]});
    sum = __builtin_amdgcn_readlane(wave_scan(sum), 63);
    int basei = (int)((sum + 870u) >> 10) - 10;
    basei = basei < 0 ? 0 : (basei > 240 ? 240 : basei);
    const u32 base = (u32)basei;
    const u16x2 bsub = {(unsigned short)(base << 8), (unsigned short)(base << 8)}, top = {0x0f00, 0x0f00};
    u32 V[8], d[4];
#pragma unroll
    for (int q = 0; q < 8; ++q) {
      const u16x2 a = __builtin_elementwise_min((u16x2)(__builtin_bit_cast(u16x2, R[2 * q]) - bsub), top);
      const u16x2 b = __builtin_elementwise_min((u16x2)(__builtin_bit_cast(u16x2, R[2 * q + 1]) - bsub), top);
      V[q] = __builtin_amdgcn_perm(__builtin_bit_cast(u32, b), __builtin_bit_cast(u32, a), 0x07050301u);
    }
#pragma unroll
    for (int q = 0; q < 4; ++q) d[q] = V[2 * q] | (V[2 * q + 1] << 4);
    transpose4(d);
    u32 Xm = BOP(d[3], d[2], d[1], A & B & C);
    u32 P1 = BOP(d[3], d[2], d[1], A & ~B & C), P0 = BOP(d[3], d[2], d[0], A & ~B & C) | Xm;
    const u32 E1 = ~(P1 | P0);
    const u32 Q1 = d[3] & ~d[0], Q0 = d[2] & BOP(d[3], d[1], d[0], (A & ~C) | (~A & B & C));
    const u32 E2 = E1 & ~(Q1 | Q0);
    const Mv ma = move_masks(E1), mb = move_masks(E2);
    const u32 n1 = __builtin_popcount(E1), n2 = __builtin_popcount(E2), nx = __builtin_popcount(Xm);
    const u32 cnt = n1 | (n2 << 12) | ((nx > 3u ? 3u : nx) << 24);
    const u32 inc = wave_scan(cnt), tot = __builtin_amdgcn_readlane(inc, 63), exc = inc - cnt;
    const u32 N1 = tot & 0xfffu, NX = tot >> 24, bitw = (2 * N1 + 3 * ((tot >> 12) & 0xfffu) + 31) >> 5;
    const bool raw = (NX > XCAP) | (__ballot(nx > 3u) != 0);
    const u32 vd = raw ? 384u : bitw + NX;
    u32 voff = 0;
    if (lane == 0) voff = __hip_atomic_fetch_add(fill(par), vd, __ATOMIC_RELAXED, __HIP_MEMORY_SCOPE_WORKGROUP);
    const u32 q1 = pext(Q1, E1, ma), q0 = pext(Q0, E1, ma), r2 = pext(d[2], E2, mb), r1 = pext(d[1] | d[3], E2, mb), r0 = pext(d[0], E2, mb);
    const u32 o1 = 2 * (exc & 0xfffu), o2 = 2 * N1 + 3 * ((exc >> 12) & 0xfffu);
    if (__all(n1 <= 16u && n2 <= 10u)) { or_bits(var, o1, q1 | (q0 << n1)); or_bits(var, o2, r2 | (r1 << n2) | (r0 << (2 * n2))); }
    else { or_bits(var, o1, q1); or_bits(var, o1 + n1, q0); or_bits(var, o2, r2); or_bits(var, o2 + n2, r1); or_bits(var, o2 + 2 * n2, r0); }
    if (NX) {
      u32 E[8];
#pragma unroll
      for (int q = 0; q < 8; ++q) E[q] = __builtin_amdgcn_perm(R[2 * q + 1], R[2 * q], 0x07050301u);
      if (!raw) {
        u32 xo = bitw + (exc >> 24);
        do {
          if (Xm != 0u) {
            const u32 j = __builtin_ctz(Xm); Xm &= Xm - 1;
            const u32 vi = 2 * (j & 3) + ((j >> 2) & 1), by = j >> 3;
            const u32 k0 = 0u - (vi & 1u), k1 = 0u - ((vi >> 1) & 1u), k2 = 0u - (vi >> 2);
            const u32 vs = BFI(k2, BFI(k1, BFI(k0, E[7], E[6]), BFI(k0, E[5], E[4])), BFI(k1, BFI(k0, E[3], E[2]), BFI(k0, E[1], E[0])));
            var[xo] = (u32)lane | (vi << 6) | (by << 9) | ((((vs >> (8 * by)) & 255u) ^ (base + 9u)) << 16);
            ++xo;
          }
        } while (__any(Xm != 0u));
      } else {
        P1 = E[0]; P0 = E[1];
#pragma unroll
        for (int q = 2; q < 8; ++q) var[(q - 2) * 64 + lane] = E[q];
      }
    }
    asm volatile("" ::: "memory");
    u32 v0 = var[lane];
    voff = rfl(voff);
    if (PIPE) {
      asm volatile("" : "+v"(v0));
      if (par) finish();
      mid();
      if (par) st4x2(enc, so + rb * 2048 + lane * 16, (u32x4){S[0], S[1], S[2], S[3]}, (u32x4){S[4], S[5], S[6], S[7]});
    }
    asm volatile("global_store_dwordx2 %0, %1, %4 sc0 sc1\n\ts_mov_b64 exec, 1\n\tglobal_store_dword %2, %3, %4 sc0 sc1\n\ts_mov_b64 exec, -1"
                 :: "v"(so + SEG_P + rb * 512 + lane * 8), "v"((u32x2){P1, P0}), "v"(so + SEG_H + rb * 4), "v"(base | ((raw ? 255u : NX) << 8) | (voff << 16)), "s"(enc) : "memory");
    const u32 vo = so + SEG_V + (voff + lane) * 4;
    if ((u32)lane < vd) asm volatile("global_store_dword %0, %1, %2 sc0 sc1" :: "v"(vo), "v"(v0), "s"(enc) : "memory");
    for (u32 i = 64; i < vd; i += 64) if (i + lane < vd) asm volatile("global_store_dword %0, %1, %2 sc0 sc1" :: "v"(vo + i * 4), "v"(var[i + lane]), "s"(enc) : "memory");
    pseg = seg; pmask = mask; ++par;
  }
};
constexpr int XBUF = 65536;
__device__ __forceinline__ void load_job(const char* px, const char* pw, LAS char* lds, int J, int buf, int w, int lane) {
  if (J >= NJOB) return;
  const int c = J >> 5, pn = (J >> 1) & 15, st = (J & 1) * NB + (w & (NB - 1));
  const char* q; u32 off, step;
  if (pn < 8) { q = px; off = ((u32)((pn * 8 + st) * 16 + (lane & 15)) * KD + c * 128 + (lane >> 4) * 8) * 2; step = 64; }
  else { q = pw; off = (u32)(c * 128 + (lane >> 1)) * (NS * 2) + (pn - 8) * 256 + st * 32 + (lane & 1) * 16; step = 65536; }
  q = uni(q);
  const u32 dst = (u32)(size_t)lds + XBUF + buf * 32768 + w * 4096;
#pragma unroll
  for (int k = 0; k < 4; ++k) asm volatile("s_mov_b32 m0, %2\n\ts_nop 0\n\tglobal_load_lds_dwordx4 %0, %1" :: "v"(off + k * step), "s"(q), "s"(dst + k * 1024) : "memory");
}
constexpr int JOB1 = NPC + UPR * 2 * NPC, JOB2 = JOB1 + MIDR * 2 * NPB;

struct Jobs {
  int nr, u, sg;
  __device__ __forceinline__ void init(int u_, int sg_) {
    u = u_; sg = sg_;
    nr = u < 0 ? 0 : (u >= NPB ? 1 + UPR : 1 + UPR + MIDR + (u >= NPA || JOB2 + 2 * u >= NJOB ? 0 : (NJOB - JOB2 - 2 * u + 2 * NPA - 1) / (2 * NPA)));
  }
  __device__ __forceinline__ int job(int r) const {
    if (r >= nr) return NJOB;
    if (r == 0) return sg ? NJOB : u;
    return 2 * u + sg + (r <= UPR ? NPC + (r - 1) * 2 * NPC : (r <= UPR + MIDR ? JOB1 + (r - 1 - UPR) * 2 * NPB : JOB2 + (r - 1 - UPR - MIDR) * 2 * NPA));
  }
};
__device__ __forceinline__ void encode_jobs(CParams& p, LAS char* lds, u32 call, int w, int lane, const Jobs& jb) {
  Enc e; e.init(p, lds, call, w, lane);
  const int rank = p.rank;
  asm volatile("" : "+v"(e.row));
  if (jb.job(1) < NJOB) WAIT_V(4); else WAIT_V(0);
  for (int r = 0; r < jb.nr; ++r) {
    const int J = jb.job(r), J2 = jb.job(r + 2);
    auto mid = [&] { if (r != 0) load_job(p.x, p.w, lds, J2, r & 1, w, lane); };
    if (J < NJOB) {
      u32x4 X[4];
      const u32 pn = (J >> 1) & 15;
      LAS char* xb = lds + XBUF + (r & 1) * 32768 + w * 4096;
      if (pn < 8u) {
#pragma unroll
        for (int k = 0; k < 4; ++k) X[k] = *(const LAS u32x4*)(xb + k * 1024 + lane * 16);
      } else {
#pragma unroll
        for (int k = 0; k < 4; ++k) {
          LAS char* q = xb + k * 1024 + (lane >> 4) * 256 + (lane & 15) * 8;
          const s16x4 lo = __builtin_amdgcn_ds_read_tr16_b64_v4i16((LAS s16x4*)q), hi = __builtin_amdgcn_ds_read_tr16_b64_v4i16((LAS s16x4*)(q + 128));
          X[k] = __builtin_bit_cast(u32x4, __builtin_shufflevector(lo, hi, 0, 1, 2, 3, 4, 5, 6, 7));
        }
      }
      e.round<true>(X, pn * 128 + (J >> 5) * 2 + (J & 1), input_mask(rank, pn, J >> 5), mid);
    } else e.idle(mid);
    if (r == 0) { e.finish(); load_job(p.x, p.w, lds, J2, 0, w, lane); }
  }
  e.finish();
}

__device__ __forceinline__ void lds_post(LAS u32* q, u32 v) { asm volatile("ds_write_b32 %0, %1" :: "v"((u32)(size_t)q), "v"(v) : "memory"); }
__device__ __forceinline__ u32 lds_peek(LAS u32* q) { u32 v; asm volatile("ds_read_b32 %0, %1\n\ts_waitcnt lgkmcnt(0)" : "=&v"(v) : "v"((u32)(size_t)q) : "memory"); return rfl(v); }
__device__ __forceinline__ void lds_add(LAS u32* q, u32 n) { asm volatile("s_mov_b64 exec, 1\n\tds_add_u32 %0, %1\n\ts_mov_b64 exec, -1" :: "v"((u32)(size_t)q), "v"(n) : "memory"); }
template <int WIN, int I, int E> __device__ __forceinline__ void pieces(const char* gs, u32 ls, const u32 (&v16)[3], LAS u32* landed, u32 k) {
  if constexpr (I < E) {
    if ((I & 3) == 0) asm volatile("s_mov_b32 m0, %0" :: "s"(ls + I * 1024) : "memory");
    wait_v<WIN - 1>();
    asm volatile("global_load_lds_dwordx4 %0, %1 offset:%2 sc0 sc1" :: "v"(v16[I >> 2]), "s"(gs), "i"((I & 3) * 1024) : "memory");
    if (I == WIN - 1) lds_post(landed, k);
    pieces<WIN, I + 1, E>(gs, ls, v16, landed, k);
  }
}
constexpr u32 FIN = 0x40000000u;
__host__ __device__ __forceinline__ u32 c_offset(int rank, u32 pc) {
  const u32 o = (pc >> 8) & 1u;
  return ((u32)(slot_row(rank ^ (o + 1), (pc >> 3) & 31u) + (pc & 1u) * 64) * NS + o * 512 + ((pc >> 2) & 1u) * 256 + ((pc >> 1) & 1u) * 128) * 2;
}

template <int WIN> __device__ __forceinline__ void loader(CParams& p, LAS char* lds, int d, u32 call, int lane) {
  LAS u32* landed = (LAS u32*)(lds + LDS_MAIN + 128); LAS u32* posted = landed + 1; LAS u32* started = landed + 2;
  LAS u32* done = (LAS u32*)(lds + LDS_MAIN + 192); LAS u32* qitem = (LAS u32*)(lds + LDS_MAIN + 256);
  const char* rsrc = uni(p.ws[d] + ENC_OFF + (call & 1) * ENC_SZ);
  __builtin_amdgcn_s_waitcnt(0x0f70);
  __builtin_amdgcn_s_setprio(3);
  const u32 v16[3] = {(u32)lane * 16, (u32)lane * 16 + 4096, (u32)lane * 16 + 8192};
  u32 k = 0, item = 0, len = 0; bool have = false;
  static_assert(WIN <= 11, "window");
#define PEEK(kk) do { u32 a_, d_; \
    asm volatile("ds_read_b32 %0, %4\n\tds_read_b32 %1, %5\n\tds_read_b32 %2, %6\n\tds_read_b32 %3, %6 offset:64\n\ts_waitcnt lgkmcnt(0)" \
                 : "=&v"(a_), "=&v"(d_), "=&v"(item), "=&v"(len) \
                 : "v"((u32)(size_t)posted), "v"((u32)(size_t)(done + (kk) % NSLOT)), "v"((u32)(size_t)(qitem + (kk) % QD)) : "memory"); \
    have = (rfl(a_) > (kk)) & (rfl(d_) >= (kk) / NSLOT * NB); item = rfl(item); len = rfl(len); } while (0)
  for (;; ++k) {
    for (u32 it = 0; !have; ++it) {
      PEEK(k);
      if (have) break;
      if (it == 2) { WAIT_V(0); lds_post(landed, k); }
      __builtin_amdgcn_s_sleep(1);
    }
    if (len == 0) break;
    const char* gs = rsrc + (size_t)(item >> 16) * SEG_STRIDE;
    const u32 ls = (u32)(size_t)lds + k % NSLOT * SEG_STRIDE, n = len;
    pieces<WIN, 0, 1>(gs, ls, v16, landed, k);
    lds_post(started, k + 1);
    pieces<WIN, 1, 8>(gs, ls, v16, landed, k);
    const u64 tail = n >= 11264u ? ~0ull : (1ull << ((n - 10240u) >> 4)) - 1;
    pieces<WIN, 8, 9>(gs, ls, v16, landed, k);
    PEEK(k + 1);
    pieces<WIN, 9, 10>(gs, ls, v16, landed, k);
    wait_v<WIN - 1>();
    asm volatile("s_mov_b64 exec, %2\n\tglobal_load_lds_dwordx4 %0, %1 offset:2048 sc0 sc1\n\ts_mov_b64 exec, -1" :: "v"(v16[2]), "s"(gs), "s"(tail) : "memory");
    if (WIN == 11) lds_post(landed, k);
    for (u32 o = 11264; o < n; o += 1024) {
      wait_v<WIN - 1>();
      if (o + lane * 16 < n) asm volatile("s_mov_b32 m0, %2\n\ts_nop 0\n\tglobal_load_lds_dwordx4 %0, %1 sc0 sc1" :: "v"(o + lane * 16), "s"(gs), "s"(ls + o) : "memory");
    }
  }
  WAIT_V(0); lds_post(landed, k | FIN);
}
template <int NJ> __device__ __forceinline__ u32 decoder(CParams& p, LAS char* lds, int d, u32 call, int lane, u32 k0) {
  LAS u32* landed = (LAS u32*)(lds + LDS_MAIN + 128); LAS u32* done = (LAS u32*)(lds + LDS_MAIN + 192);
  LAS u32* qitem = (LAS u32*)(lds + LDS_MAIN + 256); LAS u32* qlen = qitem + QD; LAS u32* qdst = qitem + 2 * QD;
  char* dec = uni(p.ws[0] + DEC_OFF);
  char* enc = uni(p.ws[0] + ENC_OFF + (call & 1) * ENC_SZ);
  char* out = uni(p.out);
  gu64* row = flag_row(p, call, lane);
  const u32 vC = (lane & 15) * (NS * 2) + ((lane & 16) << 1) + ((lane & 32) >> 1);
  u32 have = 0, pend = 0, fseg = 0, flen = 0;
#define PUBLISH() do { if (pend) { \
    asm volatile("s_waitcnt vmcnt(0)\n\ts_mov_b64 exec, %3\n\tglobal_store_dword %0, %1, %2 sc0 sc1\n\ts_mov_b64 exec, -1" :: "v"(pend + lane * (NCH * 4)), "v"(call), "s"(dec), "i"(2 * NJ - 1) : "memory"); \
    if (flen) post_seg(row, call, fseg, flen, 1u << (d ^ 3), lane); pend = 0; flen = 0; } } while (0)
#define BOTH _Pragma("unroll") for (int z = 0; z < NJ; ++z)
  for (;;) {
    u32 q; u32x2 hv;
    asm volatile("s_mov_b64 exec, 1\n\tds_add_rtn_u32 %0, %2, %3 offset:%4\n\tds_read2_b32 %1, %2 offset1:5\n\ts_waitcnt lgkmcnt(0)\n\ts_mov_b64 exec, -1"
                 : "=&v"(q), "=&v"(hv) : "v"((u32)(size_t)landed), "v"(1u), "i"(NJ == 2 ? 12 : 16) : "memory");
    q = rfl(q); have = rfl(hv[0]);
    const u32 k = k0 + q / (NB / NJ), hf = q % (NB / NJ) * NJ, slot = k % NSLOT;
    u32 fine = NJ == 2 ? ~rfl(hv[1]) : ~0u;
    if ((have & ~FIN) <= k || k >= fine) {
      PUBLISH();
      while (have <= k && k < fine) { __builtin_amdgcn_s_sleep(1); have = lds_peek(landed); if (NJ == 2) fine = ~lds_peek(landed + 5); }
      if ((have & ~FIN) <= k || k >= fine) return fine;
    }
    const LAS char* ls = lds + slot * SEG_STRIDE;
    const u32 item_v = qitem[k % QD], dst_v = qdst[k % QD];
    u32 hdr_v[2]; u32x2 Pp[2]; u32x4 S[2][2];
    BOTH hdr_v[z] = *(const LAS u32*)(ls + SEG_H + (hf + z) * 4);
    BOTH Pp[z] = *(const LAS u32x2*)(ls + SEG_P + hf * 512 + z * 512 + lane * 8);
    BOTH { S[z][0] = *(const LAS u32x4*)(ls + hf * 2048 + z * 2048 + lane * 16); S[z][1] = *(const LAS u32x4*)(ls + hf * 2048 + z * 2048 + 1024 + lane * 16); }
    const u32 item = rfl(item_v), dst = rfl(dst_v);
    if (NJ == 1 && (item & D_REL)) {
      const u32 len = rfl(qlen[k % QD]), sg = 24 * 128 + (item & 511u), to = d ^ (((item >> 8) & 1u) + 1), o = hf * 1024 + lane * 16;
      u32x4 t[5];
#pragma unroll
      for (int i = 0; i < 5; ++i) t[i] = *(const LAS u32x4*)(ls + o + i * 4096);
#pragma unroll
      for (int i = 0; i < 5; ++i) if (o + i * 4096 < len) st4x1(enc, sg * SEG_STRIDE + o + i * 4096, t[i]);
      const u32 place = mail_place(dec + (MBX_OFF - DEC_OFF), hf == 0 ? 1u << to : 0u, lane);
      if (hf == 0) qdst[k % QD] = __builtin_amdgcn_readlane(place, to);
      u32 n;
      asm volatile("s_mov_b64 exec, 1\n\tds_add_rtn_u32 %0, %1, %2\n\ts_waitcnt lgkmcnt(0)\n\ts_mov_b64 exec, -1" : "=&v"(n) : "v"((u32)(size_t)(done + NSLOT + slot)), "v"(1u) : "memory");
      if ((rfl(n) & (NB - 1)) == NB - 1) post_mail(row, call, lds_peek(qdst + k % QD), sg, len, 1u << to, lane);
      lds_add(done + slot, 1);
      continue;
    }
    u32 hdr[2], vb[2], E1[2], n1[2], N1[2], o1[2], q1[2], q0[2], Q1[2], Q0[2], E2[2], n2[2], N2[2], o2[2], r2[2], r1[2], r0[2], E[2][8]; Mv ma[2], mb[2];
    BOTH { hdr[z] = rfl(hdr_v[z]); vb[z] = ((u32)(size_t)ls + SEG_V + ((hdr[z] >> 16) & 2047u) * 4) * 8; }
    BOTH { E1[z] = ~(Pp[z][0] | Pp[z][1]); ma[z] = move_masks(E1[z]); n1[z] = __builtin_popcount(E1[z]); }
    if (NJ == 1) { const u32 i = wave_scan(n1[0]); N1[0] = __builtin_amdgcn_readlane(i, 63); o1[0] = vb[0] + 2 * (i - n1[0]); n1[1] = 0; }
    else { const u32 i = wave_scan(n1[0] | (n1[1] << 16)), t = __builtin_amdgcn_readlane(i, 63);
      N1[0] = t & 0xffffu; N1[1] = t >> 16; o1[0] = vb[0] + 2 * ((i & 0xffffu) - n1[0]); o1[1] = vb[1] + 2 * ((i >> 16) - n1[1]); }
    BOTH { q1[z] = get_bits(o1[z]); q0[z] = q1[z] >> (n1[z] & 31u); }
    if (__any((n1[0] > n1[1] ? n1[0] : n1[1]) > 16u)) BOTH q0[z] = get_bits(o1[z] + n1[z]);
    BOTH { Q1[z] = pdep(q1[z], ma[z]); Q0[z] = pdep(q0[z], ma[z]); E2[z] = BOP(E1[z], Q1[z], Q0[z], A & ~(B | C)); mb[z] = move_masks(E2[z]); n2[z] = __builtin_popcount(E2[z]); }
    if (NJ == 1) { const u32 i = wave_scan(n2[0]); N2[0] = __builtin_amdgcn_readlane(i, 63); o2[0] = mad3(i - n2[0], vb[0] + 2 * N1[0]); n2[1] = 0; }
    else { const u32 i = wave_scan(n2[0] | (n2[1] << 16)), t = __builtin_amdgcn_readlane(i, 63);
      N2[0] = t & 0xffffu; N2[1] = t >> 16; o2[0] = mad3((i & 0xffffu) - n2[0], vb[0] + 2 * N1[0]); o2[1] = mad3((i >> 16) - n2[1], vb[1] + 2 * N1[1]); }
    u32 xe[2];
    BOTH { r2[z] = get_bits(o2[z]); r1[z] = r2[z] >> (n2[z] & 31u); r0[z] = r1[z] >> (n2[z] & 31u);
           xe[z] = *(const LAS u32*)(size_t)((vb[z] >> 3) + (((2 * N1[z] + 3 * N2[z] + 31) >> 5) + lane) * 4); }
    if (__any((n2[0] > n2[1] ? n2[0] : n2[1]) > 10u)) BOTH { r1[z] = get_bits(o2[z] + n2[z]); r0[z] = get_bits(o2[z] + 2 * n2[z]); }
    if (NJ == 2) PUBLISH();
    BOTH {
      const u32 R2 = pdep(r2[z], mb[z]) & E2[z], R1 = pdep(r1[z], mb[z]), R0 = pdep(r0[z], mb[z]);
      const u32 t7 = BOP(R2, R1, R0, A & B & C), q7 = BOP(E1[z], Q1[z], Q0[z], A & ~B & C);
      u32 dd[4] = {BOP(Pp[z][1] | q7, R0, E2[z], A | (B & C)), BOP(Pp[z][0], q7, BOP(R1, E2[z], t7, A & B & ~C), A | B | C), BOP(Q0[z], E1[z], R2, (A & B) | C), BOP(E1[z], Q1[z], t7, ~A | B | C)};
      transpose4(dd);
      const u32 b4 = (hdr[z] & 255u) * 0x01010101u;
#pragma unroll
      for (int i = 0; i < 4; ++i) { E[z][2 * i] = (dd[i] & 0x0f0f0f0fu) + b4; E[z][2 * i + 1] = ((dd[i] >> 4) & 0x0f0f0f0fu) + b4; }
    }
    BOTH asm volatile("" : "+v"(xe[z]));
    BOTH {
      const u32 NX = (hdr[z] >> 8) & 255u;
      if (NX <= XCAP) {
        for (u32 i = 0; i < NX; ++i) {
          const u32 en = __builtin_amdgcn_readlane(xe[z], i);
          const u32 dv = (u32)lane == (en & 63u) ? ((en >> 16) & 255u) << (8 * ((en >> 9) & 3u)) : 0u;
          const int hot = (int)(0x80000000u >> ((en >> 6) & 7u));
#pragma unroll
          for (int i2 = 0; i2 < 8; ++i2) E[z][i2] = BOP(E[z][i2], dv, (u32)((hot << i2) >> 31), A ^ (B & C));
        }
      } else {
        E[z][0] = Pp[z][0]; E[z][1] = Pp[z][1];
#pragma unroll
        for (int i2 = 2; i2 < 8; ++i2) E[z][i2] = *(const LAS u32*)(size_t)((vb[z] >> 3) + ((i2 - 2) * 64 + lane) * 4);
      }
    }
    u32x4 O[2][4];
    BOTH {
#pragma unroll
      for (int i = 0; i < 8; ++i) {
        const u32 sb = S[z][i >> 2][i & 3];
        const u32 a = __builtin_amdgcn_perm(E[z][i], sb, 0x05010400u), b = __builtin_amdgcn_perm(E[z][i], sb, 0x07030602u);
        O[z][i >> 1][(i & 1) * 2] = __builtin_amdgcn_alignbit(a, a, 1); O[z][i >> 1][(i & 1) * 2 + 1] = __builtin_amdgcn_alignbit(b, b, 1);
      }
      asm volatile("" : "+v"(O[z][0]), "+v"(O[z][1]), "+v"(O[z][2]), "+v"(O[z][3]));
    }
    if (NJ == 1) {
      char* o = out + (dst + hf * 64 + vC);
#pragma unroll
      for (int m = 0; m < 4; ++m) BOTH st_out(o + m * 16 * NS * 2 + z * 64, O[z][m]);
    } else {
      BOTH st4x4(dec, dst + (hf + z) * 4096u + lane * 16, O[z][0], O[z][1], O[z][2], O[z][3]);
      pend = (u32)(F2_OFF - DEC_OFF) + ((((item >> 7) & 31u) * 8 + (item & 1u) * NB + hf) * NCH + ((item >> 1) & 63u)) * 4;
      if ((item & D_FWD) && hf == 0) {
        flen = lds_peek(qlen + (k % QD)); fseg = (12 + d) * 128 + (item & 127u);
        for (u32 o0 = 0; o0 < flen; o0 += 6144) {
          const u32 o = o0 + lane * 16; u32x4 t[6];
#pragma unroll
          for (int i = 0; i < 6; ++i) t[i] = *(const LAS u32x4*)(ls + o + i * 1024);
#pragma unroll
          for (int i = 0; i < 6; ++i) if (o + i * 1024 < flen) st4x1(enc, fseg * SEG_STRIDE + o + i * 1024, t[i]);
        }
      }
    }
    lds_add(done + slot, NJ);
  }
}

__device__ __forceinline__ void pull_role(CParams& p, LAS char* lds, int d, int j, int ncu, u32 call, int w, int lane) {
  LAS u32* posted = (LAS u32*)(lds + LDS_MAIN + 132); LAS u32* started = posted + 1;
  LAS u32* qitem = (LAS u32*)(lds + LDS_MAIN + 256); LAS u32* qlen = qitem + QD; LAS u32* qdst = qitem + 2 * QD;
  LAS u32* list = (LAS u32*)(lds + NSLOT * SEG_STRIDE);
  static_assert(NSLOT * SEG_STRIDE + LMAX * 4 <= LDS_MAIN, "list");
  const int rank = rfl(p.rank), lt = list_type(d);
  const u32 nl = rfl(LISTS.n[lt]), n = nl + (lt == 0 ? NMAIL0 : (lt == 2 ? NMAIL2 : 0u));
  for (u32 i = threadIdx.x; i < nl; i += 512) list[i] = resolve(d, rank, LISTS.e[lt][i]);
  LBAR;
  if (w == 1) {
    const u64* f1 = uni((const u64*)(p.ws[0] + F1_OFF + (call & 1) * F1_SZ) + d * NSRC);
    u32* tk = uni((u32*)(p.ws[0] + TKT_OFF + d * 128));
    const u32 tbase = call * 1024u, nin = nl - 1;
    __builtin_amdgcn_s_setprio(2);
    if (lane == 0) __hip_atomic_fetch_max(tk, tbase, __ATOMIC_RELAXED, __HIP_MEMORY_SCOPE_AGENT);
    WAIT_V(0);
#define FIDX(i) (list[(i) < nl ? (i) : nin] >> 16)
#define ENQUEUE(item, len, dst) do { lds_post(qdst + (k % QD), dst); lds_post(qitem + (k % QD), item); lds_post(qlen + (k % QD), len); lds_post(posted, ++k); } while (0)
#define CLAMP_LEN(l) ((l) > (u32)SEG_STRIDE ? (u32)SEG_STRIDE : ((l) < (u32)SEG_V ? (u32)SEG_V : (l) & ~15u))
    u32 pos = j, k = 0, t, wa = ~0u, wb, win = ~63u, asked = ~WREF; u64 v = 0, vn; bool dead = false, more = true;
    asm volatile("v_mov_b32 v248, %0" :: "s"(tbase + j - ncu) : "v248");
#define DRAW(ww, wo, TN, VN, TO, VO) do { while (more && lds_peek(started) + LAP <= k + 1) __builtin_amdgcn_s_sleep(1); \
      ww = pos - asked >= WREF ? pos : ~0u; if (ww != ~0u) asked = pos;        \
      asm volatile("s_mov_b64 exec, 1\n\tglobal_atomic_add " TN ", %2, %3, %4 sc0\n\ts_mov_b64 exec, %7\n\tglobal_load_dwordx2 " VN ", %5, %6 sc0 sc1\n\ts_mov_b64 exec, -1\n\t" \
                   "s_waitcnt vmcnt(2)\n\tv_mov_b32 %0, " TO "\n\tv_mov_b64 %1, " VO \
                   : "=&v"(t), "=&v"(vn) : "v"(0u), "v"(more ? 1u : 0u), "s"(tk), "v"(FIDX(pos + lane) * 8u), "s"(f1), "s"(ww != ~0u ? ~0ull : 1ull) \
                   : "memory", "v248", "v249", "v250", "v251", "v252", "v253"); \
      if (wo != ~0u) { win = wo; v = vn; } } while (0)
#define SETTLE(t, vv, ww) do { const u32 q_ = rfl(t) - tbase + ncu; \
      if (q_ >= nl) more = false; \
      else { pos = q_; const u32 item = rfl(list[pos]); u32 len = 0; \
        if (pos - ww < 64u && ((__ballot((u32)(vv >> 32) >= call) >> (pos - ww)) & 1)) len = __builtin_amdgcn_readlane((u32)vv, pos - ww); \
        else { const u64 t0 = __builtin_amdgcn_s_memrealtime(); asked = pos - WREF; \
          for (u32 it = 1;; ++it) { const u64 u = __hip_atomic_load(f1 + (item >> 16), __ATOMIC_RELAXED, __HIP_MEMORY_SCOPE_SYSTEM); \
            if (rfl((u32)(u >> 32)) >= call) { len = rfl((u32)u); break; } \
            if ((it & 63u) == 0 && give_up(p, call, t0)) { dead = true; more = false; break; } \
            __builtin_amdgcn_s_sleep(1); } } \
        if (!dead) { ENQUEUE(item, CLAMP_LEN(len), ((item >> 7) & 31u) * (u32)PANEL_B + (item & 127u) * 16384u); } } } while (0)
    for (;;) {
      bool last = !more;
      DRAW(wb, wa, "v249", "v[252:253]", "v248", "v[250:251]"); SETTLE(t, v, win);
      if (last || dead) break;
      last = !more;
      DRAW(wa, wb, "v248", "v[250:251]", "v249", "v[252:253]"); SETTLE(t, v, win);
      if (last || dead) break;
    }
    lds_post(posted + 4, ~k);
    const u32 place = j + lane * ncu;
    u64 todo = dead ? 0 : __ballot(place < n - nl);
    const u64* fp = f1 + MAIL0 + (place < n - nl ? place : 0u);
    u64 t0 = __builtin_amdgcn_s_memrealtime();
    for (u32 it = 1; todo; ++it) {
      v = __hip_atomic_load(fp, __ATOMIC_RELAXED, __HIP_MEMORY_SCOPE_SYSTEM);
      u64 rdy = __ballot((u32)(v >> 32) >= call) & todo;
      if (rdy) {
        todo &= ~rdy;
        const u32 sg_ = (((u32)v >> 16) - MAIL0) & 1023u, ln = ((u32)v & 0xffffu) << 4;
        const u32 item = ((MAIL0 + sg_) << 16) | (sg_ & 511u) | (d > 3 && sg_ < 512u ? D_REL : 0u), len = CLAMP_LEN(ln), dst = c_offset(rank, sg_);
        const u64 first = __ballot(item & D_REL);
        do {
          while (lds_peek(started) + LAM <= k) __builtin_amdgcn_s_sleep(1);
          const u32 l = __builtin_ctzll(rdy & first ? rdy & first : rdy); rdy &= ~(1ull << l);
          ENQUEUE(__builtin_amdgcn_readlane(item, l), __builtin_amdgcn_readlane(len, l), __builtin_amdgcn_readlane(dst, l));
        } while (rdy);
      } else {
        if ((it & 63u) == 0 && give_up(p, call, t0)) { dead = true; break; }
        __builtin_amdgcn_s_sleep(1);
      }
    }
    if (dead)
      for (u32 pc = lane; pc < 512u; pc += 64) { const u32 l = piece_link(pc >> 8, pc & 255u);
        if ((l > 3 ? l ^ ((pc >> 8) + 1) : l) == (u32)d) *(u32*)(p.out + c_offset(rank, pc)) = 0x7fc07fc0u; }
    while (lds_peek(started) + LAM <= k) __builtin_amdgcn_s_sleep(1);
    ENQUEUE(0u, 0u, 0u);
    return;
  }
  if (w == 0) {
    if (ncu > NPULL / 7) loader<INFL>(p, lds, d, call, lane); else loader<INFL5>(p, lds, d, call, lane);
    return;
  }
  decoder<1>(p, lds, d, call, lane, decoder<2>(p, lds, d, call, lane, 0));
}

__device__ __forceinline__ int wait_chunks(CParams& p, const u32* f, u32 o0, u32 o1, u32 o2, int need, u32 call) {
  const u64 t0 = __builtin_amdgcn_s_memrealtime();
  __builtin_amdgcn_s_waitcnt(0xc07f);
  for (u32 it = 1;; ++it) {
    const u32 a = __hip_atomic_load(f + o0, __ATOMIC_RELAXED, __HIP_MEMORY_SCOPE_SYSTEM);
    const u32 b = __hip_atomic_load(f + o1, __ATOMIC_RELAXED, __HIP_MEMORY_SCOPE_SYSTEM);
    const u32 c = __hip_atomic_load(f + o2, __ATOMIC_RELAXED, __HIP_MEMORY_SCOPE_SYSTEM);
    const u64 miss = __ballot(a < call || b < call || c < call);
    const int n = miss ? __builtin_ctzll(miss) : NCH;
    if (n >= need) return n;
    if ((it & 63u) == 0 && give_up(p, call, t0)) return NCH;
    __builtin_amdgcn_s_sleep(2);
  }
}
#define FL_ADDR(next) do { fbase = (next); const u32 c_ = fbase + (lane & 15); fvo = foff + 4 * (c_ < 63u ? c_ : 63u); } while (0)
#define FL_ISSUE() asm volatile("s_mov_b32 m0, %2\n\ts_nop 0\n\tglobal_load_lds_dword %0, %1 sc0 sc1" :: "v"(fvo), "s"(f2), "s"(lds0 + LDS_FLG + w * 256) : "memory")
#define FL_READ() fval = *(const volatile LAS u32*)(lds + LDS_FLG + w * 256 + lane * 4)
#define FL_USE() do { u64 m_ = __ballot(fval < call); m_ |= m_ >> 32; const u32 h_ = (u32)m_ | ((u32)m_ >> 16) | 0x10000u; \
    const int n_ = fbase + __builtin_ctz(h_); avail = n_ > avail ? n_ : avail; } while (0)
__device__ __forceinline__ u32 cvt_pk(float lo, float hi) { u32 r; asm("v_cvt_pk_bf16_f32 %0, %1, %2" : "=v"(r) : "v"(lo), "v"(hi)); return r; }
#define QNAN __builtin_nanf("")
typedef float f32x2 __attribute__((ext_vector_type(2)));
__device__ __forceinline__ u32 silu2(float a, float b) {
  const f32x2 t = (f32x2){__builtin_amdgcn_fmed3f(a, -60.0f, 60.0f), __builtin_amdgcn_fmed3f(b, -60.0f, 60.0f)} * -1.44269504088896341f;
  const f32x2 e = (f32x2){__builtin_amdgcn_exp2f(t.x), __builtin_amdgcn_exp2f(t.y)} + 1.0f;
  const f32x2 r = (f32x2){a, b} * (f32x2){__builtin_amdgcn_rcpf(e.x), __builtin_amdgcn_rcpf(e.y)};
  return cvt_pk(r.x, r.y);
}
__device__ __forceinline__ u32x4 finish8(const f32x4& u, const f32x4& v) {
  const auto s0 = __builtin_amdgcn_permlane16_swap(silu2(u[0], u[1]), silu2(v[0], v[1]), false, false);
  const auto s1 = __builtin_amdgcn_permlane16_swap(silu2(u[2], u[3]), silu2(v[2], v[3]), false, false);
  return (u32x4){s0[0], s1[0], s0[1], s1[1]};
}
__device__ __forceinline__ bool timed_out(CParams& p, u32 call) {
  return rfl(__hip_atomic_load((u32*)(p.ws[0] + TMO_OFF), __ATOMIC_RELAXED, __HIP_MEMORY_SCOPE_AGENT)) == call;
}
constexpr int HT = 16384;
#define A_SETUP(own) \
  const u32 sCa = (own) ? 256u : 32768u, sKa = (own) ? 128u : 2048u; \
  const u32 voffA0 = (own) ? (u32)(w * 16 + (lane >> 3)) * (KD * 2) + (((lane & 7) ^ (lane >> 4)) << 4) : (u32)(w * 4096 + lane * 16); \
  const u32 voffA1 = (own) ? (voffA0 ^ 64u) + 8 * KD * 2 - 1024 : voffA0;        \
  const u32 lds0 = (u32)(size_t)lds, ldsA = lds0 + w * 2048; \
  u32 aoff[2]; \
  aoff[0] = lds0 + wr * 8192 + ((own) ? (lane & 15) * 128 + (((lane >> 4) ^ ((lane >> 1) & 7)) << 4) : lane * 16); \
  aoff[1] = (own) ? aoff[0] ^ 64u : aoff[0] + 1024;
#define B_READ_OFF(n, t) (256 * (8 * (lane >> 4) + 4 * (t) + ((lane & 15) >> 2)) + 8 * (lane & 1) + \
    16 * ((4 * wc + 2 * (n) + ((lane & 3) >> 1)) ^ ((((lane & 15) >> 2) << 2) | ((2 * (lane >> 4) + (t)) & 3))))
#define VOFF_W ((u32)(4 * w + (lane >> 4)) * 2048u + (((lane & 15) ^ (((lane >> 4) << 2) | (w & 3))) << 4))
#define GB " sc0 sc1"
#define LDA(dst, ad, imm) do { _Pragma("unroll") for (int m = 0; m < MF; ++m) _Pragma("unroll") for (int k = 0; k < 2; ++k) \
    dst[m][k] = *(const LAS s16x8*)(size_t)(ad[k] + (imm) + m * 2048); } while (0)
#define LDB(dst, ad, imm) do { _Pragma("unroll") for (int n = 0; n < 2; ++n) _Pragma("unroll") for (int k = 0; k < 2; ++k) { \
    const s16x4 lo_ = __builtin_amdgcn_ds_read_tr16_b64_v4i16((LAS s16x4*)(size_t)(ad[n][0] + (imm) + k * 8192)); \
    const s16x4 hi_ = __builtin_amdgcn_ds_read_tr16_b64_v4i16((LAS s16x4*)(size_t)(ad[n][1] + (imm) + k * 8192)); \
    dst[n][k] = __builtin_shufflevector(lo_, hi_, 0, 1, 2, 3, 4, 5, 6, 7); } } while (0)
#define MMAX(ac, At, Bt, EXTRA) do { __builtin_amdgcn_s_setprio(1); EXTRA; _Pragma("unroll") for (int m = 0; m < MF; ++m) _Pragma("unroll") for (int n = 0; n < 2; ++n) \
    _Pragma("unroll") for (int k = 0; k < 2; ++k) ac[m][n] = __builtin_amdgcn_mfma_f32_16x16x32_bf16(__builtin_bit_cast(bf8, Bt[n][k]), __builtin_bit_cast(bf8, At[m][k]), ac[m][n], 0, 0, 0); \
    __builtin_amdgcn_s_setprio(0); \
    _Pragma("unroll") for (int m = 0; m < MF; m += 2) asm volatile("" : "+v"(ac[m][0]), "+v"(ac[m][1]), "+v"(ac[m + 1][0]), "+v"(ac[m + 1][1])); } while (0)
#define WEAVE do { _Pragma("unroll") for (int i_ = 0; i_ < 4 * MF - 1; ++i_) { __builtin_amdgcn_sched_group_barrier(0x8, 1, 0); __builtin_amdgcn_sched_group_barrier(0x6, 1, 0); } } while (0)
#define WEAVE2 do { _Pragma("unroll") for (int i_ = 0; i_ < 4 * MF - 1; ++i_) { __builtin_amdgcn_sched_group_barrier(0x8, 1, 0); __builtin_amdgcn_sched_group_barrier(0x6, 2, 0); } } while (0)
template <int MF, bool ALLOWN> __device__ __forceinline__ void gemm_wide(CParams& p, LAS char* lds, int F0, int tn, u32 call, int w, int lane) {
  const int wr = w >> 2, wc = w & 3;
  const bool own = ALLOWN || F0 < 64;
  const char* Ab = uni(own ? p.x : p.ws[0] + DEC_OFF);
  const char* Bb = uni(p.w + tn * 512);
  constexpr int NA = MF / 2;
  constexpr u32 AB = MF * 4096, TB = AB + 2 * HT;
  const u32 sCa = own ? 256u : 32768u, sKa = own ? 128u : 2048u, lds0 = (u32)(size_t)lds;
  u32 vA[NA];
#pragma unroll
  for (int i = 0; i < NA; ++i) {
    const int pi = w * NA + i, f = F0 + (pi >> 1), h = pi & 1;
    vA[i] = (own ? (u32)(f * 16 + h * 8 + (lane >> 3)) * (KD * 2) + (((lane & 7) ^ (lane >> 4) ^ (h << 2)) << 4)
                 : (f >> 3) * (u32)PANEL_B + (f & 7) * 4096u + h * 1024 + lane * 16) - i * 1024;
  }
  const u32 vB = VOFF_W, vB2 = vB + 65536;
  const int f0 = F0 + ((w * NA) >> 1), f1 = F0 + ((w * NA + NA - 1) >> 1);
  const u32* flg = (const u32*)(p.ws[0] + F2_OFF) + f0 * NCH + lane;
  const char* f2 = uni(p.ws[0] + F2_OFF);
  const u32 foff = (lane & 32 ? f1 : f0) * NCH * 4;
  int fbase = 0; u32 fval = 0, fvo = 0;
  u32 aoff[2][2], boff[2][2][2];
#pragma unroll
  for (int b = 0; b < 2; ++b) {
    aoff[b][0] = lds0 + b * TB + wr * MF * 2048 + (own ? (lane & 15) * 128 + (((lane >> 4) ^ ((lane >> 1) & 7)) << 4) : lane * 16);
    aoff[b][1] = own ? aoff[b][0] ^ 64u : aoff[b][0] + 1024;
    asm volatile("" : "+v"(aoff[b][0]), "+v"(aoff[b][1]));
#pragma unroll
    for (int i = 0; i < 4; ++i) { boff[b][i >> 1][i & 1] = lds0 + b * TB + AB + B_READ_OFF(i >> 1, i & 1); asm volatile("" : "+v"(boff[b][i >> 1][i & 1])); }
  }
#define LDAW(b, gp) do { asm volatile("s_mov_b32 m0, %0\n\ts_nop 0" :: "s"(lds0 + (b) * TB + w * NA * 1024) : "memory"); \
    _Pragma("unroll") for (int i = 0; i < NA; ++i) { \
      if (i == 0) asm volatile("global_load_lds_dwordx4 %0, %1" GB :: "v"(vA[0]), "s"(gp) : "memory"); \
      if (i == 1) asm volatile("global_load_lds_dwordx4 %0, %1 offset:1024" GB :: "v"(vA[1]), "s"(gp) : "memory"); \
      if (i == 2) asm volatile("global_load_lds_dwordx4 %0, %1 offset:2048" GB :: "v"(vA[NA - 1]), "s"(gp) : "memory"); } } while (0)
#define LDBW(b, h, gp) asm volatile("s_mov_b32 m0, %3\n\ts_nop 0\n\tglobal_load_lds_dwordx4 %0, %2\n\ts_add_u32 m0, m0, 0x2000\n\ts_nop 0\n\tglobal_load_lds_dwordx4 %1, %2" \
    :: "v"(vB), "v"(vB2), "s"(gp), "s"(lds0 + (b) * TB + AB + (h) * HT + w * 1024) : "memory", "scc")
  f32x4 acc[2][MF][2];
#pragma unroll
  for (int i = 0; i < 4 * MF; ++i) acc[i & 1][i >> 2][(i >> 1) & 1] = (f32x4){0.f, 0.f, 0.f, 0.f};
  s16x8 At[MF][2], B0[2][2], B1[2][2];
  int avail = own ? NCH : wait_chunks(p, flg, 0, (f1 - f0) * NCH, 0, 1, call);
  __builtin_amdgcn_s_waitcnt(0x0f70);
  constexpr int FL = !ALLOWN, V = NA + 4;
  LDBW(0, 0, Bb); LDAW(0, Ab);
  if (FL) { FL_ADDR(1); FL_ISSUE(); }
  LDBW(0, 1, Bb + 256); LDBW(1, 0, Bb + 131072); LDAW(1, Ab + sKa);
  wait_v<V>(); BAR;
  LDB(B0, boff[0], 0);
  WAIT_L(0); BAR;
  if (wr == 1) BAR;
  const char* gA = Ab + sCa; const char* gB1 = Bb + 131072 + 256; const char* gB0 = Bb + 262144;
#define WTILE(b, dA, dB1, dB0, V1, V2, RD, X1, CHECK, ISS) do { \
    LDA(At, aoff[b], 0); RD; SCHED; LDBW((b) ^ 1, 1, gB1); LDBW(b, 0, gB0); \
    wait_v<V1>(); WAIT_L(0); BAR; SCHED; \
    MMAX(acc[0], At, B0, gB1 += (dB1); gB0 += (dB0); X1; WEAVE); asm volatile("" : "+s"(gB1), "+s"(gB0), "+s"(avail)); asm volatile("" : "+v"(fvo)); BAR; SCHED; \
    CHECK; \
    LDB(B1, boff[b], HT); LDB(B0, boff[(b) ^ 1], 0); SCHED; LDAW(b, gA); ISS; \
    wait_v<V2>(); WAIT_L(0); BAR; SCHED; \
    MMAX(acc[1], At, B1, gA += (dA); WEAVE); asm volatile("" : "+s"(gA)); BAR; SCHED; } while (0)
#pragma unroll 1
  for (int c = 0; c < NCH; ++c) {
    const int cn = c + 1 < NCH ? c + 1 : c;
    const int n1 = c + 1 < NCH ? 131072 : 0, n2 = c + 2 < NCH;
    WTILE(0, (int)sKa, n1, n1, V, V + FL, if (FL) FL_READ(), if (FL) { FL_USE(); FL_ADDR(cn + 1); },
          if (FL && avail <= cn) avail = wait_chunks(p, flg, 0, (f1 - f0) * NCH, 0, cn + 1, call), if (FL) FL_ISSUE());
    WTILE(1, (int)(n2 ? sCa : 0u) - (int)sKa, n1, n2 ? 131072 : 0, V + FL, V, (void)0, (void)0, (void)0, (void)0);
  }
  if (wr == 0) BAR;
  if (!own && timed_out(p, call)) {
#pragma unroll
    for (int i = 0; i < 4 * MF; ++i) acc[i & 1][i >> 2][(i >> 1) & 1] = (f32x4){QNAN, QNAN, QNAN, QNAN};
  }
  int tz = threadIdx.x; asm volatile("" : "+v"(tz));
  char* ob = p.out + ((size_t)(tz & 15) * NS + tn * 256 + wc * 32 + (tz & 16) + ((tz & 32) >> 2)) * 2;
#pragma unroll
  for (int m = 0; m < MF; ++m) {
    const int f = F0 + MF * wr + m;
    char* oa = ob + (size_t)(slot_row(p.rank, f >> 3) + (f & 7) * 16) * (NS * 2);
#pragma unroll
    for (int bj = 0; bj < 2; ++bj) st_out(oa + bj * 256, finish8(acc[bj][m][0], acc[bj][m][1]));
  }
}

__device__ __forceinline__ void gemm_small(CParams& p, LAS char* lds, int s, int cp, u32 call, int w, int lane) {
  const int wr = w >> 2, wc = w & 3;
  constexpr int MF = 4;
  const bool own = s < 8;
  const char* Ab = uni(own ? p.x + (size_t)s * PANEL_B : p.ws[0] + DEC_OFF + (size_t)s * PANEL_B);
  const char* Bb = uni(p.ws[0] + DEC_OFF + (size_t)cp * 2 * PANEL_B);
  A_SETUP(own)
  const u32 voffB = w * 4096 + lane * 16;
  const u32* flg = (const u32*)(p.ws[0] + F2_OFF) + w * NCH + lane;
  const u32 fb0 = cp * 2 * 8 * NCH, fb1 = fb0 + 8 * NCH, fa = own ? fb0 : s * 8 * NCH;
  const char* f2 = uni(p.ws[0] + F2_OFF);
  const u32 foff = ((lane & 32 ? fa : (lane & 16 ? fb1 : fb0)) + w * NCH) * 4;
  int fbase = 0; u32 fval = 0, fvo = 0;
  constexpr u32 TB = 3 * HT;
  const u32 boff = lds0 + wc * 4096 + lane * 16;
  u32 bc = boff, bn = boff + TB, ac[2] = {aoff[0], aoff[1]};
  s16x8 At[4][2], B0[2][2], B1[2][2];
  Enc e; e.init(p, lds, call, w, lane);
  asm volatile("" : "+v"(e.row));
  f32x4 acc[2][4][2];
#pragma unroll
  for (int b = 0; b < 2; ++b)
#pragma unroll
    for (int m = 0; m < 4; ++m)
#pragma unroll
      for (int n = 0; n < 2; ++n) acc[b][m][n] = (f32x4){0.f, 0.f, 0.f, 0.f};
  int avail = wait_chunks(p, flg, fa, fb0, fb1, 2, call);
  __builtin_amdgcn_s_waitcnt(0x0f70);
#define LD2(m0v, v0, v1, gp) asm volatile("s_mov_b32 m0, %3\n\ts_nop 0\n\tglobal_load_lds_dwordx4 %0, %2" GB "\n\tglobal_load_lds_dwordx4 %1, %2 offset:1024" GB \
    :: "v"(v0), "v"(v1), "s"(gp), "s"(m0v) : "memory")
#define LD1(m0v, v0, gp, o) asm volatile("s_mov_b32 m0, %2\n\ts_nop 0\n\tglobal_load_lds_dwordx4 %0, %1 offset:" #o GB :: "v"(v0), "s"(gp), "s"(m0v) : "memory")
#define LDF(dst, ad, imm) do { _Pragma("unroll") for (int n = 0; n < 2; ++n) _Pragma("unroll") for (int k = 0; k < 2; ++k) \
    dst[n][k] = *(const LAS s16x8*)(size_t)((ad) + (imm) + n * 2048 + k * 1024); } while (0)
  FL_ADDR(2);
  LD2(ldsA + HT, voffB, voffB, Bb); LD2(ldsA, voffA0, voffA1, Ab); LD2(ldsA + 2 * HT, voffB, voffB, Bb + PANEL_B); LD2(ldsA + TB + HT, voffB, voffB, Bb + 2048);
  LD2(ldsA + TB, voffA0, voffA1, Ab + sKa); FL_ISSUE(); LD2(ldsA + TB + 2 * HT, voffB, voffB, Bb + 2048 + PANEL_B); LD2(ldsA + 2 * TB + HT, voffB, voffB, Bb + 32768);
  WAIT_V(11); BAR;
  LDF(B0, bc, HT);
  WAIT_L(0);
  if (wr == 1) BAR;
  u32 s0 = 0, s1 = TB, s2 = 2 * TB;
  const char* gA = Ab + sCa; const char* gB1 = Bb + 32768 + PANEL_B; const char* gB0 = Bb + 32768 + 2048;
  u32 mA = ldsA + s2, mB1 = mA + 2 * HT, mB0 = ldsA + HT;
#define PIN do { asm volatile("" : "+s"(gA), "+s"(gB1), "+s"(gB0), "+s"(mA), "+s"(mB1), "+s"(mB0), "+s"(avail)); \
    asm volatile("" : "+v"(fvo), "+v"(bc), "+v"(bn), "+v"(ac[0]), "+v"(ac[1])); } while (0)
#define KTILE(dA, dB1, dB0, V1, V2, ISS, CHECK, RD, X2) do { \
    LD2(mA, voffA0, voffA1, gA); LD1(mB1, voffB, gB1, 0); ISS; LDA(At, ac, 0); \
    WAIT_V(V1); BAR; WAIT_L(0); SCHED; \
    MMAX(acc[0], At, B0, gA += (dA); mA = ldsA + s0; ac[0] = aoff[0] + s1; ac[1] = aoff[1] + s1; WEAVE); PIN; BAR; SCHED; \
    CHECK; \
    LD1(mB1, voffB, gB1, 1024); LD2(mB0, voffB, voffB, gB0); LDF(B1, bc, 2 * HT); SCHED; LDF(B0, bn, HT); \
    WAIT_V(V2); WAIT_L(4); RD; BAR; WAIT_L(0); SCHED; \
    MMAX(acc[1], At, B1, gB1 += (dB1); gB0 += (dB0); mB1 = mA + 2 * HT; mB0 = ldsA + s1 + HT; bc = bn; bn = boff + s2; \
         { const u32 s_ = s0; s0 = s1; s1 = s2; s2 = s_; } X2; WEAVE2); PIN; BAR; SCHED; } while (0)
  for (int c = 0; c < NCH; ++c) {
    const int c2 = c + 2 < NCH ? c + 2 : NCH - 1;
    const int nx = c + 2 < NCH;
    KTILE((int)sKa, 2048, (nx ? 32768 : 0) - 2048, 10, 9, (void)0, (void)0, FL_READ(), FL_USE(); FL_ADDR(c2 + 1));
    KTILE((int)(nx ? sCa : 0u) - (int)sKa, (nx ? 32768 : 0) - 2048, 2048, 10, 10, FL_ISSUE(),
          if (avail <= c2) { avail = wait_chunks(p, flg, fa, fb0, fb1, c2 + 1, call); }, (void)0, (void)0);
  }
  if (wr == 0) BAR;
  if (wr == 0) __builtin_amdgcn_s_setprio(3); else __builtin_amdgcn_s_setprio(2);
  const u32 tmo = __hip_atomic_load((u32*)(p.ws[0] + TMO_OFF), __ATOMIC_RELAXED, __HIP_MEMORY_SCOPE_AGENT);
  int tz = threadIdx.x; asm volatile("" : "+v"(tz));
  u32x4 X[4];
#pragma unroll
  for (int m = 0; m < 4; ++m) { X[m] = finish8(acc[0][m][0], acc[0][m][1]); asm volatile("" : "+v"(X[m])); }
  if (rfl(tmo) == call) {
#pragma unroll
    for (int m = 0; m < 4; ++m) { X[m] = (u32x4){0x7fc07fc0u, 0x7fc07fc0u, 0x7fc07fc0u, 0x7fc07fc0u}; acc[1][m][0] = acc[1][m][1] = (f32x4){QNAN, QNAN, QNAN, QNAN}; }
  }
  e.lane = tz & 63;
  const u32 code = (s * 2 + (cp & 1)) * 4 + wr, seg = MAIL0 + (cp >> 1) * 256 + code;
#pragma unroll 1
  for (int bj = 0; bj < 2; ++bj) {
    e.round<false>(X, seg + 2 * bj, 1u << piece_link(cp >> 1, code + 2 * bj), [] {});
    if (wr == 0) __builtin_amdgcn_s_setprio(1); else __builtin_amdgcn_s_setprio(0);
    e.finish_mail(f2 + (MBX_OFF - F2_OFF));
    if (bj == 0) {
#pragma unroll
      for (int m = 0; m < 4; ++m) { asm volatile("" : "+v"(acc[1][m][0]), "+v"(acc[1][m][1])); X[m] = finish8(acc[1][m][0], acc[1][m][1]); }
    }
  }
}

__global__ __launch_bounds__(512, 2) void ag_gemm(Params) {
  CParams& p = *(CParams*)__builtin_amdgcn_kernarg_segment_ptr();
  extern __shared__ __attribute__((aligned(16))) char smem[];
  LAS char* lds = (LAS char*)smem + 1024;
  const int lane = threadIdx.x & 63, w = rfl(threadIdx.x >> 6), sg = w / NB;
  const int b = blockIdx.x;
  u64 px, pw, ws0;
  asm volatile("s_load_dwordx2 %0, %3, 0x40\n\ts_load_dwordx2 %1, %3, 0x48\n\ts_load_dwordx2 %2, %3, 0x0" : "=&s"(px), "=&s"(pw), "=&s"(ws0) : "s"(&p));
  static_assert(__builtin_offsetof(Params, x) == 0x40 && __builtin_offsetof(Params, w) == 0x48, "arguments");
  const Role ro = role_of(b);
  Jobs jb; jb.init(ro.u, sg);
  int j0 = jb.job(0), j1 = jb.job(1);
  asm volatile("s_waitcnt lgkmcnt(0)" : "+s"(px), "+s"(pw), "+s"(ws0), "+s"(j0), "+s"(j1));
  u32* cnt = (u32*)(__attribute__((address_space(1))) u32*)(ws0 + CNT_OFF + b * 128);
  u32 call;
  asm volatile("s_load_dword %0, %1, 0x0" : "=&s"(call) : "s"(cnt));
  load_job((const char*)px, (const char*)pw, lds, j0, 0, w, lane);
  if (w == 4) ((LAS u32*)(lds + LDS_MAIN))[lane] = 0;
  if (w < 4) ((LAS u32*)smem)[threadIdx.x] = mask_entry(threadIdx.x);
  asm volatile("s_waitcnt lgkmcnt(0)" : "+s"(call) :: "memory");
  call += 1;
  LBAR;
  load_job((const char*)px, (const char*)pw, lds, j1, 1, w, lane);
  if (threadIdx.x == 511) *cnt = call;
  if (jb.nr) encode_jobs(p, lds, call, w, lane, jb);
  if (ro.kind == 0) gemm_small(p, lds, ro.a, ro.c, call, w, lane);
  else if (ro.kind == 1) gemm_wide<6, false>(p, lds, ro.a, ro.c, call, w, lane);
  else if (ro.kind == 2) gemm_wide<4, true>(p, lds, ro.a, ro.c, call, w, lane);
  else pull_role(p, lds, link_of(ro.a), ro.a / 7, (NPULL - ro.a % 7 + 6) / 7, call, w, lane);
}
}

extern "C" void kernel_launch(int rank, int nranks, ncclComm_t comm, hipStream_t stream,
                              void* const* d_in, const int* in_sizes, int n_in,
                              void* d_out, int out_size, const int* in_dtypes, int out_dtype,
                              void* const* d_out_all, void* const* d_ws_all, size_t ws_bytes) {
  using namespace agg;
  if (nranks != NR || (unsigned)rank >= (unsigned)NR || ws_bytes < WS_NEED || n_in < 2 || in_sizes[0] != NS * KD || in_sizes[1] != KD * NS ||
      out_size != KD * NS) { fprintf(stderr, "ag_gemm: unexpected configuration\n"); return; }
  Params p;
  for (int i = 0; i < NR; ++i) p.ws[i] = (char*)d_ws_all[rank ^ i];
  p.x = (const char*)d_in[0]; p.w = (const char*)d_in[1]; p.out = (char*)d_out; p.rank = rank; p.pad = 0;
  hipError_t e = hipFuncSetAttribute((const void*)ag_gemm, hipFuncAttributeMaxDynamicSharedMemorySize, LDS_BYTES);
  if (e != hipSuccess) { fprintf(stderr, "HIP error %s\n", hipGetErrorString(e)); return; }
  hipLaunchKernelGGL(ag_gemm, dim3(GRID), dim3(512), LDS_BYTES, stream, p);
  e = hipGetLastError();
  if (e != hipSuccess) fprintf(stderr, "HIP error %s\n", hipGetErrorString(e));
}
```
